```python
import numpy as np

kernel_name = "gpu_ag_gemm_m4096_k4096_n4096_bf16_none"


ABSMAX_RTOL = 0.008

COMPUTE_DTYPE = "bfloat16"
EPILOGUE = "none"
IN_QUANT = None
OUT_QUANT = None
IN_DTYPES = {"x": "bfloat16", "w_mat": "bfloat16"}
WORLD_SIZE = 8

def _to_bf16(x):
    u = np.ascontiguousarray(x, dtype=np.float32).view(np.uint32)
    return ((u + 0x7FFF + ((u >> 16) & 1)) & 0xFFFF0000).view(np.float32)

def setup_inputs():

    x = _to_bf16(np.random.randn(4096, 4096).astype(np.float32))
    w_mat = _to_bf16((np.random.randn(4096, 4096) / np.sqrt(4096)).astype(np.float32))
    return {"x": x, "w_mat": w_mat}

def _epilogue(y):
    return y

def reference(x, w_mat):
    y = (x @ w_mat).astype(np.float32)
    return _epilogue(y)
```

```cpp
#include <hip/hip_runtime.h>
#include <rccl/rccl.h>
#include <cstdio>
#include <cstdint>

#define HIP_CHECK(expr) do { hipError_t _e = (expr); if (_e != hipSuccess) { \
    fprintf(stderr, "HIP error %s at %s:%d\n", hipGetErrorString(_e), __FILE__, __LINE__); \
    return; } } while (0)

namespace agk {

#define GAS __attribute__((address_space(1)))
#define LAS __attribute__((address_space(3)))
typedef unsigned u32x4 __attribute__((ext_vector_type(4)));
typedef unsigned u32x2 __attribute__((ext_vector_type(2)));
typedef short s16x4 __attribute__((ext_vector_type(4)));
typedef short s16x8 __attribute__((ext_vector_type(8)));
typedef __bf16 bf16x8 __attribute__((ext_vector_type(8)));
typedef __bf16 bf16x2 __attribute__((ext_vector_type(2)));
typedef float f32x4 __attribute__((ext_vector_type(4)));

constexpr int NR = 8;
constexpr int NKS = 64;
constexpr int NALPHA = 5, NBETA = 3;
constexpr int NEARLY = 2;
constexpr int NWG = 256;
constexpr int NTASK = 320;
#define AG_NPW 24
constexpr int NPW = AG_NPW;
#define AG_HOLD 1
static_assert(7 * NPW <= NWG, "pusher waves");
constexpr unsigned SHARD_BYTES = 512u * 4096u * 2u;

constexpr unsigned OFF_CTR = 0;
constexpr unsigned OFF_TMO = 32u << 10;
constexpr unsigned OFF_HOLD = 33u << 10;
constexpr unsigned OFF_FA = 64u << 10;
constexpr unsigned OFF_FW = 80u << 10;
constexpr unsigned OFF_DONE = 96u << 10;
constexpr unsigned OFF_DBG = 128u << 10;
constexpr unsigned OFF_DATA = 1u << 20;
constexpr unsigned ABLK_BYTES = NKS * 8192u;
constexpr unsigned WQ_BYTES = NKS * 16384u;
constexpr unsigned OFF_WST = 32u * ABLK_BYTES;
constexpr unsigned PAR_BYTES = OFF_WST + 4u * WQ_BYTES;
constexpr unsigned WS_NEED = OFF_DATA + 2u * PAR_BYTES;

constexpr int SA = 6, SB = 5;
constexpr int INFA = 4, INFB = 3;
constexpr unsigned L_A = 0, L_B = SA * 8192u, L_ST = L_B + SB * 16384u;
constexpr unsigned ST_ROW = 144, ST_WAVE = 32 * ST_ROW;
constexpr unsigned L_CTL = L_ST + 4 * ST_WAVE;
constexpr unsigned C_FULLA = L_CTL, C_FULLB = L_CTL + 4, C_ABORT = L_CTL + 8, C_CNT = L_CTL + 12, C_FREE = L_CTL + 16;
constexpr unsigned LDS_BYTES = L_CTL + 64;
static_assert(LDS_BYTES <= 160u * 1024u, "LDS");

constexpr unsigned long long DEADLINE = 30000000ull;

struct Params {
    const void* x;
    const void* w;
    char* ws[NR];
    char* out[NR];
    int rank;
    unsigned tile[NWG];
};

__device__ __forceinline__ unsigned long long rt() { return __builtin_amdgcn_s_memrealtime(); }
__device__ __forceinline__ unsigned rfl(unsigned v) { return (unsigned)__builtin_amdgcn_readfirstlane((int)v); }
__device__ __forceinline__ __amdgpu_buffer_rsrc_t mk_rsrc(const void* p, unsigned bytes) {
    return __builtin_amdgcn_make_buffer_rsrc((void*)p, (short)0, (int)bytes, 0x00020000);
}
#define VM_WAIT(n) asm volatile("s_waitcnt vmcnt(" #n ")" ::: "memory")
__device__ __forceinline__ void lds_st(unsigned addr, unsigned v) {
    asm volatile("ds_write_b32 %0, %1" :: "v"(addr), "v"(v) : "memory");
}
__device__ __forceinline__ unsigned lds_ld(unsigned addr) {
    unsigned v;
    asm volatile("ds_read_b32 %0, %1\n\ts_waitcnt lgkmcnt(0)" : "=&v"(v) : "v"(addr) : "memory");
    return rfl(v);
}
__device__ __forceinline__ u32x2 lds_ld2(unsigned addr) {
    u32x2 v;
    asm volatile("ds_read_b64 %0, %1\n\ts_waitcnt lgkmcnt(0)" : "=&v"(v) : "v"(addr) : "memory");
    u32x2 r; r.x = rfl(v.x); r.y = rfl(v.y);
    return r;
}
__device__ __forceinline__ unsigned lds_min4(unsigned addr) {
    u32x4 v;
    asm volatile("ds_read_b128 %0, %1\n\ts_waitcnt lgkmcnt(0)" : "=&v"(v) : "v"(addr) : "memory");
    return rfl(min(min(v.x, v.y), min(v.z, v.w)));
}
__device__ __forceinline__ unsigned ld_agent(const char* p) {
    return __hip_atomic_load((const GAS unsigned*)p, __ATOMIC_RELAXED, __HIP_MEMORY_SCOPE_AGENT);
}
__device__ __forceinline__ void st_agent(char* p, unsigned v) {
    __hip_atomic_store((GAS unsigned*)p, v, __ATOMIC_RELAXED, __HIP_MEMORY_SCOPE_AGENT);
}
__device__ __forceinline__ void st_sys(char* p, unsigned v) {
    __hip_atomic_store((GAS unsigned*)p, v, __ATOMIC_RELAXED, __HIP_MEMORY_SCOPE_SYSTEM);
}
__device__ __forceinline__ unsigned poll32(const char* p) {
    unsigned v;
    asm volatile("global_load_dword %0, %1, off sc0 sc1\n\ts_waitcnt vmcnt(0)" : "=&v"(v) : "v"(p) : "memory");
    return v;
}
__device__ __forceinline__ unsigned poll64_min(const char* p) {
    u32x2 v;
    asm volatile("global_load_dwordx2 %0, %1, off sc0 sc1\n\ts_waitcnt vmcnt(0)" : "=&v"(v) : "v"(p) : "memory");
    return min(v.x, v.y);
}

#define DMA_P(V, POL) "buffer_load_dwordx4 " V ", %5, %1 offen " POL "lds\n\ts_add_u32 m0, m0, 0x400\n\ts_add_u32 %1, %1, %7\n\t"
#define DMA_HEAD "s_nop 4\n\ts_mov_b32 %0, m0\n\ts_mov_b32 m0, %2\n\ts_mov_b32 %1, %6\n\t"
#define DMA_TAIL "s_mov_b32 m0, %0"
#define DMA8(PA, PB, PC, PD, POL) \
    asm volatile(DMA_HEAD DMA_P(PA, POL) DMA_P(PB, POL) DMA_P(PC, POL) DMA_P(PD, POL) \
                 DMA_P(PA, POL) DMA_P(PB, POL) DMA_P(PC, POL) DMA_P(PD, POL) DMA_TAIL \
                 : "=&s"(keep), "=&s"(t) : "s"(lds), "v"(v0), "v"(v1), "s"(rs), "s"(soff), "s"(js) : "scc", "memory")
template <int PAT>
__device__ __forceinline__ void dma8(unsigned lds, unsigned v0, unsigned v1, __amdgpu_buffer_rsrc_t rs,
                                     unsigned soff, unsigned js, bool sys) {
    unsigned keep, t;
    if (PAT == 0) {
        if (sys) DMA8("%3", "%4", "%3", "%4", "sc0 sc1 "); else DMA8("%3", "%4", "%3", "%4", "");
    } else {
        if (sys) DMA8("%3", "%3", "%4", "%4", "sc0 sc1 "); else DMA8("%3", "%3", "%4", "%4", "");
    }
}

#define STAMP(i) do {} while (0)

__device__ __forceinline__ unsigned x_lane_off(int l) { return (unsigned)((l >> 3) * 8192 + (((l & 7) ^ (l >> 4)) << 4)); }
__device__ __forceinline__ unsigned w_lane_off(int l) { return (unsigned)((l >> 4) * 1024 + ((((l & 15) >> 1) ^ (l >> 4)) << 5) + (l & 1) * 16); }

template <int PIECES8, int NSLOT, int INFL, int PAT>
__device__ __forceinline__ void loader(char* myws, unsigned call, int lane, bool staged,
                                       __amdgpu_buffer_rsrc_t rs, unsigned sbase, unsigned kstride, unsigned js,
                                       unsigned v0, unsigned v1, const char* flagp,
                                       unsigned lring, unsigned cfull, int stamp0) {
    unsigned ready = staged ? 0u : (unsigned)NKS;
    unsigned pub = 0, freed = 0, slot = 0;
    bool dead = false;
    for (unsigned s = 0; s < (unsigned)NKS; ++s) {
        if (s >= ready) {
            const unsigned long long t0 = rt();
            unsigned spins = 0;
            for (;;) {
                const unsigned f = PIECES8 == 1 ? poll32(flagp) : poll64_min(flagp);
                if (pub < s) { pub = s; lds_st(cfull, pub); }
                const unsigned long long m = ~__ballot(f >= call);
                ready = m ? (unsigned)__builtin_ctzll(m) : (unsigned)NKS;
                if (ready > s) break;
                if ((++spins & 15u) == 0u) {
                    if (rfl(ld_agent(myws + OFF_TMO)) == call || lds_ld(C_ABORT) != 0u || rt() - t0 > DEADLINE) { dead = true; break; }
                }
                __builtin_amdgcn_s_sleep(1);
            }
            if (dead) break;
            if (s == 0) STAMP(stamp0);
        }
        if (s >= (unsigned)NSLOT && freed < s - NSLOT + 1u) {
            freed = lds_min4(C_FREE);
            if (freed < s - NSLOT + 1u) {
                if (pub < s) { VM_WAIT(0); pub = s; lds_st(cfull, pub); }
                const unsigned long long t0 = rt();
                unsigned spins = 0;
                for (;;) {
                    freed = lds_min4(C_FREE);
                    if (freed >= s - NSLOT + 1u) break;
                    if ((++spins & 255u) == 0u) {
                        if (lds_ld(C_ABORT) != 0u || rt() - t0 > 2 * DEADLINE) { dead = true; break; }
                    }
                    __builtin_amdgcn_s_sleep(1);
                }
                if (dead) break;
            }
        }
        const unsigned ldst = lring + slot * (PIECES8 * 8192u);
        const unsigned soff = sbase + s * kstride;
        dma8<PAT>(ldst, v0, v1, rs, soff, js, staged);
        if (PIECES8 == 2) dma8<PAT>(ldst + 8192u, v0, v1, rs, soff + 8u * js, js, staged);
        slot = slot + 1u == (unsigned)NSLOT ? 0u : slot + 1u;
        if (s + 1u - pub > (unsigned)INFL) {
            static_assert(PIECES8 * 8 * INFL == 32 || PIECES8 * 8 * INFL == 48, "vmcnt ladder");
            if (PIECES8 * 8 * INFL == 32) VM_WAIT(32); else VM_WAIT(48);
            pub = s + 1u - INFL;
            lds_st(cfull, pub);
        }
    }
    VM_WAIT(0);
    if (dead) {
        lds_st(C_ABORT, 1u);
        if (lane == 0) st_agent(myws + OFF_TMO, call);
    }
    lds_st(cfull, (unsigned)NKS);
    STAMP(stamp0 + 1);
}

__global__ void __launch_bounds__(512, 2) ag_gemm_kernel(const Params p) {
    __shared__ __attribute__((aligned(1024))) unsigned char lds_raw[LDS_BYTES];
    LAS unsigned char* L = (LAS unsigned char*)lds_raw;
    const int lane = threadIdx.x & 63;
    const int wave = (int)rfl(threadIdx.x >> 6);
    const int rank = p.rank;
    const int gidx = rank & 3, gbase = rank & 4, obase = gbase ^ 4;
    char* myws = p.ws[rank];
    const unsigned bid = blockIdx.x;
    if ((unsigned)(size_t)L != 0u) return;
    if (wave == 3 && lane < 16) *(LAS unsigned*)(L + L_CTL + lane * 4) = 0u;
    asm volatile("s_waitcnt lgkmcnt(0)" ::: "memory");
    __builtin_amdgcn_s_barrier();

    auto draw_call = [&]() -> unsigned {
        unsigned ticket = 0;
        if (lane == 0)
            ticket = __hip_atomic_fetch_add((GAS unsigned*)(myws + OFF_CTR + bid * 128u + wave * 4u), 1u,
                                            __ATOMIC_RELAXED, __HIP_MEMORY_SCOPE_AGENT);
        return rfl(ticket) + 1u;
    };

    if (wave == 0) {
        STAMP(0);
        if (bid >= 7u * NPW) return;
        const int link = (int)(bid % 7u), pi = (int)(bid / 7u);
        const int peer = (rank + 1 + link) & 7;
        const bool same = (peer & 4) == gbase;
        const __amdgpu_buffer_rsrc_t rx = mk_rsrc(p.x, SHARD_BYTES), rw = mk_rsrc(p.w, SHARD_BYTES);
        char* pws = p.ws[peer];
        const __amdgpu_buffer_rsrc_t rd = mk_rsrc(pws, WS_NEED);
        const unsigned xa0 = x_lane_off(lane), xa1 = xa0 ^ 64u;
        const unsigned wb0 = w_lane_off(lane), wb1 = wb0 ^ 128u;
        const unsigned dl = (unsigned)lane * 16u;

        auto load_task = [&](int t, u32x4 (&b)[8]) {
            if (!same && t < 128) {
                const unsigned so = (unsigned)(t * 32) * 1024u + (unsigned)(peer & 3) * 256u;
#pragma unroll
                for (int j = 0; j < 8; ++j)
                    b[j] = __builtin_amdgcn_raw_buffer_load_b128(rw, ((j >> 1) & 1) ? wb1 : wb0, so + j * 4096u, 0);
            } else {
                int ks, blk;
                if (t < 128) { ks = t >> 1; blk = t & 1; }
                else { const int u = t - 128; ks = u / 3; blk = (same ? NEARLY : NALPHA) + (u - ks * 3); }
                const unsigned so = (unsigned)blk * (64u * 8192u) + (unsigned)ks * 128u;
#pragma unroll
                for (int j = 0; j < 8; ++j)
                    b[j] = __builtin_amdgcn_raw_buffer_load_b128(rx, (j & 1) ? xa1 : xa0, so + j * 65536u, 0);
            }
        };
        u32x4 b0[8], b1[8];
        load_task(pi, b0);
        const unsigned call = draw_call();
        __builtin_amdgcn_s_waitcnt(0x0F70);
        const unsigned par = OFF_DATA + (call & 1u) * PAR_BYTES;
        auto store_task = [&](int t, const u32x4 (&b)[8]) {
            unsigned dof, fof;
            if (!same && t < 128) {
                dof = OFF_WST + (unsigned)gidx * WQ_BYTES + (unsigned)t * 8192u;
                fof = OFF_FW + (unsigned)(gidx * 128 + t) * 4u;
            } else {
                int ks, ai;
                if (t < 128) { ks = t >> 1; ai = gidx * NALPHA + (t & 1); }
                else {
                    const int u = t - 128; ks = u / 3;
                    const int r = u - ks * 3;
                    ai = same ? gidx * NALPHA + NEARLY + r : 4 * NALPHA + gidx * NBETA + r;
                }
                dof = (unsigned)ai * ABLK_BYTES + (unsigned)ks * 8192u;
                fof = OFF_FA + (unsigned)(ai * 64 + ks) * 4u;
            }
            dof += par;
#pragma unroll
            for (int j = 0; j < 8; ++j)
                __builtin_amdgcn_raw_buffer_store_b128(b[j], rd, dl, dof + j * 1024u, 17);
            VM_WAIT(0);
            __builtin_amdgcn_s_waitcnt(0x0F70);
            asm volatile("" ::: "memory");
            if (lane == 0) st_sys(pws + fof, call);
        };
        for (int t = pi; t < NTASK; t += 2 * NPW) {
            const bool has1 = t + NPW < NTASK;
            if (has1) load_task(t + NPW, b1);
            store_task(t, b0);
            if (!has1) break;
            if (t + 2 * NPW < NTASK) load_task(t + 2 * NPW, b0);
            store_task(t + NPW, b1);
        }
        if (!same && lane == 0)
            __hip_atomic_fetch_add((GAS unsigned*)(myws + OFF_HOLD + (unsigned)(peer & 3) * 128u), 1u,
                                   __ATOMIC_RELAXED, __HIP_MEMORY_SCOPE_AGENT);
        STAMP(1);
        return;
    }

    if (wave == 3 && bid != NWG - 1) return;
    const unsigned call = draw_call();
    const unsigned par = OFF_DATA + (call & 1u) * PAR_BYTES;
    const int rb = (int)(p.tile[bid] & 63u), cb = (int)((p.tile[bid] >> 8) & 7u);
    int src, blk;
    if (rb < 4 * NALPHA) { src = gbase + rb / NALPHA; blk = rb % NALPHA; }
    else if (rb < 32) { src = obase + (rb - 4 * NALPHA) / NBETA; blk = NALPHA + (rb - 4 * NALPHA) % NBETA; }
    else { src = rank; blk = NALPHA + (rb - 32); }

    if (wave == 1) {
        const bool staged = src != rank;
        const __amdgpu_buffer_rsrc_t rs = staged ? mk_rsrc(myws, WS_NEED) : mk_rsrc(p.x, SHARD_BYTES);
        const unsigned v0 = staged ? (unsigned)lane * 16u : x_lane_off(lane);
        const unsigned v1 = staged ? v0 : v0 ^ 64u;
        const unsigned sbase = staged ? par + (unsigned)rb * ABLK_BYTES : (unsigned)blk * (64u * 8192u);
        loader<1, SA, INFA, 0>(myws, call, lane, staged, rs, sbase, staged ? 8192u : 128u, staged ? 1024u : 65536u,
                               v0, v1, myws + OFF_FA + (unsigned)((rb & 31) * 64 + lane) * 4u, L_A, C_FULLA, 2);
        return;
    }
    if (wave == 2) {
        const bool staged = cb >= 4;
        const __amdgpu_buffer_rsrc_t rs = staged ? mk_rsrc(myws, WS_NEED) : mk_rsrc(p.w, SHARD_BYTES);
        const unsigned v0 = staged ? (unsigned)lane * 16u : w_lane_off(lane);
        const unsigned v1 = staged ? v0 : v0 ^ 128u;
        const unsigned sbase = staged ? par + OFF_WST + (unsigned)(cb & 3) * WQ_BYTES : (unsigned)cb * 256u;
        loader<2, SB, INFB, 1>(myws, call, lane, staged, rs, sbase, staged ? 16384u : 65536u, staged ? 1024u : 4096u,
                               v0, v1, myws + OFF_FW + (unsigned)((cb & 3) * 128 + lane * 2) * 4u, L_B, C_FULLB, 4);
        return;
    }
    if (wave == 3) {
        const int i0 = (lane * 2) & 31, ex = 4 * NALPHA + gidx * NBETA;
        const bool skip0 = i0 >= ex && i0 < ex + NBETA, skip1 = i0 + 1 >= ex && i0 + 1 < ex + NBETA;
        const char* fp = myws + OFF_DONE + (unsigned)lane * 8u;
        const unsigned long long t0 = rt();
        unsigned spins = 0;
        bool dead = false;
        for (;;) {
            u32x2 v;
            asm volatile("global_load_dwordx2 %0, %1, off sc0 sc1\n\ts_waitcnt vmcnt(0)" : "=&v"(v) : "v"(fp) : "memory");
            const bool ok = (skip0 || v.x >= call) && (skip1 || v.y >= call);
            if (__all(ok)) break;
            if ((++spins & 15u) == 0u) {
                if (rfl(ld_agent(myws + OFF_TMO)) == call || rt() - t0 > 2 * DEADLINE) { dead = true; break; }
            }
            __builtin_amdgcn_s_sleep(1);
        }
        __builtin_amdgcn_fence(__ATOMIC_ACQUIRE, "");
        if (dead) {
            if (lane == 0) st_agent(myws + OFF_TMO, call);
            const unsigned long long t1 = rt();
            while (rt() - t1 < DEADLINE / 8) __builtin_amdgcn_s_sleep(64);
            if (lane == 0) st_sys(p.out[rank] + (size_t)rank * 512u * 1024u, 0x7FC07FC0u);
        }
        STAMP(6);
        return;
    }

    const int cw = wave - 4, wm = cw >> 1, wn = cw & 1;
    const int g = lane >> 4, r16 = lane & 15;
    const unsigned a0 = L_A + (unsigned)((wm * 32 + r16) * 128 + ((g ^ (r16 >> 1)) << 4));
    const unsigned a1 = a0 ^ 64u;
    const int q = (lane >> 2) & 3, pp = lane & 3, fsw = q | ((g & 1) << 2);
    unsigned bofs[4];
#pragma unroll
    for (int nt = 0; nt < 4; ++nt)
        bofs[nt] = L_B + (unsigned)((8 * g + q) * 256 + (((wn * 4 + nt) ^ fsw) << 5) + pp * 8);

    f32x4 acc[2][4];
#pragma unroll
    for (int mt = 0; mt < 2; ++mt)
#pragma unroll
        for (int nt = 0; nt < 4; ++nt) acc[mt][nt] = f32x4{0.f, 0.f, 0.f, 0.f};

    unsigned full = 0, sa = 0, sb = 0;
    for (unsigned s = 0; s < (unsigned)NKS; ++s) {
        if (full <= s) {
            const unsigned long long t0 = rt();
            unsigned spins = 0;
            for (;;) {
                const u32x2 f = lds_ld2(C_FULLA);
                full = min(f.x, f.y);
                if (full > s) break;
                if ((++spins & 1023u) == 0u && rt() - t0 > 3 * DEADLINE) { lds_st(C_ABORT, 1u); full = NKS; break; }
                __builtin_amdgcn_s_sleep(1);
            }
        }
        const unsigned ab = sa * 8192u, bb = sb * 16384u;
        s16x8 af[2][2];
        s16x4 bl[2][4], bh[2][4];
#pragma unroll
        for (int ks = 0; ks < 2; ++ks) {
#pragma unroll
            for (int mt = 0; mt < 2; ++mt)
                af[ks][mt] = *(const LAS s16x8*)(size_t)((ks ? a1 : a0) + ab + mt * 2048u);
#pragma unroll
            for (int nt = 0; nt < 4; ++nt) {
                bl[ks][nt] = __builtin_amdgcn_ds_read_tr16_b64_v4i16((LAS s16x4*)(size_t)(bofs[nt] + bb + ks * 8192u));
                bh[ks][nt] = __builtin_amdgcn_ds_read_tr16_b64_v4i16((LAS s16x4*)(size_t)(bofs[nt] + bb + ks * 8192u + 1024u));
            }
        }
#pragma unroll
        for (int nt = 0; nt < 4; ++nt) {
            const bf16x8 bf = __builtin_bit_cast(bf16x8, __builtin_shufflevector(bl[0][nt], bh[0][nt], 0, 1, 2, 3, 4, 5, 6, 7));
#pragma unroll
            for (int mt = 0; mt < 2; ++mt)
                acc[mt][nt] = __builtin_amdgcn_mfma_f32_16x16x32_bf16(bf, __builtin_bit_cast(bf16x8, af[0][mt]), acc[mt][nt], 0, 0, 0);
        }
        asm volatile("s_waitcnt lgkmcnt(0)\n\tds_write_b32 %8, %9"
                     : "+v"(af[1][0]), "+v"(af[1][1]), "+v"(bl[1][0]), "+v"(bl[1][1]), "+v"(bl[1][2]), "+v"(bl[1][3]),
                       "+v"(bh[1][0]), "+v"(bh[1][1])
                     : "v"(C_FREE + (unsigned)cw * 4u), "v"(s + 1u) : "memory");
        asm volatile("" : "+v"(bh[1][2]), "+v"(bh[1][3]) :: "memory");
#pragma unroll
        for (int nt = 0; nt < 4; ++nt) {
            const bf16x8 bf = __builtin_bit_cast(bf16x8, __builtin_shufflevector(bl[1][nt], bh[1][nt], 0, 1, 2, 3, 4, 5, 6, 7));
#pragma unroll
            for (int mt = 0; mt < 2; ++mt)
                acc[mt][nt] = __builtin_amdgcn_mfma_f32_16x16x32_bf16(bf, __builtin_bit_cast(bf16x8, af[1][mt]), acc[mt][nt], 0, 0, 0);
        }
        sa = sa + 1u == (unsigned)SA ? 0u : sa + 1u;
        sb = sb + 1u == (unsigned)SB ? 0u : sb + 1u;
    }
    if (cw == 0) STAMP(7);

    const bool bad = lds_ld(C_ABORT) != 0u;
    const unsigned stb = L_ST + (unsigned)cw * ST_WAVE;
#pragma unroll
    for (int mt = 0; mt < 2; ++mt)
#pragma unroll
        for (int nt = 0; nt < 4; ++nt) {
            bf16x2 lo, hi;
            lo[0] = (__bf16)acc[mt][nt][0]; lo[1] = (__bf16)acc[mt][nt][1];
            hi[0] = (__bf16)acc[mt][nt][2]; hi[1] = (__bf16)acc[mt][nt][3];
            u32x2 v; v.x = __builtin_bit_cast(unsigned, lo); v.y = __builtin_bit_cast(unsigned, hi);
            if (bad) { v.x = 0x7FC07FC0u; v.y = 0x7FC07FC0u; }
            *(LAS u32x2*)(size_t)(stb + (unsigned)((mt * 16 + r16) * ST_ROW + nt * 32 + g * 8)) = v;
        }
    u32x4 o[4];
#pragma unroll
    for (int i = 0; i < 4; ++i)
        o[i] = *(const LAS u32x4*)(size_t)(stb + (unsigned)((i * 8 + (lane >> 3)) * ST_ROW + (lane & 7) * 16));

    const bool remote = cb >= 4;
    const int dstr = remote ? obase + (cb & 3) : rank;
    const unsigned colb = (unsigned)(remote ? gidx : cb) * 256u + (unsigned)wn * 128u;
    const unsigned orow = (unsigned)(src * 512 + blk * 64 + wm * 32);
    const __amdgpu_buffer_rsrc_t ro = mk_rsrc(p.out[dstr], SHARD_BYTES);
    const unsigned ovo = (unsigned)(lane >> 3) * 1024u + (unsigned)(lane & 7) * 16u;
    const unsigned oso = orow * 1024u + colb;
    if (!remote) {
#pragma unroll
        for (int i = 0; i < 4; ++i) __builtin_amdgcn_raw_buffer_store_b128(o[i], ro, ovo, oso + i * 8192u, 0);
    } else {
        if (src == rank || (rb < 4 * NALPHA && blk < NEARLY)) {
            const unsigned long long t0 = rt();
            const char* hp = myws + OFF_HOLD + (unsigned)(cb & 3) * 128u;
            while (rfl(ld_agent(hp)) < call * (unsigned)NPW && rt() - t0 < DEADLINE) __builtin_amdgcn_s_sleep(8);
        }
#pragma unroll
        for (int i = 0; i < 4; ++i) __builtin_amdgcn_raw_buffer_store_b128(o[i], ro, ovo, oso + i * 8192u, 17);
        VM_WAIT(0);
        unsigned old;
        asm volatile("ds_add_rtn_u32 %0, %1, %2\n\ts_waitcnt lgkmcnt(0)" : "=&v"(old) : "v"(C_CNT), "v"(lane == 0 ? 1u : 0u) : "memory");
        if (rfl(old) == 3u && lane == 0) st_sys(p.ws[dstr] + OFF_DONE + (unsigned)(gidx * 32 + rb) * 4u, call);
    }
    if (cw == 0) STAMP(8);
}

}

extern "C" void kernel_launch(int rank, int nranks, ncclComm_t comm,
                              hipStream_t stream,
                              void* const* d_in, const int* in_sizes, int n_in,
                              void* d_out, int out_size,
                              const int* in_dtypes, int out_dtype,
                              void* const* d_out_all,
                              void* const* d_ws_all, size_t ws_bytes) {
    using namespace agk;
    (void)comm; (void)in_dtypes; (void)out_dtype;
    if (nranks != NR || (unsigned)rank >= (unsigned)NR || n_in < 2 || ws_bytes < (size_t)WS_NEED ||
        in_sizes[0] != 512 * 4096 || in_sizes[1] != 512 * 4096 || out_size != 512 * 4096) {
        fprintf(stderr, "kernel_launch: unexpected problem shape\n");
        return;
    }
    Params p;
    p.x = d_in[0];
    p.w = d_in[1];
    for (int r = 0; r < NR; ++r) { p.ws[r] = (char*)d_ws_all[r]; p.out[r] = (char*)d_out_all[r]; }
    p.out[rank] = (char*)d_out;
    p.rank = rank;
    unsigned char lrb[NWG], lcb[NWG];
    int n = 0;
    for (int cg = 0; cg < 2; ++cg)
        for (int rb = 0; rb < 35; ++rb)
            for (int c = 0; c < 4; ++c) {
                if (cg == 1 && (rb >= 32 || (rb >= 4 * NALPHA && (rb - 4 * NALPHA) / NBETA == c))) continue;
                if (n < NWG) { lrb[n] = (unsigned char)rb; lcb[n] = (unsigned char)(cg * 4 + c); }
                ++n;
            }
    if (n != NWG) { fprintf(stderr, "kernel_launch: tile count %d\n", n); return; }
    for (int b = 0; b < NWG; ++b) {
        const int t = (b % 8) * 32 + b / 8;
        p.tile[b] = (unsigned)lrb[t] | ((unsigned)lcb[t] << 8);
    }
    hipLaunchKernelGGL(ag_gemm_kernel, dim3(NWG), dim3(512), 0, stream, p);
    HIP_CHECK(hipGetLastError());
}
```

```cpp
#include <hip/hip_runtime.h>
#include <rccl/rccl.h>
#include <cstdio>
#include <cstdint>

#define HIP_CHECK(expr) do { hipError_t _e = (expr); if (_e != hipSuccess) { \
    fprintf(stderr, "HIP error %s at %s:%d\n", hipGetErrorString(_e), __FILE__, __LINE__); \
    return; } } while (0)

namespace agk {

#define GAS __attribute__((address_space(1)))
#define LAS __attribute__((address_space(3)))
typedef unsigned u32x4 __attribute__((ext_vector_type(4)));
typedef unsigned u32x2 __attribute__((ext_vector_type(2)));
typedef short s16x4 __attribute__((ext_vector_type(4)));
typedef short s16x8 __attribute__((ext_vector_type(8)));
typedef __bf16 bf16x8 __attribute__((ext_vector_type(8)));
typedef __bf16 bf16x2 __attribute__((ext_vector_type(2)));
typedef float f32x4 __attribute__((ext_vector_type(4)));

constexpr int NR = 8;
constexpr int NKS = 64;
constexpr int NALPHA = 5, NBETA = 3;
constexpr int NEARLY = 2;
constexpr int NWG = 256;
constexpr int NTILE = 256;
constexpr int NTASK = 320;
#define AG_CPL 8
#define AG_WPP 2
#define AG_XMAP 1
constexpr int CPL = AG_CPL;
constexpr int WPP = AG_WPP;
constexpr int NPW = CPL * WPP;
constexpr int NPWG = 7 * CPL;
constexpr int PMOD = AG_XMAP ? 8 : 7;
#define AG_POLL_SLEEP 2
#define AG_HOLD 0
#define AG_HOLD_LEAD 0
static_assert(NPWG <= 56 && WPP <= 9, "pushing workgroups: at most one per early tile with a local output");
constexpr unsigned SHARD_BYTES = 512u * 4096u * 2u;

constexpr unsigned OFF_CTR = 0;
constexpr unsigned OFF_TMO = 32u << 10;
constexpr unsigned OFF_Q = 33u << 10;
constexpr unsigned OFF_GO = 34u << 10;
constexpr unsigned OFF_FA = 64u << 10;
constexpr unsigned OFF_FW = 80u << 10;
constexpr unsigned OFF_DONE = 96u << 10;
constexpr unsigned OFF_DBG = 128u << 10;
constexpr unsigned OFF_DATA = 1u << 20;
constexpr unsigned ABLK_BYTES = NKS * 8192u;
constexpr unsigned WQ_BYTES = NKS * 16384u;
constexpr unsigned OFF_WST = 32u * ABLK_BYTES;
constexpr unsigned PAR_BYTES = OFF_WST + 4u * WQ_BYTES;
constexpr unsigned WS_NEED = OFF_DATA + 2u * PAR_BYTES;

constexpr int NSLOT = 6;
constexpr unsigned SLOT_BYTES = 24576u;
constexpr unsigned L_ST = NSLOT * SLOT_BYTES;
constexpr unsigned ST_ROW = 144, ST_WAVE = 16 * ST_ROW;
constexpr unsigned L_CTL = L_ST + 4 * ST_WAVE;
constexpr unsigned C_FREE = L_CTL;
constexpr unsigned C_FULL = L_CTL + 16;
constexpr unsigned C_ABORT = L_CTL + 40;
constexpr unsigned C_CNT = L_CTL + 44;
constexpr unsigned LDS_BYTES = L_CTL + 64;
static_assert(LDS_BYTES <= 160u * 1024u, "LDS");
constexpr int NTHREADS = (NSLOT + 4) * 64;

constexpr unsigned long long DEADLINE = 30000000ull;

struct Params {
    const void* x;
    const void* w;
    char* ws[NR];
    char* out[NR];
    int rank;
    unsigned tile[NWG];
};

__device__ __forceinline__ unsigned long long rt() { return __builtin_amdgcn_s_memrealtime(); }
__device__ __forceinline__ bool expired(unsigned long long& t0, unsigned long long limit) {
    const unsigned long long now = __builtin_amdgcn_s_memrealtime() | 1ull;
    if (t0 == 0ull) { t0 = now; return false; }
    return now - t0 > limit;
}
__device__ __forceinline__ unsigned rfl(unsigned v) { return (unsigned)__builtin_amdgcn_readfirstlane((int)v); }
__device__ __forceinline__ __amdgpu_buffer_rsrc_t mk_rsrc(const void* p, unsigned bytes) {
    return __builtin_amdgcn_make_buffer_rsrc((void*)p, (short)0, (int)bytes, 0x00020000);
}
#define VM_WAIT(n) asm volatile("s_waitcnt vmcnt(" #n ")" ::: "memory")
__device__ __forceinline__ void lds_st(unsigned addr, unsigned v) {
    asm volatile("ds_write_b32 %0, %1" :: "v"(addr), "v"(v) : "memory");
}
__device__ __forceinline__ unsigned lds_ld(unsigned addr) {
    unsigned v;
    asm volatile("ds_read_b32 %0, %1\n\ts_waitcnt lgkmcnt(0)" : "=&v"(v) : "v"(addr) : "memory");
    return rfl(v);
}
__device__ __forceinline__ u32x4 lds_ld4(unsigned addr) {
    u32x4 v;
    asm volatile("ds_read_b128 %0, %1\n\ts_waitcnt lgkmcnt(0)" : "=&v"(v) : "v"(addr) : "memory");
    return v;
}
__device__ __forceinline__ unsigned ld_agent(const char* p) {
    return __hip_atomic_load((const GAS unsigned*)p, __ATOMIC_RELAXED, __HIP_MEMORY_SCOPE_AGENT);
}
__device__ __forceinline__ void st_agent(char* p, unsigned v) {
    __hip_atomic_store((GAS unsigned*)p, v, __ATOMIC_RELAXED, __HIP_MEMORY_SCOPE_AGENT);
}
__device__ __forceinline__ void st_sys(char* p, unsigned v) {
    __hip_atomic_store((GAS unsigned*)p, v, __ATOMIC_RELAXED, __HIP_MEMORY_SCOPE_SYSTEM);
}
#define DMA_P(V) "buffer_load_dwordx4 " V ", %5, %1 offen sc0 sc1 lds\n\ts_add_u32 m0, m0, 0x400\n\ts_add_u32 %1, %1, %7\n\t"
#define DMA_HEAD "s_nop 4\n\ts_mov_b32 %0, m0\n\ts_mov_b32 m0, %2\n\ts_mov_b32 %1, %6\n\t"
#define DMA_TAIL "s_mov_b32 m0, %0"
#define DMA8(PA, PB, PC, PD) \
    asm volatile(DMA_HEAD DMA_P(PA) DMA_P(PB) DMA_P(PC) DMA_P(PD) DMA_P(PA) DMA_P(PB) DMA_P(PC) DMA_P(PD) DMA_TAIL \
                 : "=&s"(keep), "=&s"(t) : "s"(lds), "v"(v0), "v"(v1), "s"(rs), "s"(soff), "s"(js) : "scc", "memory")
template <int PAT>
__device__ __forceinline__ void dma8(unsigned lds, unsigned v0, unsigned v1, __amdgpu_buffer_rsrc_t rs,
                                     unsigned soff, unsigned js) {
    unsigned keep, t;
    if (PAT == 0) DMA8("%3", "%4", "%3", "%4"); else DMA8("%3", "%3", "%4", "%4");
}

#define STAMP(i) do {} while (0)

__device__ __forceinline__ unsigned x_lane_off(int l) { return (unsigned)((l >> 3) * 8192 + (((l & 7) ^ (l >> 4)) << 4)); }
__device__ __forceinline__ unsigned w_lane_off(int l) { return (unsigned)((l >> 4) * 1024 + ((((l & 15) >> 1) ^ (l >> 4)) << 5) + (l & 1) * 16); }

struct Tile { int rb, cb, src, blk; };
__device__ __forceinline__ Tile tile_of(unsigned code, int rank) {
    const int gidx = rank & 3, gbase = rank & 4, obase = gbase ^ 4;
    Tile t;
    t.rb = (int)(code & 63u); t.cb = (int)((code >> 6) & 7u);
    if (t.rb < 4 * NALPHA) { t.src = gbase + t.rb / NALPHA; t.blk = t.rb % NALPHA; }
    else if (t.rb < 32) { t.src = obase + (t.rb - 4 * NALPHA) / NBETA; t.blk = NALPHA + (t.rb - 4 * NALPHA) % NBETA; }
    else { t.src = rank; t.blk = NALPHA + (t.rb - 32); }
    return t;
}

struct Frag { s16x8 a[2], b[4]; };

__global__ void __launch_bounds__(NTHREADS, 3) ag_gemm_kernel(const Params p) {
    __shared__ __attribute__((aligned(1024))) unsigned char lds_raw[LDS_BYTES];
    LAS unsigned char* L = (LAS unsigned char*)lds_raw;
    const int lane = threadIdx.x & 63;
    const int wave = (int)rfl(threadIdx.x >> 6);
    const int rank = p.rank;
    const int gidx = rank & 3, gbase = rank & 4, obase = gbase ^ 4;
    char* myws = p.ws[rank];
    const unsigned bid = blockIdx.x;

    auto draw_call = [&]() -> unsigned {
        unsigned ticket = 0;
        if (lane == 0)
            ticket = __hip_atomic_fetch_add((GAS unsigned*)(myws + OFF_CTR + bid * 128u + (unsigned)wave * 4u), 1u,
                                            __ATOMIC_RELAXED, __HIP_MEMORY_SCOPE_AGENT);
        return rfl(ticket) + 1u;
    };

    if (bid < (unsigned)(PMOD * CPL) && bid % (unsigned)PMOD < 7u) {
        if (wave < WPP) {
            if (wave == 0) STAMP(0);
            const int link = (int)(bid % (unsigned)PMOD), pi = wave * CPL + (int)(bid / (unsigned)PMOD);
            const int peer = (rank + 1 + link) & 7;
            const bool same = (peer & 4) == gbase;
            const __amdgpu_buffer_rsrc_t rx = mk_rsrc(p.x, SHARD_BYTES), rw = mk_rsrc(p.w, SHARD_BYTES);
            char* pws = p.ws[peer];
            const __amdgpu_buffer_rsrc_t rd = mk_rsrc(pws, WS_NEED);
            const unsigned xa0 = x_lane_off(lane), xa1 = xa0 ^ 64u;
            const unsigned wb0 = w_lane_off(lane), wb1 = wb0 ^ 128u;
            const unsigned dl = (unsigned)lane * 16u;

            auto load_task = [&](int t, u32x4 (&b)[8]) {
                if (!same && t < 128) {
                    const unsigned so = (unsigned)(t * 32) * 1024u + (unsigned)(peer & 3) * 256u;
#pragma unroll
                    for (int j = 0; j < 8; ++j)
                        b[j] = __builtin_amdgcn_raw_buffer_load_b128(rw, ((j >> 1) & 1) ? wb1 : wb0, so + j * 4096u, 0);
                } else {
                    int ks, blk;
                    if (t < 128) { ks = t >> 1; blk = t & 1; }
                    else { const int u = t - 128; ks = u / 3; blk = (same ? NEARLY : NALPHA) + (u - ks * 3); }
                    const unsigned so = (unsigned)blk * (64u * 8192u) + (unsigned)ks * 128u;
#pragma unroll
                    for (int j = 0; j < 8; ++j)
                        b[j] = __builtin_amdgcn_raw_buffer_load_b128(rx, (j & 1) ? xa1 : xa0, so + j * 65536u, 0);
                }
            };
            u32x4 b0[8];
            load_task(pi, b0);
            char* qp = myws + OFF_Q + (unsigned)link * 128u;
            unsigned tk0 = 0;
            if (lane == 0)
                tk0 = __hip_atomic_fetch_add((GAS unsigned*)qp, 1u, __ATOMIC_RELAXED, __HIP_MEMORY_SCOPE_AGENT);
            const unsigned call = rfl(tk0) / (unsigned)NTASK + 1u;
            __builtin_amdgcn_s_waitcnt(0x0F70);
            const unsigned par = OFF_DATA + (call & 1u) * PAR_BYTES;
            const unsigned qbase = (call - 1u) * (unsigned)NTASK - (unsigned)NPW;
            unsigned fof;
            auto store_task = [&](int t, const u32x4 (&b)[8]) {
                unsigned dof;
                if (!same && t < 128) {
                    dof = OFF_WST + (unsigned)gidx * WQ_BYTES + (unsigned)t * 8192u;
                    fof = OFF_FW + (unsigned)(gidx * 128 + t) * 4u;
                } else {
                    int ks, ai;
                    if (t < 128) { ks = t >> 1; ai = gidx * NALPHA + (t & 1); }
                    else {
                        const int u = t - 128; ks = u / 3;
                        const int r = u - ks * 3;
                        ai = same ? gidx * NALPHA + NEARLY + r : 4 * NALPHA + gidx * NBETA + r;
                    }
                    dof = (unsigned)ai * ABLK_BYTES + (unsigned)ks * 8192u;
                    fof = OFF_FA + (unsigned)(ai * 64 + ks) * 4u;
                }
                dof += par;
#pragma unroll
                for (int j = 0; j < 8; ++j)
                    __builtin_amdgcn_raw_buffer_store_b128(b[j], rd, dl, dof + j * 1024u, 17);
            };
            auto draw = [&](unsigned& tk) {
                unsigned long long sv;
                asm volatile("s_mov_b64 %1, exec\n\ts_mov_b64 exec, 1\n\tglobal_atomic_add %0, %2, %3, off sc0\n\ts_mov_b64 exec, %1"
                             : "=&v"(tk), "=&s"(sv) : "v"(qp), "v"(1u) : "memory");
            };
            for (int t = pi;;) {
                const unsigned nx = rfl(tk0) - qbase;
                store_task(t, b0);
                if (nx == (unsigned)(NTASK - AG_HOLD_LEAD) && lane == 0) st_agent(myws + OFF_GO + (unsigned)link * 128u, call);
                if (nx < (unsigned)NTASK) { load_task((int)nx, b0); draw(tk0); }
                asm volatile("s_waitcnt vmcnt(0)" : "+v"(tk0) :: "memory");
                __builtin_amdgcn_s_waitcnt(0x0F70);
                asm volatile("" ::: "memory");
                if (lane == 0) st_sys(pws + fof, call);
                if (nx >= (unsigned)NTASK) break;
                t = (int)nx;
            }
            if (wave == WPP - 1) STAMP(9 + link);
            return;
        }
        if (wave != 9 || bid != 0u) return;
        const unsigned call = draw_call();
        const int i0 = (lane * 2) & 31, ex = 4 * NALPHA + gidx * NBETA;
        const bool skip0 = i0 >= ex && i0 < ex + NBETA, skip1 = i0 + 1 >= ex && i0 + 1 < ex + NBETA;
        const char* fp = myws + OFF_DONE + (unsigned)lane * 8u;
        unsigned long long t0 = 0;
        unsigned spins = 0;
        bool dead = false;
        for (;;) {
            u32x2 v;
            asm volatile("global_load_dwordx2 %0, %1, off sc0 sc1\n\ts_waitcnt vmcnt(0)" : "=&v"(v) : "v"(fp) : "memory");
            const bool ok = (skip0 || v.x >= call) && (skip1 || v.y >= call);
            if (__all(ok)) break;
            if ((++spins & 255u) == 0u) {
                if (rfl(ld_agent(myws + OFF_TMO)) == call || expired(t0, 2 * DEADLINE)) { dead = true; break; }
            }
            __builtin_amdgcn_s_sleep(2);
        }
        __builtin_amdgcn_fence(__ATOMIC_ACQUIRE, "");
        if (dead) {
            if (lane == 0) st_agent(myws + OFF_TMO, call);
            const unsigned long long t1 = rt();
            while (rt() - t1 < DEADLINE / 8) __builtin_amdgcn_s_sleep(64);
            if (lane == 0) st_sys(p.out[rank] + (size_t)rank * 512u * 1024u, 0x7FC07FC0u);
        }
        STAMP(6);
        return;
    }

    if ((unsigned)(size_t)L != 0u) return;
    if (wave == 0 && lane < 16) *(LAS unsigned*)(L + L_CTL + lane * 4) = 0u;
    const unsigned call = draw_call();
    const unsigned tcode = p.tile[bid];
    const unsigned ntile = (tcode >> 31) ? 2u : 1u;
    asm volatile("s_waitcnt lgkmcnt(0)" ::: "memory");
    __builtin_amdgcn_s_barrier();
    const unsigned par = OFF_DATA + (call & 1u) * PAR_BYTES;

    if (wave < NSLOT) {
        const unsigned lslot = (unsigned)wave * SLOT_BYTES, cfull = C_FULL + (unsigned)wave * 4u;
        const __amdgpu_buffer_rsrc_t rws = mk_rsrc(myws, WS_NEED);
        unsigned g = (unsigned)wave, spins = 0;
        unsigned long long t0 = 0;
        bool dead = false;
        for (unsigned ti = 0; ti < ntile && !dead; ++ti) {
            const Tile t = tile_of(tcode >> (16u * ti), rank);
            const bool sa = t.src != rank, sb = t.cb >= 4;
            const __amdgpu_buffer_rsrc_t ra = sa ? rws : mk_rsrc(p.x, SHARD_BYTES);
            const __amdgpu_buffer_rsrc_t rb = sb ? rws : mk_rsrc(p.w, SHARD_BYTES);
            const unsigned va0 = sa ? (unsigned)lane * 16u : x_lane_off(lane), va1 = sa ? va0 : va0 ^ 64u;
            const unsigned vb0 = sb ? (unsigned)lane * 16u : w_lane_off(lane), vb1 = sb ? vb0 : vb0 ^ 128u;
            const unsigned abase = sa ? par + (unsigned)t.rb * ABLK_BYTES : (unsigned)t.blk * (64u * 8192u);
            const unsigned bbase = sb ? par + OFF_WST + (unsigned)(t.cb & 3) * WQ_BYTES : (unsigned)t.cb * 256u;
            const unsigned aks = sa ? 8192u : 128u, ajs = sa ? 1024u : 65536u;
            const unsigned bks = sb ? 16384u : 65536u, bjs = sb ? 1024u : 4096u, bh = sb ? 8192u : 32768u;
            const bool poll = lane == 0 ? sa : (lane < 3 && sb);
            const unsigned fl0 = lane == 0 ? OFF_FA + (unsigned)(t.rb & 31) * 256u : OFF_FW + (unsigned)(t.cb & 3) * 512u + (unsigned)(lane & 2) * 2u;
            const unsigned fks = lane == 0 ? 4u : 8u;
            for (; g < (ti + 1u) * (unsigned)NKS; g += (unsigned)NSLOT) {
                const unsigned ks = g & (unsigned)(NKS - 1);
                if (sa || sb) {
                    const unsigned fo = fl0 + ks * fks;
                    for (;;) {
                        unsigned v = call;
                        if (poll) v = __builtin_amdgcn_raw_buffer_load_b32(rws, fo, 0, 17);
                        if (__all(v >= call)) break;
                        if ((++spins & 255u) == 0u &&
                            (rfl(ld_agent(myws + OFF_TMO)) == call || lds_ld(C_ABORT) != 0u || expired(t0, DEADLINE))) {
                            dead = true;
                            break;
                        }
                        __builtin_amdgcn_s_sleep(AG_POLL_SLEEP);
                    }
                    asm volatile("" ::: "memory");
                }
                while (g >= (unsigned)NSLOT && !dead) {
                    const u32x4 fr = lds_ld4(C_FREE);
                    if (rfl(min(min(fr.x, fr.y), min(fr.z, fr.w))) + (unsigned)NSLOT > g) break;
                    if ((++spins & 1023u) == 0u && (lds_ld(C_ABORT) != 0u || expired(t0, 2 * DEADLINE))) dead = true;
                    __builtin_amdgcn_s_sleep(1);
                }
                if (dead) break;
                if (g == (ntile - 1u) * (unsigned)NKS + (unsigned)wave && wave == 0) STAMP(2);
                dma8<0>(lslot, va0, va1, ra, abase + ks * aks, ajs);
                dma8<1>(lslot + 8192u, vb0, vb1, rb, bbase + ks * bks, bjs);
                dma8<1>(lslot + 16384u, vb0, vb1, rb, bbase + ks * bks + bh, bjs);
                VM_WAIT(0);
                lds_st(cfull, g + 1u);
            }
        }
        if (dead) {
            lds_st(C_ABORT, 1u);
            lds_st(cfull, 0x10000u);
            if (lane == 0) st_agent(myws + OFF_TMO, call);
        }
        if (wave == 3) STAMP(3);
        return;
    }

    const int cw = wave - NSLOT, wm = cw >> 1, wn = cw & 1;
    const int g4 = lane >> 4, r16 = lane & 15;
    const unsigned a0 = (unsigned)((wm * 32 + r16) * 128 + ((g4 ^ (r16 >> 1)) << 4));
    const unsigned a1 = a0 ^ 64u;
    const int q = (lane >> 2) & 3, pp = lane & 3, fsw = q | ((g4 & 1) << 2);
    unsigned bofs[4];
#pragma unroll
    for (int nt = 0; nt < 4; ++nt)
        bofs[nt] = 8192u + (unsigned)((8 * g4 + q) * 256 + (((wn * 4 + nt) ^ fsw) << 5) + pp * 8);
    const unsigned cfree = C_FREE + (unsigned)cw * 4u;

    unsigned spins = 0;
    unsigned long long t0 = 0;
    auto wait_full = [&](unsigned slot, unsigned g) {
        while (lds_ld(C_FULL + slot * 4u) <= g) {
            if ((++spins & 1023u) == 0u && expired(t0, 3 * DEADLINE)) { lds_st(C_ABORT, 1u); break; }
            __builtin_amdgcn_s_sleep(1);
        }
    };
    auto rd = [&](Frag& f, unsigned sb, int h) {
#pragma unroll
        for (int mt = 0; mt < 2; ++mt)
            f.a[mt] = *(const LAS s16x8*)(size_t)((h ? a1 : a0) + sb + mt * 2048u);
#pragma unroll
        for (int nt = 0; nt < 4; ++nt) {
            const s16x4 lo = __builtin_amdgcn_ds_read_tr16_b64_v4i16((LAS s16x4*)(size_t)(bofs[nt] + sb + h * 8192u));
            const s16x4 hi = __builtin_amdgcn_ds_read_tr16_b64_v4i16((LAS s16x4*)(size_t)(bofs[nt] + sb + h * 8192u + 1024u));
            f.b[nt] = __builtin_shufflevector(lo, hi, 0, 1, 2, 3, 4, 5, 6, 7);
        }
    };
    f32x4 acc[2][4];
    auto mm = [&](const Frag& f) {
#pragma unroll
        for (int nt = 0; nt < 4; ++nt) {
            const bf16x8 bf = __builtin_bit_cast(bf16x8, f.b[nt]);
#pragma unroll
            for (int mt = 0; mt < 2; ++mt)
                acc[mt][nt] = __builtin_amdgcn_mfma_f32_16x16x32_bf16(bf, __builtin_bit_cast(bf16x8, f.a[mt]), acc[mt][nt], 0, 0, 0);
        }
    };

    unsigned sl = 0;
    for (unsigned ti = 0; ti < ntile; ++ti) {
#pragma unroll
        for (int mt = 0; mt < 2; ++mt)
#pragma unroll
            for (int nt = 0; nt < 4; ++nt) acc[mt][nt] = f32x4{0.f, 0.f, 0.f, 0.f};
        Frag f0, f1;
        const unsigned gend = (ti + 1u) * (unsigned)NKS;
        unsigned g = ti * (unsigned)NKS;
        wait_full(sl, g);
        rd(f0, sl * SLOT_BYTES, 0);
#define RELEASE() asm volatile("s_waitcnt lgkmcnt(0)\n\tds_write_b32 %7, %8" \
                     : "+v"(f1.a[0]), "+v"(f1.a[1]), "+v"(f1.b[0]), "+v"(f1.b[1]), "+v"(f1.b[2]), "+v"(f1.b[3]), "+v"(fw) \
                     : "v"(cfree), "v"(g + 1u) : "memory")
        unsigned fw = 0;
        for (; g + 1u < gend; ) {
            const unsigned sn = sl + 1u == (unsigned)NSLOT ? 0u : sl + 1u;
            rd(f1, sl * SLOT_BYTES, 1);
            fw = *(const volatile LAS unsigned*)(size_t)(C_FULL + sn * 4u);
            __builtin_amdgcn_sched_barrier(0);
            mm(f0);
            __builtin_amdgcn_sched_barrier(0);
            RELEASE();
            ++g;
            sl = sn;
            if (rfl(fw) <= g) wait_full(sl, g);
            rd(f0, sl * SLOT_BYTES, 0);
            __builtin_amdgcn_sched_barrier(0);
            mm(f1);
            __builtin_amdgcn_sched_barrier(0);
        }
        rd(f1, sl * SLOT_BYTES, 1);
        mm(f0);
        RELEASE();
        mm(f1);
        sl = sl + 1u == (unsigned)NSLOT ? 0u : sl + 1u;
        if (cw == 0) STAMP(7);

        const Tile t = tile_of(tcode >> (16u * ti), rank);
        const bool bad = lds_ld(C_ABORT) != 0u;
        const unsigned stb = L_ST + (unsigned)cw * ST_WAVE;
        u32x4 o[4];
#pragma unroll
        for (int mt = 0; mt < 2; ++mt) {
#pragma unroll
            for (int nt = 0; nt < 4; ++nt) {
                bf16x2 lo, hi;
                lo[0] = (__bf16)acc[mt][nt][0]; lo[1] = (__bf16)acc[mt][nt][1];
                hi[0] = (__bf16)acc[mt][nt][2]; hi[1] = (__bf16)acc[mt][nt][3];
                u32x2 v; v.x = __builtin_bit_cast(unsigned, lo); v.y = __builtin_bit_cast(unsigned, hi);
                if (bad) { v.x = 0x7FC07FC0u; v.y = 0x7FC07FC0u; }
                *(volatile LAS u32x2*)(size_t)(stb + (unsigned)(r16 * ST_ROW + nt * 32 + g4 * 8)) = v;
            }
#pragma unroll
            for (int i = 0; i < 2; ++i)
                o[mt * 2 + i] = *(const volatile LAS u32x4*)(size_t)(stb + (unsigned)((i * 8 + (lane >> 3)) * ST_ROW + (lane & 7) * 16));
        }

        const bool remote = t.cb >= 4;
        const int dstr = remote ? obase + (t.cb & 3) : rank;
        const unsigned colb = (unsigned)(remote ? gidx : t.cb) * 256u + (unsigned)wn * 128u;
        const unsigned orow = (unsigned)(t.src * 512 + t.blk * 64 + wm * 32);
        const __amdgpu_buffer_rsrc_t ro = mk_rsrc(p.out[dstr], SHARD_BYTES);
        const unsigned ovo = (unsigned)(lane >> 3) * 1024u + (unsigned)(lane & 7) * 16u;
        const unsigned oso = orow * 1024u + colb;
        if (!remote) {
#pragma unroll
            for (int i = 0; i < 4; ++i) __builtin_amdgcn_raw_buffer_store_b128(o[i], ro, ovo, oso + i * 8192u, 0);
        } else {
#pragma unroll
            for (int i = 0; i < 4; ++i) __builtin_amdgcn_raw_buffer_store_b128(o[i], ro, ovo, oso + i * 8192u, 17);
            VM_WAIT(0);
            unsigned old;
            asm volatile("ds_add_rtn_u32 %0, %1, %2\n\ts_waitcnt lgkmcnt(0)" : "=&v"(old) : "v"(C_CNT), "v"(lane == 0 ? 1u : 0u) : "memory");
            if (rfl(old) == 3u && lane == 0) st_sys(p.ws[dstr] + OFF_DONE + (unsigned)(gidx * 32 + t.rb) * 4u, call);
        }
        if (cw == 0) STAMP(8);
    }
}

}

extern "C" void kernel_launch(int rank, int nranks, ncclComm_t comm,
                              hipStream_t stream,
                              void* const* d_in, const int* in_sizes, int n_in,
                              void* d_out, int out_size,
                              const int* in_dtypes, int out_dtype,
                              void* const* d_out_all,
                              void* const* d_ws_all, size_t ws_bytes) {
    using namespace agk;
    (void)comm; (void)in_dtypes; (void)out_dtype;
    if (nranks != NR || (unsigned)rank >= (unsigned)NR || n_in < 2 || ws_bytes < (size_t)WS_NEED ||
        in_sizes[0] != 512 * 4096 || in_sizes[1] != 512 * 4096 || out_size != 512 * 4096) {
        fprintf(stderr, "kernel_launch: unexpected problem shape\n");
        return;
    }
    Params p;
    p.x = d_in[0];
    p.w = d_in[1];
    for (int r = 0; r < NR; ++r) { p.ws[r] = (char*)d_ws_all[r]; p.out[r] = (char*)d_out_all[r]; }
    p.out[rank] = (char*)d_out;
    p.rank = rank;
    const int gidx = rank & 3, obase = (rank & 4) ^ 4;
    auto is_late = [&](int rb) { return rb < 32 && (rb >= 4 * NALPHA || (rb % NALPHA >= NEARLY && rb / NALPHA != gidx)); };
    auto code_of = [](int rb, int cb) { return (unsigned)rb | ((unsigned)cb << 6) | 0x8000u; };
    int next[8];
    for (int x = 0; x < 8; ++x) next[x] = x < 7 ? x + 8 * CPL : x;
    for (int b = 0; b < NWG; ++b) p.tile[b] = 0u;
    int ntile = 0;
    for (int c = 0; c < 4; ++c) {
        const int x = (obase + c - rank - 1) & 7;
        for (int pass = 0; pass < 2; ++pass)
            for (int rb = 0; rb < 32; ++rb) {
                if ((rb >= 4 * NALPHA && (rb - 4 * NALPHA) / NBETA == c) || is_late(rb) != (pass == 0)) continue;
                const int xx = next[x] < NWG ? x : 7;
                if (next[xx] >= NWG) { fprintf(stderr, "kernel_launch: placement\n"); return; }
                p.tile[next[xx]] = code_of(rb, 4 + c);
                next[xx] += 8;
                ++ntile;
            }
    }
    unsigned rest[140], first[64];
    int nr = 0, nf = 0;
    for (int pass = 0; pass < 3; ++pass)
        for (int rb = 0; rb < 35; ++rb)
            for (int c = 0; c < 4; ++c) {
                const int cls = is_late(rb) ? 2 : (rb >= 32 || rb / NALPHA == gidx) ? 0 : 1;
                if (cls != pass) continue;
                if (cls < 2 && nf < NPWG) first[nf++] = code_of(rb, c); else rest[nr++] = code_of(rb, c);
            }
    int cand[NWG], nc = 0;
    for (int x = 0, k = nr - 1; x < 8; ++x)
        for (; next[x] < NWG && k >= 0; next[x] += 8, --k) {
            p.tile[next[x]] = rest[k];
            if (is_late((int)(rest[k] & 63u))) cand[nc++] = next[x];
            ++ntile;
        }
    if (nc < nf || ntile + nf != NTILE || nf != NPWG) { fprintf(stderr, "kernel_launch: pairing %d %d %d\n", nc, nf, ntile); return; }
    for (int k = 0; k < nf; ++k) {
        const int b = cand[k * nc / nf];
        p.tile[b] = first[k] | (p.tile[b] << 16);
    }
    hipLaunchKernelGGL(ag_gemm_kernel, dim3(NWG), dim3(NTHREADS), 0, stream, p);
    HIP_CHECK(hipGetLastError());
}
```

```cpp
#include <hip/hip_runtime.h>
#include <rccl/rccl.h>
#include <cstdio>
#include <cstdint>

#define HIP_CHECK(expr) do { hipError_t _e = (expr); if (_e != hipSuccess) { \
    fprintf(stderr, "HIP error %s at %s:%d\n", hipGetErrorString(_e), __FILE__, __LINE__); \
    return; } } while (0)

namespace agk {

#define GAS __attribute__((address_space(1)))
#define LAS __attribute__((address_space(3)))
typedef unsigned u32x4 __attribute__((ext_vector_type(4)));
typedef unsigned u32x2 __attribute__((ext_vector_type(2)));
typedef short s16x4 __attribute__((ext_vector_type(4)));
typedef short s16x8 __attribute__((ext_vector_type(8)));
typedef __bf16 bf16x8 __attribute__((ext_vector_type(8)));
typedef __bf16 bf16x2 __attribute__((ext_vector_type(2)));
typedef float f32x16 __attribute__((ext_vector_type(16)));

constexpr int NR = 8;
constexpr int NKS = 64;
constexpr int NALPHA = 5, NBETA = 3;
constexpr int NEARLY = 2;
constexpr int NWG = 256;
constexpr int NTILE = 256;
constexpr int NTASK = 320;
#define AG_CPL 8
#define AG_WPP 1
#define AG_TOPO 1
constexpr int CPL = AG_CPL;
constexpr int WPP = AG_WPP;
constexpr int NPW = CPL * WPP;
constexpr int NPWG = 7 * CPL;
#define AG_POLL_SLEEP 2
#define AG_FREE_SLEEP 1
#define AG_STAGGER 5
#define AG_LOCAL_AUX 17
#define AG_HOLD 1
#define AG_HOLD_LEAD (-6)
static_assert(NPWG <= 56 && WPP <= 7, "pushing workgroups: at most one per early tile with a local output");
constexpr unsigned SHARD_BYTES = 512u * 4096u * 2u;

constexpr unsigned OFF_CTR = 0;
constexpr unsigned OFF_TMO = 32u << 10;
constexpr unsigned OFF_Q = 33u << 10;
constexpr unsigned OFF_GO = 34u << 10;
constexpr unsigned OFF_FA = 64u << 10;
constexpr unsigned OFF_FW = 80u << 10;
constexpr unsigned OFF_DONE = 96u << 10;
constexpr unsigned OFF_DBG = 128u << 10;
constexpr unsigned OFF_DATA = 1u << 20;
constexpr unsigned ABLK_BYTES = NKS * 8192u;
constexpr unsigned WQ_BYTES = NKS * 16384u;
constexpr unsigned OFF_WST = 32u * ABLK_BYTES;
constexpr unsigned PAR_BYTES = OFF_WST + 4u * WQ_BYTES;
constexpr unsigned WS_NEED = OFF_DATA + 2u * PAR_BYTES;

constexpr int NSLOT = 6;
constexpr int NLOAD = 4;
constexpr unsigned SLOT_BYTES = 24576u;
constexpr unsigned L_ST = NSLOT * SLOT_BYTES;
constexpr unsigned ST_ROW = 80, ST_WAVE = 32 * ST_ROW;
constexpr unsigned L_CTL = L_ST + 4 * ST_WAVE;
constexpr unsigned C_FREE = L_CTL;
constexpr unsigned C_FULL = L_CTL + 16;
constexpr unsigned C_ABORT = L_CTL + 40;
constexpr unsigned C_CNT = L_CTL + 44;
constexpr unsigned C_GO = L_CTL + 48;
constexpr unsigned LDS_BYTES = L_CTL + 64;
static_assert(LDS_BYTES <= 160u * 1024u, "LDS");
constexpr int NTHREADS = (NLOAD + 4) * 64;

constexpr unsigned long long DEADLINE = 30000000ull;

struct Params {
    const void* x;
    const void* w;
    char* ws[NR];
    char* out[NR];
    int rank;
    unsigned xpeer;
    unsigned tile[NWG];
};

__device__ __forceinline__ unsigned long long rt() { return __builtin_amdgcn_s_memrealtime(); }
__device__ __forceinline__ bool expired(unsigned long long& t0, unsigned long long limit) {
    const unsigned long long now = __builtin_amdgcn_s_memrealtime() | 1ull;
    if (t0 == 0ull) { t0 = now; return false; }
    return now - t0 > limit;
}
__device__ __forceinline__ unsigned rfl(unsigned v) { return (unsigned)__builtin_amdgcn_readfirstlane((int)v); }
__device__ __forceinline__ __amdgpu_buffer_rsrc_t mk_rsrc(const void* p, unsigned bytes) {
    return __builtin_amdgcn_make_buffer_rsrc((void*)p, (short)0, (int)bytes, 0x00020000);
}
#define VM_WAIT(n) asm volatile("s_waitcnt vmcnt(" #n ")" ::: "memory")
__device__ __forceinline__ void lds_st(unsigned addr, unsigned v) {
    asm volatile("ds_write_b32 %0, %1" :: "v"(addr), "v"(v) : "memory");
}
__device__ __forceinline__ unsigned lds_ld(unsigned addr) {
    unsigned v;
    asm volatile("ds_read_b32 %0, %1\n\ts_waitcnt lgkmcnt(0)" : "=&v"(v) : "v"(addr) : "memory");
    return rfl(v);
}
__device__ __forceinline__ u32x4 lds_ld4(unsigned addr) {
    u32x4 v;
    asm volatile("ds_read_b128 %0, %1\n\ts_waitcnt lgkmcnt(0)" : "=&v"(v) : "v"(addr) : "memory");
    return v;
}
__device__ __forceinline__ unsigned ld_agent(const char* p) {
    return __hip_atomic_load((const GAS unsigned*)p, __ATOMIC_RELAXED, __HIP_MEMORY_SCOPE_AGENT);
}
__device__ __forceinline__ void st_agent(char* p, unsigned v) {
    __hip_atomic_store((GAS unsigned*)p, v, __ATOMIC_RELAXED, __HIP_MEMORY_SCOPE_AGENT);
}
__device__ __forceinline__ void st_sys(char* p, unsigned v) {
    __hip_atomic_store((GAS unsigned*)p, v, __ATOMIC_RELAXED, __HIP_MEMORY_SCOPE_SYSTEM);
}
#define DMA_P(V) "buffer_load_dwordx4 " V ", %5, %1 offen sc0 sc1 lds\n\ts_add_u32 m0, m0, 0x400\n\ts_add_u32 %1, %1, %7\n\t"
#define DMA_HEAD "s_nop 4\n\ts_mov_b32 %0, m0\n\ts_mov_b32 m0, %2\n\ts_mov_b32 %1, %6\n\t"
#define DMA_TAIL "s_mov_b32 m0, %0"
#define DMA8(PA, PB, PC, PD) \
    asm volatile(DMA_HEAD DMA_P(PA) DMA_P(PB) DMA_P(PC) DMA_P(PD) DMA_P(PA) DMA_P(PB) DMA_P(PC) DMA_P(PD) DMA_TAIL \
                 : "=&s"(keep), "=&s"(t) : "s"(lds), "v"(v0), "v"(v1), "s"(rs), "s"(soff), "s"(js) : "scc", "memory")
__device__ __forceinline__ void dma8(unsigned lds, unsigned v0, unsigned v1, __amdgpu_buffer_rsrc_t rs,
                                     unsigned soff, unsigned js) {
    unsigned keep, t;
    DMA8("%3", "%4", "%3", "%4");
}

#define STAMP(i) do {} while (0)

#define FST(i) do {} while (0)
#define FFL() do {} while (0)
#define FCO(i) do {} while (0)
__device__ __forceinline__ unsigned x_lane_off(int l) { return (unsigned)((l >> 3) * 8192 + (((l & 7) ^ (l >> 4)) << 4)); }
__device__ __forceinline__ unsigned w_lane_off(int l) { return (unsigned)((l >> 4) * 1024 + ((((l & 15) >> 1) ^ ((l >> 4) << 1)) << 5) + (l & 1) * 16); }

struct Tile { int rb, cb, src, blk; };
__device__ __forceinline__ Tile tile_of(unsigned code, int rank) {
    const int gidx = rank & 3, gbase = rank & 4, obase = gbase ^ 4;
    Tile t;
    t.rb = (int)(code & 63u); t.cb = (int)((code >> 6) & 7u);
    if (t.rb < 4 * NALPHA) { t.src = gbase + t.rb / NALPHA; t.blk = t.rb % NALPHA; }
    else if (t.rb < 32) { t.src = obase + (t.rb - 4 * NALPHA) / NBETA; t.blk = NALPHA + (t.rb - 4 * NALPHA) % NBETA; }
    else { t.src = rank; t.blk = NALPHA + (t.rb - 32); }
    return t;
}

struct Frag { s16x8 a[4][2], b[4]; };

__global__ void __launch_bounds__(NTHREADS, 2) ag_gemm_kernel(const Params p) {
    __shared__ __attribute__((aligned(1024))) unsigned char lds_raw[LDS_BYTES];
    LAS unsigned char* L = (LAS unsigned char*)lds_raw;
    const int lane = threadIdx.x & 63;
    const int wave = (int)rfl(threadIdx.x >> 6);
    const int rank = p.rank;
    const int gidx = rank & 3, gbase = rank & 4, obase = gbase ^ 4;
    char* myws = p.ws[rank];
    const unsigned bid = blockIdx.x;

    auto draw_call = [&]() -> unsigned {
        unsigned ticket = 0;
        if (lane == 0)
            ticket = __hip_atomic_fetch_add((GAS unsigned*)(myws + OFF_CTR + bid * 128u + (unsigned)wave * 4u), 1u,
                                            __ATOMIC_RELAXED, __HIP_MEMORY_SCOPE_AGENT);
        return rfl(ticket) + 1u;
    };

    const int peer = (int)((p.xpeer >> ((bid & 7u) * 4u)) & 15u);
    if (bid < 8u * CPL && peer < NR) {
        if (wave < WPP) {
            if (wave == 0) STAMP(0);
            const int pi = wave * CPL + (int)(bid >> 3);
            const bool same = (peer & 4) == gbase;
            const __amdgpu_buffer_rsrc_t rx = mk_rsrc(p.x, SHARD_BYTES), rw = mk_rsrc(p.w, SHARD_BYTES);
            char* pws = p.ws[peer];
            const __amdgpu_buffer_rsrc_t rd = mk_rsrc(pws, WS_NEED);
            const unsigned xa0 = x_lane_off(lane), xa1 = xa0 ^ 64u;
            const unsigned wb0 = w_lane_off(lane);
            const unsigned dl = (unsigned)lane * 16u;

            auto load_task = [&](int t, u32x4 (&b)[8]) {
                if (!same && t < 128) {
                    const unsigned so = (unsigned)(t * 32) * 1024u + (unsigned)(peer & 3) * 256u;
#pragma unroll
                    for (int j = 0; j < 8; ++j)
                        b[j] = __builtin_amdgcn_raw_buffer_load_b128(rw, wb0, so + j * 4096u, 0);
                } else {
                    int ks, blk;
                    if (t < 128) { ks = t >> 1; blk = t & 1; }
                    else { const int u = t - 128; ks = u / 3; blk = (same ? NEARLY : NALPHA) + (u - ks * 3); }
                    const unsigned so = (unsigned)blk * (64u * 8192u) + (unsigned)ks * 128u;
#pragma unroll
                    for (int j = 0; j < 8; ++j)
                        b[j] = __builtin_amdgcn_raw_buffer_load_b128(rx, (j & 1) ? xa1 : xa0, so + j * 65536u, 0);
                }
            };
            u32x4 b0[8];
            load_task(pi, b0);
            char* qp = myws + OFF_Q + (unsigned)peer * 128u;
            unsigned tk0 = 0;
            if (lane == 0)
                tk0 = __hip_atomic_fetch_add((GAS unsigned*)qp, 1u, __ATOMIC_RELAXED, __HIP_MEMORY_SCOPE_AGENT);
            const unsigned call = rfl(tk0) / (unsigned)NTASK + 1u;
            __builtin_amdgcn_s_waitcnt(0x0F70);
            const unsigned par = OFF_DATA + (call & 1u) * PAR_BYTES;
            const unsigned qbase = (call - 1u) * (unsigned)NTASK - (unsigned)NPW;
            unsigned fof;
            auto store_task = [&](int t, const u32x4 (&b)[8]) {
                unsigned dof;
                if (!same && t < 128) {
                    dof = OFF_WST + (unsigned)gidx * WQ_BYTES + (unsigned)t * 8192u;
                    fof = OFF_FW + (unsigned)(gidx * 128 + t) * 4u;
                } else {
                    int ks, ai;
                    if (t < 128) { ks = t >> 1; ai = gidx * NALPHA + (t & 1); }
                    else {
                        const int u = t - 128; ks = u / 3;
                        const int r = u - ks * 3;
                        ai = same ? gidx * NALPHA + NEARLY + r : 4 * NALPHA + gidx * NBETA + r;
                    }
                    dof = (unsigned)ai * ABLK_BYTES + (unsigned)ks * 8192u;
                    fof = OFF_FA + (unsigned)(ai * 64 + ks) * 4u;
                }
                dof += par;
#pragma unroll
                for (int j = 0; j < 8; ++j)
                    __builtin_amdgcn_raw_buffer_store_b128(b[j], rd, dl, dof + j * 1024u, 17);
            };
            auto draw = [&](unsigned& tk) {
                unsigned long long sv;
                asm volatile("s_mov_b64 %1, exec\n\ts_mov_b64 exec, 1\n\tglobal_atomic_add %0, %2, %3, off sc0\n\ts_mov_b64 exec, %1"
                             : "=&v"(tk), "=&s"(sv) : "v"(qp), "v"(1u) : "memory");
            };
            for (int i = 0; i < pi; ++i) __builtin_amdgcn_s_sleep(AG_STAGGER);
            for (int t = pi;;) {
                const unsigned nx = rfl(tk0) - qbase;
                store_task(t, b0);
                if (nx == (unsigned)(NTASK - AG_HOLD_LEAD) && lane == 0) st_agent(myws + OFF_GO + (unsigned)peer * 128u, call);
                if (nx < (unsigned)NTASK) { load_task((int)nx, b0); draw(tk0); }
                asm volatile("s_waitcnt vmcnt(0)" : "+v"(tk0) :: "memory");
                __builtin_amdgcn_s_waitcnt(0x0F70);
                asm volatile("" ::: "memory");
                if (lane == 0) st_sys(pws + fof, call);
                if (nx >= (unsigned)NTASK) break;
                t = (int)nx;
            }
            if (wave == WPP - 1) STAMP(9 + ((peer - rank - 1) & 7));
            return;
        }
        if (wave != 7 || bid != 0u) return;
        const unsigned call = draw_call();
        const int i0 = (lane * 2) & 31, ex = 4 * NALPHA + gidx * NBETA;
        const bool skip0 = i0 >= ex && i0 < ex + NBETA, skip1 = i0 + 1 >= ex && i0 + 1 < ex + NBETA;
        const char* fp = myws + OFF_DONE + (unsigned)lane * 8u;
        unsigned long long t0 = 0;
        unsigned spins = 0;
        bool dead = false;
        for (;;) {
            u32x2 v;
            asm volatile("global_load_dwordx2 %0, %1, off sc0 sc1\n\ts_waitcnt vmcnt(0)" : "=&v"(v) : "v"(fp) : "memory");
            const bool ok = (skip0 || v.x >= call) && (skip1 || v.y >= call);
            if (__all(ok)) break;
            if ((++spins & 255u) == 0u) {
                if (rfl(ld_agent(myws + OFF_TMO)) == call || expired(t0, 2 * DEADLINE)) { dead = true; break; }
            }
            __builtin_amdgcn_s_sleep(2);
        }
        __builtin_amdgcn_fence(__ATOMIC_ACQUIRE, "");
        if (dead) {
            if (lane == 0) st_agent(myws + OFF_TMO, call);
            const unsigned long long t1 = rt();
            while (rt() - t1 < DEADLINE / 8) __builtin_amdgcn_s_sleep(64);
            if (lane == 0) st_sys(p.out[rank] + (size_t)rank * 512u * 1024u, 0x7FC07FC0u);
        }
        STAMP(6);
        return;
    }

    if ((unsigned)(size_t)L != 0u) return;
    if (wave == 0 && lane < 16) *(LAS unsigned*)(L + L_CTL + lane * 4) = 0u;
    const unsigned call = draw_call();
    const unsigned tcode = p.tile[bid];
    const unsigned ntile = (tcode >> 31) ? 2u : 1u;
    asm volatile("s_waitcnt lgkmcnt(0)" ::: "memory");
    __builtin_amdgcn_s_barrier();
    const unsigned par = OFF_DATA + (call & 1u) * PAR_BYTES;

    if (wave < NLOAD) {
        unsigned slot = (unsigned)wave;
        const __amdgpu_buffer_rsrc_t rws = mk_rsrc(myws, WS_NEED);
        unsigned g = (unsigned)wave, spins = 0;
        unsigned long long t0 = 0;
        bool dead = false;
        for (unsigned ti = 0; ti < ntile && !dead; ++ti) {
            const Tile t = tile_of(tcode >> (16u * ti), rank);
            const bool sa = t.src != rank, sb = t.cb >= 4;
            const __amdgpu_buffer_rsrc_t ra = sa ? rws : mk_rsrc(p.x, SHARD_BYTES);
            const __amdgpu_buffer_rsrc_t rb = sb ? rws : mk_rsrc(p.w, SHARD_BYTES);
            const unsigned va0 = sa ? (unsigned)lane * 16u : x_lane_off(lane), va1 = sa ? va0 : va0 ^ 64u;
            const unsigned vb0 = sb ? (unsigned)lane * 16u : w_lane_off(lane);
            const unsigned abase = sa ? par + (unsigned)t.rb * ABLK_BYTES : (unsigned)t.blk * (64u * 8192u);
            const unsigned bbase = sb ? par + OFF_WST + (unsigned)(t.cb & 3) * WQ_BYTES : (unsigned)t.cb * 256u;
            const unsigned aks = sa ? 8192u : 128u, ajs = sa ? 1024u : 65536u;
            const unsigned bks = sb ? 16384u : 65536u, bjs = sb ? 1024u : 4096u, bh = sb ? 8192u : 32768u;
            const bool poll = lane == 0 ? sa : (lane < 3 && sb);
            const unsigned fl0 = lane == 0 ? OFF_FA + (unsigned)(t.rb & 31) * 256u : OFF_FW + (unsigned)(t.cb & 3) * 512u + (unsigned)(lane & 2) * 2u;
            const unsigned fks = lane == 0 ? 4u : 8u;
            for (; g < (ti + 1u) * (unsigned)NKS; g += (unsigned)NLOAD) {
                const unsigned ks = g & (unsigned)(NKS - 1);
                const unsigned lslot = slot * SLOT_BYTES, cfull = C_FULL + slot * 4u;
                slot = slot + (unsigned)NLOAD >= (unsigned)NSLOT ? slot + (unsigned)NLOAD - (unsigned)NSLOT : slot + (unsigned)NLOAD;
                if (sa || sb) {
                    const unsigned fo = fl0 + ks * fks;
                    for (;;) {
                        unsigned v = call;
                        if (poll) v = __builtin_amdgcn_raw_buffer_load_b32(rws, fo, 0, 17);
                        if (__all(v >= call)) break;
                        if ((++spins & 255u) == 0u &&
                            (rfl(ld_agent(myws + OFF_TMO)) == call || lds_ld(C_ABORT) != 0u || expired(t0, DEADLINE))) {
                            dead = true;
                            break;
                        }
                        __builtin_amdgcn_s_sleep(AG_POLL_SLEEP);
                    }
                    asm volatile("" ::: "memory");
                }
                while (g >= (unsigned)NSLOT && !dead) {
                    const u32x4 fr = lds_ld4(C_FREE);
                    if (rfl(min(min(fr.x, fr.y), min(fr.z, fr.w))) + (unsigned)NSLOT > g) break;
                    if ((++spins & 1023u) == 0u && (lds_ld(C_ABORT) != 0u || expired(t0, 2 * DEADLINE))) dead = true;
                    __builtin_amdgcn_s_sleep(AG_FREE_SLEEP);
                }
                if (dead) break;
                if (g == (ntile - 1u) * (unsigned)NKS + (unsigned)wave && wave == 0) STAMP(2);
                FST(0);
                dma8(lslot, va0, va1, ra, abase + ks * aks, ajs);
                dma8(lslot + 8192u, vb0, vb0, rb, bbase + ks * bks, bjs);
                dma8(lslot + 16384u, vb0, vb0, rb, bbase + ks * bks + bh, bjs);
                FST(1);
                VM_WAIT(0);
                lds_st(cfull, g + 1u);
                FST(2);
                FFL();
            }
        }
        if (dead) {
            lds_st(C_ABORT, 1u);
            if (lane == 0) st_agent(myws + OFF_TMO, call);
        }
        if (wave == 0) {
            const int cb = tile_of(tcode >> (16u * (ntile - 1u)), rank).cb;
            if (cb >= 4) {
                const char* hp = myws + OFF_GO + (unsigned)(obase + (cb & 3)) * 128u;
                t0 = 0;
                while (!dead && rfl(ld_agent(hp)) < call) {
                    if ((++spins & 63u) == 0u && expired(t0, DEADLINE)) break;
                    __builtin_amdgcn_s_sleep(2);
                }
                lds_st(C_GO, 1u);
            }
        }
        if (wave == 3) STAMP(3);
        return;
    }

    const int cw = wave - NLOAD;
    const int hh = lane >> 5, r32 = lane & 31;
    const unsigned a0 = (unsigned)(r32 * 128 + ((hh ^ ((r32 >> 1) & 7)) << 4));
    const int q = (lane >> 2) & 3, pp = lane & 3;
    const unsigned b0 = 8192u + (unsigned)((8 * hh + q) * 256 + (((cw * 2 + ((lane >> 4) & 1)) ^ (q << 1)) << 5) + pp * 8);
    const unsigned cfree = C_FREE + (unsigned)cw * 4u;

    unsigned spins = 0;
    unsigned long long t0 = 0;
    auto wait_full = [&](unsigned slot, unsigned g) {
        while (lds_ld(C_FULL + slot * 4u) <= g) {
            if ((++spins & 63u) == 0u && lds_ld(C_ABORT) != 0u) break;
            if ((spins & 1023u) == 0u && expired(t0, 3 * DEADLINE)) { lds_st(C_ABORT, 1u); break; }
            __builtin_amdgcn_s_sleep(1);
        }
    };
    unsigned sl = 0, g = 0, fw = 0;
    auto rd = [&](Frag& f) {
        const unsigned sb = sl * SLOT_BYTES;
        fw = *(const volatile LAS unsigned*)(size_t)(C_FULL + (sl + 1u == (unsigned)NSLOT ? 0u : sl + 1u) * 4u);
#pragma unroll
        for (int k = 0; k < 4; ++k) {
#pragma unroll
            for (int mt = 0; mt < 2; ++mt)
                f.a[k][mt] = *(const LAS s16x8*)(size_t)(((a0 + sb) ^ (unsigned)(k * 32)) + mt * 4096u);
            const s16x4 lo = __builtin_amdgcn_ds_read_tr16_b64_v4i16((LAS s16x4*)(size_t)(b0 + sb + k * 4096u));
            const s16x4 hi = __builtin_amdgcn_ds_read_tr16_b64_v4i16((LAS s16x4*)(size_t)(b0 + sb + k * 4096u + 1024u));
            f.b[k] = __builtin_shufflevector(lo, hi, 0, 1, 2, 3, 4, 5, 6, 7);
        }
    };
    auto advance = [&]() {
        ++g;
        sl = sl + 1u == (unsigned)NSLOT ? 0u : sl + 1u;
    };
    f32x16 acc[2];
    auto mm = [&](const Frag& f) {
#pragma unroll
        for (int k = 0; k < 4; ++k)
#pragma unroll
            for (int mt = 0; mt < 2; ++mt)
                acc[mt] = __builtin_amdgcn_mfma_f32_32x32x16_bf16(__builtin_bit_cast(bf16x8, f.b[k]),
                                                                  __builtin_bit_cast(bf16x8, f.a[k][mt]), acc[mt], 0, 0, 0);
    };
    auto release = [&](unsigned n) {
        __builtin_amdgcn_sched_barrier(0);
        lds_st(cfree, n);
        __builtin_amdgcn_sched_barrier(0);
    };
    auto step = [&](const Frag& c, Frag& n) {
        const bool ready = rfl(fw) > g;
        __builtin_amdgcn_sched_barrier(0);
        rd(n);
        mm(c);
        __builtin_amdgcn_sched_group_barrier(0x100, 1, 0);
#pragma unroll
        for (int i = 0; i < 8; ++i) {
            __builtin_amdgcn_sched_group_barrier(0x008, 1, 0);
            __builtin_amdgcn_sched_group_barrier(0x100, 2, 0);
        }
        release(g);
        if (!ready) {
            wait_full(sl, g);
            rd(n);
        }
        advance();
    };

    for (unsigned ti = 0; ti < ntile; ++ti) {
#pragma unroll
        for (int i = 0; i < 16; ++i) { acc[0][i] = 0.f; acc[1][i] = 0.f; }
        Frag f0, f1;
        wait_full(sl, g);
        rd(f0);
        advance();
        for (int i = 0; i < NKS / 2 - 1; ++i) {
            step(f0, f1);
            step(f1, f0);
        }
        step(f0, f1);
        mm(f1);
        release(g);
        if (cw == 0) STAMP(7);

        const Tile t = tile_of(tcode >> (16u * ti), rank);
        const bool bad = lds_ld(C_ABORT) != 0u;
        const unsigned stb = L_ST + (unsigned)cw * ST_WAVE;
        u32x4 o[4];
#pragma unroll
        for (int mt = 0; mt < 2; ++mt) {
#pragma unroll
            for (int j = 0; j < 4; ++j) {
                bf16x2 lo, hi;
                lo[0] = (__bf16)acc[mt][4 * j]; lo[1] = (__bf16)acc[mt][4 * j + 1];
                hi[0] = (__bf16)acc[mt][4 * j + 2]; hi[1] = (__bf16)acc[mt][4 * j + 3];
                u32x2 v; v.x = __builtin_bit_cast(unsigned, lo); v.y = __builtin_bit_cast(unsigned, hi);
                if (bad) { v.x = 0x7FC07FC0u; v.y = 0x7FC07FC0u; }
                *(volatile LAS u32x2*)(size_t)(stb + (unsigned)(r32 * ST_ROW + j * 16 + hh * 8)) = v;
            }
#pragma unroll
            for (int i = 0; i < 2; ++i)
                o[mt * 2 + i] = *(const volatile LAS u32x4*)(size_t)(stb + (unsigned)((i * 16 + (lane >> 2)) * ST_ROW + (lane & 3) * 16));
        }

        const bool remote = t.cb >= 4;
        const int dstr = remote ? obase + (t.cb & 3) : rank;
        const unsigned colb = (unsigned)(remote ? gidx : t.cb) * 256u + (unsigned)cw * 64u;
        const unsigned orow = (unsigned)(t.src * 512 + t.blk * 64);
        const __amdgpu_buffer_rsrc_t ro = mk_rsrc(p.out[dstr], SHARD_BYTES);
        const unsigned ovo = (unsigned)(lane >> 2) * 1024u + (unsigned)(lane & 3) * 16u;
        const unsigned oso = orow * 1024u + colb;
        if (!remote) {
#pragma unroll
            for (int i = 0; i < 4; ++i) __builtin_amdgcn_raw_buffer_store_b128(o[i], ro, ovo, oso + i * 16384u, AG_LOCAL_AUX);
        } else {
            t0 = 0;
            while (lds_ld(C_GO) == 0u) {
                if ((++spins & 1023u) == 0u && expired(t0, 2 * DEADLINE)) break;
                __builtin_amdgcn_s_sleep(2);
            }
#pragma unroll
            for (int i = 0; i < 4; ++i) __builtin_amdgcn_raw_buffer_store_b128(o[i], ro, ovo, oso + i * 16384u, 17);
            VM_WAIT(0);
            unsigned old;
            asm volatile("ds_add_rtn_u32 %0, %1, %2\n\ts_waitcnt lgkmcnt(0)" : "=&v"(old) : "v"(C_CNT), "v"(lane == 0 ? 1u : 0u) : "memory");
            if (rfl(old) == 3u && lane == 0) st_sys(p.ws[dstr] + OFF_DONE + (unsigned)(gidx * 32 + t.rb) * 4u, call);
        }
        if (cw == 0) STAMP(8);
    }
}

}

extern "C" void kernel_launch(int rank, int nranks, ncclComm_t comm,
                              hipStream_t stream,
                              void* const* d_in, const int* in_sizes, int n_in,
                              void* d_out, int out_size,
                              const int* in_dtypes, int out_dtype,
                              void* const* d_out_all,
                              void* const* d_ws_all, size_t ws_bytes) {
    using namespace agk;
    (void)comm; (void)in_dtypes; (void)out_dtype;
    if (nranks != NR || (unsigned)rank >= (unsigned)NR || n_in < 2 || ws_bytes < (size_t)WS_NEED ||
        in_sizes[0] != 512 * 4096 || in_sizes[1] != 512 * 4096 || out_size != 512 * 4096) {
        fprintf(stderr, "kernel_launch: unexpected problem shape\n");
        return;
    }
    Params p;
    p.x = d_in[0];
    p.w = d_in[1];
    for (int r = 0; r < NR; ++r) { p.ws[r] = (char*)d_ws_all[r]; p.out[r] = (char*)d_out_all[r]; }
    p.out[rank] = (char*)d_out;
    p.rank = rank;
    const int gidx = rank & 3, obase = (rank & 4) ^ 4;
    int xcd_of[NR], spare = 3;
    {
        const unsigned setmask = 0xB2u;
        int lo = 0, hi = 4;
        for (int o = 1; o < NR; ++o) {
            const int q = (rank + o) & 7;
            xcd_of[q] = (((setmask >> q) ^ (setmask >> rank)) & 1u) ? hi++ : lo++;
        }
        xcd_of[rank] = spare;
        p.xpeer = 0u;
        for (int q = 0; q < NR; ++q) p.xpeer |= (unsigned)(q == rank ? 15 : q) << (xcd_of[q] * 4);
    }
    auto is_late = [&](int rb) { return rb < 32 && (rb >= 4 * NALPHA || (rb % NALPHA >= NEARLY && rb / NALPHA != gidx)); };
    auto code_of = [](int rb, int cb) { return (unsigned)rb | ((unsigned)cb << 6) | 0x8000u; };
    int next[8];
    for (int x = 0; x < 8; ++x) next[x] = x != spare ? x + 8 * CPL : x;
    for (int b = 0; b < NWG; ++b) p.tile[b] = 0u;
    int ntile = 0;
    for (int c = 0; c < 4; ++c) {
        const int x = xcd_of[obase + c];
        for (int pass = 0; pass < 2; ++pass)
            for (int rb = 0; rb < 32; ++rb) {
                if ((rb >= 4 * NALPHA && (rb - 4 * NALPHA) / NBETA == c) || is_late(rb) != (pass == 0)) continue;
                const int xx = next[x] < NWG ? x : spare;
                if (next[xx] >= NWG) { fprintf(stderr, "kernel_launch: placement\n"); return; }
                p.tile[next[xx]] = code_of(rb, 4 + c);
                next[xx] += 8;
                ++ntile;
            }
    }
    unsigned rest[140], first[64];
    int nr = 0, nf = 0;
    for (int pass = 0; pass < 3; ++pass)
        for (int rb = 0; rb < 35; ++rb)
            for (int c = 0; c < 4; ++c) {
                const int cls = is_late(rb) ? 2 : (rb >= 32 || rb / NALPHA == gidx) ? 0 : 1;
                if (cls != pass) continue;
                if (cls < 2 && nf < NPWG) first[nf++] = code_of(rb, c); else rest[nr++] = code_of(rb, c);
            }
    int cand[NWG], nc = 0;
    for (int x = 0, k = nr - 1; x < 8; ++x)
        for (; next[x] < NWG && k >= 0; next[x] += 8, --k) {
            p.tile[next[x]] = rest[k];
            if (is_late((int)(rest[k] & 63u))) cand[nc++] = next[x];
            ++ntile;
        }
    if (nc < nf || ntile + nf != NTILE || nf != NPWG) { fprintf(stderr, "kernel_launch: pairing %d %d %d\n", nc, nf, ntile); return; }
    for (int k = 0; k < nf; ++k) {
        const int b = cand[k * nc / nf];
        p.tile[b] = first[k] | (p.tile[b] << 16);
    }
    hipLaunchKernelGGL(ag_gemm_kernel, dim3(NWG), dim3(NTHREADS), 0, stream, p);
    HIP_CHECK(hipGetLastError());
}
```

```cpp
#include <hip/hip_runtime.h>
#include <rccl/rccl.h>
#include <cstdio>
#include <cstdint>

#define HIP_CHECK(expr) do { hipError_t _e = (expr); if (_e != hipSuccess) { \
    fprintf(stderr, "HIP error %s at %s:%d\n", hipGetErrorString(_e), __FILE__, __LINE__); \
    return; } } while (0)

namespace agk {

#define GAS __attribute__((address_space(1)))
#define LAS __attribute__((address_space(3)))
typedef unsigned u32x4 __attribute__((ext_vector_type(4)));
typedef unsigned u32x2 __attribute__((ext_vector_type(2)));
typedef short s16x4 __attribute__((ext_vector_type(4)));
typedef short s16x8 __attribute__((ext_vector_type(8)));
typedef __bf16 bf16x8 __attribute__((ext_vector_type(8)));
typedef __bf16 bf16x2 __attribute__((ext_vector_type(2)));
typedef float f32x16 __attribute__((ext_vector_type(16)));

constexpr int NR = 8;
constexpr int NKS = 64;
constexpr int NALPHA = 5, NBETA = 3;
constexpr int NEARLY = 2;
constexpr int NWG = 256;
constexpr int NTILE = 256;
constexpr int NTASK = 320;
#define AG_CPL 8
#define AG_WPP 1
#define AG_TOPO 1
constexpr int CPL = AG_CPL;
constexpr int WPP = AG_WPP;
constexpr int NPW = CPL * WPP;
constexpr int NPWG = 7 * CPL;
#define AG_POLL_SLEEP 2
#define AG_FREE_SLEEP 1
#define AG_STAGGER 5
#define AG_LOCAL_AUX 17
#define AG_HOLD 1
#define AG_GO_TASK (NTASK - 2)
static_assert(NTASK % (2 * NPW) == 0 && NPWG <= 56 && WPP <= 7, "pushing workgroups: at most one per early tile with a local output");
constexpr unsigned SHARD_BYTES = 512u * 4096u * 2u;

constexpr unsigned OFF_CTR = 0;
constexpr unsigned OFF_TMO = 32u << 10;
constexpr unsigned OFF_GO = 34u << 10;
constexpr unsigned OFF_FA = 64u << 10;
constexpr unsigned OFF_FW = 80u << 10;
constexpr unsigned FLG_PAR = 32u << 10;
constexpr unsigned OFF_DONE = 128u << 10;
constexpr unsigned OFF_DBG = 192u << 10;
constexpr unsigned OFF_DATA = 1u << 20;
constexpr unsigned CH_BYTES = 10240u;
constexpr unsigned ABLK_BYTES = NKS * CH_BYTES;
constexpr unsigned WQ_BYTES = NKS * 2u * CH_BYTES;
constexpr unsigned OFF_WST = 32u * ABLK_BYTES;
constexpr unsigned PAR_BYTES = OFF_WST + 4u * WQ_BYTES;
constexpr unsigned WS_NEED = OFF_DATA + 2u * PAR_BYTES;

constexpr int NSLOT = 6;
constexpr int NLOAD = 8;
constexpr unsigned SLOT_BYTES = 24576u;
constexpr unsigned L_ST = NSLOT * SLOT_BYTES;
constexpr unsigned ST_ROW = 80, ST_WAVE = 32 * ST_ROW;
constexpr unsigned L_CTL = L_ST + 4 * ST_WAVE;
constexpr unsigned C_FREE = L_CTL;
constexpr unsigned C_FULL = L_CTL + 16;
constexpr unsigned C_ABORT = L_CTL + 40;
constexpr unsigned C_CNT = L_CTL + 44;
constexpr unsigned C_GO = L_CTL + 48;
constexpr unsigned LDS_BYTES = L_CTL + 64;
static_assert(LDS_BYTES <= 160u * 1024u, "LDS");
constexpr int NTHREADS = (NLOAD + 4) * 64;

constexpr unsigned long long DEADLINE = 30000000ull;

struct Params {
    const void* x;
    const void* w;
    char* ws[NR];
    char* out[NR];
    int rank;
    unsigned xpeer;
    unsigned tile[NWG];
};

__device__ __forceinline__ unsigned long long rt() { return __builtin_amdgcn_s_memrealtime(); }
__device__ __forceinline__ bool expired(unsigned long long& t0, unsigned long long limit) {
    const unsigned long long now = __builtin_amdgcn_s_memrealtime() | 1ull;
    if (t0 == 0ull) { t0 = now; return false; }
    return now - t0 > limit;
}
__device__ __forceinline__ unsigned rfl(unsigned v) { return (unsigned)__builtin_amdgcn_readfirstlane((int)v); }
__device__ __forceinline__ __amdgpu_buffer_rsrc_t mk_rsrc(const void* p, unsigned bytes) {
    return __builtin_amdgcn_make_buffer_rsrc((void*)p, (short)0, (int)bytes, 0x00020000);
}
#define VM_WAIT(n) asm volatile("s_waitcnt vmcnt(" #n ")" ::: "memory")
__device__ __forceinline__ void lds_st(unsigned addr, unsigned v) {
    asm volatile("ds_write_b32 %0, %1" :: "v"(addr), "v"(v) : "memory");
}
__device__ __forceinline__ unsigned lds_ld(unsigned addr) {
    unsigned v;
    asm volatile("ds_read_b32 %0, %1\n\ts_waitcnt lgkmcnt(0)" : "=&v"(v) : "v"(addr) : "memory");
    return rfl(v);
}
__device__ __forceinline__ u32x4 lds_ld4(unsigned addr) {
    u32x4 v;
    asm volatile("ds_read_b128 %0, %1\n\ts_waitcnt lgkmcnt(0)" : "=&v"(v) : "v"(addr) : "memory");
    return v;
}
__device__ __forceinline__ unsigned ld_agent(const char* p) {
    return __hip_atomic_load((const GAS unsigned*)p, __ATOMIC_RELAXED, __HIP_MEMORY_SCOPE_AGENT);
}
__device__ __forceinline__ void st_agent(char* p, unsigned v) {
    __hip_atomic_store((GAS unsigned*)p, v, __ATOMIC_RELAXED, __HIP_MEMORY_SCOPE_AGENT);
}
__device__ __forceinline__ void st_sys(char* p, unsigned v) {
    __hip_atomic_store((GAS unsigned*)p, v, __ATOMIC_RELAXED, __HIP_MEMORY_SCOPE_SYSTEM);
}
#define DMA_P(V) "buffer_load_dwordx4 " V ", %5, %1 offen sc0 sc1 lds\n\ts_add_u32 m0, m0, 0x400\n\ts_add_u32 %1, %1, %7\n\t"
#define DMA_HEAD "s_nop 4\n\ts_mov_b32 %0, m0\n\ts_mov_b32 m0, %2\n\ts_mov_b32 %1, %6\n\t"
#define DMA_TAIL "s_mov_b32 m0, %0"
#define DMA8(PA, PB, PC, PD) \
    asm volatile(DMA_HEAD DMA_P(PA) DMA_P(PB) DMA_P(PC) DMA_P(PD) DMA_P(PA) DMA_P(PB) DMA_P(PC) DMA_P(PD) DMA_TAIL \
                 : "=&s"(keep), "=&s"(t) : "s"(lds), "v"(v0), "v"(v1), "s"(rs), "s"(soff), "s"(js) : "scc", "memory")
__device__ __forceinline__ void dma8(unsigned lds, unsigned v0, unsigned v1, __amdgpu_buffer_rsrc_t rs,
                                     unsigned soff, unsigned js) {
    unsigned keep, t;
    DMA8("%3", "%4", "%3", "%4");
}

#define STAMP(i) do {} while (0)

#define FST(i) do {} while (0)
#define FFL() do {} while (0)
#define FCO(i) do {} while (0)
__device__ __forceinline__ unsigned x_lane_off(int l) { return (unsigned)((l >> 3) * 8192 + (((l & 7) ^ (l >> 4)) << 4)); }
__device__ __forceinline__ unsigned w_lane_off(int l) { return (unsigned)((l >> 4) * 1024 + ((((l & 15) >> 1) ^ ((l >> 4) << 1)) << 5) + (l & 1) * 16); }

struct Packed { u32x4 lo[4]; u32x4 cd[2]; u32x2 hi[8]; unsigned fmt; };
__device__ __forceinline__ unsigned bfi(unsigned m, unsigned a, unsigned b) { return (a & m) | (b & ~m); }
__device__ __forceinline__ unsigned pkmax(unsigned a, unsigned b) {
    typedef unsigned short u16x2 __attribute__((ext_vector_type(2)));
    return __builtin_bit_cast(unsigned, __builtin_elementwise_max(__builtin_bit_cast(u16x2, a), __builtin_bit_cast(u16x2, b)));
}
__device__ __forceinline__ void pack_chunk(const u32x4 (&r)[8], Packed& o) {
    unsigned m = 0;
#pragma unroll
    for (int j = 0; j < 8; ++j) {
        o.lo[j >> 1][(j & 1) * 2] = __builtin_amdgcn_perm(r[j].y, r[j].x, 0x06040200u);
        o.lo[j >> 1][(j & 1) * 2 + 1] = __builtin_amdgcn_perm(r[j].w, r[j].z, 0x06040200u);
        o.hi[j].x = __builtin_amdgcn_perm(r[j].y, r[j].x, 0x07050301u);
        o.hi[j].y = __builtin_amdgcn_perm(r[j].w, r[j].z, 0x07050301u);
        const unsigned e0 = o.hi[j].x & 0x7F7F7F7Fu, e1 = o.hi[j].y & 0x7F7F7F7Fu;
        m = pkmax(pkmax(m, e0), pkmax(e0 << 8, e1));
        m = pkmax(m, e1 << 8);
    }
    m = max(m >> 24, (m >> 8) & 0xFFu);
    m = max(m, (unsigned)__builtin_amdgcn_update_dpp(0, (int)m, 0x111, 0xF, 0xF, true));
    m = max(m, (unsigned)__builtin_amdgcn_update_dpp(0, (int)m, 0x112, 0xF, 0xF, true));
    m = max(m, (unsigned)__builtin_amdgcn_update_dpp(0, (int)m, 0x114, 0xF, 0xF, true));
    m = max(m, (unsigned)__builtin_amdgcn_update_dpp(0, (int)m, 0x118, 0xF, 0xF, true));
    const unsigned top = max(max((unsigned)__builtin_amdgcn_readlane((int)m, 15), (unsigned)__builtin_amdgcn_readlane((int)m, 31)),
                             max((unsigned)__builtin_amdgcn_readlane((int)m, 47), (unsigned)__builtin_amdgcn_readlane((int)m, 63)));
    const unsigned base = top > 7u ? top - 7u : 0u;
    const unsigned kk = 0x80808080u - base * 0x01010101u;
    unsigned mask = 0;
#pragma unroll
    for (int j = 0; j < 8; ++j) {
        const unsigned t0 = (o.hi[j].x & 0x7F7F7F7Fu) + kk, t1 = (o.hi[j].y & 0x7F7F7F7Fu) + kk;
        if (__any((~(t0 & t1) & 0x80808080u) != 0u)) mask |= 1u << j;
        o.cd[j >> 2][j & 3] = bfi(0x0F0F0F0Fu, bfi(0x07070707u, t0, o.hi[j].x >> 4), bfi(0x70707070u, t1 << 4, o.hi[j].y));
    }
    if (__builtin_popcount(mask) > 4) mask = 0xFFu;
    o.fmt = base | (mask << 8);
}
struct Wire { u32x4 lo[4]; u32x4 cd[2]; u32x2 e0; };
__device__ __forceinline__ void wire_load(Wire& w, __amdgpu_buffer_rsrc_t rs, unsigned off, unsigned fmt, int lane) {
#pragma unroll
    for (int i = 0; i < 4; ++i) w.lo[i] = __builtin_amdgcn_raw_buffer_load_b128(rs, lane * 16, off + i * 1024u, 17);
#pragma unroll
    for (int i = 0; i < 2; ++i) w.cd[i] = __builtin_amdgcn_raw_buffer_load_b128(rs, lane * 16, off + 4096u + i * 1024u, 17);
    const unsigned j0 = (unsigned)__builtin_ctz((fmt >> 8) | 0x80u);
    w.e0 = __builtin_amdgcn_raw_buffer_load_b64(rs, lane * 8, off + 6144u + j0 * 512u, 17);
}
__device__ __forceinline__ void wire_unpack(const Wire& w, __amdgpu_buffer_rsrc_t rs, unsigned off, unsigned fmt, int lane, unsigned lds) {
    const unsigned br = (fmt & 0x7Fu) * 0x01010101u, mask = fmt >> 8;
    const unsigned j0 = (unsigned)__builtin_ctz(mask | 0x80u);
#pragma unroll
    for (int j = 0; j < 8; ++j) {
        const unsigned n = w.cd[j >> 2][j & 3];
        unsigned ha = ((n & 0x07070707u) + br) | ((n << 4) & 0x80808080u);
        unsigned hb = (((n >> 4) & 0x07070707u) + br) | (n & 0x80808080u);
        if ((mask >> j) & 1u) {
            u32x2 e = w.e0;
            if ((unsigned)j != j0) e = __builtin_amdgcn_raw_buffer_load_b64(rs, lane * 8, off + 6144u + j * 512u, 17);
            ha = e.x; hb = e.y;
        }
        const unsigned l0 = w.lo[j >> 1][(j & 1) * 2], l1 = w.lo[j >> 1][(j & 1) * 2 + 1];
        u32x4 v;
        v.x = __builtin_amdgcn_perm(ha, l0, 0x05010400u);
        v.y = __builtin_amdgcn_perm(ha, l0, 0x07030602u);
        v.z = __builtin_amdgcn_perm(hb, l1, 0x05010400u);
        v.w = __builtin_amdgcn_perm(hb, l1, 0x07030602u);
        *(LAS u32x4*)(size_t)(lds + (unsigned)lane * 16u + j * 1024u) = v;
    }
}

struct Tile { int rb, cb, src, blk; };
__device__ __forceinline__ Tile tile_of(unsigned code, int rank) {
    const int gidx = rank & 3, gbase = rank & 4, obase = gbase ^ 4;
    Tile t;
    t.rb = (int)(code & 63u); t.cb = (int)((code >> 6) & 7u);
    if (t.rb < 4 * NALPHA) { t.src = gbase + t.rb / NALPHA; t.blk = t.rb % NALPHA; }
    else if (t.rb < 32) { t.src = obase + (t.rb - 4 * NALPHA) / NBETA; t.blk = NALPHA + (t.rb - 4 * NALPHA) % NBETA; }
    else { t.src = rank; t.blk = NALPHA + (t.rb - 32); }
    return t;
}

struct Frag { s16x8 a[4][2], b[4]; };

__global__ void __launch_bounds__(NTHREADS, 3) ag_gemm_kernel(const Params p) {
    __shared__ __attribute__((aligned(1024))) unsigned char lds_raw[LDS_BYTES];
    LAS unsigned char* L = (LAS unsigned char*)lds_raw;
    const int lane = threadIdx.x & 63;
    const int wave = (int)rfl(threadIdx.x >> 6);
    const int rank = p.rank;
    const int gidx = rank & 3, gbase = rank & 4, obase = gbase ^ 4;
    char* myws = p.ws[rank];
    const unsigned bid = blockIdx.x;

    auto draw_call = [&]() -> unsigned {
        unsigned ticket = 0;
        if (lane == 0)
            ticket = __hip_atomic_fetch_add((GAS unsigned*)(myws + OFF_CTR + bid * 128u + (unsigned)wave * 4u), 1u,
                                            __ATOMIC_RELAXED, __HIP_MEMORY_SCOPE_AGENT);
        return rfl(ticket) + 1u;
    };

    const int peer = (int)((p.xpeer >> ((bid & 7u) * 4u)) & 15u);
    if (bid < 8u * CPL && peer < NR) {
        if (wave < WPP) {
            if (wave == 0) STAMP(0);
            const int pi = wave * CPL + (int)(bid >> 3);
            const bool same = (peer & 4) == gbase;
            const __amdgpu_buffer_rsrc_t rx = mk_rsrc(p.x, SHARD_BYTES), rw = mk_rsrc(p.w, SHARD_BYTES);
            char* pws = p.ws[peer];
            const __amdgpu_buffer_rsrc_t rd = mk_rsrc(pws, WS_NEED);
            const unsigned xa0 = x_lane_off(lane), xa1 = xa0 ^ 64u;
            const unsigned wb0 = w_lane_off(lane);
            const unsigned dl = (unsigned)lane * 16u;

            auto load_task = [&](int t, u32x4 (&b)[8]) {
                if (!same && t < 128) {
                    const unsigned so = (unsigned)(t * 32) * 1024u + (unsigned)(peer & 3) * 256u;
#pragma unroll
                    for (int j = 0; j < 8; ++j)
                        b[j] = __builtin_amdgcn_raw_buffer_load_b128(rw, wb0, so + j * 4096u, 0);
                } else {
                    int ks, blk;
                    if (t < 128) { ks = t >> 1; blk = t & 1; }
                    else { const int u = t - 128; ks = u / 3; blk = (same ? NEARLY : NALPHA) + (u - ks * 3); }
                    const unsigned so = (unsigned)blk * (64u * 8192u) + (unsigned)ks * 128u;
#pragma unroll
                    for (int j = 0; j < 8; ++j)
                        b[j] = __builtin_amdgcn_raw_buffer_load_b128(rx, (j & 1) ? xa1 : xa0, so + j * 65536u, 0);
                }
            };
            u32x4 ra[8], rb[8];
            load_task(pi, ra);
            load_task(pi + NPW, rb);
            const unsigned call = draw_call();
            const unsigned par = OFF_DATA + (call & 1u) * PAR_BYTES;
            unsigned fof;
            auto store_task = [&](int t, const Packed& k) {
                unsigned dof;
                if (!same && t < 128) {
                    dof = OFF_WST + (unsigned)gidx * WQ_BYTES + (unsigned)t * CH_BYTES;
                    fof = OFF_FW + (unsigned)(gidx * 128 + t) * 8u;
                } else {
                    int ks, ai;
                    if (t < 128) { ks = t >> 1; ai = gidx * NALPHA + (t & 1); }
                    else {
                        const int u = t - 128; ks = u / 3;
                        const int r = u - ks * 3;
                        ai = same ? gidx * NALPHA + NEARLY + r : 4 * NALPHA + gidx * NBETA + r;
                    }
                    dof = (unsigned)ai * ABLK_BYTES + (unsigned)ks * CH_BYTES;
                    fof = OFF_FA + (unsigned)(ai * 64 + ks) * 8u;
                }
                dof += par;
                fof += (call & 1u) * FLG_PAR;
#pragma unroll
                for (int j = 0; j < 4; ++j) __builtin_amdgcn_raw_buffer_store_b128(k.lo[j], rd, dl, dof + j * 1024u, 17);
                if ((k.fmt >> 8) != 0xFFu) {
#pragma unroll
                    for (int j = 0; j < 2; ++j) __builtin_amdgcn_raw_buffer_store_b128(k.cd[j], rd, dl, dof + 4096u + j * 1024u, 17);
                }
#pragma unroll
                for (int j = 0; j < 8; ++j)
                    if ((k.fmt >> (8 + j)) & 1u) __builtin_amdgcn_raw_buffer_store_b64(k.hi[j], rd, dl >> 1, dof + 6144u + j * 512u, 17);
            };
            Packed pa, pb;
            pack_chunk(ra, pa);
            for (int i = 0; i < pi; ++i) __builtin_amdgcn_s_sleep(AG_STAGGER);
            auto round = [&](int t, const Packed& c, u32x4 (&rf)[8], const u32x4 (&rn)[8], Packed& n) {
                store_task(t, c);
                if (t == AG_GO_TASK && lane == 0) st_agent(myws + OFF_GO + (unsigned)peer * 128u, call);
                const unsigned fmt = c.fmt;
                if (t + 2 * NPW < NTASK) load_task(t + 2 * NPW, rf);
                if (t + NPW < NTASK) pack_chunk(rn, n);
                asm volatile("s_waitcnt vmcnt(0)" ::: "memory");
                __builtin_amdgcn_s_waitcnt(0x0F70);
                asm volatile("" ::: "memory");
                if (lane == 0)
                    __hip_atomic_store((GAS unsigned long long*)(pws + fof), (unsigned long long)call | ((unsigned long long)fmt << 32),
                                       __ATOMIC_RELAXED, __HIP_MEMORY_SCOPE_SYSTEM);
            };
            for (int t = pi; t < NTASK; t += 2 * NPW) {
                round(t, pa, ra, rb, pb);
                round(t + NPW, pb, rb, ra, pa);
            }
            if (wave == WPP - 1) STAMP(9 + ((peer - rank - 1) & 7));
            return;
        }
        if (wave != 7 || bid != 0u) return;
        const unsigned call = draw_call();
        const int i0 = (lane * 2) & 31, ex = 4 * NALPHA + gidx * NBETA;
        const bool skip0 = i0 >= ex && i0 < ex + NBETA, skip1 = i0 + 1 >= ex && i0 + 1 < ex + NBETA;
        const char* fp = myws + OFF_DONE + (unsigned)lane * 8u;
        unsigned long long t0 = 0;
        unsigned spins = 0;
        bool dead = false;
        for (;;) {
            u32x2 v;
            asm volatile("global_load_dwordx2 %0, %1, off sc0 sc1\n\ts_waitcnt vmcnt(0)" : "=&v"(v) : "v"(fp) : "memory");
            const bool ok = (skip0 || v.x >= call) && (skip1 || v.y >= call);
            if (__all(ok)) break;
            if ((++spins & 255u) == 0u) {
                if (rfl(ld_agent(myws + OFF_TMO)) == call || expired(t0, 2 * DEADLINE)) { dead = true; break; }
            }
            __builtin_amdgcn_s_sleep(2);
        }
        __builtin_amdgcn_fence(__ATOMIC_ACQUIRE, "");
        if (dead) {
            if (lane == 0) st_agent(myws + OFF_TMO, call);
            const unsigned long long t1 = rt();
            while (rt() - t1 < DEADLINE / 8) __builtin_amdgcn_s_sleep(64);
            if (lane == 0) st_sys(p.out[rank] + (size_t)rank * 512u * 1024u, 0x7FC07FC0u);
        }
        STAMP(6);
        return;
    }

    if ((unsigned)(size_t)L != 0u) return;
    if (wave == 0 && lane < 16) *(LAS unsigned*)(L + L_CTL + lane * 4) = 0u;
    const unsigned call = draw_call();
    const unsigned tcode = p.tile[bid];
    const unsigned ntile = (tcode >> 31) ? 2u : 1u;
    asm volatile("s_waitcnt lgkmcnt(0)" ::: "memory");
    __builtin_amdgcn_s_barrier();
    const unsigned par = OFF_DATA + (call & 1u) * PAR_BYTES;

    if (wave < NLOAD) {
        unsigned slot = (unsigned)wave % (unsigned)NSLOT;
        const __amdgpu_buffer_rsrc_t rws = mk_rsrc(myws, WS_NEED);
        const __amdgpu_buffer_rsrc_t rx = mk_rsrc(p.x, SHARD_BYTES), rw = mk_rsrc(p.w, SHARD_BYTES);
        const unsigned va0 = x_lane_off(lane), va1 = va0 ^ 64u, vb0 = w_lane_off(lane);
        unsigned g = (unsigned)wave, spins = 0;
        unsigned long long t0 = 0;
        bool dead = false;
        for (unsigned ti = 0; ti < ntile && !dead; ++ti) {
            const Tile t = tile_of(tcode >> (16u * ti), rank);
            const bool sa = t.src != rank, sb = t.cb >= 4;
            const unsigned abase = sa ? par + (unsigned)t.rb * ABLK_BYTES : (unsigned)t.blk * (64u * 8192u);
            const unsigned bbase = sb ? par + OFF_WST + (unsigned)(t.cb & 3) * WQ_BYTES : (unsigned)t.cb * 256u;
            const bool poll = lane == 0 ? sa : (lane < 3 && sb);
            const unsigned fl0 = (call & 1u) * FLG_PAR +
                (lane == 0 ? OFF_FA + (unsigned)(t.rb & 31) * 512u : OFF_FW + (unsigned)(t.cb & 3) * 1024u + (unsigned)(lane & 2) * 4u);
            const unsigned fks = lane == 0 ? 8u : 16u;
            for (; g < (ti + 1u) * (unsigned)NKS; g += (unsigned)NLOAD) {
                const unsigned ks = g & (unsigned)(NKS - 1);
                const unsigned lslot = slot * SLOT_BYTES, cfull = C_FULL + slot * 4u;
                slot = (slot + (unsigned)NLOAD) % (unsigned)NSLOT;
                u32x2 fl;
                auto poll_flags = [&](unsigned need) -> unsigned {
                    for (;;) {
                        fl.x = call; fl.y = 0u;
                        if (poll) fl = __builtin_amdgcn_raw_buffer_load_b64(rws, fl0 + ks * fks, 0, 17);
                        const unsigned up = (unsigned)__ballot(fl.x >= call) & 7u;
                        asm volatile("" ::: "memory");
                        if ((up & need) == need) return up;
                        if ((++spins & 255u) == 0u &&
                            (rfl(ld_agent(myws + OFF_TMO)) == call || lds_ld(C_ABORT) != 0u || expired(t0, DEADLINE))) {
                            dead = true;
                            return 0u;
                        }
                        __builtin_amdgcn_s_sleep(AG_POLL_SLEEP);
                    }
                };
                Wire wa, wb0, wb1;
                unsigned up = 7u, fa = 0u, fb0 = 0u, fb1 = 0u;
                const unsigned aoff = abase + ks * CH_BYTES, boff = bbase + ks * (2u * CH_BYTES);
                if (sa || sb) {
                    up = poll_flags(sb ? 6u : 1u);
                    if (dead) break;
                    if (sb) {
                        fb0 = (unsigned)__builtin_amdgcn_readlane((int)fl.y, 1); fb1 = (unsigned)__builtin_amdgcn_readlane((int)fl.y, 2);
                        wire_load(wb0, rws, boff, fb0, lane);
                        wire_load(wb1, rws, boff + CH_BYTES, fb1, lane);
                    }
                    if (sa && (up & 1u)) { fa = rfl(fl.y); wire_load(wa, rws, aoff, fa, lane); }
                }
                while (g >= (unsigned)NSLOT && !dead) {
                    const u32x4 fr = lds_ld4(C_FREE);
                    if (rfl(min(min(fr.x, fr.y), min(fr.z, fr.w))) + (unsigned)NSLOT > g) break;
                    if ((++spins & 1023u) == 0u && (lds_ld(C_ABORT) != 0u || expired(t0, 2 * DEADLINE))) dead = true;
                    __builtin_amdgcn_s_sleep(AG_FREE_SLEEP);
                }
                if (dead) break;
                if (!sa) dma8(lslot, va0, va1, rx, abase + ks * 128u, 65536u);
                if (!sb) {
                    dma8(lslot + 8192u, vb0, vb0, rw, bbase + ks * 65536u, 4096u);
                    dma8(lslot + 16384u, vb0, vb0, rw, bbase + ks * 65536u + 32768u, 4096u);
                } else {
                    wire_unpack(wb0, rws, boff, fb0, lane, lslot + 8192u);
                    wire_unpack(wb1, rws, boff + CH_BYTES, fb1, lane, lslot + 16384u);
                }
                if (sa) {
                    if (!(up & 1u)) {
                        poll_flags(1u);
                        if (dead) break;
                        fa = rfl(fl.y);
                        wire_load(wa, rws, aoff, fa, lane);
                    }
                    wire_unpack(wa, rws, aoff, fa, lane, lslot);
                }
                if (g == (ntile - 1u) * (unsigned)NKS + (unsigned)wave && wave == 0) STAMP(2);
                asm volatile("s_waitcnt vmcnt(0) lgkmcnt(0)" ::: "memory");
                lds_st(cfull, g + 1u);
            }
        }
        if (dead) {
            lds_st(C_ABORT, 1u);
            if (lane == 0) st_agent(myws + OFF_TMO, call);
        }
        if (wave == 0) {
            const int cb = tile_of(tcode >> (16u * (ntile - 1u)), rank).cb;
            if (cb >= 4) {
                const char* hp = myws + OFF_GO + (unsigned)(obase + (cb & 3)) * 128u;
                t0 = 0;
                while (!dead && rfl(ld_agent(hp)) < call) {
                    if ((++spins & 63u) == 0u && expired(t0, DEADLINE)) break;
                    __builtin_amdgcn_s_sleep(2);
                }
                lds_st(C_GO, 1u);
            }
        }
        if (wave == NLOAD - 1) STAMP(3);
        return;
    }

    const int cw = wave - NLOAD;
    const int hh = lane >> 5, r32 = lane & 31;
    const unsigned a0 = (unsigned)(r32 * 128 + ((hh ^ ((r32 >> 1) & 7)) << 4));
    const int q = (lane >> 2) & 3, pp = lane & 3;
    const unsigned b0 = 8192u + (unsigned)((8 * hh + q) * 256 + (((cw * 2 + ((lane >> 4) & 1)) ^ (q << 1)) << 5) + pp * 8);
    const unsigned cfree = C_FREE + (unsigned)cw * 4u;

    unsigned spins = 0;
    unsigned long long t0 = 0;
    auto wait_full = [&](unsigned slot, unsigned g) {
        while (lds_ld(C_FULL + slot * 4u) <= g) {
            if ((++spins & 63u) == 0u && lds_ld(C_ABORT) != 0u) break;
            if ((spins & 1023u) == 0u && expired(t0, 3 * DEADLINE)) { lds_st(C_ABORT, 1u); break; }
            __builtin_amdgcn_s_sleep(1);
        }
    };
    unsigned sl = 0, g = 0, fw = 0;
    auto rd = [&](Frag& f) {
        const unsigned sb = sl * SLOT_BYTES;
        fw = *(const volatile LAS unsigned*)(size_t)(C_FULL + (sl + 1u == (unsigned)NSLOT ? 0u : sl + 1u) * 4u);
#pragma unroll
        for (int k = 0; k < 4; ++k) {
#pragma unroll
            for (int mt = 0; mt < 2; ++mt)
                f.a[k][mt] = *(const LAS s16x8*)(size_t)(((a0 + sb) ^ (unsigned)(k * 32)) + mt * 4096u);
            const s16x4 lo = __builtin_amdgcn_ds_read_tr16_b64_v4i16((LAS s16x4*)(size_t)(b0 + sb + k * 4096u));
            const s16x4 hi = __builtin_amdgcn_ds_read_tr16_b64_v4i16((LAS s16x4*)(size_t)(b0 + sb + k * 4096u + 1024u));
            f.b[k] = __builtin_shufflevector(lo, hi, 0, 1, 2, 3, 4, 5, 6, 7);
        }
    };
    auto advance = [&]() {
        ++g;
        sl = sl + 1u == (unsigned)NSLOT ? 0u : sl + 1u;
    };
    f32x16 acc[2];
    auto mm = [&](const Frag& f) {
#pragma unroll
        for (int k = 0; k < 4; ++k)
#pragma unroll
            for (int mt = 0; mt < 2; ++mt)
                acc[mt] = __builtin_amdgcn_mfma_f32_32x32x16_bf16(__builtin_bit_cast(bf16x8, f.b[k]),
                                                                  __builtin_bit_cast(bf16x8, f.a[k][mt]), acc[mt], 0, 0, 0);
    };
    auto release = [&](unsigned n) {
        __builtin_amdgcn_sched_barrier(0);
        lds_st(cfree, n);
        __builtin_amdgcn_sched_barrier(0);
    };
    auto step = [&](const Frag& c, Frag& n) {
        const bool ready = rfl(fw) > g;
        __builtin_amdgcn_sched_barrier(0);
        rd(n);
        mm(c);
        __builtin_amdgcn_sched_group_barrier(0x100, 1, 0);
#pragma unroll
        for (int i = 0; i < 8; ++i) {
            __builtin_amdgcn_sched_group_barrier(0x008, 1, 0);
            __builtin_amdgcn_sched_group_barrier(0x100, 2, 0);
        }
        release(g);
        if (!ready) {
            wait_full(sl, g);
            rd(n);
        }
        advance();
    };

    for (unsigned ti = 0; ti < ntile; ++ti) {
#pragma unroll
        for (int i = 0; i < 16; ++i) { acc[0][i] = 0.f; acc[1][i] = 0.f; }
        Frag f0, f1;
        wait_full(sl, g);
        rd(f0);
        advance();
        for (int i = 0; i < NKS / 2 - 1; ++i) {
            step(f0, f1);
            step(f1, f0);
        }
        step(f0, f1);
        mm(f1);
        release(g);
        if (cw == 0) STAMP(7);

        const Tile t = tile_of(tcode >> (16u * ti), rank);
        const bool bad = lds_ld(C_ABORT) != 0u;
        const unsigned stb = L_ST + (unsigned)cw * ST_WAVE;
        u32x4 o[4];
#pragma unroll
        for (int mt = 0; mt < 2; ++mt) {
#pragma unroll
            for (int j = 0; j < 4; ++j) {
                bf16x2 lo, hi;
                lo[0] = (__bf16)acc[mt][4 * j]; lo[1] = (__bf16)acc[mt][4 * j + 1];
                hi[0] = (__bf16)acc[mt][4 * j + 2]; hi[1] = (__bf16)acc[mt][4 * j + 3];
                u32x2 v; v.x = __builtin_bit_cast(unsigned, lo); v.y = __builtin_bit_cast(unsigned, hi);
                if (bad) { v.x = 0x7FC07FC0u; v.y = 0x7FC07FC0u; }
                *(volatile LAS u32x2*)(size_t)(stb + (unsigned)(r32 * ST_ROW + j * 16 + hh * 8)) = v;
            }
#pragma unroll
            for (int i = 0; i < 2; ++i)
                o[mt * 2 + i] = *(const volatile LAS u32x4*)(size_t)(stb + (unsigned)((i * 16 + (lane >> 2)) * ST_ROW + (lane & 3) * 16));
        }

        const bool remote = t.cb >= 4;
        const int dstr = remote ? obase + (t.cb & 3) : rank;
        const unsigned colb = (unsigned)(remote ? gidx : t.cb) * 256u + (unsigned)cw * 64u;
        const unsigned orow = (unsigned)(t.src * 512 + t.blk * 64);
        const __amdgpu_buffer_rsrc_t ro = mk_rsrc(p.out[dstr], SHARD_BYTES);
        const unsigned ovo = (unsigned)(lane >> 2) * 1024u + (unsigned)(lane & 3) * 16u;
        const unsigned oso = orow * 1024u + colb;
        if (!remote) {
#pragma unroll
            for (int i = 0; i < 4; ++i) __builtin_amdgcn_raw_buffer_store_b128(o[i], ro, ovo, oso + i * 16384u, AG_LOCAL_AUX);
        } else {
            t0 = 0;
            while (lds_ld(C_GO) == 0u) {
                if ((++spins & 1023u) == 0u && expired(t0, 2 * DEADLINE)) break;
                __builtin_amdgcn_s_sleep(2);
            }
#pragma unroll
            for (int i = 0; i < 4; ++i) __builtin_amdgcn_raw_buffer_store_b128(o[i], ro, ovo, oso + i * 16384u, 17);
            VM_WAIT(0);
            unsigned old;
            asm volatile("ds_add_rtn_u32 %0, %1, %2\n\ts_waitcnt lgkmcnt(0)" : "=&v"(old) : "v"(C_CNT), "v"(lane == 0 ? 1u : 0u) : "memory");
            if (rfl(old) == 3u && lane == 0) st_sys(p.ws[dstr] + OFF_DONE + (unsigned)(gidx * 32 + t.rb) * 4u, call);
        }
        if (cw == 0) STAMP(8);
    }
}

}

extern "C" void kernel_launch(int rank, int nranks, ncclComm_t comm,
                              hipStream_t stream,
                              void* const* d_in, const int* in_sizes, int n_in,
                              void* d_out, int out_size,
                              const int* in_dtypes, int out_dtype,
                              void* const* d_out_all,
                              void* const* d_ws_all, size_t ws_bytes) {
    using namespace agk;
    (void)comm; (void)in_dtypes; (void)out_dtype;
    if (nranks != NR || (unsigned)rank >= (unsigned)NR || n_in < 2 || ws_bytes < (size_t)WS_NEED ||
        in_sizes[0] != 512 * 4096 || in_sizes[1] != 512 * 4096 || out_size != 512 * 4096) {
        fprintf(stderr, "kernel_launch: unexpected problem shape\n");
        return;
    }
    Params p;
    p.x = d_in[0];
    p.w = d_in[1];
    for (int r = 0; r < NR; ++r) { p.ws[r] = (char*)d_ws_all[r]; p.out[r] = (char*)d_out_all[r]; }
    p.out[rank] = (char*)d_out;
    p.rank = rank;
    const int gidx = rank & 3, obase = (rank & 4) ^ 4;
    int xcd_of[NR], spare = 3;
    {
        const unsigned setmask = 0xB2u;
        int lo = 0, hi = 4;
        for (int o = 1; o < NR; ++o) {
            const int q = (rank + o) & 7;
            xcd_of[q] = (((setmask >> q) ^ (setmask >> rank)) & 1u) ? hi++ : lo++;
        }
        xcd_of[rank] = spare;
        p.xpeer = 0u;
        for (int q = 0; q < NR; ++q) p.xpeer |= (unsigned)(q == rank ? 15 : q) << (xcd_of[q] * 4);
    }
    auto is_late = [&](int rb) { return rb < 32 && (rb >= 4 * NALPHA || (rb % NALPHA >= NEARLY && rb / NALPHA != gidx)); };
    auto code_of = [](int rb, int cb) { return (unsigned)rb | ((unsigned)cb << 6) | 0x8000u; };
    int next[8];
    for (int x = 0; x < 8; ++x) next[x] = x != spare ? x + 8 * CPL : x;
    for (int b = 0; b < NWG; ++b) p.tile[b] = 0u;
    int ntile = 0;
    for (int c = 0; c < 4; ++c) {
        const int x = xcd_of[obase + c];
        for (int pass = 0; pass < 2; ++pass)
            for (int rb = 0; rb < 32; ++rb) {
                if ((rb >= 4 * NALPHA && (rb - 4 * NALPHA) / NBETA == c) || is_late(rb) != (pass == 0)) continue;
                const int xx = next[x] < NWG ? x : spare;
                if (next[xx] >= NWG) { fprintf(stderr, "kernel_launch: placement\n"); return; }
                p.tile[next[xx]] = code_of(rb, 4 + c);
                next[xx] += 8;
                ++ntile;
            }
    }
    unsigned rest[140], first[64];
    int nr = 0, nf = 0;
    for (int pass = 0; pass < 3; ++pass)
        for (int rb = 0; rb < 35; ++rb)
            for (int c = 0; c < 4; ++c) {
                const int cls = is_late(rb) ? 2 : (rb >= 32 || rb / NALPHA == gidx) ? 0 : 1;
                if (cls != pass) continue;
                if (cls < 2 && nf < NPWG) first[nf++] = code_of(rb, c); else rest[nr++] = code_of(rb, c);
            }
    int cand[NWG], nc = 0;
    for (int x = 0, k = nr - 1; x < 8; ++x)
        for (; next[x] < NWG && k >= 0; next[x] += 8, --k) {
            p.tile[next[x]] = rest[k];
            if (is_late((int)(rest[k] & 63u))) cand[nc++] = next[x];
            ++ntile;
        }
    if (nc < nf || ntile + nf != NTILE || nf != NPWG) { fprintf(stderr, "kernel_launch: pairing %d %d %d\n", nc, nf, ntile); return; }
    for (int k = 0; k < nf; ++k) {
        const int b = cand[k * nc / nf];
        p.tile[b] = first[k] | (p.tile[b] << 16);
    }
    hipLaunchKernelGGL(ag_gemm_kernel, dim3(NWG), dim3(NTHREADS), 0, stream, p);
    HIP_CHECK(hipGetLastError());
}
```

```cpp
#include <hip/hip_runtime.h>
#include <rccl/rccl.h>
#include <cstdio>
#include <cstdint>

#define HIP_CHECK(expr) do { hipError_t _e = (expr); if (_e != hipSuccess) { \
    fprintf(stderr, "HIP error %s at %s:%d\n", hipGetErrorString(_e), __FILE__, __LINE__); \
    return; } } while (0)

namespace agk {

#define GAS __attribute__((address_space(1)))
#define LAS __attribute__((address_space(3)))
typedef unsigned u32x4 __attribute__((ext_vector_type(4)));
typedef unsigned u32x2 __attribute__((ext_vector_type(2)));
typedef short s16x4 __attribute__((ext_vector_type(4)));
typedef short s16x8 __attribute__((ext_vector_type(8)));
typedef __bf16 bf16x8 __attribute__((ext_vector_type(8)));
typedef __bf16 bf16x2 __attribute__((ext_vector_type(2)));
typedef float f32x16 __attribute__((ext_vector_type(16)));

constexpr int NR = 8;
constexpr int NKS = 64;
constexpr int NALPHA = 5, NBETA = 3;
constexpr int NEARLY = 2;
constexpr int NWG = 256;
constexpr int NTILE = 256;
constexpr int NTASK = 320;
#define AG_CPL 4
#define AG_WPP 4
#define AG_TOPO 1
constexpr int CPL = AG_CPL;
constexpr int WPP = AG_WPP;
constexpr int NPW = CPL * WPP;
constexpr int NPWG = 7 * CPL;
#define AG_POLL_SLEEP 2
#define AG_FREE_SLEEP 1
#define AG_STAGGER 4
#define AG_LOCAL_AUX 17
#define AG_HOLD 1
static_assert(NPWG <= 56 && WPP <= 7, "pushing workgroups: at most one per early tile with a local output");
constexpr unsigned SHARD_BYTES = 512u * 4096u * 2u;

constexpr unsigned OFF_CTR = 0;
constexpr unsigned OFF_TMO = 32u << 10;
constexpr unsigned OFF_GO = 34u << 10;
constexpr unsigned OFF_Q = 36u << 10;
constexpr unsigned OFF_FA = 64u << 10;
constexpr unsigned OFF_FW = 80u << 10;
constexpr unsigned FLG_PAR = 32u << 10;
constexpr unsigned OFF_DONE = 128u << 10;
constexpr unsigned OFF_DBG = 192u << 10;
constexpr unsigned OFF_DATA = 1u << 20;
constexpr unsigned CH_BYTES = 10240u;
constexpr unsigned ABLK_BYTES = NKS * CH_BYTES;
constexpr unsigned WQ_BYTES = NKS * 2u * CH_BYTES;
constexpr unsigned OFF_WST = 32u * ABLK_BYTES;
constexpr unsigned PAR_BYTES = OFF_WST + 4u * WQ_BYTES;
constexpr unsigned WS_NEED = OFF_DATA + 2u * PAR_BYTES;

constexpr int NSLOT = 6;
constexpr int NLOAD = 8;
constexpr unsigned SLOT_BYTES = 24576u;
constexpr unsigned L_ST = NSLOT * SLOT_BYTES;
constexpr unsigned ST_ROW = 80, ST_WAVE = 32 * ST_ROW;
constexpr unsigned L_CTL = L_ST + 4 * ST_WAVE;
constexpr unsigned C_FREE = L_CTL;
constexpr unsigned C_FULL = L_CTL + 16;
constexpr unsigned C_ABORT = L_CTL + 64;
constexpr unsigned C_CNT = L_CTL + 68;
constexpr unsigned C_GO = L_CTL + 72;
constexpr unsigned LDS_BYTES = L_CTL + 128;
static_assert(LDS_BYTES <= 160u * 1024u, "LDS");
constexpr int NTHREADS = (NLOAD + 4) * 64;

constexpr unsigned long long DEADLINE = 30000000ull;

struct Params {
    const void* x;
    const void* w;
    char* ws[NR];
    char* out[NR];
    int rank;
    unsigned xpeer;
    unsigned tile[NWG];
};

__device__ __forceinline__ unsigned long long rt() { return __builtin_amdgcn_s_memrealtime(); }
__device__ __forceinline__ bool expired(unsigned long long& t0, unsigned long long limit) {
    const unsigned long long now = __builtin_amdgcn_s_memrealtime() | 1ull;
    if (t0 == 0ull) { t0 = now; return false; }
    return now - t0 > limit;
}
__device__ __forceinline__ unsigned rfl(unsigned v) { return (unsigned)__builtin_amdgcn_readfirstlane((int)v); }
__device__ __forceinline__ __amdgpu_buffer_rsrc_t mk_rsrc(const void* p, unsigned bytes) {
    return __builtin_amdgcn_make_buffer_rsrc((void*)p, (short)0, (int)bytes, 0x00020000);
}
#define VM_WAIT(n) asm volatile("s_waitcnt vmcnt(" #n ")" ::: "memory")
__device__ __forceinline__ void lds_st(unsigned addr, unsigned v) {
    asm volatile("ds_write_b32 %0, %1" :: "v"(addr), "v"(v) : "memory");
}
__device__ __forceinline__ unsigned lds_ld(unsigned addr) {
    unsigned v;
    asm volatile("ds_read_b32 %0, %1\n\ts_waitcnt lgkmcnt(0)" : "=&v"(v) : "v"(addr) : "memory");
    return rfl(v);
}
__device__ __forceinline__ unsigned lds_ld_min2(unsigned addr) {
    u32x2 v;
    asm volatile("ds_read_b64 %0, %1\n\ts_waitcnt lgkmcnt(0)" : "=&v"(v) : "v"(addr) : "memory");
    return rfl(min(v.x, v.y));
}
__device__ __forceinline__ u32x4 lds_ld4(unsigned addr) {
    u32x4 v;
    asm volatile("ds_read_b128 %0, %1\n\ts_waitcnt lgkmcnt(0)" : "=&v"(v) : "v"(addr) : "memory");
    return v;
}
__device__ __forceinline__ unsigned ld_agent(const char* p) {
    return __hip_atomic_load((const GAS unsigned*)p, __ATOMIC_RELAXED, __HIP_MEMORY_SCOPE_AGENT);
}
__device__ __forceinline__ void st_agent(char* p, unsigned v) {
    __hip_atomic_store((GAS unsigned*)p, v, __ATOMIC_RELAXED, __HIP_MEMORY_SCOPE_AGENT);
}
__device__ __forceinline__ void st_sys(char* p, unsigned v) {
    __hip_atomic_store((GAS unsigned*)p, v, __ATOMIC_RELAXED, __HIP_MEMORY_SCOPE_SYSTEM);
}
#define DMA_P(V) "buffer_load_dwordx4 " V ", %5, %1 offen sc0 sc1 lds\n\ts_add_u32 m0, m0, 0x400\n\ts_add_u32 %1, %1, %7\n\t"
#define DMA_HEAD "s_nop 4\n\ts_mov_b32 %0, m0\n\ts_mov_b32 m0, %2\n\ts_mov_b32 %1, %6\n\t"
#define DMA_TAIL "s_mov_b32 m0, %0"
#define DMA8(PA, PB, PC, PD) \
    asm volatile(DMA_HEAD DMA_P(PA) DMA_P(PB) DMA_P(PC) DMA_P(PD) DMA_P(PA) DMA_P(PB) DMA_P(PC) DMA_P(PD) DMA_TAIL \
                 : "=&s"(keep), "=&s"(t) : "s"(lds), "v"(v0), "v"(v1), "s"(rs), "s"(soff), "s"(js) : "scc", "memory")
__device__ __forceinline__ void dma8(unsigned lds, unsigned v0, unsigned v1, __amdgpu_buffer_rsrc_t rs,
                                     unsigned soff, unsigned js) {
    unsigned keep, t;
    DMA8("%3", "%4", "%3", "%4");
}

#define STAMP(i) do {} while (0)

#define FDECL do {} while (0)
#define FACC(i) do {} while (0)
#define FOUT() do {} while (0)
__device__ __forceinline__ unsigned x_lane_off(int l) { return (unsigned)((l >> 3) * 8192 + (((l & 7) ^ (l >> 4)) << 4)); }
__device__ __forceinline__ unsigned w_lane_off(int l) { return (unsigned)((l >> 4) * 1024 + ((((l & 15) >> 1) ^ ((l >> 4) << 1)) << 5) + (l & 1) * 16); }

__device__ __forceinline__ unsigned bfi(unsigned m, unsigned a, unsigned b) { return (a & m) | (b & ~m); }
__device__ __forceinline__ unsigned pkmax(unsigned a, unsigned b) {
    typedef unsigned short u16x2 __attribute__((ext_vector_type(2)));
    return __builtin_bit_cast(unsigned, __builtin_elementwise_max(__builtin_bit_cast(u16x2, a), __builtin_bit_cast(u16x2, b)));
}
__device__ __forceinline__ void send_low(const u32x4 (&r)[8], u32x2 (&hi)[8], __amdgpu_buffer_rsrc_t rd, unsigned dof, int lane) {
#pragma unroll
    for (int i = 0; i < 4; ++i) {
        u32x4 lo;
        lo.x = __builtin_amdgcn_perm(r[2 * i].y, r[2 * i].x, 0x06040200u);
        lo.y = __builtin_amdgcn_perm(r[2 * i].w, r[2 * i].z, 0x06040200u);
        lo.z = __builtin_amdgcn_perm(r[2 * i + 1].y, r[2 * i + 1].x, 0x06040200u);
        lo.w = __builtin_amdgcn_perm(r[2 * i + 1].w, r[2 * i + 1].z, 0x06040200u);
        __builtin_amdgcn_raw_buffer_store_b128(lo, rd, lane * 16, dof + i * 1024u, 17);
    }
#pragma unroll
    for (int j = 0; j < 8; ++j) {
        hi[j].x = __builtin_amdgcn_perm(r[j].y, r[j].x, 0x07050301u);
        hi[j].y = __builtin_amdgcn_perm(r[j].w, r[j].z, 0x07050301u);
        asm volatile("" : "+v"(hi[j]));
    }
}
__device__ __forceinline__ unsigned send_high(const u32x2 (&hi)[8], __amdgpu_buffer_rsrc_t rd, unsigned dof, int lane) {
    unsigned m = 0;
#pragma unroll
    for (int j = 0; j < 8; ++j) {
        const unsigned e0 = hi[j].x & 0x7F7F7F7Fu, e1 = hi[j].y & 0x7F7F7F7Fu;
        m = pkmax(pkmax(m, e0), pkmax(e0 << 8, e1));
        m = pkmax(m, e1 << 8);
    }
    m = max(m >> 24, (m >> 8) & 0xFFu);
    m = max(m, (unsigned)__builtin_amdgcn_update_dpp(0, (int)m, 0x111, 0xF, 0xF, true));
    m = max(m, (unsigned)__builtin_amdgcn_update_dpp(0, (int)m, 0x112, 0xF, 0xF, true));
    m = max(m, (unsigned)__builtin_amdgcn_update_dpp(0, (int)m, 0x114, 0xF, 0xF, true));
    m = max(m, (unsigned)__builtin_amdgcn_update_dpp(0, (int)m, 0x118, 0xF, 0xF, true));
    const unsigned top = max(max((unsigned)__builtin_amdgcn_readlane((int)m, 15), (unsigned)__builtin_amdgcn_readlane((int)m, 31)),
                             max((unsigned)__builtin_amdgcn_readlane((int)m, 47), (unsigned)__builtin_amdgcn_readlane((int)m, 63)));
    const unsigned base = rfl(top > 7u ? top - 7u : 0u);
    const unsigned kk = 0x80808080u - base * 0x01010101u;
    unsigned mask = 0, fix = 0, all = 0x80808080u;
    u32x4 cd[2];
#pragma unroll
    for (int j = 0; j < 8; ++j) {
        const unsigned t0 = (hi[j].x & 0x7F7F7F7Fu) + kk, t1 = (hi[j].y & 0x7F7F7F7Fu) + kk;
        all &= t0 & t1;
        const unsigned c1 = bfi(0x07070707u, t1, hi[j].y);
        cd[j >> 2][j & 3] = bfi(0x87878787u, bfi(0x07070707u, t0, hi[j].x), __builtin_amdgcn_alignbit(c1, c1, 28));
    }
    const unsigned long long lanes = __ballot(all != 0x80808080u);
    if (__builtin_popcountll(lanes) > 6) {
        mask = 0xFFu;
    } else {
        __builtin_amdgcn_raw_buffer_store_b128(cd[0], rd, lane * 16, dof + 4096u, 17);
        __builtin_amdgcn_raw_buffer_store_b128(cd[1], rd, lane * 16, dof + 5120u, 17);
        if (lanes == 0ull) return base;
        const int l = __builtin_ctzll(lanes);
#pragma unroll
        for (int j = 0; j < 8; ++j) {
            const unsigned t0 = (hi[j].x & 0x7F7F7F7Fu) + kk, t1 = (hi[j].y & 0x7F7F7F7Fu) + kk;
            const unsigned long long out = __ballot((t0 & t1 & 0x80808080u) != 0x80808080u);
            const unsigned long long t = (unsigned long long)(unsigned)__builtin_amdgcn_readlane((int)t0, l) |
                                         ((unsigned long long)(unsigned)__builtin_amdgcn_readlane((int)t1, l) << 32);
            const unsigned long long bad = ~t & 0x8080808080808080ull;
            const bool one = fix == 0u && out == (1ull << l) && (bad & (bad - 1ull)) == 0ull;
            const unsigned by = (unsigned)__builtin_ctzll(bad | (1ull << 63)) >> 3;
            const unsigned tb = (unsigned)(t >> (by * 8u)) & 0xFFu;
            fix = one ? (16u - (tb >> 3)) | ((unsigned)j << 4) | (by << 7) | ((unsigned)l << 10) : fix;
            mask |= out != 0ull && !one ? 1u << j : 0u;
        }
    }
    if (mask != 0u) {
#pragma unroll
        for (int j = 0; j < 8; ++j)
            if ((mask >> j) & 1u) __builtin_amdgcn_raw_buffer_store_b64(hi[j], rd, lane * 8, dof + 6144u + j * 512u, 17);
    }
    return base | (mask << 8) | (fix << 16);
}
struct Wire { u32x4 lo[4]; u32x4 cd[2]; u32x2 e0; };
__device__ __forceinline__ void wire_load(Wire& w, const char* ws, bool go, unsigned off, unsigned fmt, int lane) {
    const __amdgpu_buffer_rsrc_t rs = mk_rsrc(ws, go ? WS_NEED : 0u);
#pragma unroll
    for (int i = 0; i < 4; ++i) w.lo[i] = __builtin_amdgcn_raw_buffer_load_b128(rs, lane * 16, off + i * 1024u, 17);
#pragma unroll
    for (int i = 0; i < 2; ++i) w.cd[i] = __builtin_amdgcn_raw_buffer_load_b128(rs, lane * 16, off + 4096u + i * 1024u, 17);
    const unsigned mask = (fmt >> 8) & 0xFFu, j0 = (unsigned)__builtin_ctz(mask | 0x80u);
    w.e0 = __builtin_amdgcn_raw_buffer_load_b64(mk_rsrc(ws, go && mask != 0u ? WS_NEED : 0u), lane * 8, off + 6144u + j0 * 512u, 17);
}
__device__ __forceinline__ void wire_put(unsigned ha, unsigned hb, unsigned l0, unsigned l1, unsigned lds) {
    u32x4 v;
    v.x = __builtin_amdgcn_perm(ha, l0, 0x05010400u);
    v.y = __builtin_amdgcn_perm(ha, l0, 0x07030602u);
    v.z = __builtin_amdgcn_perm(hb, l1, 0x05010400u);
    v.w = __builtin_amdgcn_perm(hb, l1, 0x07030602u);
    *(volatile LAS u32x4*)(size_t)lds = v;
}
__device__ __forceinline__ void wire_unpack(const Wire& w, const char* ws, unsigned off, unsigned fmt, int lane, unsigned lds) {
    const unsigned br = (fmt & 0x7Fu) * 0x01010101u, mask = (fmt >> 8) & 0xFFu;
    const unsigned la = lds + (unsigned)lane * 16u;
#pragma unroll
    for (int j = 0; j < 8; ++j) {
        const unsigned n = w.cd[j >> 2][j & 3];
        wire_put((n & 0x87878787u) + br, (__builtin_amdgcn_alignbit(n, n, 4) & 0x87878787u) + br,
                 w.lo[j >> 1][(j & 1) * 2], w.lo[j >> 1][(j & 1) * 2 + 1], la + j * 1024u);
    }
    if ((fmt & 0xF0000u) != 0u) {
        const unsigned by = (fmt >> 23) & 7u;
        const unsigned ad = lds + ((fmt >> 20) & 7u) * 1024u + (fmt >> 26) * 16u + (by >> 1) * 4u;
        const unsigned d = ((fmt >> 13) & 0x78u) << ((by & 1u) * 16u + 8u);
        if (lane == 0) asm volatile("ds_sub_u32 %0, %1" :: "v"(ad), "v"(d) : "memory");
    }
    if (mask != 0u) {
        const unsigned j0 = (unsigned)__builtin_ctz(mask);
        const __amdgpu_buffer_rsrc_t rs = mk_rsrc(ws, WS_NEED);
#pragma unroll
        for (int j = 0; j < 8; ++j) {
            if ((mask >> j) & 1u) {
                u32x2 e = w.e0;
                if ((unsigned)j != j0)
                    asm volatile("buffer_load_dwordx2 %0, %1, %2, %3 offen sc0 sc1\n\ts_waitcnt vmcnt(0)"
                                 : "=&v"(e) : "v"(lane * 8), "s"(rs), "s"(off + 6144u + j * 512u) : "memory");
                wire_put(e.x, e.y, w.lo[j >> 1][(j & 1) * 2], w.lo[j >> 1][(j & 1) * 2 + 1], la + j * 1024u);
            }
        }
    }
}
__device__ __forceinline__ unsigned flag_pos(unsigned ks) { return (ks & 3u) * 16u + (ks >> 2); }

struct Tile { int rb, cb, src, blk; };
__device__ __forceinline__ Tile tile_of(unsigned code, int rank) {
    const int gidx = rank & 3, gbase = rank & 4, obase = gbase ^ 4;
    Tile t;
    t.rb = (int)(code & 63u); t.cb = (int)((code >> 6) & 7u);
    if (t.rb < 4 * NALPHA) { t.src = gbase + t.rb / NALPHA; t.blk = t.rb % NALPHA; }
    else if (t.rb < 32) { t.src = obase + (t.rb - 4 * NALPHA) / NBETA; t.blk = NALPHA + (t.rb - 4 * NALPHA) % NBETA; }
    else { t.src = rank; t.blk = NALPHA + (t.rb - 32); }
    return t;
}

struct Frag { s16x8 a[4][2], b[4]; };
template <int N> struct IC { static constexpr int value = N; };

__global__ void __launch_bounds__(NTHREADS, 3) ag_gemm_kernel(const Params p) {
    __shared__ __attribute__((aligned(1024))) unsigned char lds_raw[LDS_BYTES];
    LAS unsigned char* L = (LAS unsigned char*)lds_raw;
    const int lane = threadIdx.x & 63;
    const int wave = (int)rfl(threadIdx.x >> 6);
    const int rank = p.rank;
    const int gidx = rank & 3, gbase = rank & 4, obase = gbase ^ 4;
    char* myws = p.ws[rank];
    const unsigned bid = blockIdx.x;

    auto draw_call = [&]() -> unsigned {
        unsigned ticket = 0;
        if (lane == 0)
            ticket = __hip_atomic_fetch_add((GAS unsigned*)(myws + OFF_CTR + bid * 128u + (unsigned)wave * 4u), 1u,
                                            __ATOMIC_RELAXED, __HIP_MEMORY_SCOPE_AGENT);
        return rfl(ticket) + 1u;
    };

    const int peer = (int)((p.xpeer >> ((bid & 7u) * 4u)) & 15u);
    if (bid < 8u * CPL && peer < NR) {
        if (wave < WPP) {
            if (wave == 0) STAMP(0);
            const int pi = wave * CPL + (int)(bid >> 3);
            const bool same = (peer & 4) == gbase;
            const __amdgpu_buffer_rsrc_t rx = mk_rsrc(p.x, SHARD_BYTES), rw = mk_rsrc(p.w, SHARD_BYTES);
            char* pws = p.ws[peer];
            const __amdgpu_buffer_rsrc_t rd = mk_rsrc(pws, WS_NEED);
            const unsigned xa0 = x_lane_off(lane), xa1 = xa0 ^ 64u;
            const unsigned wb0 = w_lane_off(lane);

            auto load_task = [&](int t, u32x4 (&b)[8]) {
                if (!same && t < 128) {
                    const unsigned so = (unsigned)(t * 32) * 1024u + (unsigned)(peer & 3) * 256u;
#pragma unroll
                    for (int j = 0; j < 8; ++j)
                        b[j] = __builtin_amdgcn_raw_buffer_load_b128(rw, wb0, so + j * 4096u, 0);
                } else {
                    int ks, blk;
                    if (t < 128) { ks = t >> 1; blk = t & 1; }
                    else { const int u = t - 128; ks = u / 3; blk = (same ? NEARLY : NALPHA) + (u - ks * 3); }
                    const unsigned so = (unsigned)blk * (64u * 8192u) + (unsigned)ks * 128u;
#pragma unroll
                    for (int j = 0; j < 8; ++j)
                        b[j] = __builtin_amdgcn_raw_buffer_load_b128(rx, (j & 1) ? xa1 : xa0, so + j * 65536u, 0);
                }
            };
            u32x4 ra[8];
            load_task(pi, ra);
            char* qp = myws + OFF_Q + (unsigned)peer * 128u;
            unsigned tk = 0;
            if (lane == 0)
                tk = __hip_atomic_fetch_add((GAS unsigned*)qp, 1u, __ATOMIC_RELAXED, __HIP_MEMORY_SCOPE_AGENT);
            const unsigned call = rfl(tk) / (unsigned)NTASK + 1u;
            const unsigned par = OFF_DATA + (call & 1u) * PAR_BYTES;
            const unsigned qbase = (call - 1u) * (unsigned)NTASK - (unsigned)NPW;
            auto draw = [&]() {
                unsigned long long sv;
                asm volatile("s_mov_b64 %1, exec\n\ts_mov_b64 exec, 1\n\tglobal_atomic_add %0, %2, %3, off sc0\n\ts_mov_b64 exec, %1"
                             : "=&v"(tk), "=&s"(sv) : "v"(qp), "v"(1u) : "memory");
            };
            unsigned dof, fof;
            auto place = [&](int t) {
                if (!same && t < 128) {
                    dof = OFF_WST + (unsigned)gidx * WQ_BYTES + (unsigned)t * CH_BYTES;
                    fof = OFF_FW + ((unsigned)gidx * 128u + flag_pos((unsigned)t >> 1) * 2u + ((unsigned)t & 1u)) * 8u;
                } else {
                    int ks, ai;
                    if (t < 128) { ks = t >> 1; ai = gidx * NALPHA + (t & 1); }
                    else {
                        const int u = t - 128; ks = u / 3;
                        const int r = u - ks * 3;
                        ai = same ? gidx * NALPHA + NEARLY + r : 4 * NALPHA + gidx * NBETA + r;
                    }
                    dof = (unsigned)ai * ABLK_BYTES + (unsigned)ks * CH_BYTES;
                    fof = OFF_FA + ((unsigned)ai * 64u + flag_pos((unsigned)ks)) * 8u;
                }
                dof += par;
                fof += (call & 1u) * FLG_PAR;
            };
            place(pi);
            __builtin_amdgcn_s_waitcnt(0x0F70);
            for (int i = 0; i < pi; ++i) __builtin_amdgcn_s_sleep(AG_STAGGER);
            for (int t = pi; t < NTASK;) {
                const int nx = (int)min(rfl(tk) - qbase, (unsigned)NTASK);
                u32x2 hi[8];
                send_low(ra, hi, rd, dof, lane);
                if (nx < NTASK) { load_task(nx, ra); draw(); }
                else if (lane == 0)
                    __hip_atomic_fetch_add((GAS unsigned*)(myws + OFF_GO + (unsigned)peer * 128u), 1u, __ATOMIC_RELAXED, __HIP_MEMORY_SCOPE_AGENT);
                const unsigned fmt = send_high(hi, rd, dof, lane);
                char* const fp = pws + fof;
                place(nx);
                asm volatile("s_waitcnt vmcnt(0)" : "+v"(tk) :: "memory");
                __builtin_amdgcn_s_waitcnt(0x0F70);
                asm volatile("" ::: "memory");
                if (lane == 0)
                    __hip_atomic_store((GAS unsigned long long*)fp, (unsigned long long)call | ((unsigned long long)fmt << 32),
                                       __ATOMIC_RELAXED, __HIP_MEMORY_SCOPE_SYSTEM);
                t = nx;
            }
            if (wave == WPP - 1) STAMP(9 + ((peer - rank - 1) & 7));
            return;
        }
        if (wave != 7 || bid != 0u) return;
        const unsigned call = draw_call();
        const int i0 = (lane * 2) & 31, ex = 4 * NALPHA + gidx * NBETA;
        const bool skip0 = i0 >= ex && i0 < ex + NBETA, skip1 = i0 + 1 >= ex && i0 + 1 < ex + NBETA;
        const char* fp = myws + OFF_DONE + (unsigned)lane * 8u;
        unsigned long long t0 = 0;
        unsigned spins = 0;
        bool dead = false;
        for (;;) {
            u32x2 v;
            asm volatile("global_load_dwordx2 %0, %1, off sc0 sc1\n\ts_waitcnt vmcnt(0)" : "=&v"(v) : "v"(fp) : "memory");
            const bool ok = (skip0 || v.x >= call) && (skip1 || v.y >= call);
            if (__all(ok)) break;
            if ((++spins & 255u) == 0u) {
                if (rfl(ld_agent(myws + OFF_TMO)) == call || expired(t0, 2 * DEADLINE)) { dead = true; break; }
            }
            __builtin_amdgcn_s_sleep(2);
        }
        __builtin_amdgcn_fence(__ATOMIC_ACQUIRE, "");
        if (dead) {
            if (lane == 0) st_agent(myws + OFF_TMO, call);
            const unsigned long long t1 = rt();
            while (rt() - t1 < DEADLINE / 8) __builtin_amdgcn_s_sleep(64);
            if (lane == 0) st_sys(p.out[rank] + (size_t)rank * 512u * 1024u, 0x7FC07FC0u);
        }
        STAMP(6);
        return;
    }

    if ((unsigned)(size_t)L != 0u) return;
    if (wave == 0 && lane < 32) *(LAS unsigned*)(L + L_CTL + lane * 4) = 0u;
    const unsigned call = draw_call();
    const unsigned tcode = p.tile[bid];
    const unsigned ntile = (tcode >> 31) ? 2u : 1u;
    asm volatile("s_waitcnt lgkmcnt(0)" ::: "memory");
    __builtin_amdgcn_s_barrier();
    const unsigned par = OFF_DATA + (call & 1u) * PAR_BYTES;

    if (wave < NLOAD) {
        const bool isb = wave >= 4;
        const unsigned lw = (unsigned)wave & 3u;
        unsigned slot = lw, g = lw, spins = 0;
        unsigned long long t0 = 0;
        bool dead = false;
        FDECL;
        auto wait_slot = [&]() {
            while (g >= (unsigned)NSLOT) {
                const u32x4 fr = lds_ld4(C_FREE);
                if (rfl(min(min(fr.x, fr.y), min(fr.z, fr.w))) + (unsigned)NSLOT > g) break;
                if ((++spins & 1023u) == 0u && (lds_ld(C_ABORT) != 0u || expired(t0, 2 * DEADLINE))) { dead = true; break; }
                __builtin_amdgcn_s_sleep(AG_FREE_SLEEP);
            }
        };
        auto publish = [&]() {
            lds_st(C_FULL + slot * 8u + (isb ? 4u : 0u), g + 1u);
            g += 4u;
            slot = slot + 4u >= (unsigned)NSLOT ? slot + 4u - (unsigned)NSLOT : slot + 4u;
        };
        auto staged = [&](auto ncc, unsigned base, unsigned fl0, unsigned loff) {
            constexpr int NC = decltype(ncc)::value;
            constexpr unsigned M = NC == 2 ? 3u : 1u;
            const __amdgpu_buffer_rsrc_t rws = mk_rsrc(myws, WS_NEED);
            u32x2 fl = {0u, 0u};
            unsigned up = 0u;
            auto refresh = [&]() {
                if (lane < 16 * NC) fl = __builtin_amdgcn_raw_buffer_load_b64(rws, fl0 + (unsigned)lane * 8u, 0, 17);
                up = (unsigned)__ballot(fl.x >= call);
                asm volatile("" ::: "memory");
            };
            auto await = [&](unsigned need) {
                for (;;) {
                    refresh();
                    if ((up & need) == need) return;
                    if ((++spins & 255u) == 0u &&
                        (rfl(ld_agent(myws + OFF_TMO)) == call || lds_ld(C_ABORT) != 0u || expired(t0, DEADLINE))) {
                        dead = true;
                        return;
                    }
                    __builtin_amdgcn_s_sleep(AG_POLL_SLEEP);
                }
            };
            auto fetch = [&](Wire (&w)[NC], unsigned (&f)[NC], unsigned i, bool go) {
#pragma unroll
                for (int c = 0; c < NC; ++c) {
                    f[c] = (unsigned)__builtin_amdgcn_readlane((int)fl.y, (int)((i & 15u) * NC + c));
                    wire_load(w[c], myws, go, base + ((i * 4u + lw) * NC + c) * CH_BYTES, f[c], lane);
                }
            };
            auto turn = [&](const Wire (&w)[NC], const unsigned (&f)[NC], Wire (&wn)[NC], unsigned (&fn)[NC], unsigned i) {
                const bool last = i == 15u;
                const unsigned need = last ? 0u : M << ((i + 1u) * NC);
                if ((up & need) != need) refresh();
                FACC(0);
                const bool ok = (up & need) == need;
                fetch(wn, fn, i + 1u, ok && !last);
                FACC(1);
                wait_slot();
                FACC(2);
                if (dead) return;
#pragma unroll
                for (int c = 0; c < NC; ++c)
                    wire_unpack(w[c], myws, base + ((i * 4u + lw) * NC + c) * CH_BYTES, f[c], lane, slot * SLOT_BYTES + loff + c * 8192u);
                asm volatile("s_waitcnt lgkmcnt(0)" ::: "memory");
                publish();
                FACC(4);
                if (!ok) {
                    await(need);
                    fetch(wn, fn, i + 1u, !dead);
                }
                FACC(5);
            };
            Wire w0[NC], w1[NC];
            unsigned f0[NC], f1[NC];
            await(M);
            fetch(w0, f0, 0u, !dead);
            FACC(5);
            for (unsigned i = 0; i < 16u && !dead; i += 2u) {
                turn(w0, f0, w1, f1, i);
                if (dead) break;
                turn(w1, f1, w0, f0, i + 1u);
            }
        };
        for (unsigned ti = 0; ti < ntile && !dead; ++ti) {
            const Tile t = tile_of(tcode >> (16u * ti), rank);
            const unsigned fpar = (call & 1u) * FLG_PAR;
            if (ti == ntile - 1u && wave == 0) STAMP(2);
            if (!isb && t.src != rank) {
                staged(IC<1>(), par + (unsigned)t.rb * ABLK_BYTES, fpar + OFF_FA + (unsigned)t.rb * 512u + lw * 128u, 0u);
            } else if (isb && t.cb >= 4) {
                staged(IC<2>(), par + OFF_WST + (unsigned)(t.cb & 3) * WQ_BYTES, fpar + OFF_FW + (unsigned)(t.cb & 3) * 1024u + lw * 256u, 8192u);
            } else {
                const __amdgpu_buffer_rsrc_t rs = mk_rsrc(isb ? p.w : p.x, SHARD_BYTES);
                const unsigned v0 = isb ? w_lane_off(lane) : x_lane_off(lane), v1 = isb ? v0 : v0 ^ 64u;
                const unsigned base = isb ? (unsigned)t.cb * 256u : (unsigned)t.blk * (64u * 8192u);
                for (unsigned ks = lw; ks < (unsigned)NKS; ks += 4u) {
                    wait_slot();
                    FACC(2);
                    if (dead) break;
                    if (isb) {
                        dma8(slot * SLOT_BYTES + 8192u, v0, v1, rs, base + ks * 65536u, 4096u);
                        dma8(slot * SLOT_BYTES + 16384u, v0, v1, rs, base + ks * 65536u + 32768u, 4096u);
                    } else {
                        dma8(slot * SLOT_BYTES, v0, v1, rs, base + ks * 128u, 65536u);
                    }
                    FACC(1);
                    VM_WAIT(0);
                    publish();
                    FACC(3);
                }
            }
        }
        FOUT();
        if (dead) {
            lds_st(C_ABORT, 1u);
            if (lane == 0) st_agent(myws + OFF_TMO, call);
        }
        if (wave == 0) {
            const int cb = tile_of(tcode >> (16u * (ntile - 1u)), rank).cb;
            if (cb >= 4) {
                const char* hp = myws + OFF_GO + (unsigned)(obase + (cb & 3)) * 128u;
                t0 = 0;
                while (!dead && rfl(ld_agent(hp)) < call * (unsigned)NPW) {
                    if ((++spins & 63u) == 0u && expired(t0, DEADLINE)) break;
                    __builtin_amdgcn_s_sleep(2);
                }
                lds_st(C_GO, 1u);
            }
        }
        if (wave == 3) STAMP(3);
        if (wave == 7) STAMP(5);
        return;
    }

    const int cw = wave - NLOAD;
    const int hh = lane >> 5, r32 = lane & 31;
    const unsigned a0 = (unsigned)(r32 * 128 + ((hh ^ ((r32 >> 1) & 7)) << 4));
    const int q = (lane >> 2) & 3, pp = lane & 3;
    const unsigned b0 = 8192u + (unsigned)((8 * hh + q) * 256 + (((cw * 2 + ((lane >> 4) & 1)) ^ (q << 1)) << 5) + pp * 8);
    const unsigned cfree = C_FREE + (unsigned)cw * 4u;

    unsigned spins = 0;
    unsigned long long t0 = 0;
    FDECL;
    auto wait_full = [&](unsigned slot, unsigned g) {
        while (lds_ld_min2(C_FULL + slot * 8u) <= g) {
            if ((++spins & 63u) == 0u && lds_ld(C_ABORT) != 0u) break;
            if ((spins & 1023u) == 0u && expired(t0, 3 * DEADLINE)) { lds_st(C_ABORT, 1u); break; }
            __builtin_amdgcn_s_sleep(1);
        }
    };
    unsigned sl = 0, g = 0;
    u32x2 fw = {0u, 0u};
    auto rd = [&](Frag& f) {
        const unsigned sb = sl * SLOT_BYTES;
        fw = *(const volatile LAS u32x2*)(size_t)(C_FULL + (sl + 1u == (unsigned)NSLOT ? 0u : sl + 1u) * 8u);
#pragma unroll
        for (int k = 0; k < 4; ++k) {
#pragma unroll
            for (int mt = 0; mt < 2; ++mt)
                f.a[k][mt] = *(const LAS s16x8*)(size_t)(((a0 + sb) ^ (unsigned)(k * 32)) + mt * 4096u);
            const s16x4 lo = __builtin_amdgcn_ds_read_tr16_b64_v4i16((LAS s16x4*)(size_t)(b0 + sb + k * 4096u));
            const s16x4 hi = __builtin_amdgcn_ds_read_tr16_b64_v4i16((LAS s16x4*)(size_t)(b0 + sb + k * 4096u + 1024u));
            f.b[k] = __builtin_shufflevector(lo, hi, 0, 1, 2, 3, 4, 5, 6, 7);
        }
    };
    auto advance = [&]() {
        ++g;
        sl = sl + 1u == (unsigned)NSLOT ? 0u : sl + 1u;
    };
    f32x16 acc[2];
    auto mm = [&](const Frag& f) {
#pragma unroll
        for (int k = 0; k < 4; ++k)
#pragma unroll
            for (int mt = 0; mt < 2; ++mt)
                acc[mt] = __builtin_amdgcn_mfma_f32_32x32x16_bf16(__builtin_bit_cast(bf16x8, f.b[k]),
                                                                  __builtin_bit_cast(bf16x8, f.a[k][mt]), acc[mt], 0, 0, 0);
    };
    auto release = [&](unsigned n) {
        __builtin_amdgcn_sched_barrier(0);
        lds_st(cfree, n);
        __builtin_amdgcn_sched_barrier(0);
    };
    auto step = [&](const Frag& c, Frag& n) {
        const bool ready = rfl(min(fw.x, fw.y)) > g;
        __builtin_amdgcn_sched_barrier(0);
        rd(n);
        mm(c);
        __builtin_amdgcn_sched_group_barrier(0x100, 1, 0);
#pragma unroll
        for (int i = 0; i < 8; ++i) {
            __builtin_amdgcn_sched_group_barrier(0x008, 1, 0);
            __builtin_amdgcn_sched_group_barrier(0x100, 2, 0);
        }
        release(g);
#pragma unroll
        for (int k = 0; k < 4; ++k) asm volatile("" :: "v"(c.a[k][0]), "v"(c.a[k][1]), "v"(c.b[k]));
        if (__builtin_expect(!ready, 0)) {
            FACC(0);
            wait_full(sl, g);
            FACC(1);
            rd(n);
            __builtin_amdgcn_s_waitcnt(0xC07F);
        }
        advance();
    };

    for (unsigned ti = 0; ti < ntile; ++ti) {
#pragma unroll
        for (int i = 0; i < 16; ++i) { acc[0][i] = 0.f; acc[1][i] = 0.f; }
        Frag f0, f1;
        wait_full(sl, g);
        FACC(3);
        rd(f0);
        __builtin_amdgcn_s_waitcnt(0xC07F);
        advance();
        for (int i = 0; i < NKS / 2 - 1; ++i) {
            step(f0, f1);
            step(f1, f0);
        }
        step(f0, f1);
        mm(f1);
        release(g);
        if (cw == 0) STAMP(7);
        FACC(0);
        FOUT();

        const Tile t = tile_of(tcode >> (16u * ti), rank);
        const bool bad = lds_ld(C_ABORT) != 0u;
        const unsigned stb = L_ST + (unsigned)cw * ST_WAVE;
        u32x4 o[4];
#pragma unroll
        for (int mt = 0; mt < 2; ++mt) {
#pragma unroll
            for (int j = 0; j < 4; ++j) {
                bf16x2 lo, hi;
                lo[0] = (__bf16)acc[mt][4 * j]; lo[1] = (__bf16)acc[mt][4 * j + 1];
                hi[0] = (__bf16)acc[mt][4 * j + 2]; hi[1] = (__bf16)acc[mt][4 * j + 3];
                u32x2 v; v.x = __builtin_bit_cast(unsigned, lo); v.y = __builtin_bit_cast(unsigned, hi);
                if (bad) { v.x = 0x7FC07FC0u; v.y = 0x7FC07FC0u; }
                *(volatile LAS u32x2*)(size_t)(stb + (unsigned)(r32 * ST_ROW + j * 16 + hh * 8)) = v;
            }
#pragma unroll
            for (int i = 0; i < 2; ++i)
                o[mt * 2 + i] = *(const volatile LAS u32x4*)(size_t)(stb + (unsigned)((i * 16 + (lane >> 2)) * ST_ROW + (lane & 3) * 16));
        }

        const bool remote = t.cb >= 4;
        const int dstr = remote ? obase + (t.cb & 3) : rank;
        const unsigned colb = (unsigned)(remote ? gidx : t.cb) * 256u + (unsigned)cw * 64u;
        const unsigned orow = (unsigned)(t.src * 512 + t.blk * 64);
        const __amdgpu_buffer_rsrc_t ro = mk_rsrc(p.out[dstr], SHARD_BYTES);
        const unsigned ovo = (unsigned)(lane >> 2) * 1024u + (unsigned)(lane & 3) * 16u;
        const unsigned oso = orow * 1024u + colb;
        if (!remote) {
#pragma unroll
            for (int i = 0; i < 4; ++i) __builtin_amdgcn_raw_buffer_store_b128(o[i], ro, ovo, oso + i * 16384u, AG_LOCAL_AUX);
        } else {
            t0 = 0;
            while (lds_ld(C_GO) == 0u) {
                if ((++spins & 1023u) == 0u && expired(t0, 2 * DEADLINE)) break;
                __builtin_amdgcn_s_sleep(2);
            }
#pragma unroll
            for (int i = 0; i < 4; ++i) __builtin_amdgcn_raw_buffer_store_b128(o[i], ro, ovo, oso + i * 16384u, 17);
            VM_WAIT(0);
            unsigned old;
            asm volatile("ds_add_rtn_u32 %0, %1, %2\n\ts_waitcnt lgkmcnt(0)" : "=&v"(old) : "v"(C_CNT), "v"(lane == 0 ? 1u : 0u) : "memory");
            if (rfl(old) == 3u && lane == 0) st_sys(p.ws[dstr] + OFF_DONE + (unsigned)(gidx * 32 + t.rb) * 4u, call);
        }
        if (cw == 0) STAMP(8);
    }
}

}

extern "C" void kernel_launch(int rank, int nranks, ncclComm_t comm,
                              hipStream_t stream,
                              void* const* d_in, const int* in_sizes, int n_in,
                              void* d_out, int out_size,
                              const int* in_dtypes, int out_dtype,
                              void* const* d_out_all,
                              void* const* d_ws_all, size_t ws_bytes) {
    using namespace agk;
    (void)comm; (void)in_dtypes; (void)out_dtype;
    if (nranks != NR || (unsigned)rank >= (unsigned)NR || n_in < 2 || ws_bytes < (size_t)WS_NEED ||
        in_sizes[0] != 512 * 4096 || in_sizes[1] != 512 * 4096 || out_size != 512 * 4096) {
        fprintf(stderr, "kernel_launch: unexpected problem shape\n");
        return;
    }
    Params p;
    p.x = d_in[0];
    p.w = d_in[1];
    for (int r = 0; r < NR; ++r) { p.ws[r] = (char*)d_ws_all[r]; p.out[r] = (char*)d_out_all[r]; }
    p.out[rank] = (char*)d_out;
    p.rank = rank;
    const int gidx = rank & 3, obase = (rank & 4) ^ 4;
    int xcd_of[NR], spare = 3;
    {
        const unsigned setmask = 0xB2u;
        int lo = 0, hi = 4;
        for (int o = 1; o < NR; ++o) {
            const int q = (rank + o) & 7;
            xcd_of[q] = (((setmask >> q) ^ (setmask >> rank)) & 1u) ? hi++ : lo++;
        }
        xcd_of[rank] = spare;
        p.xpeer = 0u;
        for (int q = 0; q < NR; ++q) p.xpeer |= (unsigned)(q == rank ? 15 : q) << (xcd_of[q] * 4);
    }
    auto is_late = [&](int rb) { return rb < 32 && (rb >= 4 * NALPHA || (rb % NALPHA >= NEARLY && rb / NALPHA != gidx)); };
    auto code_of = [](int rb, int cb) { return (unsigned)rb | ((unsigned)cb << 6) | 0x8000u; };
    int next[8];
    for (int x = 0; x < 8; ++x) next[x] = x != spare ? x + 8 * CPL : x;
    for (int b = 0; b < NWG; ++b) p.tile[b] = 0u;
    int ntile = 0;
    for (int c = 0; c < 4; ++c) {
        const int x = xcd_of[obase + c];
        for (int pass = 0; pass < 2; ++pass)
            for (int rb = 0; rb < 32; ++rb) {
                if ((rb >= 4 * NALPHA && (rb - 4 * NALPHA) / NBETA == c) || is_late(rb) != (pass == 0)) continue;
                const int xx = next[x] < NWG ? x : spare;
                if (next[xx] >= NWG) { fprintf(stderr, "kernel_launch: placement\n"); return; }
                p.tile[next[xx]] = code_of(rb, 4 + c);
                next[xx] += 8;
                ++ntile;
            }
    }
    unsigned rest[140], first[64];
    int nr = 0, nf = 0;
    for (int pass = 0; pass < 3; ++pass)
        for (int rb = 0; rb < 35; ++rb)
            for (int c = 0; c < 4; ++c) {
                const int cls = is_late(rb) ? 2 : (rb >= 32 || rb / NALPHA == gidx) ? 0 : 1;
                if (cls != pass) continue;
                if (cls < 2 && nf < NPWG) first[nf++] = code_of(rb, c); else rest[nr++] = code_of(rb, c);
            }
    int cand[NWG], nc = 0;
    for (int x = 0, k = nr - 1; x < 8; ++x)
        for (; next[x] < NWG && k >= 0; next[x] += 8, --k) {
            p.tile[next[x]] = rest[k];
            if (is_late((int)(rest[k] & 63u))) cand[nc++] = next[x];
            ++ntile;
        }
    if (nc < nf || ntile + nf != NTILE || nf != NPWG) { fprintf(stderr, "kernel_launch: pairing %d %d %d\n", nc, nf, ntile); return; }
    for (int k = 0; k < nf; ++k) {
        const int b = cand[k * nc / nf];
        p.tile[b] = first[k] | (p.tile[b] << 16);
    }
    hipLaunchKernelGGL(ag_gemm_kernel, dim3(NWG), dim3(NTHREADS), 0, stream, p);
    HIP_CHECK(hipGetLastError());
}
```

```cpp
#include <hip/hip_runtime.h>
#include <rccl/rccl.h>
#include <cstdio>
#include <cstdint>

#define HIP_CHECK(expr) do { hipError_t _e = (expr); if (_e != hipSuccess) { \
    fprintf(stderr, "HIP error %s at %s:%d\n", hipGetErrorString(_e), __FILE__, __LINE__); \
    return; } } while (0)

namespace agk {

#define GAS __attribute__((address_space(1)))
#define LAS __attribute__((address_space(3)))
typedef unsigned u32x4 __attribute__((ext_vector_type(4)));
typedef unsigned u32x2 __attribute__((ext_vector_type(2)));
typedef short s16x4 __attribute__((ext_vector_type(4)));
typedef short s16x8 __attribute__((ext_vector_type(8)));
typedef __bf16 bf16x8 __attribute__((ext_vector_type(8)));
typedef __bf16 bf16x2 __attribute__((ext_vector_type(2)));
typedef float f32x16 __attribute__((ext_vector_type(16)));
typedef unsigned u32x16 __attribute__((ext_vector_type(16)));

constexpr int NR = 8;
constexpr int NKS = 64;
constexpr int NALPHA = 5, NBETA = 3;
constexpr int NEARLY = 2;
constexpr int NWG = 256;
constexpr int NTILE = 256;
constexpr int NTASK = 320;
#define AG_CPL 8
#define AG_WPP 3
#define AG_TOPO 1
constexpr int CPL = AG_CPL;
constexpr int WPP = AG_WPP;
constexpr int NPW = CPL * WPP;
constexpr int NPWG = 7 * CPL;
#define AG_POLL_SLEEP 2
#define AG_FREE_SLEEP 1
#define AG_STAGGER 4
#define AG_LOCAL_AUX 17
#define AG_REL 1
#define AG_NEAGER 4
#define AG_TAU 27
#define AG_GOY 4
#define AG_SPLIT 1
#define AG_LOCK 1
#define AG_CDMA 1
#define AG_GO_SLEEP 4
#define AG_GOT 40
#define AG_DELTA 320
#define AG_EAGERW 2
#define AG_W2D 8
#define AG_PRIO 2
#define AG_KEEP 2
#define AG_HOLD 1
static_assert(NPWG <= 56 && WPP <= 7 && NPW <= 32, "pushing workgroups: at most one per early tile with a local output");
constexpr unsigned SHARD_BYTES = 512u * 4096u * 2u;

constexpr unsigned OFF_CTR = 0;
constexpr unsigned OFF_TMO = 32u << 10;
constexpr unsigned OFF_GO = 34u << 10;
constexpr unsigned OFF_Q = 36u << 10;
constexpr unsigned OFF_FA = 64u << 10;
constexpr unsigned OFF_FW = 80u << 10;
constexpr unsigned FLG_PAR = 32u << 10;
constexpr unsigned OFF_DONE = 128u << 10;
constexpr unsigned OFF_DBG = 192u << 10;
constexpr unsigned OFF_DATA = 1u << 20;
constexpr unsigned CH_BYTES = 10240u;
constexpr unsigned ABLK_BYTES = NKS * CH_BYTES;
constexpr unsigned WQ_BYTES = NKS * 2u * CH_BYTES;
constexpr unsigned OFF_WST = 32u * ABLK_BYTES;
constexpr unsigned PAR_BYTES = OFF_WST + 8u * WQ_BYTES;
constexpr unsigned WS_NEED = OFF_DATA + 2u * PAR_BYTES;

constexpr int NSLOT = 16;
constexpr int NLOAD = 4;
constexpr int BD = 8;
constexpr unsigned L_ST = NSLOT * 8192u;
constexpr unsigned ST_ROW = 272;
constexpr unsigned L_REC = L_ST + 64 * ST_ROW;
constexpr unsigned L_CTL = L_REC + 2048 + 64;
constexpr unsigned C_FREE = L_CTL;
constexpr unsigned C_ABORT = L_CTL + 16;
constexpr unsigned C_CNT = L_CTL + 20;
constexpr unsigned C_GO = L_CTL + 24;
constexpr unsigned LDS_BYTES = L_CTL + 64;
static_assert(LDS_BYTES <= 160u * 1024u && (L_REC & 15u) == 0u, "LDS");
constexpr int NTHREADS = (NLOAD + 4) * 64;

constexpr unsigned long long DEADLINE = 30000000ull;

struct Params {
    const void* x;
    const void* w;
    char* ws[NR];
    char* out[NR];
    int rank;
    unsigned xpeer;
    unsigned tile[NWG];
};

__device__ __forceinline__ unsigned long long rt() { return __builtin_amdgcn_s_memrealtime(); }
__device__ __forceinline__ bool expired(unsigned long long& t0, unsigned long long limit) {
    const unsigned long long now = __builtin_amdgcn_s_memrealtime() | 1ull;
    if (t0 == 0ull) { t0 = now; return false; }
    return now - t0 > limit;
}
__device__ __forceinline__ unsigned rfl(unsigned v) { return (unsigned)__builtin_amdgcn_readfirstlane((int)v); }
__device__ __forceinline__ __amdgpu_buffer_rsrc_t mk_rsrc(const void* p, unsigned bytes) {
    return __builtin_amdgcn_make_buffer_rsrc((void*)p, (short)0, (int)bytes, 0x00020000);
}
#define VM_WAIT_(n) asm volatile("s_waitcnt vmcnt(" #n ")" ::: "memory")
#define VM_WAIT(n) VM_WAIT_(n)
__device__ __forceinline__ void lds_st(unsigned addr, unsigned v) {
    asm volatile("ds_write_b32 %0, %1" :: "v"(addr), "v"(v) : "memory");
}
__device__ __forceinline__ unsigned lds_ld(unsigned addr) {
    unsigned v;
    asm volatile("ds_read_b32 %0, %1\n\ts_waitcnt lgkmcnt(0)" : "=&v"(v) : "v"(addr) : "memory");
    return rfl(v);
}
__device__ __forceinline__ u32x4 lds_ld4(unsigned addr) {
    u32x4 v;
    asm volatile("ds_read_b128 %0, %1\n\ts_waitcnt lgkmcnt(0)" : "=&v"(v) : "v"(addr) : "memory");
    return v;
}
__device__ __forceinline__ unsigned ld_agent(const char* p) {
    return __hip_atomic_load((const GAS unsigned*)p, __ATOMIC_RELAXED, __HIP_MEMORY_SCOPE_AGENT);
}
__device__ __forceinline__ void st_agent(char* p, unsigned v) {
    __hip_atomic_store((GAS unsigned*)p, v, __ATOMIC_RELAXED, __HIP_MEMORY_SCOPE_AGENT);
}
__device__ __forceinline__ void st_sys(char* p, unsigned v) {
    __hip_atomic_store((GAS unsigned*)p, v, __ATOMIC_RELAXED, __HIP_MEMORY_SCOPE_SYSTEM);
}
#define DMA_P(V) "buffer_load_dwordx4 " V ", %5, %1 offen sc0 sc1 lds\n\ts_add_u32 m0, m0, 0x400\n\ts_add_u32 %1, %1, %7\n\t"
#define DMA_C(V) "buffer_load_dwordx4 " V ", %5, %1 offen lds\n\ts_add_u32 m0, m0, 0x400\n\ts_add_u32 %1, %1, %7\n\t"
#define DMA_HEAD "s_nop 4\n\ts_mov_b32 %0, m0\n\ts_mov_b32 m0, %2\n\ts_mov_b32 %1, %6\n\t"
#define DMA_TAIL "s_mov_b32 m0, %0"
#define DMA8(PA, PB, PC, PD) \
    asm volatile(DMA_HEAD DMA_P(PA) DMA_P(PB) DMA_P(PC) DMA_P(PD) DMA_P(PA) DMA_P(PB) DMA_P(PC) DMA_P(PD) DMA_TAIL \
                 : "=&s"(keep), "=&s"(t) : "s"(lds), "v"(v0), "v"(v1), "s"(rs), "s"(soff), "s"(js) : "scc", "memory")
__device__ __forceinline__ void dma8(unsigned lds, unsigned v0, unsigned v1, __amdgpu_buffer_rsrc_t rs,
                                     unsigned soff, unsigned js) {
    unsigned keep, t;
    DMA8("%3", "%4", "%3", "%4");
}

__device__ __forceinline__ void dma8c(unsigned lds, unsigned v0, unsigned v1, __amdgpu_buffer_rsrc_t rs,
                                      unsigned soff, unsigned js) {
    unsigned keep, t;
    asm volatile(DMA_HEAD DMA_C("%3") DMA_C("%4") DMA_C("%3") DMA_C("%4") DMA_C("%3") DMA_C("%4") DMA_C("%3") DMA_C("%4") DMA_TAIL
                 : "=&s"(keep), "=&s"(t) : "s"(lds), "v"(v0), "v"(v1), "s"(rs), "s"(soff), "s"(js) : "scc", "memory");
}

#define STAMP(i) do {} while (0)

#define FDECL do {} while (0)
#define FACC(i) do {} while (0)
#define FOUT() do {} while (0)
__device__ __forceinline__ unsigned x_lane_off(int l) { return (unsigned)((l >> 3) * 8192 + (((l & 7) ^ (l >> 4)) << 4)); }

__device__ __forceinline__ void w_fetch(unsigned lds, __amdgpu_buffer_rsrc_t rw, unsigned q, unsigned t, int lane) {
    const unsigned v = (unsigned)((lane >> 3) * 1024 + (lane & 7) * 16);
    dma8c(lds, v, v, rw, (t >> 1) * 65536u + q * 256u + (t & 1u) * 128u, 8192u);
}
__device__ __forceinline__ void w_frags(u32x4 (&r)[8], unsigned lds, int lane) {
    const unsigned wr = lds + (unsigned)((8 * (lane >> 5) + ((lane >> 2) & 3)) * 128 + ((lane >> 4) & 1) * 32 + (lane & 3) * 8);
#pragma unroll
    for (int j = 0; j < 8; ++j) {
        const unsigned ad = wr + (unsigned)((j & 3) * 2048 + (j >> 2) * 64);
        const u32x2 k0 = __builtin_bit_cast(u32x2, __builtin_amdgcn_ds_read_tr16_b64_v4i16((LAS s16x4*)(size_t)ad));
        const u32x2 k1 = __builtin_bit_cast(u32x2, __builtin_amdgcn_ds_read_tr16_b64_v4i16((LAS s16x4*)(size_t)(ad + 512u)));
        r[j] = __builtin_shufflevector(k0, k1, 0, 1, 2, 3);
    }
}

__device__ __forceinline__ unsigned bfi(unsigned m, unsigned a, unsigned b) { return (a & m) | (b & ~m); }
__device__ __forceinline__ unsigned pkmax(unsigned a, unsigned b) {
    typedef unsigned short u16x2 __attribute__((ext_vector_type(2)));
    return __builtin_bit_cast(unsigned, __builtin_elementwise_max(__builtin_bit_cast(u16x2, a), __builtin_bit_cast(u16x2, b)));
}
struct Packed { u32x4 lo[4]; u32x4 cd[2]; u32x2 hi[8]; unsigned fmt; };
__device__ __forceinline__ void pack_split(const u32x4 (&r)[8], Packed& o) {
#pragma unroll
    for (int j = 0; j < 8; ++j) {
        o.lo[j >> 1][(j & 1) * 2] = __builtin_amdgcn_perm(r[j].y, r[j].x, 0x06040200u);
        o.lo[j >> 1][(j & 1) * 2 + 1] = __builtin_amdgcn_perm(r[j].w, r[j].z, 0x06040200u);
        o.hi[j].x = __builtin_amdgcn_perm(r[j].y, r[j].x, 0x07050301u);
        o.hi[j].y = __builtin_amdgcn_perm(r[j].w, r[j].z, 0x07050301u);
        asm volatile("" : "+v"(o.hi[j]));
        if (j & 1) asm volatile("" : "+v"(o.lo[j >> 1]));
    }
}
__device__ __forceinline__ void pack_codes(Packed& o) {
    unsigned m = 0;
#pragma unroll
    for (int j = 0; j < 8; ++j) {
        const unsigned e0 = o.hi[j].x & 0x7F7F7F7Fu, e1 = o.hi[j].y & 0x7F7F7F7Fu;
        m = pkmax(pkmax(m, e0), pkmax(e0 << 8, e1));
        m = pkmax(m, e1 << 8);
    }
    m = max(m >> 24, (m >> 8) & 0xFFu);
    m = max(m, (unsigned)__builtin_amdgcn_update_dpp(0, (int)m, 0x111, 0xF, 0xF, true));
    m = max(m, (unsigned)__builtin_amdgcn_update_dpp(0, (int)m, 0x112, 0xF, 0xF, true));
    m = max(m, (unsigned)__builtin_amdgcn_update_dpp(0, (int)m, 0x114, 0xF, 0xF, true));
    m = max(m, (unsigned)__builtin_amdgcn_update_dpp(0, (int)m, 0x118, 0xF, 0xF, true));
    const unsigned top = max(max((unsigned)__builtin_amdgcn_readlane((int)m, 15), (unsigned)__builtin_amdgcn_readlane((int)m, 31)),
                             max((unsigned)__builtin_amdgcn_readlane((int)m, 47), (unsigned)__builtin_amdgcn_readlane((int)m, 63)));
    const unsigned base = rfl(top > 7u ? top - 7u : 0u);
    const unsigned kk = 0x80808080u - base * 0x01010101u;
    unsigned all = 0x80808080u;
#pragma unroll
    for (int j = 0; j < 8; ++j) {
        const unsigned t0 = (o.hi[j].x & 0x7F7F7F7Fu) + kk, t1 = (o.hi[j].y & 0x7F7F7F7Fu) + kk;
        all &= t0 & t1;
        const unsigned c1 = bfi(0x07070707u, t1, o.hi[j].y);
        o.cd[j >> 2][j & 3] = bfi(0x87878787u, bfi(0x07070707u, t0, o.hi[j].x), __builtin_amdgcn_alignbit(c1, c1, 28));
    }
    asm volatile("" : "+v"(o.cd[0]), "+v"(o.cd[1]));
    o.fmt = base;
    const unsigned long long lanes = __ballot(all != 0x80808080u);
    if (lanes == 0ull) return;
    unsigned mask = 0, one = 0, c0 = 0, c1 = 0, cj = 0;
    const unsigned long long l1 = lanes & (0ull - lanes);
#pragma unroll
    for (int j = 0; j < 8; ++j) {
        const unsigned t0 = (o.hi[j].x & 0x7F7F7F7Fu) + kk, t1 = (o.hi[j].y & 0x7F7F7F7Fu) + kk;
        const bool out = (t0 & t1 & 0x80808080u) != 0x80808080u;
        const unsigned long long ob = __ballot(out);
        mask |= ob != 0ull ? 1u << j : 0u;
        one |= ob == l1 ? 1u << j : 0u;
        c0 = out ? t0 : c0; c1 = out ? t1 : c1; cj = out ? (unsigned)j : cj;
    }
    const int l = __builtin_ctzll(lanes);
    const unsigned j = (unsigned)__builtin_amdgcn_readlane((int)cj, l);
    const unsigned long long t = (unsigned long long)(unsigned)__builtin_amdgcn_readlane((int)c0, l) |
                                 ((unsigned long long)(unsigned)__builtin_amdgcn_readlane((int)c1, l) << 32);
    const unsigned long long bad = ~t & 0x8080808080808080ull;
    unsigned fix = 0;
    if (((one >> j) & 1u) && (bad & (bad - 1ull)) == 0ull) {
        const unsigned by = (unsigned)__builtin_ctzll(bad) >> 3;
        const unsigned tb = (unsigned)(t >> (by * 8u)) & 0xFFu;
        fix = (16u - (tb >> 3)) | (j << 4) | (by << 7) | ((unsigned)l << 10);
        mask &= ~(1u << j);
    }
    if (__builtin_popcount(mask) > 4) { mask = 0xFFu; fix = 0u; }
    o.fmt = base | (mask << 8) | (fix << 16);
}
__device__ __forceinline__ void send_low(const Packed& k, __amdgpu_buffer_rsrc_t rd, unsigned dof, int lane) {
#pragma unroll
    for (int i = 0; i < 4; ++i) __builtin_amdgcn_raw_buffer_store_b128(k.lo[i], rd, lane * 16, dof + i * 1024u, 17);
}
__device__ __forceinline__ void send_high(const Packed& k, __amdgpu_buffer_rsrc_t rd, unsigned dof, int lane) {
    if ((k.fmt & 0xFF00u) != 0xFF00u) {
        __builtin_amdgcn_raw_buffer_store_b128(k.cd[0], rd, lane * 16, dof + 4096u, 17);
        __builtin_amdgcn_raw_buffer_store_b128(k.cd[1], rd, lane * 16, dof + 5120u, 17);
    }
    if (__builtin_expect((k.fmt & 0xFF00u) != 0u, 0)) {
#pragma unroll
        for (int j = 0; j < 8; ++j)
            if ((k.fmt >> (8 + j)) & 1u) __builtin_amdgcn_raw_buffer_store_b64(k.hi[j], rd, lane * 8, dof + 6144u + j * 512u, 17);
    }
}
__device__ __forceinline__ void send_chunk(const Packed& k, __amdgpu_buffer_rsrc_t rd, unsigned dof, int lane) {
    send_low(k, rd, dof, lane);
    send_high(k, rd, dof, lane);
}
struct Wire { u32x4 lo[4]; u32x4 cd[2]; u32x2 e0; };
__device__ __forceinline__ void wire_load(Wire& w, const char* ws, bool go, unsigned off, unsigned fmt, int lane) {
    const __amdgpu_buffer_rsrc_t rs = mk_rsrc(ws, go ? WS_NEED : 0u);
#pragma unroll
    for (int i = 0; i < 4; ++i) w.lo[i] = __builtin_amdgcn_raw_buffer_load_b128(rs, lane * 16, off + i * 1024u, 17);
#pragma unroll
    for (int i = 0; i < 2; ++i) w.cd[i] = __builtin_amdgcn_raw_buffer_load_b128(rs, lane * 16, off + 4096u + i * 1024u, 17);
    const unsigned mask = (fmt >> 8) & 0xFFu, j0 = (unsigned)__builtin_ctz(mask | 0x80u);
    w.e0 = __builtin_amdgcn_raw_buffer_load_b64(mk_rsrc(ws, go && mask != 0u ? WS_NEED : 0u), lane * 8, off + 6144u + j0 * 512u, 17);
}
__device__ __forceinline__ void wire_put(unsigned ha, unsigned hb, unsigned l0, unsigned l1, unsigned lds) {
    u32x4 v;
    v.x = __builtin_amdgcn_perm(ha, l0, 0x05010400u);
    v.y = __builtin_amdgcn_perm(ha, l0, 0x07030602u);
    v.z = __builtin_amdgcn_perm(hb, l1, 0x05010400u);
    v.w = __builtin_amdgcn_perm(hb, l1, 0x07030602u);
    *(volatile LAS u32x4*)(size_t)lds = v;
}
__device__ __forceinline__ void wire_unpack(const Wire& w, const char* ws, unsigned off, unsigned fmt, int lane, unsigned lds) {
    const unsigned br = (fmt & 0x7Fu) * 0x01010101u, mask = (fmt >> 8) & 0xFFu;
    const unsigned la = lds + (unsigned)lane * 16u;
#pragma unroll
    for (int j = 0; j < 8; ++j) {
        const unsigned n = w.cd[j >> 2][j & 3];
        wire_put((n & 0x87878787u) + br, (__builtin_amdgcn_alignbit(n, n, 4) & 0x87878787u) + br,
                 w.lo[j >> 1][(j & 1) * 2], w.lo[j >> 1][(j & 1) * 2 + 1], la + j * 1024u);
    }
    if ((fmt & 0xF0000u) != 0u) {
        const unsigned by = (fmt >> 23) & 7u;
        const unsigned ad = lds + ((fmt >> 20) & 7u) * 1024u + (fmt >> 26) * 16u + (by >> 1) * 4u;
        const unsigned d = ((fmt >> 13) & 0x78u) << ((by & 1u) * 16u + 8u);
        if (lane == 0) asm volatile("ds_sub_u32 %0, %1" :: "v"(ad), "v"(d) : "memory");
    }
    if (mask != 0u) {
        const unsigned j0 = (unsigned)__builtin_ctz(mask);
        const __amdgpu_buffer_rsrc_t rs = mk_rsrc(ws, WS_NEED);
#pragma unroll
        for (int j = 0; j < 8; ++j) {
            if ((mask >> j) & 1u) {
                u32x2 e = w.e0;
                if ((unsigned)j != j0)
                    asm volatile("buffer_load_dwordx2 %0, %1, %2, %3 offen sc0 sc1\n\ts_waitcnt vmcnt(0)"
                                 : "=&v"(e) : "v"(lane * 8), "s"(rs), "s"(off + 6144u + j * 512u) : "memory");
                wire_put(e.x, e.y, w.lo[j >> 1][(j & 1) * 2], w.lo[j >> 1][(j & 1) * 2 + 1], la + j * 1024u);
            }
        }
    }
}
__device__ __forceinline__ unsigned flag_pos(unsigned ks) { return (ks & 3u) * 16u + (ks >> 2); }

struct Tile { int rb, cb, src, blk; };
__device__ __forceinline__ Tile tile_of(unsigned code, int rank) {
    const int gidx = rank & 3, gbase = rank & 4, obase = gbase ^ 4;
    Tile t;
    t.rb = (int)(code & 63u); t.cb = (int)((code >> 6) & 7u);
    if (t.rb < 4 * NALPHA) { t.src = gbase + t.rb / NALPHA; t.blk = t.rb % NALPHA; }
    else if (t.rb < 32) { t.src = obase + (t.rb - 4 * NALPHA) / NBETA; t.blk = NALPHA + (t.rb - 4 * NALPHA) % NBETA; }
    else { t.src = rank; t.blk = NALPHA + (t.rb - 32); }
    return t;
}

struct Frag { s16x8 a[4][2]; };
struct PackB { u32x4 lo[2], cd; };
template <int N> struct IC { static constexpr int value = N; };

__global__ void __launch_bounds__(NTHREADS, 2) ag_gemm_kernel(const Params p) {
    __shared__ __attribute__((aligned(1024))) unsigned char lds_raw[LDS_BYTES];
    LAS unsigned char* L = (LAS unsigned char*)lds_raw;
    const int lane = threadIdx.x & 63;
    const int wave = (int)rfl(threadIdx.x >> 6);
    const int rank = p.rank;
    const int gidx = rank & 3, gbase = rank & 4, obase = gbase ^ 4;
    char* myws = p.ws[rank];
    const unsigned bid = blockIdx.x;
    if ((unsigned)(size_t)L != 0u) return;

    auto draw_call = [&]() -> unsigned {
        unsigned ticket = 0;
        if (lane == 0)
            ticket = __hip_atomic_fetch_add((GAS unsigned*)(myws + OFF_CTR + bid * 128u + (unsigned)wave * 4u), 1u,
                                            __ATOMIC_RELAXED, __HIP_MEMORY_SCOPE_AGENT);
        return rfl(ticket) + 1u;
    };

    const int peer = (int)((p.xpeer >> ((bid & 7u) * 4u)) & 15u);
    if (bid < 8u * CPL && peer < NR) {
        if (threadIdx.x < 16) *(LAS unsigned*)(L + L_CTL + threadIdx.x * 4) = 0u;
        asm volatile("s_waitcnt lgkmcnt(0)" ::: "memory");
        __builtin_amdgcn_s_barrier();
        if (wave < WPP) {
            __builtin_amdgcn_s_waitcnt(0x0F70);
            if (wave == 0) STAMP(0);
            const int pi = wave * CPL + (int)(bid >> 3);
            const bool same = (peer & 4) == gbase;
            const __amdgpu_buffer_rsrc_t rx = mk_rsrc(p.x, SHARD_BYTES), rw = mk_rsrc(p.w, SHARD_BYTES);
            char* pws = p.ws[peer];
            const unsigned xa0 = x_lane_off(lane), xa1 = xa0 ^ 64u;
            const unsigned wl = (unsigned)wave * 8192u;

            auto load_task = [&](int t) {
                if (!same && t < 128) {
                    w_fetch(wl, rw, (unsigned)(peer & 3), (unsigned)t, lane);
                } else {
                    int ks, blk;
                    if (t < 128) { ks = t >> 1; blk = t & 1; }
                    else { const int u = t - 128; ks = u / 3; blk = (same ? NEARLY : NALPHA) + (u - ks * 3); }
                    dma8c(wl, xa0, xa1, rx, (unsigned)blk * (64u * 8192u) + (unsigned)ks * 128u, 65536u);
                }
            };
            auto split_task = [&](int t, Packed& k) {
                u32x4 ra[8];
                if (!same && t < 128) w_frags(ra, wl, lane);
                else {
#pragma unroll
                    for (int j = 0; j < 8; ++j) ra[j] = *(const volatile LAS u32x4*)(size_t)(wl + (unsigned)lane * 16u + j * 1024u);
                }
                pack_split(ra, k);
            };
            char* cp = myws + OFF_CTR + bid * 128u + (unsigned)wave * 4u;
            char* qp = myws + OFF_Q + (unsigned)peer * 128u;
            unsigned tk;
            asm volatile("global_load_dword %0, %1, off sc1" : "=&v"(tk) : "v"(cp) : "memory");
            load_task(pi);
            auto drain = [&]() {
                asm volatile("s_waitcnt vmcnt(0)" : "+v"(tk) :: "memory");
                __builtin_amdgcn_s_waitcnt(0x0F70);
                asm volatile("" ::: "memory");
            };
            drain();
            const unsigned call = rfl(tk) + 1u;
            const unsigned qbase = (call - 1u) * (unsigned)(NTASK - NPW) - 2u * (unsigned)NPW;
            const __amdgpu_buffer_rsrc_t rd = mk_rsrc(pws, WS_NEED);
            auto draw = [&]() {
                unsigned long long sv;
                asm volatile("s_mov_b64 %1, exec\n\ts_mov_b64 exec, 1\n\tglobal_atomic_add %0, %2, %3, off sc0\n\ts_mov_b64 exec, %1"
                             : "=&v"(tk), "=&s"(sv) : "v"(qp), "v"(1u) : "memory");
            };
            unsigned dof, fof;
            auto place = [&](int t) {
                if (!same && t < 128) {
                    dof = OFF_WST + (unsigned)(4 + gidx) * WQ_BYTES + (unsigned)t * CH_BYTES;
                    fof = OFF_FW + (unsigned)((4 + gidx) * 128 + t) * 8u;
                } else {
                    int ks, ai;
                    if (t < 128) { ks = t >> 1; ai = gidx * NALPHA + (t & 1); }
                    else {
                        const int u = t - 128; ks = u / 3;
                        const int r = u - ks * 3;
                        ai = same ? gidx * NALPHA + NEARLY + r : 4 * NALPHA + gidx * NBETA + r;
                    }
                    dof = (unsigned)ai * ABLK_BYTES + (unsigned)ks * CH_BYTES;
                    fof = OFF_FA + ((unsigned)ai * 64u + flag_pos((unsigned)ks)) * 8u;
                }
                dof += OFF_DATA + (call & 1u) * PAR_BYTES;
                fof += (call & 1u) * FLG_PAR;
            };
            unsigned spins = 0;
            unsigned long long t0 = 0;
            for (int t = pi, nx = pi + NPW; t < NTASK;) {
                Packed k;
                t = (int)rfl((unsigned)t); nx = (int)rfl((unsigned)nx);
                place(t);
                split_task(t, k);
                const bool eager = t < AG_EAGERW * CPL;
                if (eager) send_low(k, rd, dof, lane);
                if (nx < NTASK) { load_task(nx); draw(); }
                pack_codes(k);
                if (eager) {
                    send_high(k, rd, dof, lane);
                } else {
                    if (t == pi) __builtin_amdgcn_s_sleep(AG_W2D);
                    unsigned my;
                    asm volatile("ds_add_rtn_u32 %0, %1, %2\n\ts_waitcnt lgkmcnt(0)" : "=&v"(my) : "v"(C_CNT), "v"(lane == 0 ? 1u : 0u) : "memory");
                    my = rfl(my);
                    while (lds_ld(C_GO) < my) {
                        if ((++spins & 1023u) == 0u && expired(t0, DEADLINE)) break;
                        __builtin_amdgcn_s_sleep(1);
                    }
                    send_chunk(k, rd, dof, lane);
                    VM_WAIT(AG_REL);
                    lds_st(C_GO, my + 1u);
                }
                drain();
                if (lane == 0)
                    __hip_atomic_store((GAS unsigned long long*)(pws + fof), (unsigned long long)call | ((unsigned long long)k.fmt << 32),
                                       __ATOMIC_RELAXED, __HIP_MEMORY_SCOPE_SYSTEM);
                if (t == NTASK - AG_GOT && lane == 0)
                    __hip_atomic_store((GAS unsigned long long*)(myws + OFF_GO + (unsigned)peer * 128u), rt() + AG_DELTA,
                                       __ATOMIC_RELAXED, __HIP_MEMORY_SCOPE_AGENT);
                if (nx >= NTASK && lane == 0) st_agent(cp, call);
                t = nx;
                nx = nx < NTASK ? (int)min(rfl(tk) - qbase, (unsigned)NTASK) : NTASK;
            }
            return;
        }
        if (wave != 7 || bid != 0u) return;
        const unsigned call = draw_call();
        const int i0 = (lane * 2) & 31, ex = 4 * NALPHA + gidx * NBETA;
        const bool skip0 = i0 >= ex && i0 < ex + NBETA, skip1 = i0 + 1 >= ex && i0 + 1 < ex + NBETA;
        const char* fp = myws + OFF_DONE + (unsigned)lane * 8u;
        unsigned long long t0 = 0;
        unsigned spins = 0;
        bool dead = false;
        for (;;) {
            u32x2 v;
            asm volatile("global_load_dwordx2 %0, %1, off sc0 sc1\n\ts_waitcnt vmcnt(0)" : "=&v"(v) : "v"(fp) : "memory");
            const bool ok = (skip0 || v.x >= call) && (skip1 || v.y >= call);
            if (__all(ok)) break;
            if ((++spins & 255u) == 0u) {
                if (rfl(ld_agent(myws + OFF_TMO)) == call || expired(t0, 2 * DEADLINE)) { dead = true; break; }
            }
            __builtin_amdgcn_s_sleep(2);
        }
        __builtin_amdgcn_fence(__ATOMIC_ACQUIRE, "");
        if (dead) {
            if (lane == 0) st_agent(myws + OFF_TMO, call);
            const unsigned long long t1 = rt();
            while (rt() - t1 < DEADLINE / 8) __builtin_amdgcn_s_sleep(64);
            if (lane == 0) st_sys(p.out[rank] + (size_t)rank * 512u * 1024u, 0x7FC07FC0u);
        }
        STAMP(6);
        return;
    }

    const unsigned own = (bid - 64u) * 4u + (unsigned)wave - (unsigned)NLOAD;
    const bool packs = bid >= 64u && wave >= NLOAD && own < 512u;
    if (packs) w_fetch((unsigned)(8 + wave) * 8192u, mk_rsrc(p.w, SHARD_BYTES), own & 3u, own >> 2, lane);
    if (threadIdx.x < (2048 + 128) / 8) *(LAS unsigned long long*)(L + L_REC + threadIdx.x * 8) = 0ull;
    const unsigned call = draw_call();
    const unsigned tcode = p.tile[bid];
    const unsigned ntile = (tcode >> 31) ? 2u : 1u;
    asm volatile("s_waitcnt lgkmcnt(0)" ::: "memory");
    __builtin_amdgcn_s_barrier();
    const unsigned par = OFF_DATA + (call & 1u) * PAR_BYTES;

    if (wave < NLOAD) {
        const unsigned lw = (unsigned)wave;
        const unsigned long long born = rt();
        const __amdgpu_buffer_rsrc_t rws = mk_rsrc(myws, WS_NEED);
        const unsigned fpar = (call & 1u) * FLG_PAR;
        unsigned spins = 0;
        unsigned long long t0 = 0;
        bool dead = false;
        FDECL;
        auto give_up = [&]() -> bool {
            return (++spins & 255u) == 0u &&
                   (rfl(ld_agent(myws + OFF_TMO)) == call || lds_ld(C_ABORT) != 0u || expired(t0, DEADLINE));
        };
        auto wait_slot = [&](unsigned g) {
            while (g >= (unsigned)NSLOT - 4u) {
                const u32x4 fr = lds_ld4(C_FREE);
                if (rfl(min(min(fr.x, fr.y), min(fr.z, fr.w))) + ((unsigned)NSLOT - 4u) > g) break;
                if ((++spins & 1023u) == 0u && (lds_ld(C_ABORT) != 0u || expired(t0, 2 * DEADLINE))) { dead = true; break; }
                __builtin_amdgcn_s_sleep(AG_FREE_SLEEP);
            }
        };
        for (unsigned ti = 0; ti < ntile && !dead; ++ti) {
            const Tile t = tile_of(tcode >> (16u * ti), rank);
            const unsigned g0 = ti * (unsigned)NKS + lw;
            unsigned wn = 0;
            auto publish = [&](unsigned i) {
                const unsigned need = min(i * 4u + lw + (unsigned)BD + 1u, (unsigned)NKS);
                while (wn < need) {
                    const u32x4 v = __builtin_amdgcn_raw_buffer_load_b128(rws, fpar + OFF_FW + (unsigned)t.cb * 1024u + (unsigned)lane * 16u, 0, 17);
                    const bool up = v.x >= call && v.z >= call;
                    wn = (unsigned)__builtin_ctzll(~__ballot(up) | (1ull << 63)) + (__all(up) ? 1u : 0u);
                    u32x4 f;
                    f.x = (v.y & 0x0FFFu) | ((v.y >> 22) & 1u ? 0u : v.y & 0xFFFF0000u);
                    f.y = (v.y & 0x00FFu) | ((v.y >> 4) & 0x0F00u) | ((v.y >> 22) & 1u ? v.y & 0xFFFF0000u : 0u);
                    f.z = (v.w & 0x0FFFu) | ((v.w >> 22) & 1u ? 0u : v.w & 0xFFFF0000u);
                    f.w = (v.w & 0x00FFu) | ((v.w >> 4) & 0x0F00u) | ((v.w >> 22) & 1u ? v.w & 0xFFFF0000u : 0u);
                    if (up) {
                        const u32x2 f0 = {f.x, f.y}, f1 = {f.z, f.w};
                        asm volatile("ds_or_b64 %0, %1\n\tds_or_b64 %0, %2 offset:8" :: "v"(L_REC + ti * 1024u + (unsigned)lane * 16u), "v"(f0), "v"(f1) : "memory");
                    }
                    if (wn >= need) break;
                    if (give_up()) { dead = true; return; }
                    __builtin_amdgcn_s_sleep(AG_POLL_SLEEP);
                }
                asm volatile("s_waitcnt lgkmcnt(0)" ::: "memory");
                if (lane < 4) asm volatile("ds_or_b32 %0, %1" :: "v"(L_REC + (g0 + i * 4u) * 16u + (unsigned)lane * 4u), "v"(0x80u) : "memory");
            };
            if (ti == ntile - 1u && wave == 0) STAMP(2);
            if (t.src != rank) {
                const unsigned base = par + (unsigned)t.rb * ABLK_BYTES + lw * CH_BYTES;
                const unsigned fl0 = fpar + OFF_FA + (unsigned)t.rb * 512u + lw * 128u;
                u32x2 fl = {0u, 0u};
                unsigned up = 0u;
                auto refresh = [&]() {
                    if (lane < 16) fl = __builtin_amdgcn_raw_buffer_load_b64(rws, fl0 + (unsigned)lane * 8u, 0, 17);
                    up = (unsigned)__ballot(fl.x >= call);
                    asm volatile("" ::: "memory");
                };
                auto await = [&](unsigned need) {
                    for (;;) {
                        refresh();
                        if ((up & need) == need) return;
                        if (give_up()) { dead = true; return; }
                        __builtin_amdgcn_s_sleep(AG_POLL_SLEEP);
                    }
                };
                auto fetch = [&](Wire& w, unsigned& f, unsigned i, bool go) {
                    f = (unsigned)__builtin_amdgcn_readlane((int)fl.y, (int)(i & 15u));
                    wire_load(w, myws, go, base + i * (4u * CH_BYTES), f, lane);
                };
                auto turn = [&](const Wire& w, unsigned f, Wire& wx, unsigned& fx, unsigned i) {
                    const bool last = i == 15u;
                    const unsigned need = last ? 0u : 2u << i;
                    if ((up & need) != need) refresh();
                    FACC(0);
                    const bool ok = (up & need) == need;
                    fetch(wx, fx, i + 1u, ok && !last);
                    FACC(1);
                    wait_slot(g0 + i * 4u);
                    FACC(2);
                    if (dead) return;
                    wire_unpack(w, myws, base + i * (4u * CH_BYTES), f, lane, ((g0 + i * 4u) & (unsigned)(NSLOT - 1)) * 8192u);
                    FACC(4);
                    publish(i);
                    FACC(3);
                    if (!ok && !dead) {
                        await(need);
                        fetch(wx, fx, i + 1u, !dead);
                    }
                    FACC(5);
                };
                Wire w0, w1;
                unsigned f0, f1;
                await(1u);
                fetch(w0, f0, 0u, !dead);
                FACC(5);
                for (unsigned i = 0; i < 16u && !dead; i += 2u) {
                    turn(w0, f0, w1, f1, i);
                    if (dead) break;
                    turn(w1, f1, w0, f0, i + 1u);
                }
            } else {
                const __amdgpu_buffer_rsrc_t rx = mk_rsrc(p.x, SHARD_BYTES);
                const unsigned v0 = x_lane_off(lane), v1 = v0 ^ 64u;
                const unsigned base = (unsigned)t.blk * (64u * 8192u) + lw * 128u;
                for (unsigned i = 0; i < 18u && !dead; ++i) {
                    if (i < 16u) {
                        wait_slot(g0 + i * 4u);
                        if (dead) break;
                        dma8(((g0 + i * 4u) & (unsigned)(NSLOT - 1)) * 8192u, v0, v1, rx, base + i * 512u, 65536u);
                        VM_WAIT(16);
                    } else if (i == 16u) {
                        VM_WAIT(8);
                    } else {
                        VM_WAIT(0);
                    }
                    if (i >= 2u) publish(i - 2u);
                }
                VM_WAIT(0);
            }
        }
        FOUT();
        if (dead) {
            lds_st(C_ABORT, 1u);
            if (lane == 0) st_agent(myws + OFF_TMO, call);
        }
        if (wave == 0) {
            const int cb = tile_of(tcode >> (16u * (ntile - 1u)), rank).cb;
            if (cb >= 4) {
                char* hp = myws + OFF_GO + (unsigned)(obase + (cb & 3)) * 128u;
                unsigned long long go;
                t0 = 0;
                for (;;) {
                    go = __hip_atomic_load((const GAS unsigned long long*)hp, __ATOMIC_RELAXED, __HIP_MEMORY_SCOPE_AGENT);
                    if (dead || __all(go > born)) break;
                    if ((++spins & 63u) == 0u && expired(t0, DEADLINE)) break;
                    __builtin_amdgcn_s_sleep(AG_GO_SLEEP);
                }
                const unsigned long long now = rt();
                unsigned long long at = 0;
                if (lane == 0) {
                    __hip_atomic_fetch_max((GAS unsigned long long*)(hp + 1024), go > now ? go : now, __ATOMIC_RELAXED, __HIP_MEMORY_SCOPE_AGENT);
                    at = __hip_atomic_fetch_add((GAS unsigned long long*)(hp + 1024), (unsigned long long)AG_TAU, __ATOMIC_RELAXED, __HIP_MEMORY_SCOPE_AGENT);
                }
                const unsigned wt = rfl(at > now ? (unsigned)min(at - now, 2000ull) : 0u);
                while ((unsigned)(rt() - now) < wt) __builtin_amdgcn_s_sleep(1);
                lds_st(C_GO, 1u);
            }
        }
        if (wave == 3) STAMP(3);
        return;
    }

    const int cw = wave - NLOAD;
    __builtin_amdgcn_s_setprio(AG_PRIO);
    const int hh = lane >> 5, r32 = lane & 31;
    const unsigned a0 = (unsigned)(r32 * 128 + ((hh ^ ((r32 >> 1) & 7)) << 4));
    const unsigned cfree = C_FREE + (unsigned)cw * 4u;
    if (packs) {
        u32x4 r[8];
        Packed k;
        VM_WAIT(0);
        w_frags(r, (unsigned)(8 + wave) * 8192u, lane);
        pack_split(r, k);
        lds_st(cfree, 4u);
        pack_codes(k);
        send_chunk(k, mk_rsrc(myws, WS_NEED), par + OFF_WST + (own & 3u) * WQ_BYTES + (own >> 2) * CH_BYTES, lane);
        VM_WAIT(0);
        if (lane == 0)
            __hip_atomic_store((GAS unsigned long long*)(myws + (call & 1u) * FLG_PAR + OFF_FW + ((own & 3u) * 128u + (own >> 2)) * 8u),
                               (unsigned long long)call | ((unsigned long long)k.fmt << 32), __ATOMIC_RELAXED, __HIP_MEMORY_SCOPE_SYSTEM);
    } else {
        lds_st(cfree, 4u);
    }
    const unsigned vlo = (unsigned)lane * 16u + (unsigned)(cw & 1) * 2048u, vcd = (unsigned)lane * 16u + 4096u + (unsigned)(cw & 1) * 1024u;

    unsigned spins = 0;
    unsigned long long t0 = 0;
    FDECL;
    auto wait_full = [&](unsigned g) {
        while ((lds_ld(L_REC + g * 16u) & 0x80u) == 0u) {
            if ((++spins & 63u) == 0u && lds_ld(C_ABORT) != 0u) break;
            if ((spins & 1023u) == 0u && expired(t0, 3 * DEADLINE)) { lds_st(C_ABORT, 1u); break; }
            __builtin_amdgcn_s_sleep(1);
        }
    };
    unsigned ab[4];
#pragma unroll
    for (int k = 0; k < 4; ++k) ab[k] = a0 ^ (unsigned)(k * 32);
    auto rd = [&](Frag& f, auto sc) {
        constexpr unsigned S = decltype(sc)::value;
#pragma unroll
        for (int k = 0; k < 4; ++k)
#pragma unroll
            for (int mt = 0; mt < 2; ++mt)
                f.a[k][mt] = *(const LAS s16x8*)((const LAS unsigned char*)(size_t)ab[k] + (S * 8192u + mt * 4096u));
    };
    auto ldb = [&](PackB& b, __amdgpu_buffer_rsrc_t rs, unsigned off) {
        b.lo[0] = __builtin_amdgcn_raw_buffer_load_b128(rs, vlo, off, 17);
        b.lo[1] = __builtin_amdgcn_raw_buffer_load_b128(rs, vlo + 1024u, off, 17);
        b.cd = __builtin_amdgcn_raw_buffer_load_b128(rs, vcd, off, 17);
    };
    auto put = [&](u32x16& o, int k, unsigned ha, unsigned hb, const PackB& b) {
        const unsigned l0 = b.lo[k >> 1][(k & 1) * 2], l1 = b.lo[k >> 1][(k & 1) * 2 + 1];
        o[4 * k] = __builtin_amdgcn_perm(ha, l0, 0x05010400u);
        o[4 * k + 1] = __builtin_amdgcn_perm(ha, l0, 0x07030602u);
        o[4 * k + 2] = __builtin_amdgcn_perm(hb, l1, 0x05010400u);
        o[4 * k + 3] = __builtin_amdgcn_perm(hb, l1, 0x07030602u);
    };
    auto repair = [&](u32x16& o, const PackB& b, unsigned fmt, unsigned off) {
        if ((fmt & 0xF0000u) != 0u) {
            const unsigned by = (fmt >> 23) & 7u;
            o[((fmt >> 18) & 12u) + (by >> 1)] -= (unsigned)lane == (fmt >> 26) ? ((fmt >> 13) & 0x78u) << ((by & 1u) * 16u + 8u) : 0u;
        }
        if ((fmt & 0xF00u) != 0u) {
            const __amdgpu_buffer_rsrc_t rs = mk_rsrc(myws, WS_NEED);
#pragma unroll
            for (int k = 0; k < 4; ++k) {
                if ((fmt >> (8 + k)) & 1u) {
                    u32x2 e;
                    asm volatile("buffer_load_dwordx2 %0, %1, %2, %3 offen sc0 sc1\n\ts_waitcnt vmcnt(0)"
                                 : "=&v"(e) : "v"(lane * 8), "s"(rs), "s"(off + 6144u + (unsigned)((cw & 1) * 4 + k) * 512u) : "memory");
                    put(o, k, e.x, e.y, b);
                }
            }
        }
    };
    f32x16 acc[2];
    unsigned vrec = L_REC + (unsigned)cw * 4u;
    asm volatile("" : "+v"(vrec));
#define SB() __builtin_amdgcn_sched_barrier(0)

    for (unsigned ti = 0; ti < ntile; ++ti) {
#pragma unroll
        for (int i = 0; i < 16; ++i) { acc[0][i] = 0.f; acc[1][i] = 0.f; }
        const Tile t = tile_of(tcode >> (16u * ti), rank);
        const unsigned g0 = ti * (unsigned)NKS;
        const unsigned wbase = par + OFF_WST + (unsigned)t.cb * WQ_BYTES + (unsigned)(cw >> 1) * CH_BYTES;
        Frag fa[2];
        u32x16 fb[2];
        PackB pb[BD];
        wait_full(g0);
        FACC(3);
#pragma unroll
        for (int i = 0; i < BD; ++i) { ldb(pb[i], mk_rsrc(myws, WS_NEED), wbase + (unsigned)i * (2u * CH_BYTES)); SB(); }
        rd(fa[0], IC<0>());
        unsigned rec = rfl(*(const volatile LAS unsigned*)(size_t)vrec);
#pragma unroll
        for (int k = 0; k < 4; ++k) {
            const unsigned br = (rec & 0x7Fu) * 0x01010101u, n = pb[0].cd[k];
            put(fb[0], k, (n & 0x87878787u) + br, (__builtin_amdgcn_alignbit(n, n, 4) & 0x87878787u) + br, pb[0]);
        }
        if ((rec >> 8) != 0u) repair(fb[0], pb[0], rec, wbase);
        rec = *(const volatile LAS unsigned*)((const LAS unsigned char*)(size_t)vrec + 16);
        __builtin_amdgcn_s_waitcnt(0xC07F);
        for (unsigned o = 0; o < (unsigned)(NKS / BD); ++o) {
            const __amdgpu_buffer_rsrc_t rb = mk_rsrc(myws, o + 1u < (unsigned)(NKS / BD) ? WS_NEED : 0u);
            const unsigned s0 = o * (unsigned)BD, wo = wbase + s0 * (2u * CH_BYTES);
            auto step = [&](auto uc) {
                constexpr int U = decltype(uc)::value, C = U & 1, N = C ^ 1, X = (U + 1) & (BD - 1);
                const unsigned g = g0 + s0 + (unsigned)U + 1u;
                const unsigned fmt = rfl(rec);
                const unsigned br = (fmt & 0x7Fu) * 0x01010101u;
                if (U == BD - 1) {
#pragma unroll
                    for (int k = 0; k < 4; ++k) ab[k] ^= 65536u;
                }
                rec = *(const volatile LAS unsigned*)((const LAS unsigned char*)(size_t)vrec + (U + 2) * 16);
                SB();
#pragma unroll
                for (int k = 0; k < 4; ++k) {
                    const u32x4 bk = {fb[C][4 * k], fb[C][4 * k + 1], fb[C][4 * k + 2], fb[C][4 * k + 3]};
                    acc[0] = __builtin_amdgcn_mfma_f32_32x32x16_bf16(__builtin_bit_cast(bf16x8, bk), __builtin_bit_cast(bf16x8, fa[C].a[k][0]), acc[0], 0, 0, 0);
                    SB();
                    fa[N].a[k][0] = *(const LAS s16x8*)((const LAS unsigned char*)(size_t)ab[k] + X * 8192);
                    if (k == 0) ldb(pb[U], rb, wo + (unsigned)(U + BD) * (2u * CH_BYTES));
                    const unsigned n = pb[X].cd[k];
                    const unsigned ha = (n & 0x87878787u) + br, hb = (__builtin_amdgcn_alignbit(n, n, 4) & 0x87878787u) + br;
                    asm volatile("" :: "v"(fa[C].a[k][0]), "v"(ha), "v"(hb));
                    SB();
                    acc[1] = __builtin_amdgcn_mfma_f32_32x32x16_bf16(__builtin_bit_cast(bf16x8, bk), __builtin_bit_cast(bf16x8, fa[C].a[k][1]), acc[1], 0, 0, 0);
                    SB();
                    fa[N].a[k][1] = *(const LAS s16x8*)((const LAS unsigned char*)(size_t)ab[k] + (X * 8192 + 4096));
                    put(fb[N], k, ha, hb, pb[X]);
                    asm volatile("" :: "v"(fa[C].a[k][1]), "v"(fb[N][4 * k + 3]));
                    SB();
                }
                asm volatile("" :: "v"(fb[C]));
                if ((U & 3) == 3) lds_st(cfree, g + 4u);
                if (__builtin_expect((fmt >> 7) != 1u, 0)) {
                    if (g != g0 + (unsigned)NKS) {
                        FACC(0);
                        repair(fb[N], pb[X], fmt, wo + (unsigned)(U + 1) * (2u * CH_BYTES));
                        FACC(4);
                        if ((fmt & 0x80u) == 0u) {
                            FACC(0);
                            lds_st(cfree, g + 4u);
                            wait_full(g);
                            FACC(1);
                            rd(fa[N], IC<X>());
                            rec = *(const volatile LAS unsigned*)((const LAS unsigned char*)(size_t)vrec + (U + 2) * 16);
                            __builtin_amdgcn_s_waitcnt(0xC07F);
                            FACC(5);
                        }
                    }
                }
                SB();
            };
            step(IC<0>()); step(IC<1>()); step(IC<2>()); step(IC<3>());
            step(IC<4>()); step(IC<5>()); step(IC<6>()); step(IC<7>());
            vrec += (unsigned)BD * 16u;
        }
        if (cw == 0) STAMP(7);
        FACC(0);
        FOUT();

        const bool bad = lds_ld(C_ABORT) != 0u;
#pragma unroll
        for (int mt = 0; mt < 2; ++mt) {
#pragma unroll
            for (int j = 0; j < 4; ++j) {
                bf16x2 lo, hi;
                lo[0] = (__bf16)acc[mt][4 * j]; lo[1] = (__bf16)acc[mt][4 * j + 1];
                hi[0] = (__bf16)acc[mt][4 * j + 2]; hi[1] = (__bf16)acc[mt][4 * j + 3];
                u32x2 v; v.x = __builtin_bit_cast(unsigned, lo); v.y = __builtin_bit_cast(unsigned, hi);
                if (bad) { v.x = 0x7FC07FC0u; v.y = 0x7FC07FC0u; }
                *(volatile LAS u32x2*)(size_t)(L_ST + (unsigned)((mt * 32 + r32) * ST_ROW + cw * 64 + j * 16 + hh * 8)) = v;
            }
        }
        unsigned old;
        asm volatile("ds_add_rtn_u32 %0, %1, %2\n\ts_waitcnt lgkmcnt(0)" : "=&v"(old) : "v"(C_CNT), "v"(lane == 0 ? 1u : 0u) : "memory");
        if ((rfl(old) & 3u) != 3u) continue;
        u32x4 o[16];
#pragma unroll
        for (int i = 0; i < 16; ++i)
            o[i] = *(const volatile LAS u32x4*)(size_t)(L_ST + (unsigned)((i * 4 + (lane >> 4)) * ST_ROW + (lane & 15) * 16));

        const bool remote = t.cb >= 4;
        const int dstr = remote ? obase + (t.cb & 3) : rank;
        const __amdgpu_buffer_rsrc_t ro = mk_rsrc(p.out[dstr], SHARD_BYTES);
        const unsigned ovo = (unsigned)(lane >> 4) * 1024u + (unsigned)(lane & 15) * 16u;
        const unsigned oso = (unsigned)(t.src * 512 + t.blk * 64) * 1024u + (unsigned)(remote ? gidx : t.cb) * 256u;
        if (!remote) {
#pragma unroll
            for (int i = 0; i < 16; ++i) __builtin_amdgcn_raw_buffer_store_b128(o[i], ro, ovo, oso + i * 4096u, AG_LOCAL_AUX);
        } else {
            t0 = 0;
            while (lds_ld(C_GO) == 0u) {
                if ((++spins & 1023u) == 0u && expired(t0, 2 * DEADLINE)) break;
                __builtin_amdgcn_s_sleep(1);
            }
            STAMP(4);
#pragma unroll
            for (int i = 0; i < 16; ++i) __builtin_amdgcn_raw_buffer_store_b128(o[i], ro, ovo, oso + i * 4096u, 17);
            VM_WAIT(0);
            if (lane == 0) st_sys(p.ws[dstr] + OFF_DONE + (unsigned)(gidx * 32 + t.rb) * 4u, call);
        }
        STAMP(8);
    }
}

}

extern "C" void kernel_launch(int rank, int nranks, ncclComm_t comm,
                              hipStream_t stream,
                              void* const* d_in, const int* in_sizes, int n_in,
                              void* d_out, int out_size,
                              const int* in_dtypes, int out_dtype,
                              void* const* d_out_all,
                              void* const* d_ws_all, size_t ws_bytes) {
    using namespace agk;
    (void)comm; (void)in_dtypes; (void)out_dtype;
    if (nranks != NR || (unsigned)rank >= (unsigned)NR || n_in < 2 || ws_bytes < (size_t)WS_NEED ||
        in_sizes[0] != 512 * 4096 || in_sizes[1] != 512 * 4096 || out_size != 512 * 4096) {
        fprintf(stderr, "kernel_launch: unexpected problem shape\n");
        return;
    }
    Params p;
    p.x = d_in[0];
    p.w = d_in[1];
    for (int r = 0; r < NR; ++r) { p.ws[r] = (char*)d_ws_all[r]; p.out[r] = (char*)d_out_all[r]; }
    p.out[rank] = (char*)d_out;
    p.rank = rank;
    const int gidx = rank & 3, obase = (rank & 4) ^ 4;
    int xcd_of[NR], spare = 3;
    {
        const unsigned setmask = 0xB2u;
        int lo = 0, hi = 4;
        for (int o = 1; o < NR; ++o) {
            const int q = (rank + o) & 7;
            xcd_of[q] = (((setmask >> q) ^ (setmask >> rank)) & 1u) ? hi++ : lo++;
        }
        xcd_of[rank] = spare;
        p.xpeer = 0u;
        for (int q = 0; q < NR; ++q) p.xpeer |= (unsigned)(q == rank ? 15 : q) << (xcd_of[q] * 4);
    }
    auto is_late = [&](int rb) { return rb < 32 && (rb >= 4 * NALPHA || (rb % NALPHA >= NEARLY && rb / NALPHA != gidx)); };
    auto code_of = [](int rb, int cb) { return (unsigned)rb | ((unsigned)cb << 6) | 0x8000u; };
    int next[8];
    for (int x = 0; x < 8; ++x) next[x] = x != spare ? x + 8 * CPL : x;
    for (int b = 0; b < NWG; ++b) p.tile[b] = 0u;
    int ntile = 0;
    int late_x[4], used = 0;
    for (int pass = 0; pass < 2; ++pass)
        for (int c = 0; c < 4; ++c)
            for (int o = 0; o < 4; ++o) {
                const int x = xcd_of[(rank & 4) + o];
                if ((pass == 1 && late_x[c] >= 0)) break;
                late_x[c] = -1;
                if (((used >> x) & 1) || (pass == 0 && (x >> 2) != (xcd_of[obase + c] >> 2))) continue;
                late_x[c] = x; used |= 1 << x;
                break;
            }
    for (int c = 0; c < 4; ++c)
        for (int rb = 0; rb < 32; ++rb) {
            if (rb >= 4 * NALPHA && (rb - 4 * NALPHA) / NBETA == c) continue;
            const int xx = is_late(rb) ? late_x[c] : xcd_of[obase + c];
            if (next[xx] >= NWG) { fprintf(stderr, "kernel_launch: placement\n"); return; }
            p.tile[next[xx]] = code_of(rb, 4 + c);
            next[xx] += 8;
            ++ntile;
        }
    unsigned rest[140], first[64];
    int nr = 0, nf = 0;
    for (int pass = 0; pass < 3; ++pass)
        for (int rb = 0; rb < 35; ++rb)
            for (int c = 0; c < 4; ++c) {
                const int cls = is_late(rb) ? 2 : (rb >= 32 || rb / NALPHA == gidx) ? 0 : 1;
                if (cls != pass) continue;
                if (cls < 2 && nf < NPWG) first[nf++] = code_of(rb, c); else rest[nr++] = code_of(rb, c);
            }
    int cand[NWG], nc = 0;
    for (int x = 0, k = nr - 1; x < 8; ++x)
        for (; next[x] < NWG && k >= 0; next[x] += 8, --k) {
            p.tile[next[x]] = rest[k];
            if (is_late((int)(rest[k] & 63u))) cand[nc++] = next[x];
            ++ntile;
        }
    if (nc < nf || ntile + nf != NTILE || nf != NPWG) { fprintf(stderr, "kernel_launch: pairing %d %d %d\n", nc, nf, ntile); return; }
    for (int k = 0; k < nf; ++k) {
        const int b = cand[k * nc / nf];
        p.tile[b] = first[k] | (p.tile[b] << 16);
    }
    hipLaunchKernelGGL(ag_gemm_kernel, dim3(NWG), dim3(NTHREADS), 0, stream, p);
    HIP_CHECK(hipGetLastError());
}
```

```cpp
#include <hip/hip_runtime.h>
#include <rccl/rccl.h>
#include <cstdio>
#include <cstdint>

#define HIP_CHECK(expr) do { hipError_t _e = (expr); if (_e != hipSuccess) { \
    fprintf(stderr, "HIP error %s at %s:%d\n", hipGetErrorString(_e), __FILE__, __LINE__); \
    return; } } while (0)

namespace agk {

#define GAS __attribute__((address_space(1)))
#define LAS __attribute__((address_space(3)))
typedef unsigned u32x4 __attribute__((ext_vector_type(4)));
typedef unsigned u32x2 __attribute__((ext_vector_type(2)));
typedef short s16x4 __attribute__((ext_vector_type(4)));
typedef short s16x8 __attribute__((ext_vector_type(8)));
typedef __bf16 bf16x8 __attribute__((ext_vector_type(8)));
typedef __bf16 bf16x2 __attribute__((ext_vector_type(2)));
typedef float f32x16 __attribute__((ext_vector_type(16)));
typedef unsigned u32x16 __attribute__((ext_vector_type(16)));

constexpr int NR = 8;
constexpr int NKS = 64;
constexpr int NALPHA = 5, NBETA = 3;
constexpr int NEARLY = 2;
constexpr int NWG = 256;
constexpr int NTILE = 256;
constexpr int NTASK = 320;
#define AG_CPL 8
#define AG_WPP 3
#define AG_TOPO 1
constexpr int CPL = AG_CPL;
constexpr int WPP = AG_WPP;
constexpr int NPW = CPL * WPP;
constexpr int NPWG = 7 * CPL;
#define AG_POLL_SLEEP 2
#define AG_FREE_SLEEP 1
#define AG_STAGGER 4
#define AG_LOCAL_AUX 17
#define AG_REL 4
#define AG_NEAGER 4
#define AG_TAU 27
#define AG_GOY 4
#define AG_SPLIT 1
#define AG_LOCK 1
#define AG_CDMA 1
#define AG_GO_SLEEP 4
#define AG_GOT 40
#define AG_DELTA 320
#define AG_EAGERW 2
#define AG_W2D 8
#define AG_XAUX 17
#define AG_WAUX 17
#define AG_OWNC 0
#define AG_TWO 0
#define AG_WARM 0
#define AG_TDELAY 16
#define AG_PRIO 2
#define AG_KEEP 2
#define AG_HOLD 1
static_assert(NPWG <= 56 && WPP <= 7, "pushing workgroups: at most one per early tile with a local output");
constexpr unsigned SHARD_BYTES = 512u * 4096u * 2u;

constexpr unsigned OFF_CTR = 0;
constexpr unsigned OFF_TMO = 32u << 10;
constexpr unsigned OFF_GO = 34u << 10;
constexpr unsigned OFF_Q = 36u << 10;
constexpr unsigned OFF_FA = 64u << 10;
constexpr unsigned OFF_FW = 80u << 10;
constexpr unsigned FLG_PAR = 32u << 10;
constexpr unsigned OFF_DONE = 128u << 10;
constexpr unsigned OFF_DBG = 192u << 10;
constexpr unsigned OFF_DATA = 1u << 20;
constexpr unsigned CH_BYTES = 10240u;
constexpr unsigned ABLK_BYTES = NKS * CH_BYTES;
constexpr unsigned WQ_BYTES = NKS * 2u * CH_BYTES;
constexpr unsigned OFF_WST = 32u * ABLK_BYTES;
constexpr unsigned PAR_BYTES = OFF_WST + 8u * WQ_BYTES;
constexpr unsigned WS_NEED = OFF_DATA + 2u * PAR_BYTES;

constexpr int NSLOT = 16;
constexpr int NLOAD = 4;
constexpr int BD = 8;
constexpr unsigned L_ST = NSLOT * 8192u;
constexpr unsigned ST_ROW = 272;
constexpr unsigned L_REC = L_ST + 64 * ST_ROW;
constexpr unsigned L_CTL = L_REC + 2048 + 64;
constexpr unsigned C_FREE = L_CTL;
constexpr unsigned C_ABORT = L_CTL + 16;
constexpr unsigned C_CNT = L_CTL + 20;
constexpr unsigned C_GO = L_CTL + 24;
constexpr unsigned LDS_BYTES = L_CTL + 64;
static_assert(LDS_BYTES <= 160u * 1024u && (L_REC & 15u) == 0u, "LDS");
constexpr int NTHREADS = (NLOAD + 4) * 64;

constexpr unsigned long long DEADLINE = 30000000ull;

struct Params {
    const void* x;
    const void* w;
    char* ws[NR];
    char* out[NR];
    int rank;
    unsigned xpeer;
    unsigned tile[NWG];
};

__device__ __forceinline__ unsigned long long rt() { return __builtin_amdgcn_s_memrealtime(); }
__device__ __forceinline__ bool expired(unsigned long long& t0, unsigned long long limit) {
    const unsigned long long now = __builtin_amdgcn_s_memrealtime() | 1ull;
    if (t0 == 0ull) { t0 = now; return false; }
    return now - t0 > limit;
}
__device__ __forceinline__ unsigned rfl(unsigned v) { return (unsigned)__builtin_amdgcn_readfirstlane((int)v); }
__device__ __forceinline__ __amdgpu_buffer_rsrc_t mk_rsrc(const void* p, unsigned bytes) {
    return __builtin_amdgcn_make_buffer_rsrc((void*)p, (short)0, (int)bytes, 0x00020000);
}
#define VM_WAIT_(n) asm volatile("s_waitcnt vmcnt(" #n ")" ::: "memory")
#define VM_WAIT(n) VM_WAIT_(n)
__device__ __forceinline__ void lds_st(unsigned addr, unsigned v) {
    asm volatile("ds_write_b32 %0, %1" :: "v"(addr), "v"(v) : "memory");
}
__device__ __forceinline__ unsigned lds_ld(unsigned addr) {
    unsigned v;
    asm volatile("ds_read_b32 %0, %1\n\ts_waitcnt lgkmcnt(0)" : "=&v"(v) : "v"(addr) : "memory");
    return rfl(v);
}
__device__ __forceinline__ u32x4 lds_ld4(unsigned addr) {
    u32x4 v;
    asm volatile("ds_read_b128 %0, %1\n\ts_waitcnt lgkmcnt(0)" : "=&v"(v) : "v"(addr) : "memory");
    return v;
}
__device__ __forceinline__ unsigned ld_agent(const char* p) {
    return __hip_atomic_load((const GAS unsigned*)p, __ATOMIC_RELAXED, __HIP_MEMORY_SCOPE_AGENT);
}
__device__ __forceinline__ void st_agent(char* p, unsigned v) {
    __hip_atomic_store((GAS unsigned*)p, v, __ATOMIC_RELAXED, __HIP_MEMORY_SCOPE_AGENT);
}
__device__ __forceinline__ void st_sys(char* p, unsigned v) {
    __hip_atomic_store((GAS unsigned*)p, v, __ATOMIC_RELAXED, __HIP_MEMORY_SCOPE_SYSTEM);
}
#define DMA_P(V) "buffer_load_dwordx4 " V ", %5, %1 offen sc0 sc1 lds\n\ts_add_u32 m0, m0, 0x400\n\ts_add_u32 %1, %1, %7\n\t"
#define DMA_C(V) "buffer_load_dwordx4 " V ", %5, %1 offen lds\n\ts_add_u32 m0, m0, 0x400\n\ts_add_u32 %1, %1, %7\n\t"
#define DMA_HEAD "s_nop 4\n\ts_mov_b32 %0, m0\n\ts_mov_b32 m0, %2\n\ts_mov_b32 %1, %6\n\t"
#define DMA_TAIL "s_mov_b32 m0, %0"
#define DMA8(PA, PB, PC, PD) \
    asm volatile(DMA_HEAD DMA_P(PA) DMA_P(PB) DMA_P(PC) DMA_P(PD) DMA_P(PA) DMA_P(PB) DMA_P(PC) DMA_P(PD) DMA_TAIL \
                 : "=&s"(keep), "=&s"(t) : "s"(lds), "v"(v0), "v"(v1), "s"(rs), "s"(soff), "s"(js) : "scc", "memory")
__device__ __forceinline__ void dma8(unsigned lds, unsigned v0, unsigned v1, __amdgpu_buffer_rsrc_t rs,
                                     unsigned soff, unsigned js) {
    unsigned keep, t;
    DMA8("%3", "%4", "%3", "%4");
}

__device__ __forceinline__ void dma8c(unsigned lds, unsigned v0, unsigned v1, __amdgpu_buffer_rsrc_t rs,
                                      unsigned soff, unsigned js) {
    unsigned keep, t;
    asm volatile(DMA_HEAD DMA_C("%3") DMA_C("%4") DMA_C("%3") DMA_C("%4") DMA_C("%3") DMA_C("%4") DMA_C("%3") DMA_C("%4") DMA_TAIL
                 : "=&s"(keep), "=&s"(t) : "s"(lds), "v"(v0), "v"(v1), "s"(rs), "s"(soff), "s"(js) : "scc", "memory");
}

#define STAMP(i) do {} while (0)

#define FDECL do {} while (0)
#define FACC(i) do {} while (0)
#define FOUT() do {} while (0)
__device__ __forceinline__ unsigned x_lane_off(int l) { return (unsigned)((l >> 3) * 8192 + (((l & 7) ^ (l >> 4)) << 4)); }

__device__ __forceinline__ void w_fetch(unsigned lds, __amdgpu_buffer_rsrc_t rw, unsigned q, unsigned t, int lane) {
    const unsigned v = (unsigned)((lane >> 3) * 1024 + (lane & 7) * 16);
    dma8c(lds, v, v, rw, (t >> 1) * 65536u + q * 256u + (t & 1u) * 128u, 8192u);
}
__device__ __forceinline__ void w_frags(u32x4 (&r)[8], unsigned lds, int lane) {
    const unsigned wr = lds + (unsigned)((8 * (lane >> 5) + ((lane >> 2) & 3)) * 128 + ((lane >> 4) & 1) * 32 + (lane & 3) * 8);
#pragma unroll
    for (int j = 0; j < 8; ++j) {
        const unsigned ad = wr + (unsigned)((j & 3) * 2048 + (j >> 2) * 64);
        const u32x2 k0 = __builtin_bit_cast(u32x2, __builtin_amdgcn_ds_read_tr16_b64_v4i16((LAS s16x4*)(size_t)ad));
        const u32x2 k1 = __builtin_bit_cast(u32x2, __builtin_amdgcn_ds_read_tr16_b64_v4i16((LAS s16x4*)(size_t)(ad + 512u)));
        r[j] = __builtin_shufflevector(k0, k1, 0, 1, 2, 3);
    }
}

__device__ __forceinline__ unsigned bfi(unsigned m, unsigned a, unsigned b) { return (a & m) | (b & ~m); }
__device__ __forceinline__ unsigned pkmax(unsigned a, unsigned b) {
    typedef unsigned short u16x2 __attribute__((ext_vector_type(2)));
    return __builtin_bit_cast(unsigned, __builtin_elementwise_max(__builtin_bit_cast(u16x2, a), __builtin_bit_cast(u16x2, b)));
}
struct Packed { u32x4 lo[4]; u32x4 cd[2]; u32x2 hi[8]; unsigned fmt; };
__device__ __forceinline__ void pack_split(const u32x4 (&r)[8], Packed& o) {
#pragma unroll
    for (int j = 0; j < 8; ++j) {
        o.lo[j >> 1][(j & 1) * 2] = __builtin_amdgcn_perm(r[j].y, r[j].x, 0x06040200u);
        o.lo[j >> 1][(j & 1) * 2 + 1] = __builtin_amdgcn_perm(r[j].w, r[j].z, 0x06040200u);
        o.hi[j].x = __builtin_amdgcn_perm(r[j].y, r[j].x, 0x07050301u);
        o.hi[j].y = __builtin_amdgcn_perm(r[j].w, r[j].z, 0x07050301u);
        asm volatile("" : "+v"(o.hi[j]));
        if (j & 1) asm volatile("" : "+v"(o.lo[j >> 1]));
    }
}
__device__ __forceinline__ void pack_codes(Packed& o) {
    unsigned m = 0;
#pragma unroll
    for (int j = 0; j < 8; ++j) {
        const unsigned e0 = o.hi[j].x & 0x7F7F7F7Fu, e1 = o.hi[j].y & 0x7F7F7F7Fu;
        m = pkmax(pkmax(m, e0), pkmax(e0 << 8, e1));
        m = pkmax(m, e1 << 8);
    }
    m = max(m >> 24, (m >> 8) & 0xFFu);
    m = max(m, (unsigned)__builtin_amdgcn_update_dpp(0, (int)m, 0x111, 0xF, 0xF, true));
    m = max(m, (unsigned)__builtin_amdgcn_update_dpp(0, (int)m, 0x112, 0xF, 0xF, true));
    m = max(m, (unsigned)__builtin_amdgcn_update_dpp(0, (int)m, 0x114, 0xF, 0xF, true));
    m = max(m, (unsigned)__builtin_amdgcn_update_dpp(0, (int)m, 0x118, 0xF, 0xF, true));
    const unsigned top = max(max((unsigned)__builtin_amdgcn_readlane((int)m, 15), (unsigned)__builtin_amdgcn_readlane((int)m, 31)),
                             max((unsigned)__builtin_amdgcn_readlane((int)m, 47), (unsigned)__builtin_amdgcn_readlane((int)m, 63)));
    const unsigned base = rfl(top > 7u ? top - 7u : 0u);
    const unsigned kk = 0x80808080u - base * 0x01010101u;
    unsigned all = 0x80808080u;
#pragma unroll
    for (int j = 0; j < 8; ++j) {
        const unsigned t0 = (o.hi[j].x & 0x7F7F7F7Fu) + kk, t1 = (o.hi[j].y & 0x7F7F7F7Fu) + kk;
        all &= t0 & t1;
        const unsigned c1 = bfi(0x07070707u, t1, o.hi[j].y);
        o.cd[j >> 2][j & 3] = bfi(0x87878787u, bfi(0x07070707u, t0, o.hi[j].x), __builtin_amdgcn_alignbit(c1, c1, 28));
    }
    asm volatile("" : "+v"(o.cd[0]), "+v"(o.cd[1]));
    o.fmt = base;
    const unsigned long long lanes = __ballot(all != 0x80808080u);
    if (lanes == 0ull) return;
    unsigned mask = 0, one = 0, c0 = 0, c1 = 0, cj = 0;
    const unsigned long long l1 = lanes & (0ull - lanes);
#pragma unroll
    for (int j = 0; j < 8; ++j) {
        const unsigned t0 = (o.hi[j].x & 0x7F7F7F7Fu) + kk, t1 = (o.hi[j].y & 0x7F7F7F7Fu) + kk;
        const bool out = (t0 & t1 & 0x80808080u) != 0x80808080u;
        const unsigned long long ob = __ballot(out);
        mask |= ob != 0ull ? 1u << j : 0u;
        one |= ob == l1 ? 1u << j : 0u;
        c0 = out ? t0 : c0; c1 = out ? t1 : c1; cj = out ? (unsigned)j : cj;
    }
    const int l = __builtin_ctzll(lanes);
    const unsigned j = (unsigned)__builtin_amdgcn_readlane((int)cj, l);
    const unsigned long long t = (unsigned long long)(unsigned)__builtin_amdgcn_readlane((int)c0, l) |
                                 ((unsigned long long)(unsigned)__builtin_amdgcn_readlane((int)c1, l) << 32);
    const unsigned long long bad = ~t & 0x8080808080808080ull;
    unsigned fix = 0;
    if (((one >> j) & 1u) && (bad & (bad - 1ull)) == 0ull) {
        const unsigned by = (unsigned)__builtin_ctzll(bad) >> 3;
        const unsigned tb = (unsigned)(t >> (by * 8u)) & 0xFFu;
        fix = (16u - (tb >> 3)) | (j << 4) | (by << 7) | ((unsigned)l << 10);
        mask &= ~(1u << j);
    }
    if (__builtin_popcount(mask) > 4) { mask = 0xFFu; fix = 0u; }
    o.fmt = base | (mask << 8) | (fix << 16);
}
__device__ __forceinline__ void send_low(const Packed& k, __amdgpu_buffer_rsrc_t rd, unsigned dof, int lane) {
#pragma unroll
    for (int i = 0; i < 4; ++i) __builtin_amdgcn_raw_buffer_store_b128(k.lo[i], rd, lane * 16, dof + i * 1024u, 17);
}
__device__ __forceinline__ void send_high(const Packed& k, __amdgpu_buffer_rsrc_t rd, unsigned dof, int lane) {
    if ((k.fmt & 0xFF00u) != 0xFF00u) {
        __builtin_amdgcn_raw_buffer_store_b128(k.cd[0], rd, lane * 16, dof + 4096u, 17);
        __builtin_amdgcn_raw_buffer_store_b128(k.cd[1], rd, lane * 16, dof + 5120u, 17);
    }
    if (__builtin_expect((k.fmt & 0xFF00u) != 0u, 0)) {
#pragma unroll
        for (int j = 0; j < 8; ++j)
            if ((k.fmt >> (8 + j)) & 1u) __builtin_amdgcn_raw_buffer_store_b64(k.hi[j], rd, lane * 8, dof + 6144u + j * 512u, 17);
    }
}
__device__ __forceinline__ void send_chunk(const Packed& k, __amdgpu_buffer_rsrc_t rd, unsigned dof, int lane) {
    send_low(k, rd, dof, lane);
    send_high(k, rd, dof, lane);
}
struct Wire { u32x4 lo[4]; u32x4 cd[2]; u32x2 e0; };
__device__ __forceinline__ void wire_load(Wire& w, const char* ws, bool go, unsigned off, unsigned fmt, int lane) {
    const __amdgpu_buffer_rsrc_t rs = mk_rsrc(ws, go ? WS_NEED : 0u);
#pragma unroll
    for (int i = 0; i < 4; ++i) w.lo[i] = __builtin_amdgcn_raw_buffer_load_b128(rs, lane * 16, off + i * 1024u, AG_XAUX);
#pragma unroll
    for (int i = 0; i < 2; ++i) w.cd[i] = __builtin_amdgcn_raw_buffer_load_b128(rs, lane * 16, off + 4096u + i * 1024u, AG_XAUX);
    const unsigned mask = (fmt >> 8) & 0xFFu, j0 = (unsigned)__builtin_ctz(mask | 0x80u);
    w.e0 = __builtin_amdgcn_raw_buffer_load_b64(mk_rsrc(ws, go && mask != 0u ? WS_NEED : 0u), lane * 8, off + 6144u + j0 * 512u, 17);
}
__device__ __forceinline__ void wire_put(unsigned ha, unsigned hb, unsigned l0, unsigned l1, unsigned lds) {
    u32x4 v;
    v.x = __builtin_amdgcn_perm(ha, l0, 0x05010400u);
    v.y = __builtin_amdgcn_perm(ha, l0, 0x07030602u);
    v.z = __builtin_amdgcn_perm(hb, l1, 0x05010400u);
    v.w = __builtin_amdgcn_perm(hb, l1, 0x07030602u);
    *(volatile LAS u32x4*)(size_t)lds = v;
}
__device__ __forceinline__ void wire_unpack(const Wire& w, const char* ws, unsigned off, unsigned fmt, int lane, unsigned lds) {
    const unsigned br = (fmt & 0x7Fu) * 0x01010101u, mask = (fmt >> 8) & 0xFFu;
    const unsigned la = lds + (unsigned)lane * 16u;
#pragma unroll
    for (int j = 0; j < 8; ++j) {
        const unsigned n = w.cd[j >> 2][j & 3];
        wire_put((n & 0x87878787u) + br, (__builtin_amdgcn_alignbit(n, n, 4) & 0x87878787u) + br,
                 w.lo[j >> 1][(j & 1) * 2], w.lo[j >> 1][(j & 1) * 2 + 1], la + j * 1024u);
    }
    if ((fmt & 0xF0000u) != 0u) {
        const unsigned by = (fmt >> 23) & 7u;
        const unsigned ad = lds + ((fmt >> 20) & 7u) * 1024u + (fmt >> 26) * 16u + (by >> 1) * 4u;
        const unsigned d = ((fmt >> 13) & 0x78u) << ((by & 1u) * 16u + 8u);
        if (lane == 0) asm volatile("ds_sub_u32 %0, %1" :: "v"(ad), "v"(d) : "memory");
    }
    if (mask != 0u) {
        const unsigned j0 = (unsigned)__builtin_ctz(mask);
        const __amdgpu_buffer_rsrc_t rs = mk_rsrc(ws, WS_NEED);
#pragma unroll
        for (int j = 0; j < 8; ++j) {
            if ((mask >> j) & 1u) {
                u32x2 e = w.e0;
                if ((unsigned)j != j0)
                    asm volatile("buffer_load_dwordx2 %0, %1, %2, %3 offen sc0 sc1\n\ts_waitcnt vmcnt(0)"
                                 : "=&v"(e) : "v"(lane * 8), "s"(rs), "s"(off + 6144u + j * 512u) : "memory");
                wire_put(e.x, e.y, w.lo[j >> 1][(j & 1) * 2], w.lo[j >> 1][(j & 1) * 2 + 1], la + j * 1024u);
            }
        }
    }
}
__device__ __forceinline__ unsigned flag_pos(unsigned ks) { return (ks & 3u) * 16u + (ks >> 2); }

struct Tile { int rb, cb, src, blk; };
__device__ __forceinline__ Tile tile_of(unsigned code, int rank) {
    const int gidx = rank & 3, gbase = rank & 4, obase = gbase ^ 4;
    Tile t;
    t.rb = (int)(code & 63u); t.cb = (int)((code >> 6) & 7u);
    if (t.rb < 4 * NALPHA) { t.src = gbase + t.rb / NALPHA; t.blk = t.rb % NALPHA; }
    else if (t.rb < 32) { t.src = obase + (t.rb - 4 * NALPHA) / NBETA; t.blk = NALPHA + (t.rb - 4 * NALPHA) % NBETA; }
    else { t.src = rank; t.blk = NALPHA + (t.rb - 32); }
    return t;
}

struct Frag { s16x8 a[4][2]; };
struct PackB { u32x4 lo[2], cd; };
template <int N> struct IC { static constexpr int value = N; };

__global__ void __launch_bounds__(NTHREADS, 2) ag_gemm_kernel(const Params p) {
    __shared__ __attribute__((aligned(1024))) unsigned char lds_raw[LDS_BYTES];
    LAS unsigned char* L = (LAS unsigned char*)lds_raw;
    const int lane = threadIdx.x & 63;
    const int wave = (int)rfl(threadIdx.x >> 6);
    const int rank = p.rank;
    const int gidx = rank & 3, gbase = rank & 4, obase = gbase ^ 4;
    char* myws = p.ws[rank];
    const unsigned bid = blockIdx.x;
    if ((unsigned)(size_t)L != 0u) return;

    auto draw_call = [&]() -> unsigned {
        unsigned ticket = 0;
        if (lane == 0)
            ticket = __hip_atomic_fetch_add((GAS unsigned*)(myws + OFF_CTR + bid * 128u + (unsigned)wave * 4u), 1u,
                                            __ATOMIC_RELAXED, __HIP_MEMORY_SCOPE_AGENT);
        return rfl(ticket) + 1u;
    };

    const int peer = (int)((p.xpeer >> ((bid & 7u) * 4u)) & 15u);
    if (bid < 8u * CPL && peer < NR) {
        if (wave < WPP) {
            __builtin_amdgcn_s_waitcnt(0x0F70);
            if (wave == 0) STAMP(0);
            const int pi = wave * CPL + (int)(bid >> 3);
            const bool same = (peer & 4) == gbase;
            const __amdgpu_buffer_rsrc_t rx = mk_rsrc(p.x, SHARD_BYTES), rw = mk_rsrc(p.w, SHARD_BYTES);
            char* pws = p.ws[peer];
            const unsigned xa0 = x_lane_off(lane), xa1 = xa0 ^ 64u;
            const unsigned wl = (unsigned)wave * 8192u;

            auto load_task = [&](int t) {
                if (!same && t < 128) {
                    w_fetch(wl, rw, (unsigned)(peer & 3), (unsigned)t, lane);
                } else {
                    int ks, blk;
                    if (t < 128) { ks = t >> 1; blk = t & 1; }
                    else { const int u = t - 128; ks = u / 3; blk = (same ? NEARLY : NALPHA) + (u - ks * 3); }
                    dma8c(wl, xa0, xa1, rx, (unsigned)blk * (64u * 8192u) + (unsigned)ks * 128u, 65536u);
                }
            };
            auto split_task = [&](int t, Packed& k) {
                u32x4 ra[8];
                if (!same && t < 128) w_frags(ra, wl, lane);
                else {
#pragma unroll
                    for (int j = 0; j < 8; ++j) ra[j] = *(const volatile LAS u32x4*)(size_t)(wl + (unsigned)lane * 16u + j * 1024u);
                }
                pack_split(ra, k);
            };
            char* cp = myws + OFF_CTR + bid * 128u + (unsigned)wave * 4u;
            char* qp = myws + OFF_Q + (unsigned)peer * 128u;
            unsigned tk;
            load_task(pi);
            asm volatile("global_load_dword %0, %1, off sc1" : "=&v"(tk) : "v"(cp) : "memory");
            if (threadIdx.x < 16) *(LAS unsigned*)(L + L_CTL + threadIdx.x * 4) = 0u;
            asm volatile("s_waitcnt lgkmcnt(0)" ::: "memory");
            __builtin_amdgcn_s_barrier();
            const __amdgpu_buffer_rsrc_t rd = mk_rsrc(pws, WS_NEED);
            auto draw = [&]() {
                unsigned long long sv;
                asm volatile("s_mov_b64 %1, exec\n\ts_mov_b64 exec, 1\n\tglobal_atomic_add %0, %2, %3, off sc0\n\ts_mov_b64 exec, %1"
                             : "=&v"(tk), "=&s"(sv) : "v"(qp), "v"(1u) : "memory");
            };
            unsigned dof, fof;
            auto place = [&](int t) {
                if (!same && t < 128) {
                    dof = OFF_WST + (unsigned)(4 + gidx) * WQ_BYTES + (unsigned)t * CH_BYTES;
                    fof = OFF_FW + (unsigned)((4 + gidx) * 128 + t) * 8u;
                } else {
                    int ks, ai;
                    if (t < 128) { ks = t >> 1; ai = gidx * NALPHA + (t & 1); }
                    else {
                        const int u = t - 128; ks = u / 3;
                        const int r = u - ks * 3;
                        ai = same ? gidx * NALPHA + NEARLY + r : 4 * NALPHA + gidx * NBETA + r;
                    }
                    dof = (unsigned)ai * ABLK_BYTES + (unsigned)ks * CH_BYTES;
                    fof = OFF_FA + ((unsigned)ai * 64u + flag_pos((unsigned)ks)) * 8u;
                }
            };
            unsigned spins = 0, call = 0, fmt = 0, fpv = 0;
            bool cast = false;
            unsigned long long t0 = 0;
            for (int t = pi, nx = pi + NPW;;) {
                place(t);
                asm volatile("s_waitcnt vmcnt(0)" : "+v"(tk) :: "memory");
                __builtin_amdgcn_s_waitcnt(0x0F70);
                asm volatile("" ::: "memory");
                if (call == 0u) {
                    call = rfl(tk) + 1u;
                } else {
                    if (lane == 0)
                        __hip_atomic_store((GAS unsigned long long*)(pws + fpv), (unsigned long long)call | ((unsigned long long)fmt << 32),
                                           __ATOMIC_RELAXED, __HIP_MEMORY_SCOPE_SYSTEM);
                    if (cast && lane == 0)
                        __hip_atomic_store((GAS unsigned long long*)(myws + OFF_GO + (unsigned)peer * 128u), rt() + AG_DELTA,
                                           __ATOMIC_RELAXED, __HIP_MEMORY_SCOPE_AGENT);
                    if (t >= NTASK) break;
                    nx = (int)min(rfl(tk) - (call - 1u) * (unsigned)(NTASK - NPW) + 2u * (unsigned)NPW, (unsigned)NTASK);
                }
                dof += OFF_DATA + (call & 1u) * PAR_BYTES;
                fpv = fof + (call & 1u) * FLG_PAR;
                Packed k;
                split_task(t, k);
                const bool eager = t < AG_EAGERW * CPL;
                if (eager) send_low(k, rd, dof, lane);
                if (nx < NTASK) { load_task(nx); draw(); }
                pack_codes(k);
                fmt = k.fmt;
                unsigned my = 0;
                if (!eager) {
                    if (t == pi) __builtin_amdgcn_s_sleep(AG_W2D);
                    asm volatile("ds_add_rtn_u32 %0, %1, %2\n\ts_waitcnt lgkmcnt(0)" : "=&v"(my) : "v"(C_CNT), "v"(lane == 0 ? 1u : 0u) : "memory");
                    my = rfl(my);
                    while (lds_ld(C_GO) < my) {
                        if ((++spins & 1023u) == 0u && expired(t0, DEADLINE)) break;
                        __builtin_amdgcn_s_sleep(1);
                    }
                    send_low(k, rd, dof, lane);
                }
                send_high(k, rd, dof, lane);
                if (!eager) {
                    VM_WAIT(AG_REL);
                    lds_st(C_GO, my + 1u);
                }
                cast = t == NTASK - AG_GOT;
                t = (int)rfl((unsigned)nx);
            }
            if (lane == 0) st_agent(cp, call);
            return;
        }
        __builtin_amdgcn_s_barrier();
        if (wave != 7 || bid != 0u) return;
        const unsigned call = draw_call();
        const int i0 = (lane * 2) & 31, ex = 4 * NALPHA + gidx * NBETA;
        const bool skip0 = i0 >= ex && i0 < ex + NBETA, skip1 = i0 + 1 >= ex && i0 + 1 < ex + NBETA;
        const char* fp = myws + OFF_DONE + (unsigned)lane * 8u;
        unsigned long long t0 = 0;
        unsigned spins = 0;
        bool dead = false;
        for (;;) {
            u32x2 v;
            asm volatile("global_load_dwordx2 %0, %1, off sc0 sc1\n\ts_waitcnt vmcnt(0)" : "=&v"(v) : "v"(fp) : "memory");
            const bool ok = (skip0 || v.x >= call) && (skip1 || v.y >= call);
            if (__all(ok)) break;
            if ((++spins & 255u) == 0u) {
                if (rfl(ld_agent(myws + OFF_TMO)) == call || expired(t0, 2 * DEADLINE)) { dead = true; break; }
            }
            __builtin_amdgcn_s_sleep(2);
        }
        __builtin_amdgcn_fence(__ATOMIC_ACQUIRE, "");
        if (dead) {
            if (lane == 0) st_agent(myws + OFF_TMO, call);
            const unsigned long long t1 = rt();
            while (rt() - t1 < DEADLINE / 8) __builtin_amdgcn_s_sleep(64);
            if (lane == 0) st_sys(p.out[rank] + (size_t)rank * 512u * 1024u, 0x7FC07FC0u);
        }
        STAMP(6);
        return;
    }

    __builtin_amdgcn_s_sleep(AG_TDELAY);
    const unsigned own = (bid - 64u) * 4u + (unsigned)wave - (unsigned)NLOAD;
    const bool packs = bid >= 64u && wave >= NLOAD && own < 512u;
    if (packs) w_fetch((unsigned)(8 + wave) * 8192u, mk_rsrc(p.w, SHARD_BYTES), own & 3u, own >> 2, lane);
    if (threadIdx.x < (2048 + 128) / 8) *(LAS unsigned long long*)(L + L_REC + threadIdx.x * 8) = 0ull;
    const unsigned call = draw_call();
    const unsigned tcode = p.tile[bid];
    const unsigned ntile = (tcode >> 31) ? 2u : 1u;
    asm volatile("s_waitcnt lgkmcnt(0)" ::: "memory");
    __builtin_amdgcn_s_barrier();
    const unsigned par = OFF_DATA + (call & 1u) * PAR_BYTES;

    if (wave < NLOAD) {
        const unsigned lw = (unsigned)wave;
        const unsigned long long born = rt();
        const __amdgpu_buffer_rsrc_t rws = mk_rsrc(myws, WS_NEED);
        const unsigned fpar = (call & 1u) * FLG_PAR;
        unsigned spins = 0;
        unsigned long long t0 = 0;
        bool dead = false;
        FDECL;
        auto give_up = [&]() -> bool {
            return (++spins & 255u) == 0u &&
                   (rfl(ld_agent(myws + OFF_TMO)) == call || lds_ld(C_ABORT) != 0u || expired(t0, DEADLINE));
        };
        auto wait_slot = [&](unsigned g) {
            while (g >= (unsigned)NSLOT - 4u) {
                const u32x4 fr = lds_ld4(C_FREE);
                if (rfl(min(min(fr.x, fr.y), min(fr.z, fr.w))) + ((unsigned)NSLOT - 4u) > g) break;
                if ((++spins & 1023u) == 0u && (lds_ld(C_ABORT) != 0u || expired(t0, 2 * DEADLINE))) { dead = true; break; }
                __builtin_amdgcn_s_sleep(AG_FREE_SLEEP);
            }
        };
        for (unsigned ti = 0; ti < ntile && !dead; ++ti) {
            const Tile t = tile_of(tcode >> (16u * ti), rank);
            const unsigned g0 = ti * (unsigned)NKS + lw;
            unsigned wn = 0;
            auto publish = [&](unsigned i) {
                const unsigned need = min(i * 4u + lw + (unsigned)BD + 1u, (unsigned)NKS);
                while (wn < need) {
                    const u32x4 v = __builtin_amdgcn_raw_buffer_load_b128(rws, fpar + OFF_FW + (unsigned)t.cb * 1024u + (unsigned)lane * 16u, 0, 17);
                    const bool up = v.x >= call && v.z >= call;
                    wn = (unsigned)__builtin_ctzll(~__ballot(up) | (1ull << 63)) + (__all(up) ? 1u : 0u);
                    u32x4 f;
                    f.x = (v.y & 0x0FFFu) | ((v.y >> 22) & 1u ? 0u : v.y & 0xFFFF0000u);
                    f.y = (v.y & 0x00FFu) | ((v.y >> 4) & 0x0F00u) | ((v.y >> 22) & 1u ? v.y & 0xFFFF0000u : 0u);
                    f.z = (v.w & 0x0FFFu) | ((v.w >> 22) & 1u ? 0u : v.w & 0xFFFF0000u);
                    f.w = (v.w & 0x00FFu) | ((v.w >> 4) & 0x0F00u) | ((v.w >> 22) & 1u ? v.w & 0xFFFF0000u : 0u);
                    if (up) {
                        const u32x2 f0 = {f.x, f.y}, f1 = {f.z, f.w};
                        asm volatile("ds_or_b64 %0, %1\n\tds_or_b64 %0, %2 offset:8" :: "v"(L_REC + ti * 1024u + (unsigned)lane * 16u), "v"(f0), "v"(f1) : "memory");
                    }
                    if (wn >= need) break;
                    if (give_up()) { dead = true; return; }
                    __builtin_amdgcn_s_sleep(AG_POLL_SLEEP);
                }
                asm volatile("s_waitcnt lgkmcnt(0)" ::: "memory");
                if (lane < 4) asm volatile("ds_or_b32 %0, %1" :: "v"(L_REC + (g0 + i * 4u) * 16u + (unsigned)lane * 4u), "v"(0x80u) : "memory");
            };
            if (ti == ntile - 1u && wave == 0) STAMP(2);
            if (t.src != rank) {
                const unsigned base = par + (unsigned)t.rb * ABLK_BYTES + lw * CH_BYTES;
                const unsigned fl0 = fpar + OFF_FA + (unsigned)t.rb * 512u + lw * 128u;
                u32x2 fl = {0u, 0u};
                unsigned up = 0u;
                auto refresh = [&]() {
                    if (lane < 16) fl = __builtin_amdgcn_raw_buffer_load_b64(rws, fl0 + (unsigned)lane * 8u, 0, 17);
                    up = (unsigned)__ballot(fl.x >= call);
                    asm volatile("" ::: "memory");
                };
                auto await = [&](unsigned need) {
                    for (;;) {
                        refresh();
                        if ((up & need) == need) return;
                        if (give_up()) { dead = true; return; }
                        __builtin_amdgcn_s_sleep(AG_POLL_SLEEP);
                    }
                };
                auto fetch = [&](Wire& w, unsigned& f, unsigned i, bool go) {
                    f = (unsigned)__builtin_amdgcn_readlane((int)fl.y, (int)(i & 15u));
                    wire_load(w, myws, go, base + i * (4u * CH_BYTES), f, lane);
                };
                auto turn = [&](const Wire& w, unsigned f, Wire& wx, unsigned& fx, unsigned i) {
                    const bool last = i == 15u;
                    const unsigned need = last ? 0u : 2u << i;
                    if ((up & need) != need) refresh();
                    FACC(0);
                    const bool ok = (up & need) == need;
                    fetch(wx, fx, i + 1u, ok && !last);
                    FACC(1);
                    wait_slot(g0 + i * 4u);
                    FACC(2);
                    if (dead) return;
                    wire_unpack(w, myws, base + i * (4u * CH_BYTES), f, lane, ((g0 + i * 4u) & (unsigned)(NSLOT - 1)) * 8192u);
                    FACC(4);
                    publish(i);
                    FACC(3);
                    if (!ok && !dead) {
                        await(need);
                        fetch(wx, fx, i + 1u, !dead);
                    }
                    FACC(5);
                };
                Wire w0, w1;
                unsigned f0, f1;
                await(1u);
                fetch(w0, f0, 0u, !dead);
                FACC(5);
                for (unsigned i = 0; i < 16u && !dead; i += 2u) {
                    turn(w0, f0, w1, f1, i);
                    if (dead) break;
                    turn(w1, f1, w0, f0, i + 1u);
                }
            } else {
                const __amdgpu_buffer_rsrc_t rx = mk_rsrc(p.x, SHARD_BYTES);
                const unsigned v0 = x_lane_off(lane), v1 = v0 ^ 64u;
                const unsigned base = (unsigned)t.blk * (64u * 8192u) + lw * 128u;
                for (unsigned i = 0; i < 18u && !dead; ++i) {
                    if (i < 16u) {
                        wait_slot(g0 + i * 4u);
                        if (dead) break;

                        dma8(((g0 + i * 4u) & (unsigned)(NSLOT - 1)) * 8192u, v0, v1, rx, base + i * 512u, 65536u);
                        VM_WAIT(16);
                    } else if (i == 16u) {
                        VM_WAIT(8);
                    } else {
                        VM_WAIT(0);
                    }
                    if (i >= 2u) publish(i - 2u);
                }
                VM_WAIT(0);
            }
        }
        FOUT();
        if (dead) {
            lds_st(C_ABORT, 1u);
            if (lane == 0) st_agent(myws + OFF_TMO, call);
        }
        if (wave == 0) {
            const int cb = tile_of(tcode >> (16u * (ntile - 1u)), rank).cb;
            if (cb >= 4) {
                char* hp = myws + OFF_GO + (unsigned)(obase + (cb & 3)) * 128u;
                unsigned long long go;
                t0 = 0;
                for (;;) {
                    go = __hip_atomic_load((const GAS unsigned long long*)hp, __ATOMIC_RELAXED, __HIP_MEMORY_SCOPE_AGENT);
                    if (dead || __all(go > born)) break;
                    if ((++spins & 63u) == 0u && expired(t0, DEADLINE)) break;
                    __builtin_amdgcn_s_sleep(AG_GO_SLEEP);
                }
                const unsigned long long now = rt();
                unsigned long long at = 0;
                if (lane == 0) {
                    __hip_atomic_fetch_max((GAS unsigned long long*)(hp + 1024), go > now ? go : now, __ATOMIC_RELAXED, __HIP_MEMORY_SCOPE_AGENT);
                    at = __hip_atomic_fetch_add((GAS unsigned long long*)(hp + 1024), (unsigned long long)AG_TAU, __ATOMIC_RELAXED, __HIP_MEMORY_SCOPE_AGENT);
                }
                const unsigned wt = rfl(at > now ? (unsigned)min(at - now, 2000ull) : 0u);
                while ((unsigned)(rt() - now) < wt) __builtin_amdgcn_s_sleep(1);
                lds_st(C_GO, 1u);
            }
        }
        if (wave == 3) STAMP(3);
        return;
    }

    const int cw = wave - NLOAD;
    __builtin_amdgcn_s_setprio(AG_PRIO);
    const int hh = lane >> 5, r32 = lane & 31;
    const unsigned a0 = (unsigned)(r32 * 128 + ((hh ^ ((r32 >> 1) & 7)) << 4));
    const unsigned cfree = C_FREE + (unsigned)cw * 4u;
    if (packs) {
        u32x4 r[8];
        Packed k;
        VM_WAIT(0);
        w_frags(r, (unsigned)(8 + wave) * 8192u, lane);
        pack_split(r, k);
        lds_st(cfree, 4u);
        pack_codes(k);
        send_chunk(k, mk_rsrc(myws, WS_NEED), par + OFF_WST + (own & 3u) * WQ_BYTES + (own >> 2) * CH_BYTES, lane);
        VM_WAIT(0);
        if (lane == 0)
            __hip_atomic_store((GAS unsigned long long*)(myws + (call & 1u) * FLG_PAR + OFF_FW + ((own & 3u) * 128u + (own >> 2)) * 8u),
                               (unsigned long long)call | ((unsigned long long)k.fmt << 32), __ATOMIC_RELAXED, __HIP_MEMORY_SCOPE_SYSTEM);
    } else {
        lds_st(cfree, 4u);
    }
    const unsigned vlo = (unsigned)lane * 16u + (unsigned)(cw & 1) * 2048u, vcd = (unsigned)lane * 16u + 4096u + (unsigned)(cw & 1) * 1024u;

    unsigned spins = 0;
    unsigned long long t0 = 0;
    FDECL;
    auto wait_full = [&](unsigned g) {
        while ((lds_ld(L_REC + g * 16u) & 0x80u) == 0u) {
            if ((++spins & 63u) == 0u && lds_ld(C_ABORT) != 0u) break;
            if ((spins & 1023u) == 0u && expired(t0, 3 * DEADLINE)) { lds_st(C_ABORT, 1u); break; }
            __builtin_amdgcn_s_sleep(1);
        }
    };
    unsigned ab[4];
#pragma unroll
    for (int k = 0; k < 4; ++k) ab[k] = a0 ^ (unsigned)(k * 32);
    auto rd = [&](Frag& f, auto sc) {
        constexpr unsigned S = decltype(sc)::value;
#pragma unroll
        for (int k = 0; k < 4; ++k)
#pragma unroll
            for (int mt = 0; mt < 2; ++mt)
                f.a[k][mt] = *(const LAS s16x8*)((const LAS unsigned char*)(size_t)ab[k] + (S * 8192u + mt * 4096u));
    };
    auto ldb = [&](PackB& b, __amdgpu_buffer_rsrc_t rs, unsigned off) {
        b.lo[0] = __builtin_amdgcn_raw_buffer_load_b128(rs, vlo, off, AG_WAUX);
        b.lo[1] = __builtin_amdgcn_raw_buffer_load_b128(rs, vlo + 1024u, off, AG_WAUX);
        b.cd = __builtin_amdgcn_raw_buffer_load_b128(rs, vcd, off, AG_WAUX);
    };
    auto put = [&](u32x16& o, int k, unsigned ha, unsigned hb, const PackB& b) {
        const unsigned l0 = b.lo[k >> 1][(k & 1) * 2], l1 = b.lo[k >> 1][(k & 1) * 2 + 1];
        o[4 * k] = __builtin_amdgcn_perm(ha, l0, 0x05010400u);
        o[4 * k + 1] = __builtin_amdgcn_perm(ha, l0, 0x07030602u);
        o[4 * k + 2] = __builtin_amdgcn_perm(hb, l1, 0x05010400u);
        o[4 * k + 3] = __builtin_amdgcn_perm(hb, l1, 0x07030602u);
    };
    auto repair = [&](u32x16& o, const PackB& b, unsigned fmt, unsigned off) {
        if ((fmt & 0xF0000u) != 0u) {
            const unsigned by = (fmt >> 23) & 7u;
            o[((fmt >> 18) & 12u) + (by >> 1)] -= (unsigned)lane == (fmt >> 26) ? ((fmt >> 13) & 0x78u) << ((by & 1u) * 16u + 8u) : 0u;
        }
        if ((fmt & 0xF00u) != 0u) {
            const __amdgpu_buffer_rsrc_t rs = mk_rsrc(myws, WS_NEED);
#pragma unroll
            for (int k = 0; k < 4; ++k) {
                if ((fmt >> (8 + k)) & 1u) {
                    u32x2 e;
                    asm volatile("buffer_load_dwordx2 %0, %1, %2, %3 offen sc0 sc1\n\ts_waitcnt vmcnt(0)"
                                 : "=&v"(e) : "v"(lane * 8), "s"(rs), "s"(off + 6144u + (unsigned)((cw & 1) * 4 + k) * 512u) : "memory");
                    put(o, k, e.x, e.y, b);
                }
            }
        }
    };
    f32x16 acc[2];
    unsigned vrec = L_REC + (unsigned)cw * 4u;
    asm volatile("" : "+v"(vrec));
#define SB() __builtin_amdgcn_sched_barrier(0)

    for (unsigned ti = 0; ti < ntile; ++ti) {
#pragma unroll
        for (int i = 0; i < 16; ++i) { acc[0][i] = 0.f; acc[1][i] = 0.f; }
        const Tile t = tile_of(tcode >> (16u * ti), rank);
        const unsigned g0 = ti * (unsigned)NKS;
        const unsigned wbase = par + OFF_WST + (unsigned)t.cb * WQ_BYTES + (unsigned)(cw >> 1) * CH_BYTES;
        Frag fa[2];
        u32x16 fb[2];
        PackB pb[BD];
        wait_full(g0);
        FACC(3);
#pragma unroll
        for (int i = 0; i < BD; ++i) { ldb(pb[i], mk_rsrc(myws, WS_NEED), wbase + (unsigned)i * (2u * CH_BYTES)); SB(); }
        rd(fa[0], IC<0>());
        unsigned rec = rfl(*(const volatile LAS unsigned*)(size_t)vrec);
#pragma unroll
        for (int k = 0; k < 4; ++k) {
            const unsigned br = (rec & 0x7Fu) * 0x01010101u, n = pb[0].cd[k];
            put(fb[0], k, (n & 0x87878787u) + br, (__builtin_amdgcn_alignbit(n, n, 4) & 0x87878787u) + br, pb[0]);
        }
        if ((rec >> 8) != 0u) repair(fb[0], pb[0], rec, wbase);
        rec = *(const volatile LAS unsigned*)((const LAS unsigned char*)(size_t)vrec + 16);
        __builtin_amdgcn_s_waitcnt(0xC07F);
        for (unsigned o = 0; o < (unsigned)(NKS / BD); ++o) {
            const __amdgpu_buffer_rsrc_t rb = mk_rsrc(myws, o + 1u < (unsigned)(NKS / BD) ? WS_NEED : 0u);
            const unsigned s0 = o * (unsigned)BD, wo = wbase + s0 * (2u * CH_BYTES);
            auto step = [&](auto uc) {
                constexpr int U = decltype(uc)::value, C = U & 1, N = C ^ 1, X = (U + 1) & (BD - 1);
                const unsigned g = g0 + s0 + (unsigned)U + 1u;
                const unsigned fmt = rfl(rec);
                const unsigned br = (fmt & 0x7Fu) * 0x01010101u;
                if (U == BD - 1) {
#pragma unroll
                    for (int k = 0; k < 4; ++k) ab[k] ^= 65536u;
                }
                rec = *(const volatile LAS unsigned*)((const LAS unsigned char*)(size_t)vrec + (U + 2) * 16);
                SB();
#pragma unroll
                for (int k = 0; k < 4; ++k) {
                    const u32x4 bk = {fb[C][4 * k], fb[C][4 * k + 1], fb[C][4 * k + 2], fb[C][4 * k + 3]};
                    acc[0] = __builtin_amdgcn_mfma_f32_32x32x16_bf16(__builtin_bit_cast(bf16x8, bk), __builtin_bit_cast(bf16x8, fa[C].a[k][0]), acc[0], 0, 0, 0);
                    SB();
                    fa[N].a[k][0] = *(const LAS s16x8*)((const LAS unsigned char*)(size_t)ab[k] + X * 8192);
                    if (k == 0) ldb(pb[U], rb, wo + (unsigned)(U + BD) * (2u * CH_BYTES));
                    const unsigned n = pb[X].cd[k];
                    const unsigned ha = (n & 0x87878787u) + br, hb = (__builtin_amdgcn_alignbit(n, n, 4) & 0x87878787u) + br;
                    asm volatile("" :: "v"(fa[C].a[k][0]), "v"(ha), "v"(hb));
                    SB();
                    acc[1] = __builtin_amdgcn_mfma_f32_32x32x16_bf16(__builtin_bit_cast(bf16x8, bk), __builtin_bit_cast(bf16x8, fa[C].a[k][1]), acc[1], 0, 0, 0);
                    SB();
                    fa[N].a[k][1] = *(const LAS s16x8*)((const LAS unsigned char*)(size_t)ab[k] + (X * 8192 + 4096));
                    put(fb[N], k, ha, hb, pb[X]);
                    asm volatile("" :: "v"(fa[C].a[k][1]), "v"(fb[N][4 * k + 3]));
                    SB();
                }
                asm volatile("" :: "v"(fb[C]));
                if ((U & 3) == 3) lds_st(cfree, g + 4u);
                if (__builtin_expect((fmt >> 7) != 1u, 0)) {
                    if (g != g0 + (unsigned)NKS) {
                        FACC(0);
                        repair(fb[N], pb[X], fmt, wo + (unsigned)(U + 1) * (2u * CH_BYTES));
                        FACC(4);
                        if ((fmt & 0x80u) == 0u) {
                            FACC(0);
                            lds_st(cfree, g + 4u);
                            wait_full(g);
                            FACC(1);
                            rd(fa[N], IC<X>());
                            rec = *(const volatile LAS unsigned*)((const LAS unsigned char*)(size_t)vrec + (U + 2) * 16);
                            __builtin_amdgcn_s_waitcnt(0xC07F);
                            FACC(5);
                        }
                    }
                }
                SB();
            };
            step(IC<0>()); step(IC<1>()); step(IC<2>()); step(IC<3>());
            step(IC<4>()); step(IC<5>()); step(IC<6>()); step(IC<7>());
            vrec += (unsigned)BD * 16u;
        }
        if (cw == 0) STAMP(7);
        FACC(0);
        FOUT();

        const bool bad = lds_ld(C_ABORT) != 0u;
#pragma unroll
        for (int mt = 0; mt < 2; ++mt) {
#pragma unroll
            for (int j = 0; j < 4; ++j) {
                bf16x2 lo, hi;
                lo[0] = (__bf16)acc[mt][4 * j]; lo[1] = (__bf16)acc[mt][4 * j + 1];
                hi[0] = (__bf16)acc[mt][4 * j + 2]; hi[1] = (__bf16)acc[mt][4 * j + 3];
                u32x2 v; v.x = __builtin_bit_cast(unsigned, lo); v.y = __builtin_bit_cast(unsigned, hi);
                if (bad) { v.x = 0x7FC07FC0u; v.y = 0x7FC07FC0u; }
                *(volatile LAS u32x2*)(size_t)(L_ST + (unsigned)((mt * 32 + r32) * ST_ROW + cw * 64 + j * 16 + hh * 8)) = v;
            }
        }
        unsigned old;
        asm volatile("ds_add_rtn_u32 %0, %1, %2\n\ts_waitcnt lgkmcnt(0)" : "=&v"(old) : "v"(C_CNT), "v"(lane == 0 ? 1u : 0u) : "memory");
        if ((rfl(old) & 3u) != 3u) continue;
        u32x4 o[16];
#pragma unroll
        for (int i = 0; i < 16; ++i)
            o[i] = *(const volatile LAS u32x4*)(size_t)(L_ST + (unsigned)((i * 4 + (lane >> 4)) * ST_ROW + (lane & 15) * 16));

        const bool remote = t.cb >= 4;
        const int dstr = remote ? obase + (t.cb & 3) : rank;
        const __amdgpu_buffer_rsrc_t ro = mk_rsrc(p.out[dstr], SHARD_BYTES);
        const unsigned ovo = (unsigned)(lane >> 4) * 1024u + (unsigned)(lane & 15) * 16u;
        const unsigned oso = (unsigned)(t.src * 512 + t.blk * 64) * 1024u + (unsigned)(remote ? gidx : t.cb) * 256u;
        if (!remote) {
#pragma unroll
            for (int i = 0; i < 16; ++i) __builtin_amdgcn_raw_buffer_store_b128(o[i], ro, ovo, oso + i * 4096u, AG_LOCAL_AUX);
        } else {
            t0 = 0;
            while (lds_ld(C_GO) == 0u) {
                if ((++spins & 1023u) == 0u && expired(t0, 2 * DEADLINE)) break;
                __builtin_amdgcn_s_sleep(1);
            }
            STAMP(4);
#pragma unroll
            for (int i = 0; i < 16; ++i) __builtin_amdgcn_raw_buffer_store_b128(o[i], ro, ovo, oso + i * 4096u, 17);
            VM_WAIT(0);
            if (lane == 0) st_sys(p.ws[dstr] + OFF_DONE + (unsigned)(gidx * 32 + t.rb) * 4u, call);
        }
        STAMP(8);
    }
}

}

extern "C" void kernel_launch(int rank, int nranks, ncclComm_t comm,
                              hipStream_t stream,
                              void* const* d_in, const int* in_sizes, int n_in,
                              void* d_out, int out_size,
                              const int* in_dtypes, int out_dtype,
                              void* const* d_out_all,
                              void* const* d_ws_all, size_t ws_bytes) {
    using namespace agk;
    (void)comm; (void)in_dtypes; (void)out_dtype;
    if (nranks != NR || (unsigned)rank >= (unsigned)NR || n_in < 2 || ws_bytes < (size_t)WS_NEED ||
        in_sizes[0] != 512 * 4096 || in_sizes[1] != 512 * 4096 || out_size != 512 * 4096) {
        fprintf(stderr, "kernel_launch: unexpected problem shape\n");
        return;
    }
    Params p;
    p.x = d_in[0];
    p.w = d_in[1];
    for (int r = 0; r < NR; ++r) { p.ws[r] = (char*)d_ws_all[r]; p.out[r] = (char*)d_out_all[r]; }
    p.out[rank] = (char*)d_out;
    p.rank = rank;
    const int gidx = rank & 3, obase = (rank & 4) ^ 4;
    int xcd_of[NR], spare = 3;
    {
        const unsigned setmask = 0xB2u;
        int lo = 0, hi = 4;
        for (int o = 1; o < NR; ++o) {
            const int q = (rank + o) & 7;
            xcd_of[q] = (((setmask >> q) ^ (setmask >> rank)) & 1u) ? hi++ : lo++;
        }
        xcd_of[rank] = spare;
        p.xpeer = 0u;
        for (int q = 0; q < NR; ++q) p.xpeer |= (unsigned)(q == rank ? 15 : q) << (xcd_of[q] * 4);
    }
    auto is_late = [&](int rb) { return rb < 32 && (rb >= 4 * NALPHA || (rb % NALPHA >= NEARLY && rb / NALPHA != gidx)); };
    auto code_of = [](int rb, int cb) { return (unsigned)rb | ((unsigned)cb << 6) | 0x8000u; };
    int next[8];
    for (int x = 0; x < 8; ++x) next[x] = x != spare ? x + 8 * CPL : x;
    for (int b = 0; b < NWG; ++b) p.tile[b] = 0u;
    int ntile = 0;
    int late_x[4], used = 0;
    for (int pass = 0; pass < 2; ++pass)
        for (int c = 0; c < 4; ++c)
            for (int o = 0; o < 4; ++o) {
                const int x = xcd_of[(rank & 4) + o];
                if ((pass == 1 && late_x[c] >= 0)) break;
                late_x[c] = -1;
                if (((used >> x) & 1) || (pass == 0 && (x >> 2) != (xcd_of[obase + c] >> 2))) continue;
                late_x[c] = x; used |= 1 << x;
                break;
            }
    for (int c = 0; c < 4; ++c)
        for (int rb = 0; rb < 32; ++rb) {
            if (rb >= 4 * NALPHA && (rb - 4 * NALPHA) / NBETA == c) continue;
            const int xx = is_late(rb) ? late_x[c] : xcd_of[obase + c];
            if (next[xx] >= NWG) { fprintf(stderr, "kernel_launch: placement\n"); return; }
            p.tile[next[xx]] = code_of(rb, 4 + c);
            next[xx] += 8;
            ++ntile;
        }
    unsigned rest[140], first[64];
    int nr = 0, nf = 0;
    for (int pass = 0; pass < 3; ++pass)
        for (int rb = 0; rb < 35; ++rb)
            for (int c = 0; c < 4; ++c) {
                const int cls = is_late(rb) ? 2 : (rb >= 32 || rb / NALPHA == gidx) ? 0 : 1;
                if (cls != pass) continue;
                if (cls < 2 && nf < NPWG) first[nf++] = code_of(rb, c); else rest[nr++] = code_of(rb, c);
            }
    int cand[NWG], nc = 0;
    for (int x = 0, k = nr - 1; x < 8; ++x)
        for (; next[x] < NWG && k >= 0; next[x] += 8, --k) {
            p.tile[next[x]] = rest[k];
            if (is_late((int)(rest[k] & 63u))) cand[nc++] = next[x];
            ++ntile;
        }
    if (nc < nf || ntile + nf != NTILE || nf != NPWG) { fprintf(stderr, "kernel_launch: pairing %d %d %d\n", nc, nf, ntile); return; }
    for (int k = 0; k < nf; ++k) {
        const int b = cand[k * nc / nf];
        p.tile[b] = first[k] | (p.tile[b] << 16);
    }
    hipLaunchKernelGGL(ag_gemm_kernel, dim3(NWG), dim3(NTHREADS), 0, stream, p);
    HIP_CHECK(hipGetLastError());
}
```

```cpp
#include <hip/hip_runtime.h>
#include <rccl/rccl.h>
#include <cstdio>
#include <cstdint>

#define HIP_CHECK(expr) do { hipError_t _e = (expr); if (_e != hipSuccess) { \
    fprintf(stderr, "HIP error %s at %s:%d\n", hipGetErrorString(_e), __FILE__, __LINE__); \
    return; } } while (0)

namespace agk {

#define GAS __attribute__((address_space(1)))
#define LAS __attribute__((address_space(3)))
typedef unsigned u32x4 __attribute__((ext_vector_type(4)));
typedef unsigned u32x2 __attribute__((ext_vector_type(2)));
typedef short s16x4 __attribute__((ext_vector_type(4)));
typedef short s16x8 __attribute__((ext_vector_type(8)));
typedef __bf16 bf16x8 __attribute__((ext_vector_type(8)));
typedef __bf16 bf16x2 __attribute__((ext_vector_type(2)));
typedef float f32x16 __attribute__((ext_vector_type(16)));
typedef unsigned u32x16 __attribute__((ext_vector_type(16)));

constexpr int NR = 8;
constexpr int NKS = 64;
constexpr int NALPHA = 5, NBETA = 3;
constexpr int NEARLY = 2;
constexpr int NWG = 256;
constexpr int NTILE = 256;
constexpr int NTASK = 320;
#define AG_CPL 8
#define AG_WPP 3
#define AG_TOPO 1
constexpr int CPL = AG_CPL;
constexpr int WPP = AG_WPP;
constexpr int NPW = CPL * WPP;
constexpr int NPWG = 7 * CPL;
#define AG_POLL_SLEEP 2
#define AG_FREE_SLEEP 1
#define AG_STAGGER 4
#define AG_LOCAL_AUX 17
#define AG_REL 5
#define AG_NEAGER 4
#define AG_TAU 27
#define AG_GOY 4
#define AG_SPLIT 1
#define AG_LOCK 1
#define AG_CDMA 1
#define AG_GO_SLEEP 4
#define AG_GOT 40
#define AG_DELTA 320
#define AG_EAGERW 2
#define AG_W2D 8
#define AG_XAUX 17
#define AG_WAUX 17
#define AG_OWNC 0
#define AG_TWO 0
#define AG_WARM 0
#define AG_TDELAY 16
#define AG_PRIO 2
#define AG_KEEP 2
#define AG_HOLD 1
static_assert(NPWG <= 56 && WPP <= 7, "pushing workgroups: at most one per early tile with a local output");
constexpr unsigned SHARD_BYTES = 512u * 4096u * 2u;

constexpr unsigned OFF_CTR = 0;
constexpr unsigned OFF_TMO = 32u << 10;
constexpr unsigned OFF_GO = 34u << 10;
constexpr unsigned OFF_Q = 36u << 10;
constexpr unsigned OFF_FA = 64u << 10;
constexpr unsigned OFF_FW = 80u << 10;
constexpr unsigned FLG_PAR = 32u << 10;
constexpr unsigned OFF_DONE = 128u << 10;
constexpr unsigned OFF_DBG = 192u << 10;
constexpr unsigned OFF_DATA = 1u << 20;
constexpr unsigned CH_BYTES = 10240u;
constexpr unsigned ABLK_BYTES = NKS * CH_BYTES;
constexpr unsigned WQ_BYTES = NKS * 2u * CH_BYTES;
constexpr unsigned OFF_WST = 32u * ABLK_BYTES;
constexpr unsigned PAR_BYTES = OFF_WST + 8u * WQ_BYTES;
constexpr unsigned WS_NEED = OFF_DATA + 2u * PAR_BYTES;

constexpr int NSLOT = 16;
constexpr int NLOAD = 4;
constexpr int BD = 8;
constexpr unsigned L_ST = NSLOT * 8192u;
constexpr unsigned ST_ROW = 272;
constexpr unsigned L_REC = L_ST + 64 * ST_ROW;
constexpr unsigned L_CTL = L_REC + 2048 + 64;
constexpr unsigned C_FREE = L_CTL;
constexpr unsigned C_ABORT = L_CTL + 16;
constexpr unsigned C_CNT = L_CTL + 20;
constexpr unsigned C_GO = L_CTL + 24;
constexpr unsigned LDS_BYTES = L_CTL + 64;
static_assert(LDS_BYTES <= 160u * 1024u && (L_REC & 15u) == 0u, "LDS");
constexpr int NTHREADS = (NLOAD + 4) * 64;

constexpr unsigned long long DEADLINE = 30000000ull;

struct Params {
    const void* x;
    const void* w;
    char* ws[NR];
    char* out[NR];
    int rank;
    unsigned xpeer;
    unsigned tile[NWG];
};

__device__ __forceinline__ unsigned long long rt() { return __builtin_amdgcn_s_memrealtime(); }
__device__ __forceinline__ bool expired(unsigned long long& t0, unsigned long long limit) {
    const unsigned long long now = __builtin_amdgcn_s_memrealtime() | 1ull;
    if (t0 == 0ull) { t0 = now; return false; }
    return now - t0 > limit;
}
__device__ __forceinline__ unsigned rfl(unsigned v) { return (unsigned)__builtin_amdgcn_readfirstlane((int)v); }
__device__ __forceinline__ __amdgpu_buffer_rsrc_t mk_rsrc(const void* p, unsigned bytes) {
    return __builtin_amdgcn_make_buffer_rsrc((void*)p, (short)0, (int)bytes, 0x00020000);
}
#define VM_WAIT_(n) asm volatile("s_waitcnt vmcnt(" #n ")" ::: "memory")
#define VM_WAIT(n) VM_WAIT_(n)
__device__ __forceinline__ void lds_st(unsigned addr, unsigned v) {
    asm volatile("ds_write_b32 %0, %1" :: "v"(addr), "v"(v) : "memory");
}
__device__ __forceinline__ unsigned lds_ld(unsigned addr) {
    unsigned v;
    asm volatile("ds_read_b32 %0, %1\n\ts_waitcnt lgkmcnt(0)" : "=&v"(v) : "v"(addr) : "memory");
    return rfl(v);
}
__device__ __forceinline__ u32x4 lds_ld4(unsigned addr) {
    u32x4 v;
    asm volatile("ds_read_b128 %0, %1\n\ts_waitcnt lgkmcnt(0)" : "=&v"(v) : "v"(addr) : "memory");
    return v;
}
__device__ __forceinline__ unsigned ld_agent(const char* p) {
    return __hip_atomic_load((const GAS unsigned*)p, __ATOMIC_RELAXED, __HIP_MEMORY_SCOPE_AGENT);
}
__device__ __forceinline__ void st_agent(char* p, unsigned v) {
    __hip_atomic_store((GAS unsigned*)p, v, __ATOMIC_RELAXED, __HIP_MEMORY_SCOPE_AGENT);
}
__device__ __forceinline__ void st_sys(char* p, unsigned v) {
    __hip_atomic_store((GAS unsigned*)p, v, __ATOMIC_RELAXED, __HIP_MEMORY_SCOPE_SYSTEM);
}
#define DMA_P(V) "buffer_load_dwordx4 " V ", %5, %1 offen sc0 sc1 lds\n\ts_add_u32 m0, m0, 0x400\n\ts_add_u32 %1, %1, %7\n\t"
#define DMA_C(V) "buffer_load_dwordx4 " V ", %5, %1 offen lds\n\ts_add_u32 m0, m0, 0x400\n\ts_add_u32 %1, %1, %7\n\t"
#define DMA_HEAD "s_nop 4\n\ts_mov_b32 %0, m0\n\ts_mov_b32 m0, %2\n\ts_mov_b32 %1, %6\n\t"
#define DMA_TAIL "s_mov_b32 m0, %0"
#define DMA8(PA, PB, PC, PD) \
    asm volatile(DMA_HEAD DMA_P(PA) DMA_P(PB) DMA_P(PC) DMA_P(PD) DMA_P(PA) DMA_P(PB) DMA_P(PC) DMA_P(PD) DMA_TAIL \
                 : "=&s"(keep), "=&s"(t) : "s"(lds), "v"(v0), "v"(v1), "s"(rs), "s"(soff), "s"(js) : "scc", "memory")
__device__ __forceinline__ void dma8(unsigned lds, unsigned v0, unsigned v1, __amdgpu_buffer_rsrc_t rs,
                                     unsigned soff, unsigned js) {
    unsigned keep, t;
    DMA8("%3", "%4", "%3", "%4");
}

__device__ __forceinline__ void dma8c(unsigned lds, unsigned v0, unsigned v1, __amdgpu_buffer_rsrc_t rs,
                                      unsigned soff, unsigned js) {
    unsigned keep, t;
    asm volatile(DMA_HEAD DMA_C("%3") DMA_C("%4") DMA_C("%3") DMA_C("%4") DMA_C("%3") DMA_C("%4") DMA_C("%3") DMA_C("%4") DMA_TAIL
                 : "=&s"(keep), "=&s"(t) : "s"(lds), "v"(v0), "v"(v1), "s"(rs), "s"(soff), "s"(js) : "scc", "memory");
}

#define STAMP(i) do {} while (0)

#define FDECL do {} while (0)
#define FACC(i) do {} while (0)
#define FOUT() do {} while (0)
__device__ __forceinline__ unsigned x_lane_off(int l) { return (unsigned)((l >> 3) * 8192 + (((l & 7) ^ (l >> 4)) << 4)); }

__device__ __forceinline__ void w_fetch(unsigned lds, __amdgpu_buffer_rsrc_t rw, unsigned q, unsigned t, int lane) {
    const unsigned v = (unsigned)((lane >> 3) * 1024 + (lane & 7) * 16);
    dma8c(lds, v, v, rw, (t >> 1) * 65536u + q * 256u + (t & 1u) * 128u, 8192u);
}
__device__ __forceinline__ void w_frags(u32x4 (&r)[8], unsigned lds, int lane) {
    const unsigned wr = lds + (unsigned)((8 * (lane >> 5) + ((lane >> 2) & 3)) * 128 + ((lane >> 4) & 1) * 32 + (lane & 3) * 8);
#pragma unroll
    for (int j = 0; j < 8; ++j) {
        const unsigned ad = wr + (unsigned)((j & 3) * 2048 + (j >> 2) * 64);
        const u32x2 k0 = __builtin_bit_cast(u32x2, __builtin_amdgcn_ds_read_tr16_b64_v4i16((LAS s16x4*)(size_t)ad));
        const u32x2 k1 = __builtin_bit_cast(u32x2, __builtin_amdgcn_ds_read_tr16_b64_v4i16((LAS s16x4*)(size_t)(ad + 512u)));
        r[j] = __builtin_shufflevector(k0, k1, 0, 1, 2, 3);
    }
}

__device__ __forceinline__ unsigned bfi(unsigned m, unsigned a, unsigned b) { return (a & m) | (b & ~m); }
__device__ __forceinline__ unsigned pkmax(unsigned a, unsigned b) {
    typedef unsigned short u16x2 __attribute__((ext_vector_type(2)));
    return __builtin_bit_cast(unsigned, __builtin_elementwise_max(__builtin_bit_cast(u16x2, a), __builtin_bit_cast(u16x2, b)));
}
struct Packed { u32x4 lo[4]; u32x4 cd[2]; u32x2 hi[8]; unsigned fmt; };
__device__ __forceinline__ void pack_split(const u32x4 (&r)[8], Packed& o) {
#pragma unroll
    for (int j = 0; j < 8; ++j) {
        o.lo[j >> 1][(j & 1) * 2] = __builtin_amdgcn_perm(r[j].y, r[j].x, 0x06040200u);
        o.lo[j >> 1][(j & 1) * 2 + 1] = __builtin_amdgcn_perm(r[j].w, r[j].z, 0x06040200u);
        o.hi[j].x = __builtin_amdgcn_perm(r[j].y, r[j].x, 0x07050301u);
        o.hi[j].y = __builtin_amdgcn_perm(r[j].w, r[j].z, 0x07050301u);
        asm volatile("" : "+v"(o.hi[j]));
        if (j & 1) asm volatile("" : "+v"(o.lo[j >> 1]));
    }
}
__device__ __forceinline__ void pack_codes(Packed& o) {
    unsigned m = 0;
#pragma unroll
    for (int j = 0; j < 8; ++j) {
        const unsigned e0 = o.hi[j].x & 0x7F7F7F7Fu, e1 = o.hi[j].y & 0x7F7F7F7Fu;
        m = pkmax(pkmax(m, e0), pkmax(e0 << 8, e1));
        m = pkmax(m, e1 << 8);
    }
    m = max(m >> 24, (m >> 8) & 0xFFu);
    m = max(m, (unsigned)__builtin_amdgcn_update_dpp(0, (int)m, 0x111, 0xF, 0xF, true));
    m = max(m, (unsigned)__builtin_amdgcn_update_dpp(0, (int)m, 0x112, 0xF, 0xF, true));
    m = max(m, (unsigned)__builtin_amdgcn_update_dpp(0, (int)m, 0x114, 0xF, 0xF, true));
    m = max(m, (unsigned)__builtin_amdgcn_update_dpp(0, (int)m, 0x118, 0xF, 0xF, true));
    const unsigned top = max(max((unsigned)__builtin_amdgcn_readlane((int)m, 15), (unsigned)__builtin_amdgcn_readlane((int)m, 31)),
                             max((unsigned)__builtin_amdgcn_readlane((int)m, 47), (unsigned)__builtin_amdgcn_readlane((int)m, 63)));
    const unsigned base = rfl(top > 7u ? top - 7u : 0u);
    const unsigned kk = 0x80808080u - base * 0x01010101u;
    unsigned all = 0x80808080u;
#pragma unroll
    for (int j = 0; j < 8; ++j) {
        const unsigned t0 = (o.hi[j].x & 0x7F7F7F7Fu) + kk, t1 = (o.hi[j].y & 0x7F7F7F7Fu) + kk;
        all &= t0 & t1;
        const unsigned c1 = bfi(0x07070707u, t1, o.hi[j].y);
        o.cd[j >> 2][j & 3] = bfi(0x87878787u, bfi(0x07070707u, t0, o.hi[j].x), __builtin_amdgcn_alignbit(c1, c1, 28));
    }
    asm volatile("" : "+v"(o.cd[0]), "+v"(o.cd[1]));
    o.fmt = base;
    const unsigned long long lanes = __ballot(all != 0x80808080u);
    if (lanes == 0ull) return;
    unsigned mask = 0, one = 0, c0 = 0, c1 = 0, cj = 0;
    const unsigned long long l1 = lanes & (0ull - lanes);
#pragma unroll
    for (int j = 0; j < 8; ++j) {
        const unsigned t0 = (o.hi[j].x & 0x7F7F7F7Fu) + kk, t1 = (o.hi[j].y & 0x7F7F7F7Fu) + kk;
        const bool out = (t0 & t1 & 0x80808080u) != 0x80808080u;
        const unsigned long long ob = __ballot(out);
        mask |= ob != 0ull ? 1u << j : 0u;
        one |= ob == l1 ? 1u << j : 0u;
        c0 = out ? t0 : c0; c1 = out ? t1 : c1; cj = out ? (unsigned)j : cj;
    }
    const int l = __builtin_ctzll(lanes);
    const unsigned j = (unsigned)__builtin_amdgcn_readlane((int)cj, l);
    const unsigned long long t = (unsigned long long)(unsigned)__builtin_amdgcn_readlane((int)c0, l) |
                                 ((unsigned long long)(unsigned)__builtin_amdgcn_readlane((int)c1, l) << 32);
    const unsigned long long bad = ~t & 0x8080808080808080ull;
    unsigned fix = 0;
    if (((one >> j) & 1u) && (bad & (bad - 1ull)) == 0ull) {
        const unsigned by = (unsigned)__builtin_ctzll(bad) >> 3;
        const unsigned tb = (unsigned)(t >> (by * 8u)) & 0xFFu;
        fix = (16u - (tb >> 3)) | (j << 4) | (by << 7) | ((unsigned)l << 10);
        mask &= ~(1u << j);
    }
    if (__builtin_popcount(mask) > 4) { mask = 0xFFu; fix = 0u; }
    o.fmt = base | (mask << 8) | (fix << 16);
}
__device__ __forceinline__ void send_low(const Packed& k, __amdgpu_buffer_rsrc_t rd, unsigned dof, int lane) {
#pragma unroll
    for (int i = 0; i < 4; ++i) __builtin_amdgcn_raw_buffer_store_b128(k.lo[i], rd, lane * 16, dof + i * 1024u, 17);
}
__device__ __forceinline__ void send_high(const Packed& k, __amdgpu_buffer_rsrc_t rd, unsigned dof, int lane) {
    if ((k.fmt & 0xFF00u) != 0xFF00u) {
        __builtin_amdgcn_raw_buffer_store_b128(k.cd[0], rd, lane * 16, dof + 4096u, 17);
        __builtin_amdgcn_raw_buffer_store_b128(k.cd[1], rd, lane * 16, dof + 5120u, 17);
    }
    if (__builtin_expect((k.fmt & 0xFF00u) != 0u, 0)) {
#pragma unroll
        for (int j = 0; j < 8; ++j)
            if ((k.fmt >> (8 + j)) & 1u) __builtin_amdgcn_raw_buffer_store_b64(k.hi[j], rd, lane * 8, dof + 6144u + j * 512u, 17);
    }
}
__device__ __forceinline__ void send_chunk(const Packed& k, __amdgpu_buffer_rsrc_t rd, unsigned dof, int lane) {
    send_low(k, rd, dof, lane);
    send_high(k, rd, dof, lane);
}
struct Wire { u32x4 lo[4]; u32x4 cd[2]; u32x2 e0; };
__device__ __forceinline__ void wire_load(Wire& w, const char* ws, bool go, unsigned off, unsigned fmt, int lane) {
    const __amdgpu_buffer_rsrc_t rs = mk_rsrc(ws, go ? WS_NEED : 0u);
#pragma unroll
    for (int i = 0; i < 4; ++i) w.lo[i] = __builtin_amdgcn_raw_buffer_load_b128(rs, lane * 16, off + i * 1024u, AG_XAUX);
#pragma unroll
    for (int i = 0; i < 2; ++i) w.cd[i] = __builtin_amdgcn_raw_buffer_load_b128(rs, lane * 16, off + 4096u + i * 1024u, AG_XAUX);
    const unsigned mask = (fmt >> 8) & 0xFFu, j0 = (unsigned)__builtin_ctz(mask | 0x80u);
    w.e0 = __builtin_amdgcn_raw_buffer_load_b64(mk_rsrc(ws, go && mask != 0u ? WS_NEED : 0u), lane * 8, off + 6144u + j0 * 512u, 17);
}
__device__ __forceinline__ void wire_put(unsigned ha, unsigned hb, unsigned l0, unsigned l1, unsigned lds) {
    u32x4 v;
    v.x = __builtin_amdgcn_perm(ha, l0, 0x05010400u);
    v.y = __builtin_amdgcn_perm(ha, l0, 0x07030602u);
    v.z = __builtin_amdgcn_perm(hb, l1, 0x05010400u);
    v.w = __builtin_amdgcn_perm(hb, l1, 0x07030602u);
    *(volatile LAS u32x4*)(size_t)lds = v;
}
__device__ __forceinline__ void wire_unpack(const Wire& w, const char* ws, unsigned off, unsigned fmt, int lane, unsigned lds) {
    const unsigned br = (fmt & 0x7Fu) * 0x01010101u, mask = (fmt >> 8) & 0xFFu;
    const unsigned la = lds + (unsigned)lane * 16u;
#pragma unroll
    for (int j = 0; j < 8; ++j) {
        const unsigned n = w.cd[j >> 2][j & 3];
        wire_put((n & 0x87878787u) + br, (__builtin_amdgcn_alignbit(n, n, 4) & 0x87878787u) + br,
                 w.lo[j >> 1][(j & 1) * 2], w.lo[j >> 1][(j & 1) * 2 + 1], la + j * 1024u);
    }
    if ((fmt & 0xF0000u) != 0u) {
        const unsigned by = (fmt >> 23) & 7u;
        const unsigned ad = lds + ((fmt >> 20) & 7u) * 1024u + (fmt >> 26) * 16u + (by >> 1) * 4u;
        const unsigned d = ((fmt >> 13) & 0x78u) << ((by & 1u) * 16u + 8u);
        if (lane == 0) asm volatile("ds_sub_u32 %0, %1" :: "v"(ad), "v"(d) : "memory");
    }
    if (mask != 0u) {
        const unsigned j0 = (unsigned)__builtin_ctz(mask);
        const __amdgpu_buffer_rsrc_t rs = mk_rsrc(ws, WS_NEED);
#pragma unroll
        for (int j = 0; j < 8; ++j) {
            if ((mask >> j) & 1u) {
                u32x2 e = w.e0;
                if ((unsigned)j != j0)
                    asm volatile("buffer_load_dwordx2 %0, %1, %2, %3 offen sc0 sc1\n\ts_waitcnt vmcnt(0)"
                                 : "=&v"(e) : "v"(lane * 8), "s"(rs), "s"(off + 6144u + j * 512u) : "memory");
                wire_put(e.x, e.y, w.lo[j >> 1][(j & 1) * 2], w.lo[j >> 1][(j & 1) * 2 + 1], la + j * 1024u);
            }
        }
    }
}
__device__ __forceinline__ unsigned flag_pos(unsigned ks) { return (ks & 3u) * 16u + (ks >> 2); }

struct Tile { int rb, cb, src, blk; };
__device__ __forceinline__ Tile tile_of(unsigned code, int rank) {
    const int gidx = rank & 3, gbase = rank & 4, obase = gbase ^ 4;
    Tile t;
    t.rb = (int)(code & 63u); t.cb = (int)((code >> 6) & 7u);
    if (t.rb < 4 * NALPHA) { t.src = gbase + t.rb / NALPHA; t.blk = t.rb % NALPHA; }
    else if (t.rb < 32) { t.src = obase + (t.rb - 4 * NALPHA) / NBETA; t.blk = NALPHA + (t.rb - 4 * NALPHA) % NBETA; }
    else { t.src = rank; t.blk = NALPHA + (t.rb - 32); }
    return t;
}

struct Frag { s16x8 a[4][2]; };
struct PackB { u32x4 lo[2], cd; };
template <int N> struct IC { static constexpr int value = N; };

__global__ void __launch_bounds__(NTHREADS, 2) ag_gemm_kernel(const Params p) {
    __shared__ __attribute__((aligned(1024))) unsigned char lds_raw[LDS_BYTES];
    LAS unsigned char* L = (LAS unsigned char*)lds_raw;
    const int lane = threadIdx.x & 63;
    const int wave = (int)rfl(threadIdx.x >> 6);
    const int rank = p.rank;
    const int gidx = rank & 3, gbase = rank & 4, obase = gbase ^ 4;
    char* myws = p.ws[rank];
    const unsigned bid = blockIdx.x;
    if ((unsigned)(size_t)L != 0u) return;

    auto draw_call = [&]() -> unsigned {
        unsigned ticket = 0;
        if (lane == 0)
            ticket = __hip_atomic_fetch_add((GAS unsigned*)(myws + OFF_CTR + bid * 128u + (unsigned)wave * 4u), 1u,
                                            __ATOMIC_RELAXED, __HIP_MEMORY_SCOPE_AGENT);
        return rfl(ticket) + 1u;
    };

    const int peer = (int)((p.xpeer >> ((bid & 7u) * 4u)) & 15u);
    if (bid < 8u * CPL && peer < NR) {
        if (wave < WPP) {
            __builtin_amdgcn_s_waitcnt(0x0F70);
            if (wave == 0) STAMP(0);
            const int pi = wave * CPL + (int)(bid >> 3);
            const bool same = (peer & 4) == gbase;
            const __amdgpu_buffer_rsrc_t rx = mk_rsrc(p.x, SHARD_BYTES), rw = mk_rsrc(p.w, SHARD_BYTES);
            char* pws = p.ws[peer];
            const unsigned xa0 = x_lane_off(lane), xa1 = xa0 ^ 64u;
            const unsigned wl = (unsigned)wave * 8192u;

            auto load_task = [&](int t) {
                if (!same && t < 128) {
                    w_fetch(wl, rw, (unsigned)(peer & 3), (unsigned)t, lane);
                } else {
                    int ks, blk;
                    if (t < 128) { ks = t >> 1; blk = t & 1; }
                    else { const int u = t - 128; ks = u / 3; blk = (same ? NEARLY : NALPHA) + (u - ks * 3); }
                    dma8c(wl, xa0, xa1, rx, (unsigned)blk * (64u * 8192u) + (unsigned)ks * 128u, 65536u);
                }
            };
            auto split_task = [&](int t, Packed& k) {
                u32x4 ra[8];
                if (!same && t < 128) w_frags(ra, wl, lane);
                else {
#pragma unroll
                    for (int j = 0; j < 8; ++j) ra[j] = *(const volatile LAS u32x4*)(size_t)(wl + (unsigned)lane * 16u + j * 1024u);
                }
                pack_split(ra, k);
            };
            char* cp = myws + OFF_CTR + bid * 128u + (unsigned)wave * 4u;
            char* qp = myws + OFF_Q + (unsigned)peer * 128u;
            unsigned tk;
            load_task(pi);
            asm volatile("global_load_dword %0, %1, off sc1" : "=&v"(tk) : "v"(cp) : "memory");
            if (threadIdx.x < 16) *(LAS unsigned*)(L + L_CTL + threadIdx.x * 4) = 0u;
            asm volatile("s_waitcnt lgkmcnt(0)" ::: "memory");
            __builtin_amdgcn_s_barrier();
            const __amdgpu_buffer_rsrc_t rd = mk_rsrc(pws, WS_NEED);
            auto draw = [&]() {
                unsigned long long sv;
                asm volatile("s_mov_b64 %1, exec\n\ts_mov_b64 exec, 1\n\tglobal_atomic_add %0, %2, %3, off sc0\n\ts_mov_b64 exec, %1"
                             : "=&v"(tk), "=&s"(sv) : "v"(qp), "v"(1u) : "memory");
            };
            unsigned dof, fof;
            auto place = [&](int t) {
                if (!same && t < 128) {
                    dof = OFF_WST + (unsigned)(4 + gidx) * WQ_BYTES + (unsigned)t * CH_BYTES;
                    fof = OFF_FW + (unsigned)((4 + gidx) * 128 + t) * 8u;
                } else {
                    int ks, ai;
                    if (t < 128) { ks = t >> 1; ai = gidx * NALPHA + (t & 1); }
                    else {
                        const int u = t - 128; ks = u / 3;
                        const int r = u - ks * 3;
                        ai = same ? gidx * NALPHA + NEARLY + r : 4 * NALPHA + gidx * NBETA + r;
                    }
                    dof = (unsigned)ai * ABLK_BYTES + (unsigned)ks * CH_BYTES;
                    fof = OFF_FA + ((unsigned)ai * 64u + flag_pos((unsigned)ks)) * 8u;
                }
            };
            unsigned spins = 0, call = 0, fmt = 0, fpv = 0;
            bool cast = false;
            unsigned long long t0 = 0;
            for (int t = pi, nx = pi + NPW;;) {
                place(t);
                asm volatile("s_waitcnt vmcnt(0)" : "+v"(tk) :: "memory");
                __builtin_amdgcn_s_waitcnt(0x0F70);
                asm volatile("" ::: "memory");
                if (call == 0u) {
                    call = rfl(tk) + 1u;
                } else {
                    if (lane == 0)
                        __hip_atomic_store((GAS unsigned long long*)(pws + fpv), (unsigned long long)call | ((unsigned long long)fmt << 32),
                                           __ATOMIC_RELAXED, __HIP_MEMORY_SCOPE_SYSTEM);
                    if (cast && lane == 0)
                        __hip_atomic_store((GAS unsigned long long*)(myws + OFF_GO + (unsigned)peer * 128u), rt() + AG_DELTA,
                                           __ATOMIC_RELAXED, __HIP_MEMORY_SCOPE_AGENT);
                    if (t >= NTASK) break;
                    nx = (int)min(rfl(tk) - (call - 1u) * (unsigned)(NTASK - NPW) + 2u * (unsigned)NPW, (unsigned)NTASK);
                }
                dof += OFF_DATA + (call & 1u) * PAR_BYTES;
                fpv = fof + (call & 1u) * FLG_PAR;
                Packed k;
                split_task(t, k);
                const bool eager = t < AG_EAGERW * CPL;
                if (eager) send_low(k, rd, dof, lane);
                if (nx < NTASK) { load_task(nx); draw(); }
                pack_codes(k);
                fmt = k.fmt;
                unsigned my = 0;
                if (!eager) {
                    if (t == pi) __builtin_amdgcn_s_sleep(AG_W2D);
                    asm volatile("ds_add_rtn_u32 %0, %1, %2\n\ts_waitcnt lgkmcnt(0)" : "=&v"(my) : "v"(C_CNT), "v"(lane == 0 ? 1u : 0u) : "memory");
                    my = rfl(my);
                    while (lds_ld(C_GO) < my) {
                        if ((++spins & 1023u) == 0u && expired(t0, DEADLINE)) break;
                        __builtin_amdgcn_s_sleep(1);
                    }
                    send_low(k, rd, dof, lane);
                }
                send_high(k, rd, dof, lane);
                if (!eager) {
                    VM_WAIT(AG_REL);
                    lds_st(C_GO, my + 1u);
                }
                cast = t == NTASK - AG_GOT;
                t = (int)rfl((unsigned)nx);
            }
            if (lane == 0) st_agent(cp, call);
            return;
        }
        __builtin_amdgcn_s_barrier();
        if (wave != 7 || bid != 0u) return;
        const unsigned call = draw_call();
        const int i0 = (lane * 2) & 31, ex = 4 * NALPHA + gidx * NBETA;
        const bool skip0 = i0 >= ex && i0 < ex + NBETA, skip1 = i0 + 1 >= ex && i0 + 1 < ex + NBETA;
        const char* fp = myws + OFF_DONE + (unsigned)lane * 8u;
        unsigned long long t0 = 0;
        unsigned spins = 0;
        bool dead = false;
        for (;;) {
            u32x2 v;
            asm volatile("global_load_dwordx2 %0, %1, off sc0 sc1\n\ts_waitcnt vmcnt(0)" : "=&v"(v) : "v"(fp) : "memory");
            const bool ok = (skip0 || v.x >= call) && (skip1 || v.y >= call);
            if (__all(ok)) break;
            if ((++spins & 255u) == 0u) {
                if (rfl(ld_agent(myws + OFF_TMO)) == call || expired(t0, 2 * DEADLINE)) { dead = true; break; }
            }
            __builtin_amdgcn_s_sleep(2);
        }
        __builtin_amdgcn_fence(__ATOMIC_ACQUIRE, "");
        if (dead) {
            if (lane == 0) st_agent(myws + OFF_TMO, call);
            const unsigned long long t1 = rt();
            while (rt() - t1 < DEADLINE / 8) __builtin_amdgcn_s_sleep(64);
            if (lane == 0) st_sys(p.out[rank] + (size_t)rank * 512u * 1024u, 0x7FC07FC0u);
        }
        STAMP(6);
        return;
    }

    __builtin_amdgcn_s_sleep(AG_TDELAY);
    const unsigned own = (bid - 64u) * 4u + (unsigned)wave - (unsigned)NLOAD;
    const bool packs = bid >= 64u && wave >= NLOAD && own < 512u;
    if (packs) w_fetch((unsigned)(8 + wave) * 8192u, mk_rsrc(p.w, SHARD_BYTES), own & 3u, own >> 2, lane);
    if (threadIdx.x < (2048 + 128) / 8) *(LAS unsigned long long*)(L + L_REC + threadIdx.x * 8) = 0ull;
    const unsigned call = draw_call();
    const unsigned tcode = p.tile[bid];
    const unsigned ntile = (tcode >> 31) ? 2u : 1u;
    asm volatile("s_waitcnt lgkmcnt(0)" ::: "memory");
    __builtin_amdgcn_s_barrier();
    const unsigned par = OFF_DATA + (call & 1u) * PAR_BYTES;

    if (wave < NLOAD) {
        const unsigned lw = (unsigned)wave;
        const unsigned long long born = rt();
        const __amdgpu_buffer_rsrc_t rws = mk_rsrc(myws, WS_NEED);
        const unsigned fpar = (call & 1u) * FLG_PAR;
        unsigned spins = 0;
        unsigned long long t0 = 0;
        bool dead = false;
        FDECL;
        auto give_up = [&]() -> bool {
            return (++spins & 255u) == 0u &&
                   (rfl(ld_agent(myws + OFF_TMO)) == call || lds_ld(C_ABORT) != 0u || expired(t0, DEADLINE));
        };
        auto wait_slot = [&](unsigned g) {
            while (g >= (unsigned)NSLOT - 4u) {
                const u32x4 fr = lds_ld4(C_FREE);
                if (rfl(min(min(fr.x, fr.y), min(fr.z, fr.w))) + ((unsigned)NSLOT - 4u) > g) break;
                if ((++spins & 1023u) == 0u && (lds_ld(C_ABORT) != 0u || expired(t0, 2 * DEADLINE))) { dead = true; break; }
                __builtin_amdgcn_s_sleep(AG_FREE_SLEEP);
            }
        };
        for (unsigned ti = 0; ti < ntile && !dead; ++ti) {
            const Tile t = tile_of(tcode >> (16u * ti), rank);
            const unsigned g0 = ti * (unsigned)NKS + lw;
            unsigned wn = 0;
            auto publish = [&](unsigned i) {
                const unsigned need = min(i * 4u + lw + (unsigned)BD + 1u, (unsigned)NKS);
                while (wn < need) {
                    const u32x4 v = __builtin_amdgcn_raw_buffer_load_b128(rws, fpar + OFF_FW + (unsigned)t.cb * 1024u + (unsigned)lane * 16u, 0, 17);
                    const bool up = v.x >= call && v.z >= call;
                    wn = (unsigned)__builtin_ctzll(~__ballot(up) | (1ull << 63)) + (__all(up) ? 1u : 0u);
                    u32x4 f;
                    f.x = (v.y & 0x0FFFu) | ((v.y >> 22) & 1u ? 0u : v.y & 0xFFFF0000u);
                    f.y = (v.y & 0x00FFu) | ((v.y >> 4) & 0x0F00u) | ((v.y >> 22) & 1u ? v.y & 0xFFFF0000u : 0u);
                    f.z = (v.w & 0x0FFFu) | ((v.w >> 22) & 1u ? 0u : v.w & 0xFFFF0000u);
                    f.w = (v.w & 0x00FFu) | ((v.w >> 4) & 0x0F00u) | ((v.w >> 22) & 1u ? v.w & 0xFFFF0000u : 0u);
                    if (up) {
                        const u32x2 f0 = {f.x, f.y}, f1 = {f.z, f.w};
                        asm volatile("ds_or_b64 %0, %1\n\tds_or_b64 %0, %2 offset:8" :: "v"(L_REC + ti * 1024u + (unsigned)lane * 16u), "v"(f0), "v"(f1) : "memory");
                    }
                    if (wn >= need) break;
                    if (give_up()) { dead = true; return; }
                    __builtin_amdgcn_s_sleep(AG_POLL_SLEEP);
                }
                asm volatile("s_waitcnt lgkmcnt(0)" ::: "memory");
                if (lane < 4) asm volatile("ds_or_b32 %0, %1" :: "v"(L_REC + (g0 + i * 4u) * 16u + (unsigned)lane * 4u), "v"(0x80u) : "memory");
            };
            if (ti == ntile - 1u && wave == 0) STAMP(2);
            if (t.src != rank) {
                const unsigned base = par + (unsigned)t.rb * ABLK_BYTES + lw * CH_BYTES;
                const unsigned fl0 = fpar + OFF_FA + (unsigned)t.rb * 512u + lw * 128u;
                u32x2 fl = {0u, 0u};
                unsigned up = 0u;
                auto refresh = [&]() {
                    if (lane < 16) fl = __builtin_amdgcn_raw_buffer_load_b64(rws, fl0 + (unsigned)lane * 8u, 0, 17);
                    up = (unsigned)__ballot(fl.x >= call);
                    asm volatile("" ::: "memory");
                };
                auto await = [&](unsigned need) {
                    for (;;) {
                        refresh();
                        if ((up & need) == need) return;
                        if (give_up()) { dead = true; return; }
                        __builtin_amdgcn_s_sleep(AG_POLL_SLEEP);
                    }
                };
                auto fetch = [&](Wire& w, unsigned& f, unsigned i, bool go) {
                    f = (unsigned)__builtin_amdgcn_readlane((int)fl.y, (int)(i & 15u));
                    wire_load(w, myws, go, base + i * (4u * CH_BYTES), f, lane);
                };
                auto turn = [&](const Wire& w, unsigned f, Wire& wx, unsigned& fx, unsigned i) {
                    const bool last = i == 15u;
                    const unsigned need = last ? 0u : 2u << i;
                    if ((up & need) != need) refresh();
                    FACC(0);
                    const bool ok = (up & need) == need;
                    fetch(wx, fx, i + 1u, ok && !last);
                    FACC(1);
                    wait_slot(g0 + i * 4u);
                    FACC(2);
                    if (dead) return;
                    wire_unpack(w, myws, base + i * (4u * CH_BYTES), f, lane, ((g0 + i * 4u) & (unsigned)(NSLOT - 1)) * 8192u);
                    FACC(4);
                    publish(i);
                    FACC(3);
                    if (!ok && !dead) {
                        await(need);
                        fetch(wx, fx, i + 1u, !dead);
                    }
                    FACC(5);
                };
                Wire w0, w1;
                unsigned f0, f1;
                await(1u);
                fetch(w0, f0, 0u, !dead);
                FACC(5);
                for (unsigned i = 0; i < 16u && !dead; i += 2u) {
                    turn(w0, f0, w1, f1, i);
                    if (dead) break;
                    turn(w1, f1, w0, f0, i + 1u);
                }
            } else {
                const __amdgpu_buffer_rsrc_t rx = mk_rsrc(p.x, SHARD_BYTES);
                const unsigned v0 = x_lane_off(lane), v1 = v0 ^ 64u;
                const unsigned base = (unsigned)t.blk * (64u * 8192u) + lw * 128u;
                for (unsigned i = 0; i < 18u && !dead; ++i) {
                    if (i < 16u) {
                        wait_slot(g0 + i * 4u);
                        if (dead) break;

                        dma8(((g0 + i * 4u) & (unsigned)(NSLOT - 1)) * 8192u, v0, v1, rx, base + i * 512u, 65536u);
                        VM_WAIT(16);
                    } else if (i == 16u) {
                        VM_WAIT(8);
                    } else {
                        VM_WAIT(0);
                    }
                    if (i >= 2u) publish(i - 2u);
                }
                VM_WAIT(0);
            }
        }
        FOUT();
        if (dead) {
            lds_st(C_ABORT, 1u);
            if (lane == 0) st_agent(myws + OFF_TMO, call);
        }
        if (wave == 0) {
            const int cb = tile_of(tcode >> (16u * (ntile - 1u)), rank).cb;
            if (cb >= 4) {
                char* hp = myws + OFF_GO + (unsigned)(obase + (cb & 3)) * 128u;
                unsigned long long go;
                t0 = 0;
                for (;;) {
                    go = __hip_atomic_load((const GAS unsigned long long*)hp, __ATOMIC_RELAXED, __HIP_MEMORY_SCOPE_AGENT);
                    if (dead || __all(go > born)) break;
                    if ((++spins & 63u) == 0u && expired(t0, DEADLINE)) break;
                    __builtin_amdgcn_s_sleep(AG_GO_SLEEP);
                }
                const unsigned long long now = rt();
                unsigned long long at = 0;
                if (lane == 0) {
                    __hip_atomic_fetch_max((GAS unsigned long long*)(hp + 1024), go > now ? go : now, __ATOMIC_RELAXED, __HIP_MEMORY_SCOPE_AGENT);
                    at = __hip_atomic_fetch_add((GAS unsigned long long*)(hp + 1024), (unsigned long long)AG_TAU, __ATOMIC_RELAXED, __HIP_MEMORY_SCOPE_AGENT);
                }
                const unsigned wt = rfl(at > now ? (unsigned)min(at - now, 2000ull) : 0u);
                while ((unsigned)(rt() - now) < wt) __builtin_amdgcn_s_sleep(1);
                lds_st(C_GO, 1u);
            }
        }
        if (wave == 3) STAMP(3);
        return;
    }

    const int cw = wave - NLOAD;
    __builtin_amdgcn_s_setprio(AG_PRIO);
    const int hh = lane >> 5, r32 = lane & 31;
    const unsigned a0 = (unsigned)(r32 * 128 + ((hh ^ ((r32 >> 1) & 7)) << 4));
    const unsigned cfree = C_FREE + (unsigned)cw * 4u;
    if (packs) {
        u32x4 r[8];
        Packed k;
        VM_WAIT(0);
        w_frags(r, (unsigned)(8 + wave) * 8192u, lane);
        pack_split(r, k);
        lds_st(cfree, 4u);
        pack_codes(k);
        send_chunk(k, mk_rsrc(myws, WS_NEED), par + OFF_WST + (own & 3u) * WQ_BYTES + (own >> 2) * CH_BYTES, lane);
        VM_WAIT(0);
        if (lane == 0)
            __hip_atomic_store((GAS unsigned long long*)(myws + (call & 1u) * FLG_PAR + OFF_FW + ((own & 3u) * 128u + (own >> 2)) * 8u),
                               (unsigned long long)call | ((unsigned long long)k.fmt << 32), __ATOMIC_RELAXED, __HIP_MEMORY_SCOPE_SYSTEM);
    } else {
        lds_st(cfree, 4u);
    }
    const unsigned vlo = (unsigned)lane * 16u + (unsigned)(cw & 1) * 2048u, vcd = (unsigned)lane * 16u + 4096u + (unsigned)(cw & 1) * 1024u;

    unsigned spins = 0;
    unsigned long long t0 = 0;
    FDECL;
    auto wait_full = [&](unsigned g) {
        while ((lds_ld(L_REC + g * 16u) & 0x80u) == 0u) {
            if ((++spins & 63u) == 0u && lds_ld(C_ABORT) != 0u) break;
            if ((spins & 1023u) == 0u && expired(t0, 3 * DEADLINE)) { lds_st(C_ABORT, 1u); break; }
            __builtin_amdgcn_s_sleep(1);
        }
    };
    unsigned ab[4];
#pragma unroll
    for (int k = 0; k < 4; ++k) ab[k] = a0 ^ (unsigned)(k * 32);
    auto rd = [&](Frag& f, auto sc) {
        constexpr unsigned S = decltype(sc)::value;
#pragma unroll
        for (int k = 0; k < 4; ++k)
#pragma unroll
            for (int mt = 0; mt < 2; ++mt)
                f.a[k][mt] = *(const LAS s16x8*)((const LAS unsigned char*)(size_t)ab[k] + (S * 8192u + mt * 4096u));
    };
    auto ldb = [&](PackB& b, __amdgpu_buffer_rsrc_t rs, unsigned off) {
        b.lo[0] = __builtin_amdgcn_raw_buffer_load_b128(rs, vlo, off, AG_WAUX);
        b.lo[1] = __builtin_amdgcn_raw_buffer_load_b128(rs, vlo + 1024u, off, AG_WAUX);
        b.cd = __builtin_amdgcn_raw_buffer_load_b128(rs, vcd, off, AG_WAUX);
    };
    auto put = [&](u32x16& o, int k, unsigned ha, unsigned hb, const PackB& b) {
        const unsigned l0 = b.lo[k >> 1][(k & 1) * 2], l1 = b.lo[k >> 1][(k & 1) * 2 + 1];
        o[4 * k] = __builtin_amdgcn_perm(ha, l0, 0x05010400u);
        o[4 * k + 1] = __builtin_amdgcn_perm(ha, l0, 0x07030602u);
        o[4 * k + 2] = __builtin_amdgcn_perm(hb, l1, 0x05010400u);
        o[4 * k + 3] = __builtin_amdgcn_perm(hb, l1, 0x07030602u);
    };
    auto repair = [&](u32x16& o, const PackB& b, unsigned fmt, unsigned off) {
        if ((fmt & 0xF0000u) != 0u) {
            const unsigned by = (fmt >> 23) & 7u;
            o[((fmt >> 18) & 12u) + (by >> 1)] -= (unsigned)lane == (fmt >> 26) ? ((fmt >> 13) & 0x78u) << ((by & 1u) * 16u + 8u) : 0u;
        }
        if ((fmt & 0xF00u) != 0u) {
            const __amdgpu_buffer_rsrc_t rs = mk_rsrc(myws, WS_NEED);
#pragma unroll
            for (int k = 0; k < 4; ++k) {
                if ((fmt >> (8 + k)) & 1u) {
                    u32x2 e;
                    asm volatile("buffer_load_dwordx2 %0, %1, %2, %3 offen sc0 sc1\n\ts_waitcnt vmcnt(0)"
                                 : "=&v"(e) : "v"(lane * 8), "s"(rs), "s"(off + 6144u + (unsigned)((cw & 1) * 4 + k) * 512u) : "memory");
                    put(o, k, e.x, e.y, b);
                }
            }
        }
    };
    f32x16 acc[2];
    unsigned vrec = L_REC + (unsigned)cw * 4u;
    asm volatile("" : "+v"(vrec));
#define SB() __builtin_amdgcn_sched_barrier(0)

    for (unsigned ti = 0; ti < ntile; ++ti) {
#pragma unroll
        for (int i = 0; i < 16; ++i) { acc[0][i] = 0.f; acc[1][i] = 0.f; }
        const Tile t = tile_of(tcode >> (16u * ti), rank);
        const unsigned g0 = ti * (unsigned)NKS;
        const unsigned wbase = par + OFF_WST + (unsigned)t.cb * WQ_BYTES + (unsigned)(cw >> 1) * CH_BYTES;
        Frag fa[2];
        u32x16 fb[2];
        PackB pb[BD];
        wait_full(g0);
        FACC(3);
#pragma unroll
        for (int i = 0; i < BD; ++i) { ldb(pb[i], mk_rsrc(myws, WS_NEED), wbase + (unsigned)i * (2u * CH_BYTES)); SB(); }
        rd(fa[0], IC<0>());
        unsigned rec = rfl(*(const volatile LAS unsigned*)(size_t)vrec);
#pragma unroll
        for (int k = 0; k < 4; ++k) {
            const unsigned br = (rec & 0x7Fu) * 0x01010101u, n = pb[0].cd[k];
            put(fb[0], k, (n & 0x87878787u) + br, (__builtin_amdgcn_alignbit(n, n, 4) & 0x87878787u) + br, pb[0]);
        }
        if ((rec >> 8) != 0u) repair(fb[0], pb[0], rec, wbase);
        rec = *(const volatile LAS unsigned*)((const LAS unsigned char*)(size_t)vrec + 16);
        __builtin_amdgcn_s_waitcnt(0xC07F);
        for (unsigned o = 0; o < (unsigned)(NKS / BD); ++o) {
            const __amdgpu_buffer_rsrc_t rb = mk_rsrc(myws, o + 1u < (unsigned)(NKS / BD) ? WS_NEED : 0u);
            const unsigned s0 = o * (unsigned)BD, wo = wbase + s0 * (2u * CH_BYTES);
            auto step = [&](auto uc) {
                constexpr int U = decltype(uc)::value, C = U & 1, N = C ^ 1, X = (U + 1) & (BD - 1);
                const unsigned g = g0 + s0 + (unsigned)U + 1u;
                const unsigned fmt = rfl(rec);
                const unsigned br = (fmt & 0x7Fu) * 0x01010101u;
                if (U == BD - 1) {
#pragma unroll
                    for (int k = 0; k < 4; ++k) ab[k] ^= 65536u;
                }
                rec = *(const volatile LAS unsigned*)((const LAS unsigned char*)(size_t)vrec + (U + 2) * 16);
                SB();
#pragma unroll
                for (int k = 0; k < 4; ++k) {
                    const u32x4 bk = {fb[C][4 * k], fb[C][4 * k + 1], fb[C][4 * k + 2], fb[C][4 * k + 3]};
                    acc[0] = __builtin_amdgcn_mfma_f32_32x32x16_bf16(__builtin_bit_cast(bf16x8, bk), __builtin_bit_cast(bf16x8, fa[C].a[k][0]), acc[0], 0, 0, 0);
                    SB();
                    fa[N].a[k][0] = *(const LAS s16x8*)((const LAS unsigned char*)(size_t)ab[k] + X * 8192);
                    if (k == 0) ldb(pb[U], rb, wo + (unsigned)(U + BD) * (2u * CH_BYTES));
                    const unsigned n = pb[X].cd[k];
                    const unsigned ha = (n & 0x87878787u) + br, hb = (__builtin_amdgcn_alignbit(n, n, 4) & 0x87878787u) + br;
                    asm volatile("" :: "v"(fa[C].a[k][0]), "v"(ha), "v"(hb));
                    SB();
                    acc[1] = __builtin_amdgcn_mfma_f32_32x32x16_bf16(__builtin_bit_cast(bf16x8, bk), __builtin_bit_cast(bf16x8, fa[C].a[k][1]), acc[1], 0, 0, 0);
                    SB();
                    fa[N].a[k][1] = *(const LAS s16x8*)((const LAS unsigned char*)(size_t)ab[k] + (X * 8192 + 4096));
                    put(fb[N], k, ha, hb, pb[X]);
                    asm volatile("" :: "v"(fa[C].a[k][1]), "v"(fb[N][4 * k + 3]));
                    SB();
                }
                asm volatile("" :: "v"(fb[C]));
                if ((U & 3) == 3) lds_st(cfree, g + 4u);
                if (__builtin_expect((fmt >> 7) != 1u, 0)) {
                    if (g != g0 + (unsigned)NKS) {
                        FACC(0);
                        repair(fb[N], pb[X], fmt, wo + (unsigned)(U + 1) * (2u * CH_BYTES));
                        FACC(4);
                        if ((fmt & 0x80u) == 0u) {
                            FACC(0);
                            lds_st(cfree, g + 4u);
                            wait_full(g);
                            FACC(1);
                            rd(fa[N], IC<X>());
                            rec = *(const volatile LAS unsigned*)((const LAS unsigned char*)(size_t)vrec + (U + 2) * 16);
                            __builtin_amdgcn_s_waitcnt(0xC07F);
                            FACC(5);
                        }
                    }
                }
                SB();
            };
            step(IC<0>()); step(IC<1>()); step(IC<2>()); step(IC<3>());
            step(IC<4>()); step(IC<5>()); step(IC<6>()); step(IC<7>());
            vrec += (unsigned)BD * 16u;
        }
        if (cw == 0) STAMP(7);
        FACC(0);
        FOUT();

        const bool bad = lds_ld(C_ABORT) != 0u;
#pragma unroll
        for (int mt = 0; mt < 2; ++mt) {
#pragma unroll
            for (int j = 0; j < 4; ++j) {
                bf16x2 lo, hi;
                lo[0] = (__bf16)acc[mt][4 * j]; lo[1] = (__bf16)acc[mt][4 * j + 1];
                hi[0] = (__bf16)acc[mt][4 * j + 2]; hi[1] = (__bf16)acc[mt][4 * j + 3];
                u32x2 v; v.x = __builtin_bit_cast(unsigned, lo); v.y = __builtin_bit_cast(unsigned, hi);
                if (bad) { v.x = 0x7FC07FC0u; v.y = 0x7FC07FC0u; }
                *(volatile LAS u32x2*)(size_t)(L_ST + (unsigned)((mt * 32 + r32) * ST_ROW + cw * 64 + j * 16 + hh * 8)) = v;
            }
        }
        unsigned old;
        asm volatile("ds_add_rtn_u32 %0, %1, %2\n\ts_waitcnt lgkmcnt(0)" : "=&v"(old) : "v"(C_CNT), "v"(lane == 0 ? 1u : 0u) : "memory");
        if ((rfl(old) & 3u) != 3u) continue;
        u32x4 o[16];
#pragma unroll
        for (int i = 0; i < 16; ++i)
            o[i] = *(const volatile LAS u32x4*)(size_t)(L_ST + (unsigned)((i * 4 + (lane >> 4)) * ST_ROW + (lane & 15) * 16));

        const bool remote = t.cb >= 4;
        const int dstr = remote ? obase + (t.cb & 3) : rank;
        const __amdgpu_buffer_rsrc_t ro = mk_rsrc(p.out[dstr], SHARD_BYTES);
        const unsigned ovo = (unsigned)(lane >> 4) * 1024u + (unsigned)(lane & 15) * 16u;
        const unsigned oso = (unsigned)(t.src * 512 + t.blk * 64) * 1024u + (unsigned)(remote ? gidx : t.cb) * 256u;
        if (!remote) {
#pragma unroll
            for (int i = 0; i < 16; ++i) __builtin_amdgcn_raw_buffer_store_b128(o[i], ro, ovo, oso + i * 4096u, AG_LOCAL_AUX);
        } else {
            t0 = 0;
            while (lds_ld(C_GO) == 0u) {
                if ((++spins & 1023u) == 0u && expired(t0, 2 * DEADLINE)) break;
                __builtin_amdgcn_s_sleep(1);
            }
            STAMP(4);
#pragma unroll
            for (int i = 0; i < 16; ++i) __builtin_amdgcn_raw_buffer_store_b128(o[i], ro, ovo, oso + i * 4096u, 17);
            VM_WAIT(0);
            if (lane == 0) st_sys(p.ws[dstr] + OFF_DONE + (unsigned)(gidx * 32 + t.rb) * 4u, call);
        }
        STAMP(8);
    }
}

}

extern "C" void kernel_launch(int rank, int nranks, ncclComm_t comm,
                              hipStream_t stream,
                              void* const* d_in, const int* in_sizes, int n_in,
                              void* d_out, int out_size,
                              const int* in_dtypes, int out_dtype,
                              void* const* d_out_all,
                              void* const* d_ws_all, size_t ws_bytes) {
    using namespace agk;
    (void)comm; (void)in_dtypes; (void)out_dtype;
    if (nranks != NR || (unsigned)rank >= (unsigned)NR || n_in < 2 || ws_bytes < (size_t)WS_NEED ||
        in_sizes[0] != 512 * 4096 || in_sizes[1] != 512 * 4096 || out_size != 512 * 4096) {
        fprintf(stderr, "kernel_launch: unexpected problem shape\n");
        return;
    }
    {
        int dev = -1, bus = -1, cnt = -1;
        (void)hipGetDevice(&dev);
        (void)hipGetDeviceCount(&cnt);
        (void)hipDeviceGetAttribute(&bus, hipDeviceAttributePciBusId, dev);
        char buf[512];
        int n = snprintf(buf, sizeof buf, "TOPO rank %d dev %d of %d bus %x ws:", rank, dev, cnt, bus);
        for (int r = 0; r < NR; ++r) {
            hipPointerAttribute_t at;
            const hipError_t e = hipPointerGetAttributes(&at, d_ws_all[r]);
            n += snprintf(buf + n, sizeof buf - n, " %d/%d/%p", (int)e, e == hipSuccess ? at.device : -1, d_ws_all[r]);
        }
        (void)hipGetLastError();
        fprintf(stderr, "%s out %p x %p\n", buf, d_out, d_in[0]);
    }
    Params p;
    p.x = d_in[0];
    p.w = d_in[1];
    for (int r = 0; r < NR; ++r) { p.ws[r] = (char*)d_ws_all[r]; p.out[r] = (char*)d_out_all[r]; }
    p.out[rank] = (char*)d_out;
    p.rank = rank;
    const int gidx = rank & 3, obase = (rank & 4) ^ 4;
    int xcd_of[NR], spare = 3;
    {
        const unsigned setmask = 0xB2u;
        int lo = 0, hi = 4;
        for (int o = 1; o < NR; ++o) {
            const int q = (rank + o) & 7;
            xcd_of[q] = (((setmask >> q) ^ (setmask >> rank)) & 1u) ? hi++ : lo++;
        }
        xcd_of[rank] = spare;
        p.xpeer = 0u;
        for (int q = 0; q < NR; ++q) p.xpeer |= (unsigned)(q == rank ? 15 : q) << (xcd_of[q] * 4);
    }
    auto is_late = [&](int rb) { return rb < 32 && (rb >= 4 * NALPHA || (rb % NALPHA >= NEARLY && rb / NALPHA != gidx)); };
    auto code_of = [](int rb, int cb) { return (unsigned)rb | ((unsigned)cb << 6) | 0x8000u; };
    int next[8];
    for (int x = 0; x < 8; ++x) next[x] = x != spare ? x + 8 * CPL : x;
    for (int b = 0; b < NWG; ++b) p.tile[b] = 0u;
    int ntile = 0;
    int late_x[4], used = 0;
    for (int pass = 0; pass < 2; ++pass)
        for (int c = 0; c < 4; ++c)
            for (int o = 0; o < 4; ++o) {
                const int x = xcd_of[(rank & 4) + o];
                if ((pass == 1 && late_x[c] >= 0)) break;
                late_x[c] = -1;
                if (((used >> x) & 1) || (pass == 0 && (x >> 2) != (xcd_of[obase + c] >> 2))) continue;
                late_x[c] = x; used |= 1 << x;
                break;
            }
    for (int c = 0; c < 4; ++c)
        for (int rb = 0; rb < 32; ++rb) {
            if (rb >= 4 * NALPHA && (rb - 4 * NALPHA) / NBETA == c) continue;
            const int xx = is_late(rb) ? late_x[c] : xcd_of[obase + c];
            if (next[xx] >= NWG) { fprintf(stderr, "kernel_launch: placement\n"); return; }
            p.tile[next[xx]] = code_of(rb, 4 + c);
            next[xx] += 8;
            ++ntile;
        }
    unsigned rest[140], first[64];
    int nr = 0, nf = 0;
    for (int pass = 0; pass < 3; ++pass)
        for (int rb = 0; rb < 35; ++rb)
            for (int c = 0; c < 4; ++c) {
                const int cls = is_late(rb) ? 2 : (rb >= 32 || rb / NALPHA == gidx) ? 0 : 1;
                if (cls != pass) continue;
                if (cls < 2 && nf < NPWG) first[nf++] = code_of(rb, c); else rest[nr++] = code_of(rb, c);
            }
    int cand[NWG], nc = 0;
    for (int x = 0, k = nr - 1; x < 8; ++x)
        for (; next[x] < NWG && k >= 0; next[x] += 8, --k) {
            p.tile[next[x]] = rest[k];
            if (is_late((int)(rest[k] & 63u))) cand[nc++] = next[x];
            ++ntile;
        }
    if (nc < nf || ntile + nf != NTILE || nf != NPWG) { fprintf(stderr, "kernel_launch: pairing %d %d %d\n", nc, nf, ntile); return; }
    for (int k = 0; k < nf; ++k) {
        const int b = cand[k * nc / nf];
        p.tile[b] = first[k] | (p.tile[b] << 16);
    }
    hipLaunchKernelGGL(ag_gemm_kernel, dim3(NWG), dim3(NTHREADS), 0, stream, p);
    HIP_CHECK(hipGetLastError());
}
```

```cpp
#include <hip/hip_runtime.h>
#include <rccl/rccl.h>
#include <cstdio>
#include <cstdint>

#define HIP_CHECK(expr) do { hipError_t _e = (expr); if (_e != hipSuccess) { \
    fprintf(stderr, "HIP error %s at %s:%d\n", hipGetErrorString(_e), __FILE__, __LINE__); \
    return; } } while (0)

namespace agk {

#define GAS __attribute__((address_space(1)))
#define LAS __attribute__((address_space(3)))
#define INL __attribute__((always_inline))
typedef unsigned u32x4 __attribute__((ext_vector_type(4)));
typedef unsigned u32x2 __attribute__((ext_vector_type(2)));
typedef short s16x4 __attribute__((ext_vector_type(4)));
typedef short s16x8 __attribute__((ext_vector_type(8)));
typedef __bf16 bf16x8 __attribute__((ext_vector_type(8)));
typedef __bf16 bf16x2 __attribute__((ext_vector_type(2)));
typedef float f32x16 __attribute__((ext_vector_type(16)));
typedef unsigned u32x16 __attribute__((ext_vector_type(16)));

constexpr int NR = 8;
constexpr int NKS = 64;
constexpr int NALPHA = 5, NBETA = 3;
constexpr int NEARLY = 2;
constexpr int NWG = 256;
constexpr int NTILE = 256;
constexpr int NTASK = 320;
#define AG_CPL 8
#define AG_WPP 3
#define AG_TOPO 1
constexpr int CPL = AG_CPL;
constexpr int WPP = AG_WPP;
constexpr int NPW = CPL * WPP;
constexpr int NPWG = 7 * CPL;
#define AG_POLL_SLEEP 2
#define AG_FREE_SLEEP 1
#define AG_STAGGER 4
#define AG_LOCAL_AUX 17
#define AG_REL 5
#define AG_NEAGER 4
#define AG_TAU 27
#define AG_GOY 4
#define AG_SPLIT 1
#define AG_LOCK 1
#define AG_CDMA 1
#define AG_GO_SLEEP 4
#define AG_GOT 40
#define AG_DELTA 320
#define AG_EAGERW 2
#define AG_W2D 8
#define AG_XAUX 17
#define AG_WAUX 17
#define AG_OWNC 0
#define AG_TWO 0
#define AG_WARM 0
#define AG_TDELAY 16
#define AG_PRIO 2
#define AG_KEEP 2
#define AG_HOLD 1
static_assert(NPWG <= 56 && WPP <= 3, "pushing workgroups: at most one per early tile with a local output");
constexpr unsigned SHARD_BYTES = 512u * 4096u * 2u;

constexpr unsigned OFF_CTR = 0;
constexpr unsigned OFF_TMO = 32u << 10;
constexpr unsigned OFF_GO = 34u << 10;
constexpr unsigned OFF_ARR = 33u << 10;
constexpr unsigned OFF_Q = 36u << 10;
constexpr unsigned OFF_FA = 64u << 10;
constexpr unsigned OFF_FW = 80u << 10;
constexpr unsigned FLG_PAR = 32u << 10;
constexpr unsigned OFF_DONE = 128u << 10;
constexpr unsigned OFF_DBG = 192u << 10;
constexpr unsigned OFF_DATA = 1u << 20;
constexpr unsigned CH_BYTES = 10240u;
constexpr unsigned ABLK_BYTES = NKS * CH_BYTES;
constexpr unsigned WQ_BYTES = NKS * 2u * CH_BYTES;
constexpr unsigned OFF_WST = 32u * ABLK_BYTES;
constexpr unsigned PAR_BYTES = OFF_WST + 8u * WQ_BYTES;
constexpr unsigned WS_NEED = OFF_DATA + 2u * PAR_BYTES;

constexpr int NSLOT = 16;
constexpr int NLOAD = 4;
constexpr int BD = 8;
constexpr unsigned L_ST = NSLOT * 8192u;
constexpr unsigned ST_ROW = 272;
constexpr unsigned L_REC = L_ST + 64 * ST_ROW;
constexpr unsigned L_CTL = L_REC + 2048 + 64;
constexpr unsigned C_FREE = L_CTL;
constexpr unsigned C_ABORT = L_CTL + 16;
constexpr unsigned C_CNT = L_CTL + 20;
constexpr unsigned C_GO = L_CTL + 24;
constexpr unsigned C_ROLE = L_CTL + 64;
constexpr unsigned LDS_BYTES = L_CTL + 80;
static_assert(LDS_BYTES <= 160u * 1024u && (L_REC & 15u) == 0u, "LDS");
constexpr int NTHREADS = (NLOAD + 4) * 64;

constexpr unsigned long long DEADLINE = 30000000ull;

struct Params {
    const void* x;
    const void* w;
    char* ws[NR];
    char* out[NR];
    int rank;
    unsigned xpeer;
    unsigned tile[NWG];
};

__device__ __forceinline__ unsigned long long rt() { return __builtin_amdgcn_s_memrealtime(); }
__device__ __forceinline__ bool expired(unsigned long long& t0, unsigned long long limit) {
    const unsigned long long now = __builtin_amdgcn_s_memrealtime() | 1ull;
    if (t0 == 0ull) { t0 = now; return false; }
    return now - t0 > limit;
}
__device__ __forceinline__ unsigned rfl(unsigned v) { return (unsigned)__builtin_amdgcn_readfirstlane((int)v); }
__device__ __forceinline__ __amdgpu_buffer_rsrc_t mk_rsrc(const void* p, unsigned bytes) {
    return __builtin_amdgcn_make_buffer_rsrc((void*)p, (short)0, (int)bytes, 0x00020000);
}
#define VM_WAIT_(n) asm volatile("s_waitcnt vmcnt(" #n ")" ::: "memory")
#define VM_WAIT(n) VM_WAIT_(n)
__device__ __forceinline__ void lds_st(unsigned addr, unsigned v) {
    asm volatile("ds_write_b32 %0, %1" :: "v"(addr), "v"(v) : "memory");
}
__device__ __forceinline__ unsigned lds_ld(unsigned addr) {
    unsigned v;
    asm volatile("ds_read_b32 %0, %1\n\ts_waitcnt lgkmcnt(0)" : "=&v"(v) : "v"(addr) : "memory");
    return rfl(v);
}
__device__ __forceinline__ u32x4 lds_ld4(unsigned addr) {
    u32x4 v;
    asm volatile("ds_read_b128 %0, %1\n\ts_waitcnt lgkmcnt(0)" : "=&v"(v) : "v"(addr) : "memory");
    return v;
}
__device__ __forceinline__ unsigned ld_agent(const char* p) {
    return __hip_atomic_load((const GAS unsigned*)p, __ATOMIC_RELAXED, __HIP_MEMORY_SCOPE_AGENT);
}
__device__ __forceinline__ void st_agent(char* p, unsigned v) {
    __hip_atomic_store((GAS unsigned*)p, v, __ATOMIC_RELAXED, __HIP_MEMORY_SCOPE_AGENT);
}
__device__ __forceinline__ void st_sys(char* p, unsigned v) {
    __hip_atomic_store((GAS unsigned*)p, v, __ATOMIC_RELAXED, __HIP_MEMORY_SCOPE_SYSTEM);
}
#define DMA_P(V) "buffer_load_dwordx4 " V ", %5, %1 offen sc0 sc1 lds\n\ts_add_u32 m0, m0, 0x400\n\ts_add_u32 %1, %1, %7\n\t"
#define DMA_C(V) "buffer_load_dwordx4 " V ", %5, %1 offen lds\n\ts_add_u32 m0, m0, 0x400\n\ts_add_u32 %1, %1, %7\n\t"
#define DMA_HEAD "s_nop 4\n\ts_mov_b32 %0, m0\n\ts_mov_b32 m0, %2\n\ts_mov_b32 %1, %6\n\t"
#define DMA_TAIL "s_mov_b32 m0, %0"
#define DMA8(PA, PB, PC, PD) \
    asm volatile(DMA_HEAD DMA_P(PA) DMA_P(PB) DMA_P(PC) DMA_P(PD) DMA_P(PA) DMA_P(PB) DMA_P(PC) DMA_P(PD) DMA_TAIL \
                 : "=&s"(keep), "=&s"(t) : "s"(lds), "v"(v0), "v"(v1), "s"(rs), "s"(soff), "s"(js) : "scc", "memory")
__device__ __forceinline__ void dma8(unsigned lds, unsigned v0, unsigned v1, __amdgpu_buffer_rsrc_t rs,
                                     unsigned soff, unsigned js) {
    unsigned keep, t;
    DMA8("%3", "%4", "%3", "%4");
}

__device__ __forceinline__ void dma8c(unsigned lds, unsigned v0, unsigned v1, __amdgpu_buffer_rsrc_t rs,
                                      unsigned soff, unsigned js) {
    unsigned keep, t;
    asm volatile(DMA_HEAD DMA_C("%3") DMA_C("%4") DMA_C("%3") DMA_C("%4") DMA_C("%3") DMA_C("%4") DMA_C("%3") DMA_C("%4") DMA_TAIL
                 : "=&s"(keep), "=&s"(t) : "s"(lds), "v"(v0), "v"(v1), "s"(rs), "s"(soff), "s"(js) : "scc", "memory");
}

#define STAMP(i) do {} while (0)

#define FDECL do {} while (0)
#define FACC(i) do {} while (0)
#define FOUT() do {} while (0)
__device__ __forceinline__ unsigned x_lane_off(int l) { return (unsigned)((l >> 3) * 8192 + (((l & 7) ^ (l >> 4)) << 4)); }

__device__ __forceinline__ void w_fetch(unsigned lds, __amdgpu_buffer_rsrc_t rw, unsigned q, unsigned t, int lane) {
    const unsigned v = (unsigned)((lane >> 3) * 1024 + (lane & 7) * 16);
    dma8c(lds, v, v, rw, (t >> 1) * 65536u + q * 256u + (t & 1u) * 128u, 8192u);
}
__device__ __forceinline__ void w_frags(u32x4 (&r)[8], unsigned lds, int lane) {
    const unsigned wr = lds + (unsigned)((8 * (lane >> 5) + ((lane >> 2) & 3)) * 128 + ((lane >> 4) & 1) * 32 + (lane & 3) * 8);
#pragma unroll
    for (int j = 0; j < 8; ++j) {
        const unsigned ad = wr + (unsigned)((j & 3) * 2048 + (j >> 2) * 64);
        const u32x2 k0 = __builtin_bit_cast(u32x2, __builtin_amdgcn_ds_read_tr16_b64_v4i16((LAS s16x4*)(size_t)ad));
        const u32x2 k1 = __builtin_bit_cast(u32x2, __builtin_amdgcn_ds_read_tr16_b64_v4i16((LAS s16x4*)(size_t)(ad + 512u)));
        r[j] = __builtin_shufflevector(k0, k1, 0, 1, 2, 3);
    }
}

__device__ __forceinline__ unsigned bfi(unsigned m, unsigned a, unsigned b) { return (a & m) | (b & ~m); }
__device__ __forceinline__ unsigned pkmax(unsigned a, unsigned b) {
    typedef unsigned short u16x2 __attribute__((ext_vector_type(2)));
    return __builtin_bit_cast(unsigned, __builtin_elementwise_max(__builtin_bit_cast(u16x2, a), __builtin_bit_cast(u16x2, b)));
}
struct Packed { u32x4 lo[4]; u32x4 cd[2]; u32x2 hi[8]; unsigned fmt; };
__device__ __forceinline__ void pack_split(const u32x4 (&r)[8], Packed& o) {
#pragma unroll
    for (int j = 0; j < 8; ++j) {
        o.lo[j >> 1][(j & 1) * 2] = __builtin_amdgcn_perm(r[j].y, r[j].x, 0x06040200u);
        o.lo[j >> 1][(j & 1) * 2 + 1] = __builtin_amdgcn_perm(r[j].w, r[j].z, 0x06040200u);
        o.hi[j].x = __builtin_amdgcn_perm(r[j].y, r[j].x, 0x07050301u);
        o.hi[j].y = __builtin_amdgcn_perm(r[j].w, r[j].z, 0x07050301u);
        asm volatile("" : "+v"(o.hi[j]));
        if (j & 1) asm volatile("" : "+v"(o.lo[j >> 1]));
    }
}
__device__ __forceinline__ void pack_codes(Packed& o) {
    unsigned m = 0;
#pragma unroll
    for (int j = 0; j < 8; ++j) {
        const unsigned e0 = o.hi[j].x & 0x7F7F7F7Fu, e1 = o.hi[j].y & 0x7F7F7F7Fu;
        m = pkmax(pkmax(m, e0), pkmax(e0 << 8, e1));
        m = pkmax(m, e1 << 8);
    }
    m = max(m >> 24, (m >> 8) & 0xFFu);
    m = max(m, (unsigned)__builtin_amdgcn_update_dpp(0, (int)m, 0x111, 0xF, 0xF, true));
    m = max(m, (unsigned)__builtin_amdgcn_update_dpp(0, (int)m, 0x112, 0xF, 0xF, true));
    m = max(m, (unsigned)__builtin_amdgcn_update_dpp(0, (int)m, 0x114, 0xF, 0xF, true));
    m = max(m, (unsigned)__builtin_amdgcn_update_dpp(0, (int)m, 0x118, 0xF, 0xF, true));
    const unsigned top = max(max((unsigned)__builtin_amdgcn_readlane((int)m, 15), (unsigned)__builtin_amdgcn_readlane((int)m, 31)),
                             max((unsigned)__builtin_amdgcn_readlane((int)m, 47), (unsigned)__builtin_amdgcn_readlane((int)m, 63)));
    const unsigned base = rfl(top > 7u ? top - 7u : 0u);
    const unsigned kk = 0x80808080u - base * 0x01010101u;
    unsigned all = 0x80808080u;
#pragma unroll
    for (int j = 0; j < 8; ++j) {
        const unsigned t0 = (o.hi[j].x & 0x7F7F7F7Fu) + kk, t1 = (o.hi[j].y & 0x7F7F7F7Fu) + kk;
        all &= t0 & t1;
        const unsigned c1 = bfi(0x07070707u, t1, o.hi[j].y);
        o.cd[j >> 2][j & 3] = bfi(0x87878787u, bfi(0x07070707u, t0, o.hi[j].x), __builtin_amdgcn_alignbit(c1, c1, 28));
    }
    asm volatile("" : "+v"(o.cd[0]), "+v"(o.cd[1]));
    o.fmt = base;
    const unsigned long long lanes = __ballot(all != 0x80808080u);
    if (lanes == 0ull) return;
    unsigned mask = 0, one = 0, c0 = 0, c1 = 0, cj = 0;
    const unsigned long long l1 = lanes & (0ull - lanes);
#pragma unroll
    for (int j = 0; j < 8; ++j) {
        const unsigned t0 = (o.hi[j].x & 0x7F7F7F7Fu) + kk, t1 = (o.hi[j].y & 0x7F7F7F7Fu) + kk;
        const bool out = (t0 & t1 & 0x80808080u) != 0x80808080u;
        const unsigned long long ob = __ballot(out);
        mask |= ob != 0ull ? 1u << j : 0u;
        one |= ob == l1 ? 1u << j : 0u;
        c0 = out ? t0 : c0; c1 = out ? t1 : c1; cj = out ? (unsigned)j : cj;
    }
    const int l = __builtin_ctzll(lanes);
    const unsigned j = (unsigned)__builtin_amdgcn_readlane((int)cj, l);
    const unsigned long long t = (unsigned long long)(unsigned)__builtin_amdgcn_readlane((int)c0, l) |
                                 ((unsigned long long)(unsigned)__builtin_amdgcn_readlane((int)c1, l) << 32);
    const unsigned long long bad = ~t & 0x8080808080808080ull;
    unsigned fix = 0;
    if (((one >> j) & 1u) && (bad & (bad - 1ull)) == 0ull) {
        const unsigned by = (unsigned)__builtin_ctzll(bad) >> 3;
        const unsigned tb = (unsigned)(t >> (by * 8u)) & 0xFFu;
        fix = (16u - (tb >> 3)) | (j << 4) | (by << 7) | ((unsigned)l << 10);
        mask &= ~(1u << j);
    }
    if (__builtin_popcount(mask) > 4) { mask = 0xFFu; fix = 0u; }
    o.fmt = base | (mask << 8) | (fix << 16);
}
__device__ __forceinline__ void send_low(const Packed& k, __amdgpu_buffer_rsrc_t rd, unsigned dof, int lane) {
#pragma unroll
    for (int i = 0; i < 4; ++i) __builtin_amdgcn_raw_buffer_store_b128(k.lo[i], rd, lane * 16, dof + i * 1024u, 17);
}
__device__ __forceinline__ void send_high(const Packed& k, __amdgpu_buffer_rsrc_t rd, unsigned dof, int lane) {
    if ((k.fmt & 0xFF00u) != 0xFF00u) {
        __builtin_amdgcn_raw_buffer_store_b128(k.cd[0], rd, lane * 16, dof + 4096u, 17);
        __builtin_amdgcn_raw_buffer_store_b128(k.cd[1], rd, lane * 16, dof + 5120u, 17);
    }
    if (__builtin_expect((k.fmt & 0xFF00u) != 0u, 0)) {
#pragma unroll
        for (int j = 0; j < 8; ++j)
            if ((k.fmt >> (8 + j)) & 1u) __builtin_amdgcn_raw_buffer_store_b64(k.hi[j], rd, lane * 8, dof + 6144u + j * 512u, 17);
    }
}
__device__ __forceinline__ void send_chunk(const Packed& k, __amdgpu_buffer_rsrc_t rd, unsigned dof, int lane) {
    send_low(k, rd, dof, lane);
    send_high(k, rd, dof, lane);
}
struct Wire { u32x4 lo[4]; u32x4 cd[2]; u32x2 e0; };
__device__ __forceinline__ void wire_load(Wire& w, const char* ws, bool go, unsigned off, unsigned fmt, int lane) {
    const __amdgpu_buffer_rsrc_t rs = mk_rsrc(ws, go ? WS_NEED : 0u);
#pragma unroll
    for (int i = 0; i < 4; ++i) w.lo[i] = __builtin_amdgcn_raw_buffer_load_b128(rs, lane * 16, off + i * 1024u, AG_XAUX);
#pragma unroll
    for (int i = 0; i < 2; ++i) w.cd[i] = __builtin_amdgcn_raw_buffer_load_b128(rs, lane * 16, off + 4096u + i * 1024u, AG_XAUX);
    const unsigned mask = (fmt >> 8) & 0xFFu, j0 = (unsigned)__builtin_ctz(mask | 0x80u);
    w.e0 = __builtin_amdgcn_raw_buffer_load_b64(mk_rsrc(ws, go && mask != 0u ? WS_NEED : 0u), lane * 8, off + 6144u + j0 * 512u, 17);
}
__device__ __forceinline__ void wire_put(unsigned ha, unsigned hb, unsigned l0, unsigned l1, unsigned lds) {
    u32x4 v;
    v.x = __builtin_amdgcn_perm(ha, l0, 0x05010400u);
    v.y = __builtin_amdgcn_perm(ha, l0, 0x07030602u);
    v.z = __builtin_amdgcn_perm(hb, l1, 0x05010400u);
    v.w = __builtin_amdgcn_perm(hb, l1, 0x07030602u);
    *(volatile LAS u32x4*)(size_t)lds = v;
}
__device__ __forceinline__ void wire_unpack(const Wire& w, const char* ws, unsigned off, unsigned fmt, int lane, unsigned lds) {
    const unsigned br = (fmt & 0x7Fu) * 0x01010101u, mask = (fmt >> 8) & 0xFFu;
    const unsigned la = lds + (unsigned)lane * 16u;
#pragma unroll
    for (int j = 0; j < 8; ++j) {
        const unsigned n = w.cd[j >> 2][j & 3];
        wire_put((n & 0x87878787u) + br, (__builtin_amdgcn_alignbit(n, n, 4) & 0x87878787u) + br,
                 w.lo[j >> 1][(j & 1) * 2], w.lo[j >> 1][(j & 1) * 2 + 1], la + j * 1024u);
    }
    if ((fmt & 0xF0000u) != 0u) {
        const unsigned by = (fmt >> 23) & 7u;
        const unsigned ad = lds + ((fmt >> 20) & 7u) * 1024u + (fmt >> 26) * 16u + (by >> 1) * 4u;
        const unsigned d = ((fmt >> 13) & 0x78u) << ((by & 1u) * 16u + 8u);
        if (lane == 0) asm volatile("ds_sub_u32 %0, %1" :: "v"(ad), "v"(d) : "memory");
    }
    if (mask != 0u) {
        const unsigned j0 = (unsigned)__builtin_ctz(mask);
        const __amdgpu_buffer_rsrc_t rs = mk_rsrc(ws, WS_NEED);
#pragma unroll
        for (int j = 0; j < 8; ++j) {
            if ((mask >> j) & 1u) {
                u32x2 e = w.e0;
                if ((unsigned)j != j0)
                    asm volatile("buffer_load_dwordx2 %0, %1, %2, %3 offen sc0 sc1\n\ts_waitcnt vmcnt(0)"
                                 : "=&v"(e) : "v"(lane * 8), "s"(rs), "s"(off + 6144u + j * 512u) : "memory");
                wire_put(e.x, e.y, w.lo[j >> 1][(j & 1) * 2], w.lo[j >> 1][(j & 1) * 2 + 1], la + j * 1024u);
            }
        }
    }
}
__device__ __forceinline__ unsigned flag_pos(unsigned ks) { return (ks & 3u) * 16u + (ks >> 2); }

struct Tile { int rb, cb, src, blk; };
__device__ __forceinline__ Tile tile_of(unsigned code, int rank) {
    const int gidx = rank & 3, gbase = rank & 4, obase = gbase ^ 4;
    Tile t;
    t.rb = (int)(code & 63u); t.cb = (int)((code >> 6) & 7u);
    if (t.rb < 4 * NALPHA) { t.src = gbase + t.rb / NALPHA; t.blk = t.rb % NALPHA; }
    else if (t.rb < 32) { t.src = obase + (t.rb - 4 * NALPHA) / NBETA; t.blk = NALPHA + (t.rb - 4 * NALPHA) % NBETA; }
    else { t.src = rank; t.blk = NALPHA + (t.rb - 32); }
    return t;
}

struct Frag { s16x8 a[4][2]; };
struct PackB { u32x4 lo[2], cd; };
template <int N> struct IC { static constexpr int value = N; };

__global__ void __launch_bounds__(NTHREADS, 2) ag_gemm_kernel(const Params p) {
    __shared__ __attribute__((aligned(1024))) unsigned char lds_raw[LDS_BYTES];
    LAS unsigned char* L = (LAS unsigned char*)lds_raw;
    const int lane = threadIdx.x & 63;
    const int wave = (int)rfl(threadIdx.x >> 6);
    const int rank = p.rank;
    const int gidx = rank & 3, gbase = rank & 4, obase = gbase ^ 4;
    char* myws = p.ws[rank];
    if ((unsigned)(size_t)L != 0u) return;

    unsigned bid = (blockIdx.x & ~7u) | ((unsigned)__builtin_amdgcn_s_getreg(20 | (3 << 11)) & 7u);
    unsigned call = 0, own;
    int peer;
    bool pusher, packs;
    for (;;) {
        peer = (int)((p.xpeer >> ((bid & 7u) * 4u)) & 15u);
        pusher = bid < 8u * CPL && peer < NR;
        own = (bid - 64u) * 4u + (unsigned)wave - (unsigned)NLOAD;
        packs = bid >= 64u && wave >= NLOAD && own < 512u;
        if (pusher && wave == 0) STAMP(0);
        if (!pusher) __builtin_amdgcn_s_sleep(AG_TDELAY);
        if (packs || (pusher && wave < WPP)) {
            const unsigned t = packs ? own >> 2 : (unsigned)wave * CPL + (bid >> 3), lds = (unsigned)(packs ? 8 + wave : wave) * 8192u;
            if (pusher && (peer & 4) == gbase)
                dma8c(lds, x_lane_off(lane), x_lane_off(lane) ^ 64u, mk_rsrc(p.x, SHARD_BYTES), (t & 1u) * (64u * 8192u) + (t >> 1) * 128u, 65536u);
            else
                w_fetch(lds, mk_rsrc(p.w, SHARD_BYTES), (packs ? own : (unsigned)peer) & 3u, t, lane);
        }
        if (call != 0u) break;
        if (threadIdx.x < (2048 + 128) / 8) *(LAS unsigned long long*)(L + L_REC + threadIdx.x * 8) = 0ull;
        if (wave == 3) {
            unsigned v = 0, role = bid;
            if (lane < 2)
                v = __hip_atomic_fetch_add((GAS unsigned*)(myws + OFF_CTR + (lane ? bid * 128u + 64u : blockIdx.x * 128u)), 1u,
                                           __ATOMIC_RELAXED, __HIP_MEMORY_SCOPE_AGENT);
            const unsigned c = rfl(v);
            const bool got = (unsigned)__builtin_amdgcn_readlane((int)v, 1) == c;
            if (got) { const u32x2 cr = {c + 1u, role}; *(volatile LAS u32x2*)(size_t)C_ROLE = cr; }
            if (lane == 0)
                __hip_atomic_fetch_add((GAS unsigned*)(myws + OFF_ARR + (blockIdx.x & 7u) * 128u), 1u, __ATOMIC_RELAXED, __HIP_MEMORY_SCOPE_AGENT);
            if (__builtin_expect(!got, 0)) {
                if (lane == 0)
                    __hip_atomic_fetch_sub((GAS unsigned*)(myws + OFF_CTR + bid * 128u + 64u), 1u, __ATOMIC_RELAXED, __HIP_MEMORY_SCOPE_AGENT);
                unsigned long long t0 = 0;
                role = 0;
                while (!__all(ld_agent(myws + OFF_ARR + (unsigned)(lane & 7) * 128u) >= (c + 1u) * (unsigned)(NWG / 8))) {
                    if (expired(t0, DEADLINE)) { role = NWG; break; }
                    __builtin_amdgcn_s_sleep(8);
                }
                for (; role < (unsigned)NWG; ++role) {
                    unsigned e = c;
                    if (lane == 0)
                        __hip_atomic_compare_exchange_strong((GAS unsigned*)(myws + OFF_CTR + role * 128u + 64u), &e, c + 1u,
                                                             __ATOMIC_RELAXED, __ATOMIC_RELAXED, __HIP_MEMORY_SCOPE_AGENT);
                    if (rfl(e) == c) break;
                }
                if (role >= (unsigned)NWG && lane == 0) st_agent(myws + OFF_TMO, c + 1u);
                const u32x2 cr = {c + 1u, role};
                *(volatile LAS u32x2*)(size_t)C_ROLE = cr;
            }
        }
        asm volatile("s_waitcnt lgkmcnt(0)" ::: "memory");
        __builtin_amdgcn_s_barrier();
        call = lds_ld(C_ROLE);
        const unsigned role = lds_ld(C_ROLE + 4u);
        if (__builtin_expect(role == bid, 1)) break;
        VM_WAIT(0);
        if (role >= (unsigned)NWG) return;
        bid = role;
    }

    if (pusher) {
        if (wave < WPP) {
            const int pi = wave * CPL + (int)(bid >> 3);
            const bool same = (peer & 4) == gbase;
            const __amdgpu_buffer_rsrc_t rx = mk_rsrc(p.x, SHARD_BYTES), rw = mk_rsrc(p.w, SHARD_BYTES);
            char* pws = p.ws[peer];
            const unsigned xa0 = x_lane_off(lane), xa1 = xa0 ^ 64u;
            const unsigned wl = (unsigned)wave * 8192u;

            auto load_task = [&](int t) INL {
                if (!same && t < 128) {
                    w_fetch(wl, rw, (unsigned)(peer & 3), (unsigned)t, lane);
                } else {
                    int ks, blk;
                    if (t < 128) { ks = t >> 1; blk = t & 1; }
                    else { const int u = t - 128; ks = u / 3; blk = (same ? NEARLY : NALPHA) + (u - ks * 3); }
                    dma8c(wl, xa0, xa1, rx, (unsigned)blk * (64u * 8192u) + (unsigned)ks * 128u, 65536u);
                }
            };
            auto split_task = [&](int t, Packed& k) INL {
                u32x4 ra[8];
                if (!same && t < 128) w_frags(ra, wl, lane);
                else {
#pragma unroll
                    for (int j = 0; j < 8; ++j) ra[j] = *(const volatile LAS u32x4*)(size_t)(wl + (unsigned)lane * 16u + j * 1024u);
                }
                pack_split(ra, k);
            };
            char* qp = myws + OFF_Q + (unsigned)peer * 128u;
            unsigned tk = 0;
            const __amdgpu_buffer_rsrc_t rd = mk_rsrc(pws, WS_NEED);
            auto draw = [&]() INL {
                unsigned long long sv;
                asm volatile("s_mov_b64 %1, exec\n\ts_mov_b64 exec, 1\n\tglobal_atomic_add %0, %2, %3, off sc0\n\ts_mov_b64 exec, %1"
                             : "=&v"(tk), "=&s"(sv) : "v"(qp), "v"(1u) : "memory");
            };
            unsigned dof, fof;
            auto place = [&](int t) INL {
                if (!same && t < 128) {
                    dof = OFF_WST + (unsigned)(4 + gidx) * WQ_BYTES + (unsigned)t * CH_BYTES;
                    fof = OFF_FW + (unsigned)((4 + gidx) * 128 + t) * 8u;
                } else {
                    int ks, ai;
                    if (t < 128) { ks = t >> 1; ai = gidx * NALPHA + (t & 1); }
                    else {
                        const int u = t - 128; ks = u / 3;
                        const int r = u - ks * 3;
                        ai = same ? gidx * NALPHA + NEARLY + r : 4 * NALPHA + gidx * NBETA + r;
                    }
                    dof = (unsigned)ai * ABLK_BYTES + (unsigned)ks * CH_BYTES;
                    fof = OFF_FA + ((unsigned)ai * 64u + flag_pos((unsigned)ks)) * 8u;
                }
            };
            unsigned spins = 0, fmt = 0, fpv = 0;
            bool cast = false;
            unsigned long long t0 = 0;
            for (int t = pi, nx = pi + NPW;;) {
                place(t);
                asm volatile("s_waitcnt vmcnt(0)" : "+v"(tk) :: "memory");
                __builtin_amdgcn_s_waitcnt(0x0F70);
                asm volatile("" ::: "memory");
                if (t != pi) {
                    if (lane == 0)
                        __hip_atomic_store((GAS unsigned long long*)(pws + fpv), (unsigned long long)call | ((unsigned long long)fmt << 32),
                                           __ATOMIC_RELAXED, __HIP_MEMORY_SCOPE_SYSTEM);
                    if (cast && lane == 0)
                        __hip_atomic_store((GAS unsigned long long*)(myws + OFF_GO + (unsigned)peer * 128u), ((rt() + AG_DELTA) << 32) | call,
                                           __ATOMIC_RELAXED, __HIP_MEMORY_SCOPE_AGENT);
                    if (t >= NTASK) break;
                    nx = (int)min(rfl(tk) - (call - 1u) * (unsigned)(NTASK - NPW) + 2u * (unsigned)NPW, (unsigned)NTASK);
                }
                dof += OFF_DATA + (call & 1u) * PAR_BYTES;
                fpv = fof + (call & 1u) * FLG_PAR;
                Packed k;
                split_task(t, k);
                const bool eager = t < AG_EAGERW * CPL;
                if (eager) send_low(k, rd, dof, lane);
                if (nx < NTASK) { load_task(nx); draw(); }
                pack_codes(k);
                fmt = k.fmt;
                unsigned my = 0;
                if (!eager) {
                    if (t == pi) __builtin_amdgcn_s_sleep(AG_W2D);
                    asm volatile("ds_add_rtn_u32 %0, %1, %2\n\ts_waitcnt lgkmcnt(0)" : "=&v"(my) : "v"(C_CNT), "v"(lane == 0 ? 1u : 0u) : "memory");
                    my = rfl(my);
                    while (lds_ld(C_GO) < my) {
                        if ((++spins & 1023u) == 0u && expired(t0, DEADLINE)) break;
                        __builtin_amdgcn_s_sleep(1);
                    }
                    send_low(k, rd, dof, lane);
                }
                send_high(k, rd, dof, lane);
                if (!eager) {
                    VM_WAIT(AG_REL);
                    lds_st(C_GO, my + 1u);
                }
                cast = t == NTASK - AG_GOT;
                t = (int)rfl((unsigned)nx);
            }
            return;
        }
        if (wave != 7 || bid != 0u) return;
        const int i0 = (lane * 2) & 31, ex = 4 * NALPHA + gidx * NBETA;
        const bool skip0 = i0 >= ex && i0 < ex + NBETA, skip1 = i0 + 1 >= ex && i0 + 1 < ex + NBETA;
        const char* fp = myws + OFF_DONE + (unsigned)lane * 8u;
        unsigned long long t0 = 0;
        unsigned spins = 0;
        bool dead = false;
        for (;;) {
            u32x2 v;
            asm volatile("global_load_dwordx2 %0, %1, off sc0 sc1\n\ts_waitcnt vmcnt(0)" : "=&v"(v) : "v"(fp) : "memory");
            const bool ok = (skip0 || v.x >= call) && (skip1 || v.y >= call);
            if (__all(ok)) break;
            if ((++spins & 255u) == 0u) {
                if (rfl(ld_agent(myws + OFF_TMO)) == call || expired(t0, 2 * DEADLINE)) { dead = true; break; }
            }
            __builtin_amdgcn_s_sleep(2);
        }
        __builtin_amdgcn_fence(__ATOMIC_ACQUIRE, "");
        if (dead) {
            if (lane == 0) st_agent(myws + OFF_TMO, call);
            const unsigned long long t1 = rt();
            while (rt() - t1 < DEADLINE / 8) __builtin_amdgcn_s_sleep(64);
            if (lane == 0) st_sys(p.out[rank] + (size_t)rank * 512u * 1024u, 0x7FC07FC0u);
        }
        STAMP(6);
        return;
    }

    const unsigned tcode = p.tile[bid];
    const unsigned ntile = (tcode >> 31) ? 2u : 1u;
    const unsigned par = OFF_DATA + (call & 1u) * PAR_BYTES;

    if (wave < NLOAD) {
        const unsigned lw = (unsigned)wave;
        const __amdgpu_buffer_rsrc_t rws = mk_rsrc(myws, WS_NEED);
        const unsigned fpar = (call & 1u) * FLG_PAR;
        unsigned spins = 0;
        unsigned long long t0 = 0;
        bool dead = false;
        FDECL;
        auto give_up = [&]() INL -> bool {
            return (++spins & 255u) == 0u &&
                   (rfl(ld_agent(myws + OFF_TMO)) == call || lds_ld(C_ABORT) != 0u || expired(t0, DEADLINE));
        };
        auto wait_slot = [&](unsigned g) INL {
            while (g >= (unsigned)NSLOT - 4u) {
                const u32x4 fr = lds_ld4(C_FREE);
                if (rfl(min(min(fr.x, fr.y), min(fr.z, fr.w))) + ((unsigned)NSLOT - 4u) > g) break;
                if ((++spins & 1023u) == 0u && (lds_ld(C_ABORT) != 0u || expired(t0, 2 * DEADLINE))) { dead = true; break; }
                __builtin_amdgcn_s_sleep(AG_FREE_SLEEP);
            }
        };
        for (unsigned ti = 0; ti < ntile && !dead; ++ti) {
            const Tile t = tile_of(tcode >> (16u * ti), rank);
            const unsigned g0 = ti * (unsigned)NKS + lw;
            unsigned wn = 0;
            auto publish = [&](unsigned i) INL {
                const unsigned need = min(i * 4u + lw + (unsigned)BD + 1u, (unsigned)NKS);
                while (wn < need) {
                    const u32x4 v = __builtin_amdgcn_raw_buffer_load_b128(rws, fpar + OFF_FW + (unsigned)t.cb * 1024u + (unsigned)lane * 16u, 0, 17);
                    const bool up = v.x >= call && v.z >= call;
                    wn = (unsigned)__builtin_ctzll(~__ballot(up) | (1ull << 63)) + (__all(up) ? 1u : 0u);
                    u32x4 f;
                    f.x = (v.y & 0x0FFFu) | ((v.y >> 22) & 1u ? 0u : v.y & 0xFFFF0000u);
                    f.y = (v.y & 0x00FFu) | ((v.y >> 4) & 0x0F00u) | ((v.y >> 22) & 1u ? v.y & 0xFFFF0000u : 0u);
                    f.z = (v.w & 0x0FFFu) | ((v.w >> 22) & 1u ? 0u : v.w & 0xFFFF0000u);
                    f.w = (v.w & 0x00FFu) | ((v.w >> 4) & 0x0F00u) | ((v.w >> 22) & 1u ? v.w & 0xFFFF0000u : 0u);
                    if (up) {
                        const u32x2 f0 = {f.x, f.y}, f1 = {f.z, f.w};
                        asm volatile("ds_or_b64 %0, %1\n\tds_or_b64 %0, %2 offset:8" :: "v"(L_REC + ti * 1024u + (unsigned)lane * 16u), "v"(f0), "v"(f1) : "memory");
                    }
                    if (wn >= need) break;
                    if (give_up()) { dead = true; return; }
                    __builtin_amdgcn_s_sleep(AG_POLL_SLEEP);
                }
                asm volatile("s_waitcnt lgkmcnt(0)" ::: "memory");
                if (lane < 4) asm volatile("ds_or_b32 %0, %1" :: "v"(L_REC + (g0 + i * 4u) * 16u + (unsigned)lane * 4u), "v"(0x80u) : "memory");
            };
            if (ti == ntile - 1u && wave == 0) STAMP(2);
            if (t.src != rank) {
                const unsigned base = par + (unsigned)t.rb * ABLK_BYTES + lw * CH_BYTES;
                const unsigned fl0 = fpar + OFF_FA + (unsigned)t.rb * 512u + lw * 128u;
                u32x2 fl = {0u, 0u};
                unsigned up = 0u;
                auto refresh = [&]() INL {
                    if (lane < 16) fl = __builtin_amdgcn_raw_buffer_load_b64(rws, fl0 + (unsigned)lane * 8u, 0, 17);
                    up = (unsigned)__ballot(fl.x >= call);
                    asm volatile("" ::: "memory");
                };
                auto await = [&](unsigned need) INL {
                    for (;;) {
                        refresh();
                        if ((up & need) == need) return;
                        if (give_up()) { dead = true; return; }
                        __builtin_amdgcn_s_sleep(AG_POLL_SLEEP);
                    }
                };
                auto fetch = [&](Wire& w, unsigned& f, unsigned i, bool go) INL {
                    f = (unsigned)__builtin_amdgcn_readlane((int)fl.y, (int)(i & 15u));
                    wire_load(w, myws, go, base + i * (4u * CH_BYTES), f, lane);
                };
                auto turn = [&](const Wire& w, unsigned f, Wire& wx, unsigned& fx, unsigned i) INL {
                    const bool last = i == 15u;
                    const unsigned need = last ? 0u : 2u << i;
                    if ((up & need) != need) refresh();
                    FACC(0);
                    const bool ok = (up & need) == need;
                    fetch(wx, fx, i + 1u, ok && !last);
                    FACC(1);
                    wait_slot(g0 + i * 4u);
                    FACC(2);
                    if (dead) return;
                    wire_unpack(w, myws, base + i * (4u * CH_BYTES), f, lane, ((g0 + i * 4u) & (unsigned)(NSLOT - 1)) * 8192u);
                    FACC(4);
                    publish(i);
                    FACC(3);
                    if (!ok && !dead) {
                        await(need);
                        fetch(wx, fx, i + 1u, !dead);
                    }
                    FACC(5);
                };
                Wire w0, w1;
                unsigned f0, f1;
                await(1u);
                fetch(w0, f0, 0u, !dead);
                FACC(5);
                for (unsigned i = 0; i < 16u && !dead; i += 2u) {
                    turn(w0, f0, w1, f1, i);
                    if (dead) break;
                    turn(w1, f1, w0, f0, i + 1u);
                }
            } else {
                const __amdgpu_buffer_rsrc_t rx = mk_rsrc(p.x, SHARD_BYTES);
                const unsigned v0 = x_lane_off(lane), v1 = v0 ^ 64u;
                const unsigned base = (unsigned)t.blk * (64u * 8192u) + lw * 128u;
                for (unsigned i = 0; i < 18u && !dead; ++i) {
                    if (i < 16u) {
                        wait_slot(g0 + i * 4u);
                        if (dead) break;

                        dma8(((g0 + i * 4u) & (unsigned)(NSLOT - 1)) * 8192u, v0, v1, rx, base + i * 512u, 65536u);
                        VM_WAIT(16);
                    } else if (i == 16u) {
                        VM_WAIT(8);
                    } else {
                        VM_WAIT(0);
                    }
                    if (i >= 2u) publish(i - 2u);
                }
                VM_WAIT(0);
            }
        }
        FOUT();
        if (dead) {
            lds_st(C_ABORT, 1u);
            if (lane == 0) st_agent(myws + OFF_TMO, call);
        }
        if (wave == 0) {
            const int cb = tile_of(tcode >> (16u * (ntile - 1u)), rank).cb;
            if (cb >= 4) {
                char* hp = myws + OFF_GO + (unsigned)(obase + (cb & 3)) * 128u;
                unsigned long long go;
                t0 = 0;
                for (;;) {
                    go = __hip_atomic_load((const GAS unsigned long long*)hp, __ATOMIC_RELAXED, __HIP_MEMORY_SCOPE_AGENT);
                    if (dead || __all((unsigned)go == call)) break;
                    if ((++spins & 63u) == 0u && expired(t0, DEADLINE)) break;
                    __builtin_amdgcn_s_sleep(AG_GO_SLEEP);
                }
                const unsigned long long now = rt();
                const unsigned ahead = (unsigned)(go >> 32) - (unsigned)now;
                unsigned long long at = 0;
                if (lane == 0) {
                    __hip_atomic_fetch_max((GAS unsigned long long*)(hp + 1024), now + (ahead <= AG_DELTA ? ahead : 0u), __ATOMIC_RELAXED, __HIP_MEMORY_SCOPE_AGENT);
                    at = __hip_atomic_fetch_add((GAS unsigned long long*)(hp + 1024), (unsigned long long)AG_TAU, __ATOMIC_RELAXED, __HIP_MEMORY_SCOPE_AGENT);
                }
                const unsigned wt = rfl(at > now ? (unsigned)min(at - now, 2000ull) : 0u);
                while ((unsigned)(rt() - now) < wt) __builtin_amdgcn_s_sleep(1);
                lds_st(C_GO, 1u);
            }
        }
        if (wave == 3) STAMP(3);
        return;
    }

    const int cw = wave - NLOAD;
    __builtin_amdgcn_s_setprio(AG_PRIO);
    const int hh = lane >> 5, r32 = lane & 31;
    const unsigned a0 = (unsigned)(r32 * 128 + ((hh ^ ((r32 >> 1) & 7)) << 4));
    const unsigned cfree = C_FREE + (unsigned)cw * 4u;
    if (packs) {
        u32x4 r[8];
        Packed k;
        VM_WAIT(0);
        w_frags(r, (unsigned)(8 + wave) * 8192u, lane);
        pack_split(r, k);
        lds_st(cfree, 4u);
        pack_codes(k);
        send_chunk(k, mk_rsrc(myws, WS_NEED), par + OFF_WST + (own & 3u) * WQ_BYTES + (own >> 2) * CH_BYTES, lane);
        VM_WAIT(0);
        if (lane == 0)
            __hip_atomic_store((GAS unsigned long long*)(myws + (call & 1u) * FLG_PAR + OFF_FW + ((own & 3u) * 128u + (own >> 2)) * 8u),
                               (unsigned long long)call | ((unsigned long long)k.fmt << 32), __ATOMIC_RELAXED, __HIP_MEMORY_SCOPE_SYSTEM);
    } else {
        lds_st(cfree, 4u);
    }
    const unsigned vlo = (unsigned)lane * 16u + (unsigned)(cw & 1) * 2048u, vcd = (unsigned)lane * 16u + 4096u + (unsigned)(cw & 1) * 1024u;

    unsigned spins = 0;
    unsigned long long t0 = 0;
    FDECL;
    auto wait_full = [&](unsigned g) INL {
        while ((lds_ld(L_REC + g * 16u) & 0x80u) == 0u) {
            if ((++spins & 63u) == 0u && lds_ld(C_ABORT) != 0u) break;
            if ((spins & 1023u) == 0u && expired(t0, 3 * DEADLINE)) { lds_st(C_ABORT, 1u); break; }
            __builtin_amdgcn_s_sleep(1);
        }
    };
    unsigned ab[4];
#pragma unroll
    for (int k = 0; k < 4; ++k) ab[k] = a0 ^ (unsigned)(k * 32);
    auto rd = [&](Frag& f, auto sc) INL {
        constexpr unsigned S = decltype(sc)::value;
#pragma unroll
        for (int k = 0; k < 4; ++k)
#pragma unroll
            for (int mt = 0; mt < 2; ++mt)
                f.a[k][mt] = *(const LAS s16x8*)((const LAS unsigned char*)(size_t)ab[k] + (S * 8192u + mt * 4096u));
    };
    auto ldb = [&](PackB& b, __amdgpu_buffer_rsrc_t rs, unsigned off) INL {
        b.lo[0] = __builtin_amdgcn_raw_buffer_load_b128(rs, vlo, off, AG_WAUX);
        b.lo[1] = __builtin_amdgcn_raw_buffer_load_b128(rs, vlo + 1024u, off, AG_WAUX);
        b.cd = __builtin_amdgcn_raw_buffer_load_b128(rs, vcd, off, AG_WAUX);
    };
    auto put = [&](u32x16& o, int k, unsigned ha, unsigned hb, const PackB& b) INL {
        const unsigned l0 = b.lo[k >> 1][(k & 1) * 2], l1 = b.lo[k >> 1][(k & 1) * 2 + 1];
        o[4 * k] = __builtin_amdgcn_perm(ha, l0, 0x05010400u);
        o[4 * k + 1] = __builtin_amdgcn_perm(ha, l0, 0x07030602u);
        o[4 * k + 2] = __builtin_amdgcn_perm(hb, l1, 0x05010400u);
        o[4 * k + 3] = __builtin_amdgcn_perm(hb, l1, 0x07030602u);
    };
    auto repair = [&](u32x16& o, const PackB& b, unsigned fmt, unsigned off) INL {
        if ((fmt & 0xF0000u) != 0u) {
            const unsigned by = (fmt >> 23) & 7u;
            o[((fmt >> 18) & 12u) + (by >> 1)] -= (unsigned)lane == (fmt >> 26) ? ((fmt >> 13) & 0x78u) << ((by & 1u) * 16u + 8u) : 0u;
        }
        if ((fmt & 0xF00u) != 0u) {
            const __amdgpu_buffer_rsrc_t rs = mk_rsrc(myws, WS_NEED);
#pragma unroll
            for (int k = 0; k < 4; ++k) {
                if ((fmt >> (8 + k)) & 1u) {
                    u32x2 e;
                    asm volatile("buffer_load_dwordx2 %0, %1, %2, %3 offen sc0 sc1\n\ts_waitcnt vmcnt(0)"
                                 : "=&v"(e) : "v"(lane * 8), "s"(rs), "s"(off + 6144u + (unsigned)((cw & 1) * 4 + k) * 512u) : "memory");
                    put(o, k, e.x, e.y, b);
                }
            }
        }
    };
    f32x16 acc[2];
    unsigned vrec = L_REC + (unsigned)cw * 4u;
    asm volatile("" : "+v"(vrec));
#define SB() __builtin_amdgcn_sched_barrier(0)

    for (unsigned ti = 0; ti < ntile; ++ti) {
#pragma unroll
        for (int i = 0; i < 16; ++i) { acc[0][i] = 0.f; acc[1][i] = 0.f; }
        const Tile t = tile_of(tcode >> (16u * ti), rank);
        const unsigned g0 = ti * (unsigned)NKS;
        const unsigned wbase = par + OFF_WST + (unsigned)t.cb * WQ_BYTES + (unsigned)(cw >> 1) * CH_BYTES;
        Frag fa[2];
        u32x16 fb[2];
        PackB pb[BD];
        wait_full(g0);
        FACC(3);
#pragma unroll
        for (int i = 0; i < BD; ++i) { ldb(pb[i], mk_rsrc(myws, WS_NEED), wbase + (unsigned)i * (2u * CH_BYTES)); SB(); }
        rd(fa[0], IC<0>());
        unsigned rec = rfl(*(const volatile LAS unsigned*)(size_t)vrec);
#pragma unroll
        for (int k = 0; k < 4; ++k) {
            const unsigned br = (rec & 0x7Fu) * 0x01010101u, n = pb[0].cd[k];
            put(fb[0], k, (n & 0x87878787u) + br, (__builtin_amdgcn_alignbit(n, n, 4) & 0x87878787u) + br, pb[0]);
        }
        if ((rec >> 8) != 0u) repair(fb[0], pb[0], rec, wbase);
        rec = *(const volatile LAS unsigned*)((const LAS unsigned char*)(size_t)vrec + 16);
        __builtin_amdgcn_s_waitcnt(0xC07F);
        for (unsigned o = 0; o < (unsigned)(NKS / BD); ++o) {
            const __amdgpu_buffer_rsrc_t rb = mk_rsrc(myws, o + 1u < (unsigned)(NKS / BD) ? WS_NEED : 0u);
            const unsigned s0 = o * (unsigned)BD, wo = wbase + s0 * (2u * CH_BYTES);
            auto step = [&](auto uc) INL {
                constexpr int U = decltype(uc)::value, C = U & 1, N = C ^ 1, X = (U + 1) & (BD - 1);
                const unsigned g = g0 + s0 + (unsigned)U + 1u;
                const unsigned fmt = rfl(rec);
                const unsigned br = (fmt & 0x7Fu) * 0x01010101u;
                if (U == BD - 1) {
#pragma unroll
                    for (int k = 0; k < 4; ++k) ab[k] ^= 65536u;
                }
                rec = *(const volatile LAS unsigned*)((const LAS unsigned char*)(size_t)vrec + (U + 2) * 16);
                SB();
#pragma unroll
                for (int k = 0; k < 4; ++k) {
                    const u32x4 bk = {fb[C][4 * k], fb[C][4 * k + 1], fb[C][4 * k + 2], fb[C][4 * k + 3]};
                    acc[0] = __builtin_amdgcn_mfma_f32_32x32x16_bf16(__builtin_bit_cast(bf16x8, bk), __builtin_bit_cast(bf16x8, fa[C].a[k][0]), acc[0], 0, 0, 0);
                    SB();
                    fa[N].a[k][0] = *(const LAS s16x8*)((const LAS unsigned char*)(size_t)ab[k] + X * 8192);
                    if (k == 0) ldb(pb[U], rb, wo + (unsigned)(U + BD) * (2u * CH_BYTES));
                    const unsigned n = pb[X].cd[k];
                    const unsigned ha = (n & 0x87878787u) + br, hb = (__builtin_amdgcn_alignbit(n, n, 4) & 0x87878787u) + br;
                    asm volatile("" :: "v"(fa[C].a[k][0]), "v"(ha), "v"(hb));
                    SB();
                    acc[1] = __builtin_amdgcn_mfma_f32_32x32x16_bf16(__builtin_bit_cast(bf16x8, bk), __builtin_bit_cast(bf16x8, fa[C].a[k][1]), acc[1], 0, 0, 0);
                    SB();
                    fa[N].a[k][1] = *(const LAS s16x8*)((const LAS unsigned char*)(size_t)ab[k] + (X * 8192 + 4096));
                    put(fb[N], k, ha, hb, pb[X]);
                    asm volatile("" :: "v"(fa[C].a[k][1]), "v"(fb[N][4 * k + 3]));
                    SB();
                }
                asm volatile("" :: "v"(fb[C]));
                if ((U & 3) == 3) lds_st(cfree, g + 4u);
                if (__builtin_expect((fmt >> 7) != 1u, 0)) {
                    if (g != g0 + (unsigned)NKS) {
                        FACC(0);
                        repair(fb[N], pb[X], fmt, wo + (unsigned)(U + 1) * (2u * CH_BYTES));
                        FACC(4);
                        if ((fmt & 0x80u) == 0u) {
                            FACC(0);
                            lds_st(cfree, g + 4u);
                            wait_full(g);
                            FACC(1);
                            rd(fa[N], IC<X>());
                            rec = *(const volatile LAS unsigned*)((const LAS unsigned char*)(size_t)vrec + (U + 2) * 16);
                            __builtin_amdgcn_s_waitcnt(0xC07F);
                            FACC(5);
                        }
                    }
                }
                SB();
            };
            step(IC<0>()); step(IC<1>()); step(IC<2>()); step(IC<3>());
            step(IC<4>()); step(IC<5>()); step(IC<6>()); step(IC<7>());
            vrec += (unsigned)BD * 16u;
        }
        if (cw == 0) STAMP(7);
        FACC(0);
        FOUT();

        const bool bad = lds_ld(C_ABORT) != 0u;
#pragma unroll
        for (int mt = 0; mt < 2; ++mt) {
#pragma unroll
            for (int j = 0; j < 4; ++j) {
                bf16x2 lo, hi;
                lo[0] = (__bf16)acc[mt][4 * j]; lo[1] = (__bf16)acc[mt][4 * j + 1];
                hi[0] = (__bf16)acc[mt][4 * j + 2]; hi[1] = (__bf16)acc[mt][4 * j + 3];
                u32x2 v; v.x = __builtin_bit_cast(unsigned, lo); v.y = __builtin_bit_cast(unsigned, hi);
                if (bad) { v.x = 0x7FC07FC0u; v.y = 0x7FC07FC0u; }
                *(volatile LAS u32x2*)(size_t)(L_ST + (unsigned)((mt * 32 + r32) * ST_ROW + cw * 64 + j * 16 + hh * 8)) = v;
            }
        }
        unsigned old;
        asm volatile("ds_add_rtn_u32 %0, %1, %2\n\ts_waitcnt lgkmcnt(0)" : "=&v"(old) : "v"(C_CNT), "v"(lane == 0 ? 1u : 0u) : "memory");
        if ((rfl(old) & 3u) != 3u) continue;
        u32x4 o[16];
#pragma unroll
        for (int i = 0; i < 16; ++i)
            o[i] = *(const volatile LAS u32x4*)(size_t)(L_ST + (unsigned)((i * 4 + (lane >> 4)) * ST_ROW + (lane & 15) * 16));

        const bool remote = t.cb >= 4;
        const int dstr = remote ? obase + (t.cb & 3) : rank;
        const __amdgpu_buffer_rsrc_t ro = mk_rsrc(p.out[dstr], SHARD_BYTES);
        const unsigned ovo = (unsigned)(lane >> 4) * 1024u + (unsigned)(lane & 15) * 16u;
        const unsigned oso = (unsigned)(t.src * 512 + t.blk * 64) * 1024u + (unsigned)(remote ? gidx : t.cb) * 256u;
        if (!remote) {
#pragma unroll
            for (int i = 0; i < 16; ++i) __builtin_amdgcn_raw_buffer_store_b128(o[i], ro, ovo, oso + i * 4096u, AG_LOCAL_AUX);
        } else {
            t0 = 0;
            while (lds_ld(C_GO) == 0u) {
                if ((++spins & 1023u) == 0u && expired(t0, 2 * DEADLINE)) break;
                __builtin_amdgcn_s_sleep(1);
            }
            STAMP(4);
#pragma unroll
            for (int i = 0; i < 16; ++i) __builtin_amdgcn_raw_buffer_store_b128(o[i], ro, ovo, oso + i * 4096u, 17);
            VM_WAIT(0);
            if (lane == 0) st_sys(p.ws[dstr] + OFF_DONE + (unsigned)(gidx * 32 + t.rb) * 4u, call);
        }
        STAMP(8);
    }
}

}

extern "C" void kernel_launch(int rank, int nranks, ncclComm_t comm,
                              hipStream_t stream,
                              void* const* d_in, const int* in_sizes, int n_in,
                              void* d_out, int out_size,
                              const int* in_dtypes, int out_dtype,
                              void* const* d_out_all,
                              void* const* d_ws_all, size_t ws_bytes) {
    using namespace agk;
    (void)comm; (void)in_dtypes; (void)out_dtype;
    if (nranks != NR || (unsigned)rank >= (unsigned)NR || n_in < 2 || ws_bytes < (size_t)WS_NEED ||
        in_sizes[0] != 512 * 4096 || in_sizes[1] != 512 * 4096 || out_size != 512 * 4096) {
        fprintf(stderr, "kernel_launch: unexpected problem shape\n");
        return;
    }
    Params p;
    p.x = d_in[0];
    p.w = d_in[1];
    for (int r = 0; r < NR; ++r) { p.ws[r] = (char*)d_ws_all[r]; p.out[r] = (char*)d_out_all[r]; }
    p.out[rank] = (char*)d_out;
    p.rank = rank;
    const int gidx = rank & 3, obase = (rank & 4) ^ 4;
    int xcd_of[NR], spare = 3;
    {
#define AG_SETMASK 0xB2u
        const unsigned setmask = AG_SETMASK;
        int lo = 0, hi = 4;
        for (int o = 1; o < NR; ++o) {
            const int q = (rank + o) & 7;
            xcd_of[q] = (((setmask >> q) ^ (setmask >> rank)) & 1u) ? hi++ : lo++;
        }
        xcd_of[rank] = spare;
        p.xpeer = 0u;
        for (int q = 0; q < NR; ++q) p.xpeer |= (unsigned)(q == rank ? 15 : q) << (xcd_of[q] * 4);
    }
    auto is_late = [&](int rb) { return rb < 32 && (rb >= 4 * NALPHA || (rb % NALPHA >= NEARLY && rb / NALPHA != gidx)); };
    auto code_of = [](int rb, int cb) { return (unsigned)rb | ((unsigned)cb << 6) | 0x8000u; };
    int next[8];
    for (int x = 0; x < 8; ++x) next[x] = x != spare ? x + 8 * CPL : x;
    for (int b = 0; b < NWG; ++b) p.tile[b] = 0u;
    int ntile = 0;
    int late_x[4], used = 0;
    for (int pass = 0; pass < 2; ++pass)
        for (int c = 0; c < 4; ++c)
            for (int o = 0; o < 4; ++o) {
                const int x = xcd_of[(rank & 4) + o];
                if ((pass == 1 && late_x[c] >= 0)) break;
                late_x[c] = -1;
                if (((used >> x) & 1) || (pass == 0 && (x >> 2) != (xcd_of[obase + c] >> 2))) continue;
                late_x[c] = x; used |= 1 << x;
                break;
            }
    for (int c = 0; c < 4; ++c)
        for (int rb = 0; rb < 32; ++rb) {
            if (rb >= 4 * NALPHA && (rb - 4 * NALPHA) / NBETA == c) continue;
            const int xx = is_late(rb) ? late_x[c] : xcd_of[obase + c];
            if (next[xx] >= NWG) { fprintf(stderr, "kernel_launch: placement\n"); return; }
            p.tile[next[xx]] = code_of(rb, 4 + c);
            next[xx] += 8;
            ++ntile;
        }
    unsigned rest[140], first[64];
    int nr = 0, nf = 0;
    for (int pass = 0; pass < 3; ++pass)
        for (int rb = 0; rb < 35; ++rb)
            for (int c = 0; c < 4; ++c) {
                const int cls = is_late(rb) ? 2 : (rb >= 32 || rb / NALPHA == gidx) ? 0 : 1;
                if (cls != pass) continue;
                if (cls < 2 && nf < NPWG) first[nf++] = code_of(rb, c); else rest[nr++] = code_of(rb, c);
            }
    int cand[NWG], nc = 0;
    for (int x = 0, k = nr - 1; x < 8; ++x)
        for (; next[x] < NWG && k >= 0; next[x] += 8, --k) {
            p.tile[next[x]] = rest[k];
            if (is_late((int)(rest[k] & 63u))) cand[nc++] = next[x];
            ++ntile;
        }
    if (nc < nf || ntile + nf != NTILE || nf != NPWG) { fprintf(stderr, "kernel_launch: pairing %d %d %d\n", nc, nf, ntile); return; }
    for (int k = 0; k < nf; ++k) {
        const int b = cand[k * nc / nf];
        p.tile[b] = first[k] | (p.tile[b] << 16);
    }
    hipLaunchKernelGGL(ag_gemm_kernel, dim3(NWG), dim3(NTHREADS), 0, stream, p);
    HIP_CHECK(hipGetLastError());
}
```

```cpp
#include <hip/hip_runtime.h>
#include <rccl/rccl.h>
#include <cstdio>
#include <cstdint>
#include <cstddef>

#define HIP_CHECK(expr) do { hipError_t _e = (expr); if (_e != hipSuccess) { \
    fprintf(stderr, "HIP error %s at %s:%d\n", hipGetErrorString(_e), __FILE__, __LINE__); \
    return; } } while (0)

namespace agk {

#define GAS __attribute__((address_space(1)))
#define LAS __attribute__((address_space(3)))
#define INL __attribute__((always_inline))
typedef unsigned u32x4 __attribute__((ext_vector_type(4)));
typedef unsigned u32x2 __attribute__((ext_vector_type(2)));
typedef short s16x4 __attribute__((ext_vector_type(4)));
typedef short s16x8 __attribute__((ext_vector_type(8)));
typedef __bf16 bf16x8 __attribute__((ext_vector_type(8)));
typedef __bf16 bf16x2 __attribute__((ext_vector_type(2)));
typedef float f32x16 __attribute__((ext_vector_type(16)));
typedef unsigned u32x16 __attribute__((ext_vector_type(16)));

constexpr int NR = 8;
constexpr int NKS = 64;
constexpr int NALPHA = 5, NBETA = 3;
constexpr int NEARLY = 2;
constexpr int NWG = 256;
constexpr int NTILE = 256;
#define AG_NFWD 8
constexpr int NFWD = AG_NFWD;
constexpr int FBLK = NBETA - 1;
constexpr int NTASK = 320 + NFWD;
constexpr int NTASKX = 320 - NFWD / 4 * 3;
#define AG_CPL 8
#define AG_WPP 4
#define AG_TOPO 1
constexpr int CPL = AG_CPL;
constexpr int WPP = AG_WPP;
constexpr int NPW = CPL * WPP;
constexpr int NPWG = 7 * CPL;
#define AG_POLL_SLEEP 2
#define AG_FREE_SLEEP 1
#define AG_LOCAL_AUX 17
#define AG_REL 5
#define AG_TAU 27
#define AG_CDMA 1
#define AG_GO_SLEEP 4
#define AG_GOT 40
#define AG_DELTA 300
#define AG_EAGERW 2
#define AG_W2D 8
#define AG_XAUX 17
#define AG_WAUX 17
#define AG_TDELAY 32
#define AG_PRIO 0
#define AG_FREEPH 5
#define AG_PAIR0 4
#define AG_PAIR1 2
#define AG_PREX 1
#define AG_LPRIO 2
#define AG_HOLD 1
#define AG_SEG_OWN {{64, 48, 38, 0}, {0, 16, 10, NFWD / 4}, {0, 0, 16, NFWD / 4 * 3}, {0, 0, 0, 0}}
#define AG_SEG_OTH {{64, 52, 52 - NFWD, NFWD / 4}, {0, 12, 12, 0}, {0, 0, 0, 0}, {0, 0, 0, 0}}
#define AG_PACKX AG_PACKPH
#define AG_PACKPH 2
#define AG_TAUP 20
#define AG_UNP_SLEEP 4
#define AG_CONV_SLEEP 8
#define AG_NKEEP (128u - 41u)
static_assert(NFWD % 4 == 0 && NFWD < 64 && NTASK - 128 < 256, "order table");
static_assert(NPWG == 56 && WPP <= 7, "pushing workgroups: at most one per early tile with a local output");
constexpr unsigned SHARD_BYTES = 512u * 4096u * 2u;

constexpr unsigned OFF_CTR = 0;
constexpr unsigned OFF_TMO = 32u << 10;
constexpr unsigned OFF_GO = 34u << 10;
constexpr unsigned OFF_ARR = 33u << 10;
constexpr unsigned OFF_Q = 36u << 10;
constexpr unsigned OFF_FA = 64u << 10;
constexpr unsigned OFF_FW = 80u << 10;
constexpr unsigned FLG_PAR = 32u << 10;
constexpr unsigned OFF_DONE = 128u << 10;
constexpr unsigned OFF_FR = 132u << 10;
constexpr unsigned OFF_DBG = 192u << 10;
constexpr unsigned OFF_DATA = 1u << 20;
constexpr unsigned CH_BYTES = 10240u;
constexpr unsigned ABLK_BYTES = NKS * CH_BYTES;
constexpr unsigned WQ_BYTES = NKS * 2u * CH_BYTES;
constexpr unsigned OFF_WST = 32u * ABLK_BYTES;
constexpr unsigned PAR_BYTES = OFF_WST + 4u * WQ_BYTES;
constexpr unsigned OFF_RAW = OFF_DATA + 2u * PAR_BYTES;
constexpr unsigned WS_NEED = OFF_RAW + (8u << 20);

constexpr int NSLOT = 16;
constexpr int NLOAD = 4;
constexpr int BD = 8;
constexpr unsigned L_ST = NSLOT * 8192u;
constexpr unsigned ST_ROW = 272;
constexpr unsigned L_REC = L_ST + 64 * ST_ROW;
constexpr unsigned L_CTL = L_REC + 512 + 64;
constexpr unsigned C_FREE = L_CTL;
constexpr unsigned C_ABORT = L_CTL + 16;
constexpr unsigned C_CNT = L_CTL + 20;
constexpr unsigned C_GO = L_CTL + 24;
constexpr unsigned C_ROLE = L_CTL + 64;
constexpr unsigned L_SPARE = L_CTL + 128;
constexpr unsigned LDS_BYTES = L_SPARE + 256;
static_assert(256u * CH_BYTES <= NALPHA * ABLK_BYTES && NPWG * 8 >= 256, "staging / unpacking of output tiles");
static_assert(LDS_BYTES <= 160u * 1024u && (L_REC & 15u) == 0u, "LDS");
constexpr int NTHREADS = (NLOAD + 4) * 64;

constexpr unsigned long long DEADLINE = 30000000ull;

struct Params {
    const void* x;
    const void* w;
    char* ws;
    char* out;
    int rank;
    unsigned spare;
    unsigned xpeer;
    unsigned pad[5];
    char* xws[8];
    char* oth[4][2];
    unsigned tile[NWG];
    unsigned order[64][2];
};

__device__ __forceinline__ unsigned long long rt() { return __builtin_amdgcn_s_memrealtime(); }
__device__ __forceinline__ bool expired(unsigned long long& t0, unsigned long long limit) {
    const unsigned long long now = __builtin_amdgcn_s_memrealtime() | 1ull;
    if (t0 == 0ull) { t0 = now; return false; }
    return now - t0 > limit;
}
__device__ __forceinline__ unsigned rfl(unsigned v) { return (unsigned)__builtin_amdgcn_readfirstlane((int)v); }
__device__ __forceinline__ __amdgpu_buffer_rsrc_t mk_rsrc(const void* p, unsigned bytes) {
    return __builtin_amdgcn_make_buffer_rsrc((void*)p, (short)0, (int)bytes, 0x00020000);
}
#define VM_WAIT_(n) asm volatile("s_waitcnt vmcnt(" #n ")" ::: "memory")
#define VM_WAIT(n) VM_WAIT_(n)
__device__ __forceinline__ void lds_st(unsigned addr, unsigned v) {
    asm volatile("ds_write_b32 %0, %1" :: "v"(addr), "v"(v) : "memory");
}
__device__ __forceinline__ unsigned lds_ld(unsigned addr) {
    unsigned v;
    asm volatile("ds_read_b32 %0, %1\n\ts_waitcnt lgkmcnt(0)" : "=&v"(v) : "v"(addr) : "memory");
    return rfl(v);
}
__device__ __forceinline__ u32x4 lds_ld4(unsigned addr) {
    u32x4 v;
    asm volatile("ds_read_b128 %0, %1\n\ts_waitcnt lgkmcnt(0)" : "=&v"(v) : "v"(addr) : "memory");
    return v;
}
__device__ __forceinline__ unsigned ld_agent(const char* p) {
    return __hip_atomic_load((const GAS unsigned*)p, __ATOMIC_RELAXED, __HIP_MEMORY_SCOPE_AGENT);
}
__device__ __forceinline__ void st_agent(char* p, unsigned v) {
    __hip_atomic_store((GAS unsigned*)p, v, __ATOMIC_RELAXED, __HIP_MEMORY_SCOPE_AGENT);
}
__device__ __forceinline__ void st_sys(char* p, unsigned v) {
    __hip_atomic_store((GAS unsigned*)p, v, __ATOMIC_RELAXED, __HIP_MEMORY_SCOPE_SYSTEM);
}
#define DMA_P(V) "buffer_load_dwordx4 " V ", %5, %1 offen sc0 sc1 lds\n\ts_add_u32 m0, m0, 0x400\n\ts_add_u32 %1, %1, %7\n\t"
#define DMA_C(V) "buffer_load_dwordx4 " V ", %5, %1 offen lds\n\ts_add_u32 m0, m0, 0x400\n\ts_add_u32 %1, %1, %7\n\t"
#define DMA_HEAD "s_nop 4\n\ts_mov_b32 %0, m0\n\ts_mov_b32 m0, %2\n\ts_mov_b32 %1, %6\n\t"
#define DMA_TAIL "s_mov_b32 m0, %0"
#define DMA8(PA, PB, PC, PD) \
    asm volatile(DMA_HEAD DMA_P(PA) DMA_P(PB) DMA_P(PC) DMA_P(PD) DMA_P(PA) DMA_P(PB) DMA_P(PC) DMA_P(PD) DMA_TAIL \
                 : "=&s"(keep), "=&s"(t) : "s"(lds), "v"(v0), "v"(v1), "s"(rs), "s"(soff), "s"(js) : "scc", "memory")
__device__ __forceinline__ void dma8(unsigned lds, unsigned v0, unsigned v1, __amdgpu_buffer_rsrc_t rs,
                                     unsigned soff, unsigned js) {
    unsigned keep, t;
    DMA8("%3", "%4", "%3", "%4");
}

__device__ __forceinline__ void dma8c(unsigned lds, unsigned v0, unsigned v1, __amdgpu_buffer_rsrc_t rs,
                                      unsigned soff, unsigned js) {
    unsigned keep, t;
    asm volatile(DMA_HEAD DMA_C("%3") DMA_C("%4") DMA_C("%3") DMA_C("%4") DMA_C("%3") DMA_C("%4") DMA_C("%3") DMA_C("%4") DMA_TAIL
                 : "=&s"(keep), "=&s"(t) : "s"(lds), "v"(v0), "v"(v1), "s"(rs), "s"(soff), "s"(js) : "scc", "memory");
}

#define STAMP(i) do {} while (0)

#define FDECL do {} while (0)
#define FACC(i) do {} while (0)
#define FACS(i) do {} while (0)
#define FOUT() do {} while (0)
__device__ __forceinline__ unsigned x_lane_off(int l) { return (unsigned)((l >> 3) * 8192 + (((l & 7) ^ (l >> 4)) << 4)); }

__device__ __forceinline__ void w_fetch(unsigned lds, __amdgpu_buffer_rsrc_t rw, unsigned q, unsigned t, int lane) {
    const unsigned v = (unsigned)((lane >> 3) * 1024 + (lane & 7) * 16);
    dma8c(lds, v, v, rw, (t >> 1) * 65536u + q * 256u + (t & 1u) * 128u, 8192u);
}
__device__ __forceinline__ void w_frags(u32x4 (&r)[8], unsigned lds, int lane) {
    const unsigned wr = lds + (unsigned)((8 * (lane >> 5) + ((lane >> 2) & 3)) * 128 + ((lane >> 4) & 1) * 32 + (lane & 3) * 8);
#pragma unroll
    for (int j = 0; j < 8; ++j) {
        const unsigned ad = wr + (unsigned)((j & 3) * 2048 + (j >> 2) * 64);
        const u32x2 k0 = __builtin_bit_cast(u32x2, __builtin_amdgcn_ds_read_tr16_b64_v4i16((LAS s16x4*)(size_t)ad));
        const u32x2 k1 = __builtin_bit_cast(u32x2, __builtin_amdgcn_ds_read_tr16_b64_v4i16((LAS s16x4*)(size_t)(ad + 512u)));
        r[j] = __builtin_shufflevector(k0, k1, 0, 1, 2, 3);
    }
}

constexpr unsigned F_ALL = 1u << 24, F_RAW = 1u << 25;
typedef unsigned short u16x2 __attribute__((ext_vector_type(2)));
__device__ __forceinline__ unsigned bfi(unsigned m, unsigned a, unsigned b) { return (a & m) | (b & ~m); }
__device__ __forceinline__ unsigned pkmax(unsigned a, unsigned b) {
    return __builtin_bit_cast(unsigned, __builtin_elementwise_max(__builtin_bit_cast(u16x2, a), __builtin_bit_cast(u16x2, b)));
}
__device__ __forceinline__ unsigned pkadd(unsigned a, unsigned b) {
    return __builtin_bit_cast(unsigned, (u16x2)(__builtin_bit_cast(u16x2, a) + __builtin_bit_cast(u16x2, b)));
}
__device__ __forceinline__ unsigned rotr(unsigned a, unsigned n) { return __builtin_amdgcn_alignbit(a, a, n); }
struct Packed { u32x4 lo[4], hi[4], ca, cb; unsigned fmt; };
__device__ __forceinline__ void pack_low_odd(const u32x4 (&r)[8], Packed& o) {
#pragma unroll
    for (int j = 0; j < 8; ++j) {
        o.lo[j >> 1][(j & 1) * 2] = __builtin_amdgcn_perm(r[j].y, r[j].x, 0x06040200u) ^ 0x80808080u;
        o.lo[j >> 1][(j & 1) * 2 + 1] = __builtin_amdgcn_perm(r[j].w, r[j].z, 0x06040200u) ^ 0x80808080u;
    }
}
__device__ __forceinline__ unsigned pick_base(const u32x4 (&r)[8]) {
    unsigned m = 0, g = 0;
#pragma unroll
    for (int d = 0; d < 4; ++d) m = pkmax(pkmax(m, r[0][d] & 0x7FFF7FFFu), r[4][d] & 0x7FFF7FFFu);
    m = max(m >> 16, m & 0xFFFFu);
#pragma unroll
    for (unsigned bit = 0x4000u; bit >= 0x80u; bit >>= 1) {
        unsigned n = (unsigned)__builtin_popcountll(__ballot(m >= (g | bit)));
        asm volatile("" : "+s"(n));
        g |= n > 8u ? bit : 0u;
    }
    return g > 7u * 128u ? min((g >> 7) - 7u, 247u) : 0u;
}
__device__ __forceinline__ void pack(const u32x4 (&r)[8], Packed& o, unsigned base, unsigned scr, int lane) {
    const unsigned neg = ((0u - (base << 7)) & 0xFFFFu) * 0x10001u;
    unsigned h[16], ex[2] = {0u, 0u};
#pragma unroll
    for (int j = 0; j < 8; ++j) {
        const unsigned t0 = pkadd(r[j].x, neg), t1 = pkadd(r[j].y, neg), t2 = pkadd(r[j].z, neg), t3 = pkadd(r[j].w, neg);
        o.lo[j >> 1][(j & 1) * 2] = __builtin_amdgcn_perm(t1, t0, 0x06040200u);
        o.lo[j >> 1][(j & 1) * 2 + 1] = __builtin_amdgcn_perm(t3, t2, 0x06040200u);
        h[2 * j] = __builtin_amdgcn_perm(t1, t0, 0x07050301u);
        h[2 * j + 1] = __builtin_amdgcn_perm(t3, t2, 0x07050301u);
        const u32x2 hv = {h[2 * j], h[2 * j + 1]};
        *(volatile LAS u32x2*)(size_t)(scr + (unsigned)lane * 8u + j * 512u) = hv;
        ex[j >> 2] = bfi(0x80808080u, (h[2 * j] & 0x7C7C7C7Cu) + 0x7C7C7C7Cu, ex[j >> 2] >> 1);
        ex[j >> 2] = bfi(0x80808080u, (h[2 * j + 1] & 0x7C7C7C7Cu) + 0x7C7C7C7Cu, ex[j >> 2] >> 1);
    }
    unsigned c[6];
#pragma unroll
    for (int q = 0; q < 2; ++q) {
        const unsigned* e = h + 8 * q;
        c[3 * q] = bfi(0x83838383u, e[0], bfi(0x1C1C1C1Cu, rotr(e[1], 29), e[6] << 5));
        c[3 * q + 1] = bfi(0x83838383u, e[2], bfi(0x1C1C1C1Cu, rotr(e[3], 29), e[7] << 5));
        c[3 * q + 2] = bfi(0x83838383u, e[4], bfi(0x1C1C1C1Cu, rotr(e[5], 29), bfi(0x20202020u, e[6] >> 2, e[7] >> 1)));
    }
#pragma unroll
    for (int i = 0; i < 16; ++i) o.hi[i >> 2][i & 3] = h[i];
    o.ca.x = c[0]; o.ca.y = c[1]; o.ca.z = c[2]; o.ca.w = c[3];
    const u32x2 cv = {c[4], c[5]};
    *(volatile LAS u32x2*)(size_t)(scr + 4096u + (unsigned)lane * 8u) = cv;
    unsigned n[2] = {0u, 0u};
    for (;;) {
        const unsigned long long a0 = __ballot(ex[0] != 0u), a1 = __ballot(ex[1] != 0u);
        if ((a0 | a1) == 0ull || max(n[0], n[1]) > 64u) break;
#pragma unroll
        for (int q = 0; q < 2; ++q) {
            const unsigned long long a = q ? a1 : a0;
            const unsigned i = __builtin_amdgcn_mbcnt_hi((unsigned)(a >> 32), __builtin_amdgcn_mbcnt_lo((unsigned)a, 0u));
            if (ex[q] != 0u)
                *(volatile LAS unsigned short*)(size_t)(scr + 4608u + q * 2u + n[q] * 4u + i * 4u) = (unsigned short)((unsigned)lane * 32u | (unsigned)__builtin_ctz(ex[q]));
            ex[q] &= ex[q] - 1u;
            n[q] += (unsigned)__builtin_popcountll(a);
        }
    }
    const unsigned e = *(const volatile LAS unsigned*)(size_t)(scr + 4608u + (unsigned)lane * 4u);
    const unsigned pos = ((e & 0x00060006u) << 8) | ((e >> 2) & 0x01F801F8u) | ((e & 0x00010001u) << 2) | ((e >> 3) & 0x00030003u);
    const unsigned x0 = *(const volatile LAS unsigned char*)(size_t)(scr + (pos & 0xFFFFu));
    const unsigned x1 = *(const volatile LAS unsigned char*)(size_t)(scr + 2048u + (pos >> 16));
    *(volatile LAS unsigned*)(size_t)(scr + 4608u + (unsigned)lane * 4u) = pos | (((x0 | (x1 << 16)) & 0x007C007Cu) << 9);
    o.cb = *(const volatile LAS u32x4*)(size_t)(scr + 4096u + (unsigned)lane * 16u);
    o.fmt = base | (max(n[0], n[1]) > 64u ? F_ALL : (n[0] << 8) | (n[1] << 16));
}
__device__ __forceinline__ void send_low(const Packed& k, __amdgpu_buffer_rsrc_t rd, unsigned dof, int lane) {
#pragma unroll
    for (int i = 0; i < 4; ++i) __builtin_amdgcn_raw_buffer_store_b128(k.lo[i], rd, lane * 16, dof + i * 1024u, 17);
}
__device__ __forceinline__ void send_high(const Packed& k, __amdgpu_buffer_rsrc_t rd, unsigned dof, int lane) {
    if (__builtin_expect((k.fmt & F_ALL) == 0u, 1)) {
        __builtin_amdgcn_raw_buffer_store_b128(k.ca, rd, lane * 16, dof + 4096u, 17);
        const unsigned nl = 32u + ((max((k.fmt >> 8) & 0xFFu, k.fmt >> 16) + 15u) >> 4) * 4u;
        if ((unsigned)lane < nl) __builtin_amdgcn_raw_buffer_store_b128(k.cb, rd, lane * 16, dof + 5120u, 17);
    } else {
#pragma unroll
        for (int i = 0; i < 4; ++i) __builtin_amdgcn_raw_buffer_store_b128(k.hi[i], rd, lane * 16, dof + 6144u + i * 1024u, 17);
    }
}
struct Wire { u32x4 lo[4], ca; u32x2 cb; unsigned ls; };
__device__ __forceinline__ void wire_load(Wire& w, const char* ws, bool go, unsigned off, int lane) {
    const __amdgpu_buffer_rsrc_t rs = mk_rsrc(ws, go ? WS_NEED : 0u);
    w.ca = __builtin_amdgcn_raw_buffer_load_b128(rs, lane * 16, off + 4096u, AG_XAUX);
#pragma unroll
    for (int i = 0; i < 2; ++i) w.lo[i] = __builtin_amdgcn_raw_buffer_load_b128(rs, lane * 16, off + i * 1024u, AG_XAUX);
    w.ls = __builtin_amdgcn_raw_buffer_load_b32(rs, lane * 4, off + 5632u, AG_XAUX);
    w.cb = __builtin_amdgcn_raw_buffer_load_b64(rs, lane * 8, off + 5120u, AG_XAUX);
#pragma unroll
    for (int i = 2; i < 4; ++i) w.lo[i] = __builtin_amdgcn_raw_buffer_load_b128(rs, lane * 16, off + i * 1024u, AG_XAUX);
}
template <bool ALL>
__device__ __forceinline__ void wire_put(unsigned ha, unsigned hb, unsigned l0, unsigned l1, unsigned br, unsigned lds) {
    u32x4 v;
    v.x = __builtin_amdgcn_perm(ha, l0, 0x05010400u);
    v.y = __builtin_amdgcn_perm(ha, l0, 0x07030602u);
    v.z = __builtin_amdgcn_perm(hb, l1, 0x05010400u);
    v.w = __builtin_amdgcn_perm(hb, l1, 0x07030602u);
#pragma unroll
    for (int i = 0; i < 4; ++i) v[i] = ALL ? pkadd(v[i], br) : v[i] + br;
    *(volatile LAS u32x4*)(size_t)lds = v;
}
struct Fix { unsigned ad[2], old[2], ls, fmt; };
__device__ __forceinline__ void wire_spread(const Wire& w, Fix& x, const char* ws, unsigned off, unsigned fmt, int lane, unsigned lds) {
    const unsigned br = (fmt & 0xFFu) * 0x00800080u, la = lds + (unsigned)lane * 16u;
    x.ls = w.ls; x.fmt = fmt;
    if (__builtin_expect((fmt & F_ALL) != 0u, 0)) {
        const __amdgpu_buffer_rsrc_t rs = mk_rsrc(ws, WS_NEED);
#pragma unroll
        for (int i = 0; i < 4; ++i) {
            u32x4 e;
            asm volatile("buffer_load_dwordx4 %0, %1, %2, %3 offen sc0 sc1\n\ts_waitcnt vmcnt(0)"
                         : "=&v"(e) : "v"(lane * 16), "s"(rs), "s"(off + 6144u + i * 1024u) : "memory");
            wire_put<true>(e.x, e.y, w.lo[i].x, w.lo[i].y, br, la + i * 2048u);
            wire_put<true>(e.z, e.w, w.lo[i].z, w.lo[i].w, br, la + i * 2048u + 1024u);
        }
        x.ad[0] = x.ad[1] = x.old[0] = x.old[1] = 0u;
        return;
    }
#pragma unroll
    for (int q = 0; q < 2; ++q) {
        const unsigned a = q ? w.ca.w : w.ca.x, b = q ? w.cb.x : w.ca.y, c = q ? w.cb.y : w.ca.z;
        const u32x4 l0 = w.lo[2 * q], l1 = w.lo[2 * q + 1];
        wire_put<false>(a & 0x83838383u, rotr(a, 3) & 0x83838383u, l0.x, l0.y, br, la + q * 4096u);
        wire_put<false>(b & 0x83838383u, rotr(b, 3) & 0x83838383u, l0.z, l0.w, br, la + q * 4096u + 1024u);
        wire_put<false>(c & 0x83838383u, rotr(c, 3) & 0x83838383u, l1.x, l1.y, br, la + q * 4096u + 2048u);
        wire_put<false>(bfi(0x80808080u, c << 2, a >> 5) & 0x83838383u, bfi(0x80808080u, c << 1, b >> 5) & 0x83838383u, l1.z, l1.w, br, la + q * 4096u + 3072u);
        x.ad[q] = lds + q * 4096u + 1u + ((w.ls >> (16 * q)) & 0x7FFu) * 2u;
        x.old[q] = *(const volatile LAS unsigned char*)(size_t)x.ad[q];
    }
}
__device__ __forceinline__ void wire_fix(const Fix& x, int lane) {
#pragma unroll
    for (int q = 0; q < 2; ++q)
        *(volatile LAS unsigned char*)(size_t)((unsigned)lane < ((x.fmt >> (8 + 8 * q)) & 0xFFu) ? x.ad[q] : L_SPARE + (unsigned)lane * 4u) =
            (unsigned char)(x.old[q] + ((x.ls >> (9 + 16 * q)) & 0x7Cu));
}
__device__ __forceinline__ void wire_unpack(const Wire& w, const char* ws, unsigned off, unsigned fmt, int lane, unsigned lds) {
    Fix x;
    wire_spread(w, x, ws, off, fmt, lane, lds);
    wire_fix(x, lane);
}
__device__ __forceinline__ unsigned flag_pos(unsigned ks) { return (ks & 3u) * 16u + (ks >> 2); }

struct Tile { int rb, cb, src, blk; };
__device__ __forceinline__ Tile tile_of(unsigned code, int rank) {
    const int gidx = rank & 3, gbase = rank & 4, obase = gbase ^ 4;
    Tile t;
    t.rb = (int)(code & 63u); t.cb = (int)((code >> 6) & 7u);
    if (t.rb < 4 * NALPHA) { t.src = gbase + t.rb / NALPHA; t.blk = t.rb % NALPHA; }
    else if (t.rb < 32) { t.src = obase + (t.rb - 4 * NALPHA) / NBETA; t.blk = NALPHA + (t.rb - 4 * NALPHA) % NBETA; }
    else { t.src = rank; t.blk = NALPHA + (t.rb - 32); }
    return t;
}

struct Frag { s16x8 a[4][2]; };
struct FragB { u32x4 b[4]; };
template <int N> struct IC { static constexpr int value = N; };

__global__ void __launch_bounds__(NTHREADS, 2) ag_gemm_kernel(const Params p) {
    __shared__ __attribute__((aligned(1024))) unsigned char lds_raw[LDS_BYTES];
    LAS unsigned char* L = (LAS unsigned char*)lds_raw;
    const int lane = threadIdx.x & 63;
    const int wave = (int)rfl(threadIdx.x >> 6);
    unsigned bid = (blockIdx.x & ~7u) | ((unsigned)__builtin_amdgcn_s_getreg(20 | (3 << 11)) & 7u);
    const u32x2 ords = *(const GAS u32x2*)((size_t)__builtin_amdgcn_kernarg_segment_ptr() + offsetof(Params, order) + (unsigned)lane * 8u);
    const void *px = p.x, *pw = p.w;
    char *myws = p.ws, *myout = p.out, *pws = p.xws[bid & 7u];
    int rank = p.rank;
    unsigned spare = p.spare, xpeer = p.xpeer, tcode = p.tile[bid];
    asm volatile("" : "+s"(px), "+s"(pw), "+s"(myws), "+s"(myout), "+s"(pws), "+s"(rank), "+s"(spare), "+s"(xpeer), "+s"(tcode) :: "memory");
    const int gidx = rank & 3, gbase = rank & 4, obase = gbase ^ 4;
    if ((unsigned)(size_t)L != 0u) return;
    unsigned call = 0, own;
    int peer;
    bool pusher, packs;
    for (;;) {
        peer = (int)((xpeer >> ((bid & 7u) * 4u)) & 15u);
        pusher = bid < 8u * CPL && peer < NR;
        own = (bid - 64u) * 4u + (unsigned)wave - (unsigned)NLOAD;
        packs = bid >= 64u && wave >= NLOAD && own < 512u;
        if (pusher && wave == 0) STAMP(0);
        if (!pusher) __builtin_amdgcn_s_sleep(AG_TDELAY);
        if (packs || (pusher && wave < WPP)) {
            const unsigned t = packs ? own >> 2 : (unsigned)wave * CPL + (bid >> 3), lds = (unsigned)(packs ? 8 + wave : wave) * 8192u;
            if (pusher && (peer & 4) == gbase)
                dma8c(lds, x_lane_off(lane), x_lane_off(lane) ^ 64u, mk_rsrc(px, SHARD_BYTES), (t & 1u) * (64u * 8192u) + (t >> 1) * 128u, 65536u);
            else
                w_fetch(lds, mk_rsrc(pw, SHARD_BYTES), (packs ? own : (unsigned)peer) & 3u, t, lane);
        }
        if (call != 0u) break;
        if (bid >= 64u && wave < 3 && (bid - 64u) * 3u + (unsigned)wave < 512u) {
            unsigned keep;
            asm volatile("s_mov_b32 %0, m0\n\ts_mov_b32 m0, %1\n\ts_nop 0\n\tbuffer_load_dword %2, %3, %4 offen lds\n\ts_mov_b32 m0, %0"
                         : "=&s"(keep) : "s"((unsigned)wave * 8192u), "v"(lane * 128), "s"(mk_rsrc(px, SHARD_BYTES)),
                           "s"(((bid - 64u) * 3u + (unsigned)wave) * 8192u) : "memory");
        }
        if (threadIdx.x < (512 + 128) / 8) *(LAS unsigned long long*)(L + L_REC + threadIdx.x * 8) = 0ull;
        if (wave == (pusher ? 7 : 3)) {
            unsigned v = 0, role = bid;
            if (lane < 2)
                v = __hip_atomic_fetch_add((GAS unsigned*)(myws + OFF_CTR + (lane ? bid * 128u + 64u : blockIdx.x * 128u)), 1u,
                                           __ATOMIC_RELAXED, __HIP_MEMORY_SCOPE_AGENT);
            const unsigned c = rfl(v);
            const bool got = (unsigned)__builtin_amdgcn_readlane((int)v, 1) == c;
            if (got) { const u32x2 cr = {c + 1u, role}; *(volatile LAS u32x2*)(size_t)C_ROLE = cr; }
            if (lane == 0)
                __hip_atomic_fetch_add((GAS unsigned*)(myws + OFF_ARR + (blockIdx.x & 7u) * 128u), 1u, __ATOMIC_RELAXED, __HIP_MEMORY_SCOPE_AGENT);
            if (__builtin_expect(!got, 0)) {
                if (lane == 0)
                    __hip_atomic_fetch_sub((GAS unsigned*)(myws + OFF_CTR + bid * 128u + 64u), 1u, __ATOMIC_RELAXED, __HIP_MEMORY_SCOPE_AGENT);
                unsigned long long t0 = 0;
                role = 0;
                while (!__all(ld_agent(myws + OFF_ARR + (unsigned)(lane & 7) * 128u) >= (c + 1u) * (unsigned)(NWG / 8))) {
                    if (expired(t0, DEADLINE)) { role = NWG; break; }
                    __builtin_amdgcn_s_sleep(8);
                }
                for (; role < (unsigned)NWG; ++role) {
                    unsigned e = c;
                    if (lane == 0)
                        __hip_atomic_compare_exchange_strong((GAS unsigned*)(myws + OFF_CTR + role * 128u + 64u), &e, c + 1u,
                                                             __ATOMIC_RELAXED, __ATOMIC_RELAXED, __HIP_MEMORY_SCOPE_AGENT);
                    if (rfl(e) == c) break;
                }
                if (role >= (unsigned)NWG && lane == 0) st_agent(myws + OFF_TMO, c + 1u);
                const u32x2 cr = {c + 1u, role};
                *(volatile LAS u32x2*)(size_t)C_ROLE = cr;
            }
        }
        asm volatile("s_waitcnt lgkmcnt(0)" ::: "memory");
        __builtin_amdgcn_s_barrier();
        call = lds_ld(C_ROLE);
        const unsigned role = lds_ld(C_ROLE + 4u);
        if (__builtin_expect(role == bid, 1)) break;
        VM_WAIT(0);
        if (role >= (unsigned)NWG) return;
        bid = role;
        pws = p.xws[bid & 7u];
        tcode = p.tile[bid];
    }

    if (pusher) {
        if (wave < WPP) {
            const int pi = wave * CPL + (int)(bid >> 3);
            const bool same = (peer & 4) == gbase;
            const __amdgpu_buffer_rsrc_t rx = mk_rsrc(px, SHARD_BYTES), rw = mk_rsrc(pw, SHARD_BYTES);
            const unsigned xa0 = x_lane_off(lane), xa1 = xa0 ^ 64u;
            const unsigned wl = (unsigned)wave * 8192u;
            const unsigned ord = same ? ords.x : ords.y;
            const int nt = same ? NTASK : NTASKX;
            auto late = [&](int t) INL -> unsigned {
                const unsigned e = ((unsigned)__builtin_amdgcn_readlane((int)ord, (t - 128) >> 2) >> ((unsigned)(t & 3) * 8u)) & 0xFFu;
                if (e < 192u) return e;
                return same ? ((4u + (e & 3u)) << 6) | (unsigned)(NKS - NFWD + gidx) + (e & 60u)
                            : (unsigned)(FBLK * 64 + NKS - NFWD + (peer & 3)) + (e & 63u) * 4u;
            };
            unsigned dof, fof;
            auto place = [&](int t) INL {
                if (!same && t < 128) {
                    dof = OFF_WST + (unsigned)gidx * WQ_BYTES + (unsigned)t * CH_BYTES;
                    fof = OFF_FW + (unsigned)(gidx * 128 + t) * 8u;
                } else {
                    int ks, ai;
                    if (t < 128) { ks = t >> 1; ai = gidx * NALPHA + (t & 1); }
                    else {
                        const unsigned e = late(t);
                        const int r = (int)(e >> 6);
                        ks = (int)(e & 63u);
                        ai = r >= 4 ? 4 * NALPHA + (r - 4) * NBETA + FBLK : same ? gidx * NALPHA + NEARLY + r : 4 * NALPHA + gidx * NBETA + r;
                    }
                    dof = (unsigned)ai * ABLK_BYTES + (unsigned)ks * CH_BYTES;
                    fof = OFF_FA + ((unsigned)ai * 64u + flag_pos((unsigned)ks)) * 8u;
                }
            };
            const __amdgpu_buffer_rsrc_t rs = mk_rsrc(myws, WS_NEED);
            auto ask = [&](unsigned fo) INL {
                unsigned keep;
                asm volatile("s_nop 4\n\ts_mov_b32 %0, m0\n\ts_mov_b32 m0, %1\n\ts_nop 0\n\tbuffer_load_dwordx4 %2, %3, %4 offen sc0 sc1 lds\n\ts_mov_b32 m0, %0"
                             : "=&s"(keep) : "s"(wl), "v"(lane * 16), "s"(rs), "s"(fo) : "memory");
            };
            auto load_task = [&](int t) INL {
                if (!same && t < 128) {
                    w_fetch(wl, rw, (unsigned)(peer & 3), (unsigned)t, lane);
                } else {
                    int ks, blk;
                    if (t < 128) { ks = t >> 1; blk = t & 1; }
                    else { const unsigned e = late(t); ks = (int)(e & 63u); blk = (same ? NEARLY : NALPHA) + (int)(e >> 6); }
                    if (same && blk >= NEARLY + 4)
                        ask((call & 1u) * FLG_PAR + OFF_FA + (unsigned)(4 * NALPHA + (blk - NEARLY - 4) * NBETA + FBLK) * 512u);
                    else
                        dma8c(wl, xa0, xa1, rx, (unsigned)blk * (64u * 8192u) + (unsigned)ks * 128u, 65536u);
                }
            };
            auto read_task = [&](int t, u32x4 (&ra)[8]) INL {
                if (!same && t < 128) w_frags(ra, wl, lane);
                else {
#pragma unroll
                    for (int j = 0; j < 8; ++j) ra[j] = *(const volatile LAS u32x4*)(size_t)(wl + (unsigned)lane * 16u + j * 1024u);
                }
#pragma unroll
                for (int j = 0; j < 8; ++j) asm volatile("" : "+v"(ra[j]));
            };
            char* qp = myws + OFF_Q + (unsigned)peer * 128u;
            unsigned tk = 0;
            const __amdgpu_buffer_rsrc_t rd = mk_rsrc(pws, WS_NEED);
            auto draw = [&]() INL {
                unsigned long long sv;
                asm volatile("s_mov_b64 %1, exec\n\ts_mov_b64 exec, 1\n\tglobal_atomic_add %0, %2, %3, off sc0\n\ts_mov_b64 exec, %1"
                             : "+v"(tk), "=&s"(sv) : "v"(qp), "v"(1u) : "memory");
            };
            unsigned spins = 0, fmt = 0, fpv = 0, sent = call;
            bool cast = false;
            unsigned long long t0 = 0;
            int tpv = 0;
            for (int t = pi, nx = pi + NPW;;) {
                place(t);
                asm volatile("s_waitcnt vmcnt(0)" : "+v"(tk) :: "memory");
                __builtin_amdgcn_s_waitcnt(0x0F70);
                asm volatile("" ::: "memory");
                if (t != pi) {
                    if (lane == 0)
                        __hip_atomic_store((GAS unsigned long long*)(pws + fpv), (unsigned long long)sent | ((unsigned long long)fmt << 32),
                                           __ATOMIC_RELAXED, __HIP_MEMORY_SCOPE_SYSTEM);
                    if (cast && lane == 0)
                        __hip_atomic_store((GAS unsigned long long*)(myws + OFF_GO + (unsigned)peer * 128u), ((rt() + AG_DELTA) << 32) | call,
                                           __ATOMIC_RELAXED, __HIP_MEMORY_SCOPE_AGENT);
                    if (t >= NTASKX - 64) lds_st(C_ABORT, 1u);
                    if (t >= nt) break;
                    nx = (int)min(rfl(tk) - (call - 1u) * (unsigned)(nt - NPW) + 2u * (unsigned)NPW, (unsigned)nt);
                }
                dof += OFF_DATA + (call & 1u) * PAR_BYTES;
                fpv = fof + (call & 1u) * FLG_PAR;
                Packed k;
                const bool eager = t < AG_EAGERW * CPL;
                const bool pass = same && fof >= OFF_FA + 4u * NALPHA * 512u;
                u32x4 ra[8];
                if (pass) {
                    unsigned long long t1 = 0;
                    while (lds_ld(wl + (fpv & 511u)) < call) {
                        if ((++spins & 63u) == 0u && expired(t1, DEADLINE)) { sent = 0u; break; }
                        __builtin_amdgcn_s_sleep(4);
                        ask(fpv & ~511u);
                        VM_WAIT(0);
                    }
                    k.fmt = lds_ld(wl + (fpv & 511u) + 4u);
#pragma unroll
                    for (int i = 0; i < 4; ++i) k.lo[i] = __builtin_amdgcn_raw_buffer_load_b128(rs, lane * 16, dof + i * 1024u, 17);
                    k.ca = __builtin_amdgcn_raw_buffer_load_b128(rs, lane * 16, dof + 4096u, 17);
                    k.cb = __builtin_amdgcn_raw_buffer_load_b128(rs, lane * 16, dof + 5120u, 17);
                    const __amdgpu_buffer_rsrc_t rh = mk_rsrc(myws, (k.fmt & F_ALL) ? WS_NEED : 0u);
#pragma unroll
                    for (int i = 0; i < 4; ++i) k.hi[i] = __builtin_amdgcn_raw_buffer_load_b128(rh, lane * 16, dof + 6144u + i * 1024u, 17);
                    __builtin_amdgcn_s_waitcnt(0x0F70);
                } else {
                    read_task(t, ra);
                    if (eager) { pack_low_odd(ra, k); send_low(k, rd, dof, lane); }
                }
                if (nx < nt) { load_task(nx); draw(); }
                if (!pass) {
                    if (t == pi || eager || ((fmt + (AG_NKEEP * 0x10100u)) & (F_ALL | 0x808000u)) != 0u || (!same && t >= 128 && tpv < 128))
                        fmt = pick_base(ra) | (eager ? 1u : 0u);
                    pack(ra, k, fmt & 0xFFu, wl + 65536u, lane);
                }
                tpv = t;
                fmt = k.fmt;
                unsigned my = 0;
                if (!eager) {
                    if (t == pi) __builtin_amdgcn_s_sleep(AG_W2D);
                    asm volatile("ds_add_rtn_u32 %0, %1, %2\n\ts_waitcnt lgkmcnt(0)" : "=&v"(my) : "v"(C_CNT), "v"(lane == 0 ? 1u : 0u) : "memory");
                    my = rfl(my);
                    while (lds_ld(C_GO) < my) {
                        if ((++spins & 1023u) == 0u && expired(t0, DEADLINE)) break;
                        __builtin_amdgcn_s_sleep(1);
                    }
                    send_low(k, rd, dof, lane);
                }
                send_high(k, rd, dof, lane);
                if (!eager) {
                    VM_WAIT(AG_REL);
                    lds_st(C_GO, my + 1u);
                }
                cast = t == nt - AG_GOT;
                t = (int)rfl((unsigned)nx);
            }
        }
        const unsigned v = (bid >> 3) * 7u + (bid & 7u) - ((bid & 7u) > spare ? 1u : 0u) + (unsigned)(NPWG * (7 - wave));
        const unsigned u = (v & 3u) * 64u + (v >> 2);
        const Tile t = tile_of((u >> 1) & 31u, obase + (int)((u >> 6) & 3u));
        if (v >= 256u || t.src == rank) return;
        unsigned long long t0 = 0, f;
        unsigned spins = 0;
        while (lds_ld(C_ABORT) == 0u) {
            if ((++spins & 63u) == 0u && expired(t0, DEADLINE)) break;
            __builtin_amdgcn_s_sleep(16);
        }
        for (;;) {
            f = __hip_atomic_load((const GAS unsigned long long*)(myws + OFF_DONE + u * 8u), __ATOMIC_RELAXED, __HIP_MEMORY_SCOPE_SYSTEM);
            if (__all((unsigned)f >= call)) break;
            if ((++spins & 255u) == 0u && (rfl(ld_agent(myws + OFF_TMO)) == call || expired(t0, 2 * DEADLINE))) {
                if (lane == 0) st_agent(myws + OFF_TMO, call);
                const unsigned long long t1 = rt();
                while (rt() - t1 < DEADLINE / 8) __builtin_amdgcn_s_sleep(64);
                if (lane == 0) st_sys(myout + (size_t)rank * 512u * 1024u, 0x7FC07FC0u);
                return;
            }
            __builtin_amdgcn_s_sleep(AG_UNP_SLEEP);
        }
        const unsigned fmt = rfl((unsigned)(f >> 32));
        if ((fmt & F_RAW) == 0u) {
            const unsigned off = OFF_DATA + (unsigned)(gidx * NALPHA) * ABLK_BYTES + u * CH_BYTES, lds = (unsigned)wave * 8192u;
            Wire w;
            wire_load(w, myws, true, off, lane);
            wire_unpack(w, myws, off, fmt, lane, lds);
            const __amdgpu_buffer_rsrc_t ro = mk_rsrc(myout, SHARD_BYTES);
            const unsigned ovo = (unsigned)(lane >> 4) * 1024u + (unsigned)(lane & 15) * 16u;
            const unsigned oso = ((unsigned)(t.src * 512 + t.blk * 64) + (u & 1u) * 32u) * 1024u + (u >> 6) * 256u;
#pragma unroll
            for (int j = 0; j < 8; ++j)
                __builtin_amdgcn_raw_buffer_store_b128(*(const volatile LAS u32x4*)(size_t)(lds + (unsigned)lane * 16u + j * 1024u), ro, ovo, oso + j * 4096u, AG_LOCAL_AUX);
        }
        return;
    }

    const unsigned ntile = (tcode >> 31) ? 2u : 1u;
    const unsigned par = OFF_DATA + (call & 1u) * PAR_BYTES;
    const unsigned callx = call;
    {
        const unsigned lds = (unsigned)(wave < NLOAD ? wave : 8 + wave) * 8192u;
        auto put_raw = [&](const u32x4 (&r)[8], unsigned c) INL {
            const __amdgpu_buffer_rsrc_t rs = mk_rsrc(myws, WS_NEED);
#pragma unroll
            for (int j = 0; j < 8; ++j) __builtin_amdgcn_raw_buffer_store_b128(r[j], rs, lane * 16, OFF_RAW + c * 8192u + j * 1024u, 17);
            VM_WAIT(0);
            if (lane == 0) st_sys(myws + OFF_FR + c * 4u, call);
        };
        u32x4 r[8];
        if (packs) {
            VM_WAIT(0);
            w_frags(r, lds, lane);
            put_raw(r, (own & 3u) * 128u + (own >> 2));
        }
        if (ntile == 1u && (tcode >> 16) != 0u) {
            const unsigned c = ((unsigned)wave & 3u) * 128u + ((tcode >> 16) - 1u) * 2u + ((unsigned)wave >> 2);
            const char* fp = myws + (call & 1u) * FLG_PAR + OFF_FW + c * 8u;
            unsigned long long t0 = 0, f;
            unsigned spins = 0;
            for (;;) {
                f = __hip_atomic_load((const GAS unsigned long long*)fp, __ATOMIC_RELAXED, __HIP_MEMORY_SCOPE_SYSTEM);
                if (__all((unsigned)f >= callx)) break;
                if ((++spins & 63u) == 0u && expired(t0, DEADLINE)) { f = 0ull; break; }
                __builtin_amdgcn_s_sleep(AG_CONV_SLEEP);
            }
            if (rfl((unsigned)f) != 0u) {
                const unsigned fmt = rfl((unsigned)(f >> 32)), off = par + OFF_WST + c * CH_BYTES;
                Wire w;
                wire_load(w, myws, true, off, lane);
                wire_unpack(w, myws, off, fmt, lane, lds);
#pragma unroll
                for (int j = 0; j < 8; ++j) r[j] = *(const volatile LAS u32x4*)(size_t)(lds + (unsigned)lane * 16u + j * 1024u);
                put_raw(r, 512u + c);
            }
        }
    }

    if (wave < NLOAD) {
        const unsigned lw = (unsigned)wave;
        __builtin_amdgcn_s_setprio(AG_LPRIO);
        const __amdgpu_buffer_rsrc_t rws = mk_rsrc(myws, WS_NEED);
        const unsigned fpar = (call & 1u) * FLG_PAR;
        unsigned spins = 0;
        unsigned long long t0 = 0;
        unsigned want = callx, room = (unsigned)NSLOT - 4u;
        FDECL;
        auto quit = [&]() INL {
            want = 0u; room = 1u << 30;
            lds_st(C_ABORT, 1u);
            if (lane == 0) st_agent(myws + OFF_TMO, call);
        };
        auto pause = [&]() INL {
            if ((++spins & 255u) == 0u && (rfl(ld_agent(myws + OFF_TMO)) == call || lds_ld(C_ABORT) != 0u || expired(t0, DEADLINE))) quit();
            __builtin_amdgcn_s_sleep(AG_POLL_SLEEP);
        };
        auto wait_slot = [&](unsigned g) INL {
            for (;;) {
                const u32x4 fr = lds_ld4(C_FREE);
                if (rfl(min(min(fr.x, fr.y), min(fr.z, fr.w))) + room > g) break;
                if ((++spins & 1023u) == 0u && (lds_ld(C_ABORT) != 0u || expired(t0, 2 * DEADLINE))) quit();
                __builtin_amdgcn_s_sleep(AG_FREE_SLEEP);
            }
        };
        unsigned wn = 0;
        for (unsigned ti = 0; ti < ntile; ++ti) {
            const Tile t = tile_of(tcode >> (16u * ti), rank);
            const unsigned g0 = ti * (unsigned)NKS + lw;
            auto publish = [&](unsigned i) INL {
                const unsigned need = i * 4u + lw + (unsigned)BD + 1u;
                if (__builtin_expect(wn < need, 0)) {
                    for (;;) {
                        const u32x2 v = __builtin_amdgcn_raw_buffer_load_b64(rws, OFF_FR + (unsigned)t.cb * 512u + (unsigned)lane * 8u, 0, 17);
                        const unsigned long long no = ~__ballot(v.x >= want && v.y >= want);
                        wn = no ? (unsigned)__builtin_ctzll(no) : 128u;
                        if (wn >= need) break;
                        pause();
                    }
                }
                lds_st(L_REC + (g0 + i * 4u) * 4u, 1u);
            };
            if (ti == ntile - 1u && wave == 0) STAMP(2);
            if (t.src != rank) {
                const unsigned base = par + (unsigned)t.rb * ABLK_BYTES + lw * CH_BYTES;
                const unsigned fl0 = fpar + OFF_FA + (unsigned)t.rb * 512u + lw * 128u;
                u32x2 fl = {0u, 0u};
                auto ask = [&]() INL {
                    fl = __builtin_amdgcn_raw_buffer_load_b64(rws, (unsigned)(lane & 15) * 8u, fl0, 17);
                };
                auto look = [&](unsigned i) INL -> bool {
                    const bool up = (unsigned)__builtin_amdgcn_readlane((int)fl.x, (int)(i & 15u)) >= want;
                    asm volatile("" ::: "memory");
                    return up;
                };
                auto await = [&](unsigned i) INL {
                    while (!look(i)) { pause(); ask(); }
                };
                auto fetch = [&](Wire& w, unsigned& f, unsigned i, bool go) INL {
                    f = (unsigned)__builtin_amdgcn_readlane((int)fl.y, (int)(i & 15u));
                    ask();
                    __builtin_amdgcn_sched_barrier(0);
                    wire_load(w, myws, go, base + (i & 31u) * (4u * CH_BYTES), lane);
                };
                auto turn = [&](const Wire& w, unsigned f, Wire& wx, unsigned& fx, unsigned i) INL {
                    const u32x4 fr = *(const volatile LAS u32x4*)(size_t)C_FREE;
                    const bool ok = look(i + 1u);
                    fetch(wx, fx, i + 1u, ok && i != 15u);
                    if (__builtin_expect(rfl(min(min(fr.x, fr.y), min(fr.z, fr.w))) + room <= g0 + i * 4u, 0)) { FACS(0); wait_slot(g0 + i * 4u); FACS(2); }
                    wire_unpack(w, myws, base + i * (4u * CH_BYTES), f, lane, ((g0 + i * 4u) & (unsigned)(NSLOT - 1)) * 8192u);
                    publish(i);
                    if (__builtin_expect(!ok, 0)) {
                        FACS(0);
                        await(i + 1u);
                        fetch(wx, fx, i + 1u, true);
                        FACS(5);
                    }
                };
                FACS(0);
                ask();
                Wire w0, w1;
                unsigned f0, f1;
                await(0u);
                fetch(w0, f0, 0u, true);
                FACS(1);
                for (unsigned i = 0; i < 16u; i += 2u) {
                    turn(w0, f0, w1, f1, i);
                    turn(w1, f1, w0, f0, i + 1u);
                }
            } else {
                const __amdgpu_buffer_rsrc_t rx = mk_rsrc(px, SHARD_BYTES);
                const unsigned v0 = x_lane_off(lane), v1 = v0 ^ 64u;
                const unsigned base = (unsigned)t.blk * (64u * 8192u) + lw * 128u;
                for (unsigned i = 0; i < 18u; ++i) {
                    if (i < 16u) {
                        wait_slot(g0 + i * 4u);
                        dma8(((g0 + i * 4u) & (unsigned)(NSLOT - 1)) * 8192u, v0, v1, rx, base + i * 512u, 65536u);
                        VM_WAIT(16);
                    } else if (i == 16u) {
                        VM_WAIT(8);
                    } else {
                        VM_WAIT(0);
                    }
                    if (i >= 2u) publish(i - 2u);
                }
                VM_WAIT(0);
            }
        }
        FACS(0);
        FOUT();
        const bool dead = want == 0u;
        if (wave == 0) {
            const int cb = tile_of(tcode >> (16u * (ntile - 1u)), rank).cb;
            const unsigned tau = (tcode & 0x200u) ? AG_TAUP : AG_TAU;
            if (cb >= 4) {
                char* hp = myws + OFF_GO + (unsigned)(obase + (cb & 3)) * 128u;
                unsigned long long go;
                t0 = 0;
                for (;;) {
                    go = __hip_atomic_load((const GAS unsigned long long*)hp, __ATOMIC_RELAXED, __HIP_MEMORY_SCOPE_AGENT);
                    if (dead || __all((unsigned)go == call)) break;
                    if ((++spins & 63u) == 0u && expired(t0, DEADLINE)) break;
                    __builtin_amdgcn_s_sleep(AG_GO_SLEEP);
                }
                const unsigned long long now = rt();
                const unsigned ahead = (unsigned)(go >> 32) - (unsigned)now;
                unsigned long long at = 0;
                if (lane == 0) {
                    __hip_atomic_fetch_max((GAS unsigned long long*)(hp + 1024), now + (ahead <= AG_DELTA ? ahead : 0u), __ATOMIC_RELAXED, __HIP_MEMORY_SCOPE_AGENT);
                    at = __hip_atomic_fetch_add((GAS unsigned long long*)(hp + 1024), (unsigned long long)tau, __ATOMIC_RELAXED, __HIP_MEMORY_SCOPE_AGENT);
                }
                const unsigned wt = rfl(at > now ? (unsigned)min(at - now, 2000ull) : 0u);
                while ((unsigned)(rt() - now) < wt) __builtin_amdgcn_s_sleep(1);
                lds_st(C_GO, 1u);
            }
        }
        if (wave == 3) STAMP(3);
        return;
    }

    const int cw = wave - NLOAD;
    __builtin_amdgcn_s_setprio(AG_PRIO);
    const int hh = lane >> 5, r32 = lane & 31;
    const unsigned a0 = (unsigned)(r32 * 128 + ((hh ^ ((r32 >> 1) & 7)) << 4));
    const unsigned cfree = C_FREE + (unsigned)cw * 4u;
    lds_st(cfree, 4u);
    const unsigned vw = (unsigned)lane * 16u + (unsigned)cw * 4096u;

    unsigned spins = 0;
    unsigned long long t0 = 0;
    FDECL;
    auto wait_full = [&](unsigned g) INL {
        while (lds_ld(L_REC + g * 4u) == 0u) {
            if ((++spins & 63u) == 0u && lds_ld(C_ABORT) != 0u) break;
            if ((spins & 1023u) == 0u && expired(t0, 3 * DEADLINE)) { lds_st(C_ABORT, 1u); break; }
            __builtin_amdgcn_s_sleep(1);
        }
    };
    unsigned ab[4];
#pragma unroll
    for (int k = 0; k < 4; ++k) ab[k] = a0 ^ (unsigned)(k * 32);
    auto rd = [&](Frag& f, auto sc) INL {
        constexpr unsigned S = decltype(sc)::value;
#pragma unroll
        for (int k = 0; k < 4; ++k)
#pragma unroll
            for (int mt = 0; mt < 2; ++mt)
                f.a[k][mt] = *(const LAS s16x8*)((const LAS unsigned char*)(size_t)ab[k] + (S * 8192u + mt * 4096u));
    };
    f32x16 acc[2];
    unsigned vrec = L_REC;
    asm volatile("" : "+v"(vrec));
#define SB() __builtin_amdgcn_sched_barrier(0)

    const unsigned wcb = (unsigned)tile_of(tcode, rank).cb, wbase = OFF_RAW + (wcb << 20);
    char *dws = p.oth[wcb & 3u][0], *dout = p.oth[wcb & 3u][1];
    asm volatile("" : "+s"(dws), "+s"(dout));
    const __amdgpu_buffer_rsrc_t rb = mk_rsrc(myws, WS_NEED);
    Frag fa;
    FragB pb[BD];
    unsigned old = 0;
    wait_full(0u);
#pragma unroll
    for (int i = 0; i < BD; ++i) {
#pragma unroll
        for (int k = 0; k < 4; ++k) pb[i].b[k] = __builtin_amdgcn_raw_buffer_load_b128(rb, vw + k * 1024u, wbase + (unsigned)i * 16384u, AG_WAUX);
        SB();
    }
    wait_full(0u);
    FACS(3);
    rd(fa, IC<0>());
    __builtin_amdgcn_s_waitcnt(0xC07F);
    for (unsigned ti = 0; ti < ntile; ++ti) {
#pragma unroll
        for (int i = 0; i < 16; ++i) { acc[0][i] = 0.f; acc[1][i] = 0.f; }
        const Tile t = tile_of(tcode >> (16u * ti), rank);
        const unsigned g0 = ti * (unsigned)NKS;
        auto group = [&](unsigned o, auto mc) INL {
            constexpr bool MORE = decltype(mc)::value != 0;
            const unsigned s0 = o * (unsigned)BD, wo = wbase + ((s0 + (unsigned)BD) & (unsigned)(NKS - 1)) * 16384u;
            auto step = [&](auto uc) INL {
                constexpr int U = decltype(uc)::value, X = (U + 1) & (BD - 1);
                constexpr bool NEXT = MORE || U != BD - 1;
                const unsigned g = g0 + s0 + (unsigned)U + 1u;
                if (U == BD - 1) {
#pragma unroll
                    for (int k = 0; k < 4; ++k) ab[k] ^= 65536u;
                }
                unsigned rec = 1u;
                if (NEXT) rec = *(const volatile LAS unsigned*)((const LAS unsigned char*)(size_t)vrec + (U + 1) * 4);
                SB();
#pragma unroll
                for (int k = 0; k < 4; ++k) {
                    acc[0] = __builtin_amdgcn_mfma_f32_32x32x16_bf16(__builtin_bit_cast(bf16x8, pb[U].b[k]), __builtin_bit_cast(bf16x8, fa.a[k][0]), acc[0], 0, 0, 0);
                    SB();
                    if (NEXT) fa.a[k][0] = *(const LAS s16x8*)((const LAS unsigned char*)(size_t)ab[k] + X * 8192);
                    SB();
                    acc[1] = __builtin_amdgcn_mfma_f32_32x32x16_bf16(__builtin_bit_cast(bf16x8, pb[U].b[k]), __builtin_bit_cast(bf16x8, fa.a[k][1]), acc[1], 0, 0, 0);
                    SB();
                    if (MORE) pb[U].b[k] = __builtin_amdgcn_raw_buffer_load_b128(rb, vw + k * 1024u, wo + (unsigned)U * 16384u, AG_WAUX);
                    if (NEXT) fa.a[k][1] = *(const LAS s16x8*)((const LAS unsigned char*)(size_t)ab[k] + (X * 8192 + 4096));
                    SB();
                }
                if ((U & 3) == 3) lds_st(cfree, g + 4u);
                if (NEXT && __builtin_expect(rfl(rec) == 0u, 0)) {
                    FACS(0);
                    lds_st(cfree, g + 4u);
                    wait_full(g);
                    FACC(1);
                    rd(fa, IC<X>());
                    __builtin_amdgcn_s_waitcnt(0xC07F);
                    FACS(5);
                }
                SB();
            };
            step(IC<0>()); step(IC<1>()); step(IC<2>()); step(IC<3>());
            step(IC<4>()); step(IC<5>()); step(IC<6>()); step(IC<7>());
            vrec += (unsigned)BD * 4u;
        };
        const bool fin = ti + 1u == ntile;
        for (unsigned o = fin ? 1u : 0u; o < (unsigned)(NKS / BD); ++o) group(o - (fin ? 1u : 0u), IC<1>());
        if (fin) group((unsigned)(NKS / BD) - 1u, IC<0>());
        if (cw == 0) STAMP(7);
        FACS(0);
        FOUT();

        const bool bad = lds_ld(C_ABORT) != 0u;
#pragma unroll
        for (int mt = 0; mt < 2; ++mt) {
#pragma unroll
            for (int j = 0; j < 4; ++j) {
                bf16x2 lo, hi;
                lo[0] = (__bf16)acc[mt][4 * j]; lo[1] = (__bf16)acc[mt][4 * j + 1];
                hi[0] = (__bf16)acc[mt][4 * j + 2]; hi[1] = (__bf16)acc[mt][4 * j + 3];
                u32x2 v; v.x = __builtin_bit_cast(unsigned, lo); v.y = __builtin_bit_cast(unsigned, hi);
                if (bad) { v.x = 0x7FC07FC0u; v.y = 0x7FC07FC0u; }
                *(volatile LAS u32x2*)(size_t)(L_ST + (unsigned)((mt * 32 + r32) * ST_ROW + cw * 64 + j * 16 + hh * 8)) = v;
            }
        }
        asm volatile("ds_add_rtn_u32 %0, %1, %2\n\ts_waitcnt lgkmcnt(0)" : "=&v"(old) : "v"(C_CNT), "v"(lane == 0 ? 1u : 0u) : "memory");
        old = rfl(old);
        if (wcb < 4u && (old & 3u) == 3u) {
            int ln = lane;
            asm volatile("" : "+v"(ln));
            const unsigned ovo = (unsigned)(ln >> 4) * 1024u + (unsigned)(ln & 15) * 16u;
            const unsigned sto = L_ST + (unsigned)((ln >> 4) * ST_ROW + (ln & 15) * 16);
            const __amdgpu_buffer_rsrc_t ro = mk_rsrc(myout, SHARD_BYTES);
            const unsigned oso = (unsigned)(t.src * 512 + t.blk * 64) * 1024u + wcb * 256u;
#pragma unroll
            for (int h = 0; h < 2; ++h) {
                u32x4 o[8];
#pragma unroll
                for (int i = 0; i < 8; ++i) o[i] = *(const volatile LAS u32x4*)(size_t)(sto + (unsigned)((h * 8 + i) * 4 * ST_ROW));
#pragma unroll
                for (int i = 0; i < 8; ++i) __builtin_amdgcn_raw_buffer_store_b128(o[i], ro, ovo, oso + (h * 8 + i) * 4096u, AG_LOCAL_AUX);
                SB();
            }
            __builtin_amdgcn_s_waitcnt(0xC07F);
            STAMP(8);
        }
    }
    const unsigned h = old - 2u;
    if (wcb >= 4u && h <= 1u) {
        const Tile t = tile_of(tcode, rank);
        int ln = lane;
        asm volatile("" : "+v"(ln));
        const unsigned ovo = (unsigned)(ln >> 4) * 1024u + (unsigned)(ln & 15) * 16u;
        const unsigned sto = L_ST + (unsigned)((ln >> 4) * ST_ROW + (ln & 15) * 16);
        t0 = 0;
        while (lds_ld(C_CNT) < 4u) {
            if ((++spins & 1023u) == 0u && expired(t0, 4 * DEADLINE)) break;
        }
        u32x4 o[8];
#pragma unroll
        for (int i = 0; i < 8; ++i) o[i] = *(const volatile LAS u32x4*)(size_t)(sto + (h * 8u + (unsigned)i) * (4u * ST_ROW));
        const int dstr = obase + (t.cb & 3);
        const unsigned u = (unsigned)(gidx * 32 + t.rb) * 2u + h;
        auto hold = [&]() INL {
            t0 = 0;
            while ((tcode & 0x400u) == 0u && lds_ld(C_GO) == 0u) {
                if ((++spins & 1023u) == 0u && expired(t0, 2 * DEADLINE)) break;
                __builtin_amdgcn_s_sleep(1);
            }
            STAMP(4);
        };
        unsigned fmt = F_RAW;
        if ((tcode & 0x200u) != 0u) {
            Packed k;
            pack(o, k, pick_base(o), h * 8192u, ln);
            hold();
            const unsigned dof = OFF_DATA + (unsigned)((t.cb & 3) * NALPHA) * ABLK_BYTES + u * CH_BYTES;
            send_low(k, mk_rsrc(dws, WS_NEED), dof, ln);
            send_high(k, mk_rsrc(dws, WS_NEED), dof, ln);
            fmt = k.fmt;
        } else {
            const __amdgpu_buffer_rsrc_t ro = mk_rsrc(dout, SHARD_BYTES);
            const unsigned oso = ((unsigned)(t.src * 512 + t.blk * 64) + h * 32u) * 1024u + (unsigned)gidx * 256u;
            hold();
#pragma unroll
            for (int i = 0; i < 8; ++i) __builtin_amdgcn_raw_buffer_store_b128(o[i], ro, ovo, oso + i * 4096u, 17);
        }
        VM_WAIT(0);
        if (lane == 0)
            __hip_atomic_store((GAS unsigned long long*)(dws + OFF_DONE + u * 8u), (unsigned long long)call | ((unsigned long long)fmt << 32),
                               __ATOMIC_RELAXED, __HIP_MEMORY_SCOPE_SYSTEM);
        STAMP(8);
    }
}

}

extern "C" void kernel_launch(int rank, int nranks, ncclComm_t comm,
                              hipStream_t stream,
                              void* const* d_in, const int* in_sizes, int n_in,
                              void* d_out, int out_size,
                              const int* in_dtypes, int out_dtype,
                              void* const* d_out_all,
                              void* const* d_ws_all, size_t ws_bytes) {
    using namespace agk;
    (void)comm; (void)in_dtypes; (void)out_dtype;
    if (nranks != NR || (unsigned)rank >= (unsigned)NR || n_in < 2 || ws_bytes < (size_t)WS_NEED ||
        in_sizes[0] != 512 * 4096 || in_sizes[1] != 512 * 4096 || out_size != 512 * 4096) {
        fprintf(stderr, "kernel_launch: unexpected problem shape\n");
        return;
    }
    Params p;
    p.x = d_in[0];
    p.w = d_in[1];
    p.ws = (char*)d_ws_all[rank];
    p.out = (char*)d_out;
    for (int i = 0; i < 5; ++i) p.pad[i] = 0u;
    for (int c = 0; c < 4; ++c) { p.oth[c][0] = (char*)d_ws_all[((rank & 4) ^ 4) + c]; p.oth[c][1] = (char*)d_out_all[((rank & 4) ^ 4) + c]; }
    p.rank = rank;
    const int gidx = rank & 3, obase = (rank & 4) ^ 4;
    int xcd_of[NR], spare = 3;
    {
#define AG_SETMASK 0xB2u
        const unsigned setmask = AG_SETMASK;
        int lo = 0, hi = 4;
        for (int o = 1; o < NR; ++o) {
            const int q = (rank + o) & 7;
            xcd_of[q] = (((setmask >> q) ^ (setmask >> rank)) & 1u) ? hi++ : lo++;
        }
        xcd_of[rank] = spare;
        p.spare = (unsigned)spare;
        p.xpeer = 0u;
        for (int q = 0; q < NR; ++q) {
            p.xpeer |= (unsigned)(q == rank ? 15 : q) << (xcd_of[q] * 4);
            p.xws[xcd_of[q]] = (char*)d_ws_all[q];
        }
    }
    {
        static const int seg[2][4][4] = {AG_SEG_OWN, AG_SEG_OTH};
        for (int g = 0; g < 2; ++g) {
            unsigned char o[256];
            int n = 0, ks[4] = {0, 0, 0, 0};
            for (int i = 0; i < 256; ++i) o[i] = 0;
            for (int q = 0; q < 4; ++q) {
                const int* c = seg[g][q];
                int got[4] = {0, 0, 0, 0};
                for (int i = 0, tot = c[0] + c[1] + c[2] + c[3]; i < tot && n < 256; ++i) {
                    int r = 0;
                    for (int j = 1; j < 4; ++j)
                        if (c[j] * (2 * i + 1) - 2 * tot * got[j] > c[r] * (2 * i + 1) - 2 * tot * got[r]) r = j;
                    ++got[r];
                    o[n++] = (unsigned char)((r << 6) | (ks[r]++ & 63));
                }
            }
            if (ks[0] != 64 || ks[1] != 64 || ks[2] != (g ? 64 - NFWD : 64) || ks[3] != (g ? NFWD / 4 : NFWD)) {
                fprintf(stderr, "kernel_launch: order\n");
                return;
            }
            for (int i = 0; i < 64; ++i)
                p.order[i][g] = o[4 * i] | (unsigned)o[4 * i + 1] << 8 | (unsigned)o[4 * i + 2] << 16 | (unsigned)o[4 * i + 3] << 24;
        }
    }
    auto phase = [&](int rb) {
        if (rb >= 32 || (rb < 4 * NALPHA && rb / NALPHA == gidx)) return 0;
        if (rb >= 4 * NALPHA) return 2 + (rb - 4 * NALPHA) % NBETA;
        return rb % NALPHA < NEARLY ? 1 : rb % NALPHA;
    };
    auto code_of = [](int rb, int cb) { return (unsigned)rb | ((unsigned)cb << 6) | 0x8000u; };
    int next[8];
    for (int x = 0; x < 8; ++x) next[x] = x != spare ? x + 8 * CPL : x;
    for (int b = 0; b < NWG; ++b) p.tile[b] = 0u;
    int ntile = 0;
    int late_x[4], used = 0;
    for (int pass = 0; pass < 2; ++pass)
        for (int c = 0; c < 4; ++c)
            for (int o = 0; o < 4; ++o) {
                const int x = xcd_of[(rank & 4) + o];
                if ((pass == 1 && late_x[c] >= 0)) break;
                late_x[c] = -1;
                if (((used >> x) & 1) || (pass == 0 && (x >> 2) != (xcd_of[obase + c] >> 2))) continue;
                late_x[c] = x; used |= 1 << x;
                break;
            }
    for (int c = 0; c < 4; ++c)
        for (int rb = 0, early = 0; rb < 32; ++rb) {
            const bool his = rb >= 4 * NALPHA && (rb - 4 * NALPHA) / NBETA == c;
            const int ph = phase(rb);
            for (int loc = 0; loc < 2; ++loc) {
                if (loc ? ph < 2 : his) continue;
                const int xx = ph <= 2 ? xcd_of[obase + c] : his ? spare : late_x[c];
                if (next[xx] >= NWG) { fprintf(stderr, "kernel_launch: placement\n"); return; }
                unsigned code = loc ? code_of(rb, c) : code_of(rb, 4 + c) | (ph <= (rb >= 4 * NALPHA ? AG_PACKX : AG_PACKPH) ? 0x200u : 0u) | (ph >= AG_FREEPH ? 0x400u : 0u);
                if (loc && (ph == AG_PAIR0 || ph == AG_PAIR1)) {
                    int f = -1;
                    for (int want = ph == AG_PAIR1, n = 0; n < 2 && f < 0; ++n, want ^= 1)
                        for (int r = 0; r < 35 && f < 0; ++r)
                            if (phase(r) == want && !((early >> (r < 32 ? r : r - 12)) & 1)) f = r;
                    if (f < 0) { fprintf(stderr, "kernel_launch: pairing\n"); return; }
                    early |= 1 << (f < 32 ? f : f - 12);
                    code = code_of(f, c) | (code << 16);
                    ++ntile;
                }
                p.tile[next[xx]] = code;
                next[xx] += 8;
                ++ntile;
            }
        }
    if (ntile != NTILE) { fprintf(stderr, "kernel_launch: tiles %d\n", ntile); return; }
    {
        int k = NKS;
        for (int ph = 4; ph >= 2; --ph)
            for (int b = 0; b < NWG && k > 0; ++b)
                if (p.tile[b] != 0u && (p.tile[b] >> 16) == 0u && phase((int)(p.tile[b] & 63u)) == ph) p.tile[b] |= (unsigned)(k--) << 16;
        if (k != 0) { fprintf(stderr, "kernel_launch: unpackers\n"); return; }
    }
    hipLaunchKernelGGL(ag_gemm_kernel, dim3(NWG), dim3(NTHREADS), 0, stream, p);
    HIP_CHECK(hipGetLastError());
}
```
